```python
import numpy as np

kernel_name = "gpu_a2a_gemm_m4096_k4096_n16384_e4m3e4m3_dqrelu"


ABSMAX_RTOL = 0.03

COMPUTE_DTYPE = "float32"
EPILOGUE = "dequant_relu"
IN_QUANT = None
OUT_QUANT = None
IN_FP8 = "e4m3e4m3"
IN_DTYPES = {"x": "e4m3", "w_mat": "e4m3",
             "scale_x": "float32", "scale_w": "float32"}
WORLD_SIZE = 8

def _snap_fp8(x, mant_bits, max_abs, min_normal):
    a = np.abs(x.astype(np.float64))
    _m, _e = np.frexp(a)
    q = np.where(a >= min_normal, np.ldexp(1.0, _e - 1 - mant_bits),
                 min_normal * 2.0 ** -mant_bits)
    snapped = np.minimum(np.round(a / q) * q, max_abs)
    return (np.sign(x) * snapped).astype(np.float32)

def setup_inputs():

    x = _snap_fp8(np.random.randn(4096, 4096).astype(np.float32), 3, 448.0, 2.0 ** -6)
    w_mat = _snap_fp8(np.random.randn(4096, 16384).astype(np.float32), 3, 448.0, 2.0 ** -6)
    scale_x = (0.02 * (1.0 + np.random.rand(1))).astype(np.float32)
    scale_w = (0.02 * (1.0 + np.random.rand(1))).astype(np.float32)
    return {"x": x, "w_mat": w_mat, "scale_x": scale_x, "scale_w": scale_w}

def reference(x, w_mat, scale_x, scale_w):
    acc = (x @ w_mat).astype(np.float32)
    y = (acc * (scale_x * scale_w)).astype(np.float32)
    return np.maximum(y, 0.0).astype(np.float32)
```

```cpp
#include <hip/hip_runtime.h>
#include <rccl/rccl.h>
#include <cstddef>
#include <cstdio>

#define LAS __attribute__((address_space(3)))

namespace {
constexpr int NRANKS = 8, MLOC = 512, KK = 4096, NN = 16384;
constexpr unsigned X_OFF = 1u << 20, X_SLOT = 2u << 20;
constexpr size_t WS_NEEDED = X_OFF + 4 * (size_t)X_SLOT;
constexpr int LDS_BYTES = 163840;
struct Args {
  const unsigned char* x; const unsigned char* w; const float* sx; const float* sw; float* out;
  unsigned char* ws[NRANKS];
  void* pad0;
  int rank; int pad1;
};
static_assert(offsetof(Args, out) == 0x20 && offsetof(Args, ws) == 0x28 && offsetof(Args, rank) == 0x70 && sizeof(Args) == 0x78, "kernarg layout");
}

#define GEMM_ASM_TEXT \
  "START_%=:\n\t" \
  "s_load_dwordx8 s[24:31], %[karg], 0x0\n\t" \
  "s_load_dwordx2 s[48:49], %[karg], 0x20\n\t" \
  "s_load_dwordx2 s[70:71], %[karg], 0x28\n\t" \
  "s_load_dword s35, %[karg], 0x70\n\t" \
  "s_mul_i32 s12, %[bid], 1171\n\t" \
  "s_lshr_b32 s12, s12, 16\n\t" \
  "s_mul_i32 s13, s12, 56\n\t" \
  "s_sub_u32 s13, %[bid], s13\n\t" \
  "s_mul_i32 s14, s13, 37\n\t" \
  "s_lshr_b32 s14, s14, 8\n\t" \
  "s_mul_i32 s15, s14, 7\n\t" \
  "s_sub_u32 s9, s13, s15\n\t" \
  "s_add_u32 s9, s9, 1\n\t" \
  "s_lshl_b32 s15, s9, 3\n\t" \
  "s_add_u32 s15, s15, 0x28\n\t" \
  "s_load_dwordx2 s[22:23], %[karg], s15\n\t" \
  "v_lshrrev_b32 v5, 6, %[tid]\n\t" \
  "v_and_b32 v4, 63, %[tid]\n\t" \
  "v_readfirstlane_b32 s4, v5\n\t" \
  "v_mov_b32 v88, 0\n\t" \
  "v_lshrrev_b32 v5, 4, v4\n\t" \
  "v_and_b32 v6, 7, v4\n\t" \
  "v_xor_b32 v6, v5, v6\n\t" \
  "v_lshlrev_b32 v6, 4, v6\n\t" \
  "v_lshrrev_b32 v7, 3, v4\n\t" \
  "v_lshl_add_u32 v24, v7, 9, v6\n\t" \
  "v_xor_b32 v25, 64, v24\n\t" \
  "v_lshl_add_u32 v80, v7, 14, v6\n\t" \
  "v_add_u32 v81, 131072, v80\n\t" \
  "v_xor_b32 v82, 64, v80\n\t" \
  "v_add_u32 v82, 262144, v82\n\t" \
  "v_add_u32 v83, 131072, v82\n\t" \
  "s_and_b32 s15, %[bid], 3\n\t" \
  "s_cmp_eq_u32 s4, s15\n\t" \
  "s_cselect_b32 s63, s12, 255\n\t" \
  "s_cselect_b32 s3, 1, 0\n\t" \
  "s_cmp_lt_u32 %[bid], 224\n\t" \
  "s_cselect_b32 s63, s63, 255\n\t" \
  "s_add_u32 s84, %[bid], 1\n\t" \
  "s_and_b32 s84, s84, 3\n\t" \
  "s_cmp_eq_u32 s4, s84\n\t" \
  "s_cselect_b32 s3, 2, s3\n\t" \
  "s_cmp_lt_u32 %[bid], 0\n\t" \
  "s_cselect_b32 s3, s3, 0\n\t" \
  "s_sub_u32 s85, s15, s4\n\t" \
  "s_mov_b32 s84, s12\n\t" \
  "s_and_b32 s12, s12, 3\n\t" \
  "s_lshl_b32 s16, s14, 15\n\t" \
  "s_lshl_b32 s86, s12, 7\n\t" \
  "s_add_u32 s16, s16, s86\n\t" \
  "s_mul_i32 s12, s12, 7\n\t" \
  "s_sub_u32 s12, s12, s9\n\t" \
  "s_add_u32 s12, s12, 11\n\t" \
  "s_lshl_b32 s6, s12, 16\n\t" \
  "s_lshl_b32 s86, s14, 13\n\t" \
  "s_add_u32 s6, s6, s86\n\t" \
  "s_lshl_b32 s7, s12, 2\n\t" \
  "s_lshl_b32 s86, s14, 7\n\t" \
  "s_add_u32 s7, s7, s86\n\t" \
  "s_lshl_b32 s8, %[bid], 7\n\t" \
  "s_add_u32 s8, s8, 32768\n\t" \
  "s_mov_b32 s15, s84\n\t" \
  "s_mov_b32 s14, s85\n\t" \
  "s_lshr_b32 s11, %[bid], 3\n\t" \
  "s_and_b32 s10, s11, 1\n\t" \
  "s_and_b32 s12, %[bid], 7\n\t" \
  "s_lshl_b32 s12, s12, 4\n\t" \
  "s_lshr_b32 s13, s11, 1\n\t" \
  "s_add_u32 s12, s12, s13\n\t" \
  "s_lshl_b32 s12, s12, 7\n\t" \
  "s_lshl_b32 s5, s4, 19\n\t" \
  "s_add_u32 s5, s5, s12\n\t" \
  "s_lshl_b32 s13, s10, 2\n\t" \
  "s_add_u32 s13, s13, s4\n\t" \
  "s_lshl_b32 s17, s13, 15\n\t" \
  "s_lshl_b32 s62, s13, 13\n\t" \
  "s_lshl_b32 s60, s4, 12\n\t" \
  "s_mul_i32 s34, s4, 24576\n\t" \
  "s_add_u32 s34, s34, 65536\n\t" \
  "s_waitcnt lgkmcnt(0)\n\t" \
  "s_and_b32 s35, s35, 7\n\t" \
  "s_add_u32 s91, s35, s9\n\t" \
  "s_and_b32 s91, s91, 7\n\t" \
  "s_lshl_b32 s91, s91, 18\n\t" \
  "s_add_u32 s91, s91, s16\n\t" \
  "s_mov_b64 s[40:41], s[24:25]\n\t" \
  "s_and_b32 s41, s41, 0xffff\n\t" \
  "s_mov_b32 s42, 2097152\n\t" \
  "s_mov_b32 s43, 0x20000\n\t" \
  "s_mov_b64 s[36:37], s[26:27]\n\t" \
  "s_and_b32 s37, s37, 0xffff\n\t" \
  "s_mov_b32 s38, 67108864\n\t" \
  "s_mov_b32 s39, 0x20000\n\t" \
  "s_mov_b64 s[80:81], s[28:29]\n\t" \
  "s_mov_b64 s[82:83], s[30:31]\n\t" \
  "s_add_u32 s92, s70, s8\n\t" \
  "s_addc_u32 s93, s71, 0\n\t" \
  "s_lshl_b32 s61, s35, 18\n\t" \
  "s_add_u32 s61, s61, s17\n\t" \
  "s_lshl_b32 s84, s35, 23\n\t" \
  "s_add_u32 s59, s84, s5\n\t" \
  "s_lshl_b32 s26, s14, 19\n\t" \
  "s_lshl_b32 s27, s14, 15\n\t" \
  "s_lshl_b32 s84, s14, 12\n\t" \
  "s_add_u32 s24, s60, s84\n\t" \
  "s_mul_i32 s84, s14, 24576\n\t" \
  "s_add_u32 s25, s34, s84\n\t" \
  "s_cmp_lg_u32 s4, 2\n\t" \
  "s_cbranch_scc1 XW1_%=\n\t" \
  "v_lshrrev_b32 v5, 4, v4\n\t" \
  "v_lshlrev_b32 v5, 7, v5\n\t" \
  "v_and_b32 v6, 15, v4\n\t" \
  "v_lshl_add_u32 v5, v6, 9, v5\n\t" \
  "s_lshl_b32 s84, s35, 18\n\t" \
  "s_lshl_b32 s85, s11, 13\n\t" \
  "s_add_u32 s84, s84, s85\n\t" \
  "buffer_load_dword v8, v5, s[40:43], s84 offen\n\t" \
  "XW1_%=:\n\t" \
  "s_cmp_gt_u32 s63, 1\n\t" \
  "s_cbranch_scc1 PL5_%=\n\t" \
  "s_add_u32 s96, s91, 0\n\t" \
  "s_add_u32 m0, s34, 16384\n\t" \
  "s_nop 0\n\t" \
  "buffer_load_dwordx4 v24, s[40:43], s96 offen lds\n\t" \
  "s_add_u32 s96, s91, 4096\n\t" \
  "s_add_u32 m0, s34, 17408\n\t" \
  "s_nop 0\n\t" \
  "buffer_load_dwordx4 v25, s[40:43], s96 offen lds\n\t" \
  "s_add_u32 s96, s91, 8192\n\t" \
  "s_add_u32 m0, s34, 18432\n\t" \
  "s_nop 0\n\t" \
  "buffer_load_dwordx4 v24, s[40:43], s96 offen lds\n\t" \
  "s_add_u32 s96, s91, 12288\n\t" \
  "s_add_u32 m0, s34, 19456\n\t" \
  "s_nop 0\n\t" \
  "buffer_load_dwordx4 v25, s[40:43], s96 offen lds\n\t" \
  "s_add_u32 s96, s91, 16384\n\t" \
  "s_add_u32 m0, s34, 20480\n\t" \
  "s_nop 0\n\t" \
  "buffer_load_dwordx4 v24, s[40:43], s96 offen lds\n\t" \
  "s_add_u32 s96, s91, 20480\n\t" \
  "s_add_u32 m0, s34, 21504\n\t" \
  "s_nop 0\n\t" \
  "buffer_load_dwordx4 v25, s[40:43], s96 offen lds\n\t" \
  "s_add_u32 s96, s91, 24576\n\t" \
  "s_add_u32 m0, s34, 22528\n\t" \
  "s_nop 0\n\t" \
  "buffer_load_dwordx4 v24, s[40:43], s96 offen lds\n\t" \
  "s_add_u32 s96, s91, 28672\n\t" \
  "s_add_u32 m0, s34, 23552\n\t" \
  "s_nop 0\n\t" \
  "buffer_load_dwordx4 v25, s[40:43], s96 offen lds\n\t" \
  "PL5_%=:\n\t" \
  "global_load_dword v89, v88, s[92:93] sc1\n\t" \
  "s_add_u32 s84, s61, 0\n\t" \
  "s_add_u32 m0, s34, 0\n\t" \
  "s_nop 0\n\t" \
  "buffer_load_dwordx4 v24, s[40:43], s84 offen lds\n\t" \
  "s_add_u32 s84, s61, 4096\n\t" \
  "s_add_u32 m0, s34, 1024\n\t" \
  "s_nop 0\n\t" \
  "buffer_load_dwordx4 v25, s[40:43], s84 offen lds\n\t" \
  "s_add_u32 s84, s61, 8192\n\t" \
  "s_add_u32 m0, s34, 2048\n\t" \
  "s_nop 0\n\t" \
  "buffer_load_dwordx4 v24, s[40:43], s84 offen lds\n\t" \
  "s_add_u32 s84, s61, 12288\n\t" \
  "s_add_u32 m0, s34, 3072\n\t" \
  "s_nop 0\n\t" \
  "buffer_load_dwordx4 v25, s[40:43], s84 offen lds\n\t" \
  "s_add_u32 s84, s61, 16384\n\t" \
  "s_add_u32 m0, s34, 4096\n\t" \
  "s_nop 0\n\t" \
  "buffer_load_dwordx4 v24, s[40:43], s84 offen lds\n\t" \
  "s_add_u32 s84, s61, 20480\n\t" \
  "s_add_u32 m0, s34, 5120\n\t" \
  "s_nop 0\n\t" \
  "buffer_load_dwordx4 v25, s[40:43], s84 offen lds\n\t" \
  "s_add_u32 s84, s61, 24576\n\t" \
  "s_add_u32 m0, s34, 6144\n\t" \
  "s_nop 0\n\t" \
  "buffer_load_dwordx4 v24, s[40:43], s84 offen lds\n\t" \
  "s_add_u32 s84, s61, 28672\n\t" \
  "s_add_u32 m0, s34, 7168\n\t" \
  "s_nop 0\n\t" \
  "buffer_load_dwordx4 v25, s[40:43], s84 offen lds\n\t" \
  "s_add_u32 s88, s59, 0\n\t" \
  "s_add_u32 m0, s60, 0\n\t" \
  "s_nop 0\n\t" \
  "buffer_load_dwordx4 v80, s[36:39], s88 offen  lds\n\t" \
  "s_add_u32 m0, s60, 1024\n\t" \
  "s_nop 0\n\t" \
  "buffer_load_dwordx4 v81, s[36:39], s88 offen  lds\n\t" \
  "s_add_u32 m0, s60, 2048\n\t" \
  "s_nop 0\n\t" \
  "buffer_load_dwordx4 v82, s[36:39], s88 offen  lds\n\t" \
  "s_add_u32 m0, s60, 3072\n\t" \
  "s_nop 0\n\t" \
  "buffer_load_dwordx4 v83, s[36:39], s88 offen  lds\n\t" \
  "s_add_u32 s88, s59, 2097152\n\t" \
  "s_add_u32 m0, s60, 16384\n\t" \
  "s_nop 0\n\t" \
  "buffer_load_dwordx4 v80, s[36:39], s88 offen  lds\n\t" \
  "s_add_u32 m0, s60, 17408\n\t" \
  "s_nop 0\n\t" \
  "buffer_load_dwordx4 v81, s[36:39], s88 offen  lds\n\t" \
  "s_add_u32 m0, s60, 18432\n\t" \
  "s_nop 0\n\t" \
  "buffer_load_dwordx4 v82, s[36:39], s88 offen  lds\n\t" \
  "s_add_u32 m0, s60, 19456\n\t" \
  "s_nop 0\n\t" \
  "buffer_load_dwordx4 v83, s[36:39], s88 offen  lds\n\t" \
  "FBE4_%=:\n\t" \
  "s_sub_u32 s84, s15, 2\n\t" \
  "s_cmp_gt_u32 s84, 1\n\t" \
  "s_cbranch_scc1 PW6_%=\n\t" \
  "v_and_b32 v14, 15, v4\n\t" \
  "s_lshl_b32 s84, s4, 4\n\t" \
  "v_add_u32 v14, s84, v14\n\t" \
  "v_lshlrev_b32 v14, 9, v14\n\t" \
  "buffer_load_dword v8, v14, s[40:43], s91 offen\n\t" \
  "PW6_%=:\n\t" \
  "s_load_dword s94, s[80:81], 0\n\t" \
  "s_load_dword s95, s[82:83], 0\n\t" \
  "s_add_u32 s98, s70, 65536\n\t" \
  "s_addc_u32 s99, s71, 0\n\t" \
  "v_lshlrev_b32 v84, 4, v4\n\t" \
  "v_lshrrev_b32 v5, 4, v4\n\t" \
  "v_bfe_u32 v6, v4, 1, 3\n\t" \
  "v_xor_b32 v5, v5, v6\n\t" \
  "v_lshlrev_b32 v5, 4, v5\n\t" \
  "v_and_b32 v6, 15, v4\n\t" \
  "v_lshl_add_u32 v5, v6, 7, v5\n\t" \
  "v_add_u32 v28, 147456, v5\n\t" \
  "v_xor_b32 v29, 64, v28\n\t" \
  "v_add_u32 v26, s34, v5\n\t" \
  "v_xor_b32 v27, 64, v26\n\t" \
  "v_lshrrev_b32 v5, 4, v4\n\t" \
  "v_bfe_u32 v6, v4, 1, 3\n\t" \
  "v_lshl_add_u32 v6, v5, 4, v6\n\t" \
  "v_lshlrev_b32 v2, 7, v6\n\t" \
  "v_and_b32 v6, 1, v4\n\t" \
  "v_lshl_add_u32 v2, v6, 3, v2\n\t" \
  "v_bfe_u32 v6, v4, 2, 2\n\t" \
  "v_and_b32 v5, 1, v5\n\t" \
  "v_lshl_or_b32 v6, v5, 2, v6\n\t" \
  "v_lshl_add_u32 v2, v6, 4, v2\n\t" \
  "v_xor_b32 v64, 0, v2\n\t" \
  "v_add_u32 v72, 0x10000, v64\n\t" \
  "v_add_u32 v16, 0x20000, v64\n\t" \
  "v_xor_b32 v65, 16, v2\n\t" \
  "v_add_u32 v73, 0x10000, v65\n\t" \
  "v_add_u32 v17, 0x20000, v65\n\t" \
  "v_xor_b32 v66, 32, v2\n\t" \
  "v_add_u32 v74, 0x10000, v66\n\t" \
  "v_add_u32 v18, 0x20000, v66\n\t" \
  "v_xor_b32 v67, 48, v2\n\t" \
  "v_add_u32 v75, 0x10000, v67\n\t" \
  "v_add_u32 v19, 0x20000, v67\n\t" \
  "v_xor_b32 v68, 64, v2\n\t" \
  "v_add_u32 v76, 0x10000, v68\n\t" \
  "v_add_u32 v20, 0x20000, v68\n\t" \
  "v_xor_b32 v69, 80, v2\n\t" \
  "v_add_u32 v77, 0x10000, v69\n\t" \
  "v_add_u32 v21, 0x20000, v69\n\t" \
  "v_xor_b32 v70, 96, v2\n\t" \
  "v_add_u32 v78, 0x10000, v70\n\t" \
  "v_add_u32 v22, 0x20000, v70\n\t" \
  "v_xor_b32 v71, 112, v2\n\t" \
  "v_add_u32 v79, 0x10000, v71\n\t" \
  "v_add_u32 v23, 0x20000, v71\n\t" \
  "v_and_b32 v5, 31, v4\n\t" \
  "v_lshlrev_b32 v5, 2, v5\n\t" \
  "s_lshl_b32 s84, s13, 7\n\t" \
  "v_add_u32 v87, s84, v5\n\t" \
  "s_lshl_b32 s84, s10, 8\n\t" \
  "s_lshl_b32 s85, s4, 6\n\t" \
  "s_add_u32 s84, s84, s85\n\t" \
  "s_lshl_b32 s84, s84, 16\n\t" \
  "s_lshl_b32 s85, s12, 2\n\t" \
  "s_add_u32 s84, s84, s85\n\t" \
  "v_and_b32 v5, 15, v4\n\t" \
  "v_lshlrev_b32 v5, 16, v5\n\t" \
  "v_and_b32 v6, 48, v4\n\t" \
  "v_add_u32 v5, v5, v6\n\t" \
  "v_add_u32 v85, s84, v5\n\t" \
  "s_add_u32 s84, s35, 1\n\t" \
  "s_and_b32 s84, s84, 7\n\t" \
  "s_lshl_b32 s84, s84, 23\n\t" \
  "s_add_u32 s52, s84, s5\n\t" \
  "s_add_u32 s84, s35, 2\n\t" \
  "s_and_b32 s84, s84, 7\n\t" \
  "s_lshl_b32 s84, s84, 23\n\t" \
  "s_add_u32 s53, s84, s5\n\t" \
  "s_add_u32 s84, s35, 3\n\t" \
  "s_and_b32 s84, s84, 7\n\t" \
  "s_lshl_b32 s84, s84, 23\n\t" \
  "s_add_u32 s54, s84, s5\n\t" \
  "s_add_u32 s84, s35, 4\n\t" \
  "s_and_b32 s84, s84, 7\n\t" \
  "s_lshl_b32 s84, s84, 23\n\t" \
  "s_add_u32 s55, s84, s5\n\t" \
  "s_add_u32 s84, s35, 5\n\t" \
  "s_and_b32 s84, s84, 7\n\t" \
  "s_lshl_b32 s84, s84, 23\n\t" \
  "s_add_u32 s56, s84, s5\n\t" \
  "s_add_u32 s84, s35, 6\n\t" \
  "s_and_b32 s84, s84, 7\n\t" \
  "s_lshl_b32 s84, s84, 23\n\t" \
  "s_add_u32 s57, s84, s5\n\t" \
  "s_add_u32 s84, s35, 7\n\t" \
  "s_and_b32 s84, s84, 7\n\t" \
  "s_lshl_b32 s84, s84, 23\n\t" \
  "s_add_u32 s58, s84, s5\n\t" \
  "v_and_b32 v9, 3, v4\n\t" \
  "v_lshlrev_b32 v9, 7, v9\n\t" \
  "s_lshr_b32 s84, s11, 1\n\t" \
  "s_lshl_b32 s84, s84, 9\n\t" \
  "v_add_u32 v9, s84, v9\n\t" \
  "v_bfe_u32 v13, v4, 2, 2\n\t" \
  "v_lshl_add_u32 v13, v13, 16, v9\n\t" \
  "s_mov_b64 s[48:49], s[48:49]\n\t" \
  "s_and_b32 s49, s49, 0xffff\n\t" \
  "s_mov_b32 s50, 33554432\n\t" \
  "s_mov_b32 s51, 0x20000\n\t" \
  "s_mov_b32 s65, 0\n\t" \
  "s_mov_b32 s66, 0\n\t" \
  "s_mov_b32 s13, s15\n\t" \
  "s_cmp_gt_u32 s63, 1\n\t" \
  "s_cbranch_scc1 PR10_%=\n\t" \
  "s_waitcnt vmcnt(17)\n\t" \
  "ds_read_b128 v[32:35], v26 offset:16384\n\t" \
  "ds_read_b128 v[36:39], v27 offset:16384\n\t" \
  "ds_read_b128 v[40:43], v26 offset:18432\n\t" \
  "ds_read_b128 v[44:47], v27 offset:18432\n\t" \
  "ds_read_b128 v[48:51], v26 offset:20480\n\t" \
  "ds_read_b128 v[52:55], v27 offset:20480\n\t" \
  "ds_read_b128 v[56:59], v26 offset:22528\n\t" \
  "ds_read_b128 v[60:63], v27 offset:22528\n\t" \
  "PR10_%=:\n\t" \
  "s_waitcnt vmcnt(16)\n\t" \
  "v_readfirstlane_b32 s64, v89\n\t" \
  "s_nop 3\n\t" \
  "s_add_u32 s64, s64, 1\n\t" \
  "s_and_b32 s84, s64, 3\n\t" \
  "s_lshl_b32 s84, s84, 21\n\t" \
  "s_add_u32 s84, s84, 1048576\n\t" \
  "s_add_u32 s44, s70, s84\n\t" \
  "s_addc_u32 s45, s71, 0\n\t" \
  "s_and_b32 s45, s45, 0xffff\n\t" \
  "s_mov_b32 s46, 2097152\n\t" \
  "s_mov_b32 s47, 0x20000\n\t" \
  "s_add_u32 s84, s84, s6\n\t" \
  "s_mov_b64 s[72:73], s[22:23]\n\t" \
  "s_mov_b64 s[76:77], s[22:23]\n\t" \
  "s_add_u32 s72, s72, s84\n\t" \
  "s_addc_u32 s73, s73, 0\n\t" \
  "s_add_u32 s74, s72, 4096\n\t" \
  "s_addc_u32 s75, s73, 0\n\t" \
  "s_add_u32 s76, s76, s7\n\t" \
  "s_addc_u32 s77, s77, 0\n\t" \
  "s_waitcnt lgkmcnt(0)\n\t" \
  "v_mov_b32 v89, s94\n\t" \
  "v_mul_f32 v89, s95, v89\n\t" \
  "s_cmp_lg_u32 s63, 0\n\t" \
  "s_cbranch_scc1 PS11_%=\n\t" \
  "global_store_dwordx4 v84, v[32:35], s[72:73] offset:0 sc0 sc1\n\t" \
  "global_store_dwordx4 v84, v[36:39], s[72:73] offset:1024 sc0 sc1\n\t" \
  "global_store_dwordx4 v84, v[40:43], s[72:73] offset:2048 sc0 sc1\n\t" \
  "global_store_dwordx4 v84, v[44:47], s[72:73] offset:3072 sc0 sc1\n\t" \
  "global_store_dwordx4 v84, v[48:51], s[74:75] offset:0 sc0 sc1\n\t" \
  "global_store_dwordx4 v84, v[52:55], s[74:75] offset:1024 sc0 sc1\n\t" \
  "global_store_dwordx4 v84, v[56:59], s[74:75] offset:2048 sc0 sc1\n\t" \
  "global_store_dwordx4 v84, v[60:63], s[74:75] offset:3072 sc0 sc1\n\t" \
  "s_nop 1\n\t" \
  "s_waitcnt vmcnt(12)\n\t" \
  "s_branch PJ12_%=\n\t" \
  "PS11_%=:\n\t" \
  "s_waitcnt vmcnt(4)\n\t" \
  "PJ12_%=:\n\t" \
  "v_readfirstlane_b32 s78, v89\n\t" \
  "s_barrier\n\t" \
  "s_mov_b32 s79, s78\n\t" \
  "ds_read_b128 a[0:3], v26 offset:0\n\t" \
  "ds_read_b128 a[4:7], v27 offset:0\n\t" \
  "ds_read_b128 a[8:11], v26 offset:2048\n\t" \
  "ds_read_b128 a[12:15], v27 offset:2048\n\t" \
  "ds_read_b128 a[16:19], v26 offset:4096\n\t" \
  "ds_read_b128 a[20:23], v27 offset:4096\n\t" \
  "ds_read_b128 a[24:27], v26 offset:6144\n\t" \
  "ds_read_b128 a[28:31], v27 offset:6144\n\t" \
  "ds_read_b64_tr_b8 v[96:97], v64 offset:0\n\t" \
  "ds_read_b64_tr_b8 v[98:99], v64 offset:1024\n\t" \
  "ds_read_b64_tr_b8 v[100:101], v64 offset:8192\n\t" \
  "ds_read_b64_tr_b8 v[102:103], v64 offset:9216\n\t" \
  "ds_read_b64_tr_b8 v[104:105], v65 offset:0\n\t" \
  "ds_read_b64_tr_b8 v[106:107], v65 offset:1024\n\t" \
  "ds_read_b64_tr_b8 v[108:109], v65 offset:8192\n\t" \
  "ds_read_b64_tr_b8 v[110:111], v65 offset:9216\n\t" \
  "s_waitcnt lgkmcnt(4)\n\t" \
  "v_mfma_f32_16x16x128_f8f6f4 v[128:131], v[96:103], a[0:7], 0\n\t" \
  "ds_read_b64_tr_b8 v[112:113], v66 offset:0\n\t" \
  "ds_read_b64_tr_b8 v[114:115], v66 offset:1024\n\t" \
  "v_mfma_f32_16x16x128_f8f6f4 v[132:135], v[96:103], a[8:15], 0\n\t" \
  "ds_read_b64_tr_b8 v[116:117], v66 offset:8192\n\t" \
  "ds_read_b64_tr_b8 v[118:119], v66 offset:9216\n\t" \
  "s_add_u32 s84, s61, 128\n\t" \
  "s_add_u32 m0, s34, 8192\n\t" \
  "s_nop 0\n\t" \
  "buffer_load_dwordx4 v24, s[40:43], s84 offen lds\n\t" \
  "v_mfma_f32_16x16x128_f8f6f4 v[136:139], v[96:103], a[16:23], 0\n\t" \
  "s_add_u32 s84, s61, 4224\n\t" \
  "s_add_u32 m0, s34, 9216\n\t" \
  "s_nop 0\n\t" \
  "buffer_load_dwordx4 v25, s[40:43], s84 offen lds\n\t" \
  "v_mfma_f32_16x16x128_f8f6f4 v[140:143], v[96:103], a[24:31], 0\n\t" \
  "s_waitcnt lgkmcnt(4)\n\t" \
  "v_mfma_f32_16x16x128_f8f6f4 v[144:147], v[104:111], a[0:7], 0\n\t" \
  "ds_read_b64_tr_b8 v[120:121], v67 offset:0\n\t" \
  "ds_read_b64_tr_b8 v[122:123], v67 offset:1024\n\t" \
  "v_mfma_f32_16x16x128_f8f6f4 v[148:151], v[104:111], a[8:15], 0\n\t" \
  "ds_read_b64_tr_b8 v[124:125], v67 offset:8192\n\t" \
  "ds_read_b64_tr_b8 v[126:127], v67 offset:9216\n\t" \
  "s_add_u32 s84, s61, 8320\n\t" \
  "s_add_u32 m0, s34, 10240\n\t" \
  "s_nop 0\n\t" \
  "buffer_load_dwordx4 v24, s[40:43], s84 offen lds\n\t" \
  "v_mfma_f32_16x16x128_f8f6f4 v[152:155], v[104:111], a[16:23], 0\n\t" \
  "s_add_u32 s84, s61, 12416\n\t" \
  "s_add_u32 m0, s34, 11264\n\t" \
  "s_nop 0\n\t" \
  "buffer_load_dwordx4 v25, s[40:43], s84 offen lds\n\t" \
  "v_mfma_f32_16x16x128_f8f6f4 v[156:159], v[104:111], a[24:31], 0\n\t" \
  "s_waitcnt lgkmcnt(4)\n\t" \
  "v_mfma_f32_16x16x128_f8f6f4 v[160:163], v[112:119], a[0:7], 0\n\t" \
  "ds_read_b64_tr_b8 v[96:97], v68 offset:0\n\t" \
  "ds_read_b64_tr_b8 v[98:99], v68 offset:1024\n\t" \
  "v_mfma_f32_16x16x128_f8f6f4 v[164:167], v[112:119], a[8:15], 0\n\t" \
  "ds_read_b64_tr_b8 v[100:101], v68 offset:8192\n\t" \
  "ds_read_b64_tr_b8 v[102:103], v68 offset:9216\n\t" \
  "s_add_u32 s84, s61, 16512\n\t" \
  "s_add_u32 m0, s34, 12288\n\t" \
  "s_nop 0\n\t" \
  "buffer_load_dwordx4 v24, s[40:43], s84 offen lds\n\t" \
  "v_mfma_f32_16x16x128_f8f6f4 v[168:171], v[112:119], a[16:23], 0\n\t" \
  "s_add_u32 s84, s61, 20608\n\t" \
  "s_add_u32 m0, s34, 13312\n\t" \
  "s_nop 0\n\t" \
  "buffer_load_dwordx4 v25, s[40:43], s84 offen lds\n\t" \
  "v_mfma_f32_16x16x128_f8f6f4 v[172:175], v[112:119], a[24:31], 0\n\t" \
  "s_waitcnt lgkmcnt(4)\n\t" \
  "v_mfma_f32_16x16x128_f8f6f4 v[176:179], v[120:127], a[0:7], 0\n\t" \
  "ds_read_b64_tr_b8 v[104:105], v69 offset:0\n\t" \
  "ds_read_b64_tr_b8 v[106:107], v69 offset:1024\n\t" \
  "v_mfma_f32_16x16x128_f8f6f4 v[180:183], v[120:127], a[8:15], 0\n\t" \
  "ds_read_b64_tr_b8 v[108:109], v69 offset:8192\n\t" \
  "ds_read_b64_tr_b8 v[110:111], v69 offset:9216\n\t" \
  "s_add_u32 s84, s61, 24704\n\t" \
  "s_add_u32 m0, s34, 14336\n\t" \
  "s_nop 0\n\t" \
  "buffer_load_dwordx4 v24, s[40:43], s84 offen lds\n\t" \
  "v_mfma_f32_16x16x128_f8f6f4 v[184:187], v[120:127], a[16:23], 0\n\t" \
  "s_add_u32 s84, s61, 28800\n\t" \
  "s_add_u32 m0, s34, 15360\n\t" \
  "s_nop 0\n\t" \
  "buffer_load_dwordx4 v25, s[40:43], s84 offen lds\n\t" \
  "v_mfma_f32_16x16x128_f8f6f4 v[188:191], v[120:127], a[24:31], 0\n\t" \
  "s_cmp_lg_u32 s63, 0\n\t" \
  "s_cbranch_scc1 PS13_%=\n\t" \
  "s_waitcnt vmcnt(8)\n\t" \
  "v_mov_b32 v89, s64\n\t" \
  "s_mov_b64 exec, 1\n\t" \
  "global_store_dword v88, v89, s[76:77] sc0 sc1\n\t" \
  "s_mov_b64 exec, -1\n\t" \
  "PS13_%=:\n\t" \
  "s_waitcnt lgkmcnt(4)\n\t" \
  "v_mfma_f32_16x16x128_f8f6f4 v[192:195], v[96:103], a[0:7], 0\n\t" \
  "ds_read_b64_tr_b8 v[112:113], v70 offset:0\n\t" \
  "ds_read_b64_tr_b8 v[114:115], v70 offset:1024\n\t" \
  "v_mfma_f32_16x16x128_f8f6f4 v[196:199], v[96:103], a[8:15], 0\n\t" \
  "ds_read_b64_tr_b8 v[116:117], v70 offset:8192\n\t" \
  "ds_read_b64_tr_b8 v[118:119], v70 offset:9216\n\t" \
  "s_add_u32 s88, s59, 4194304\n\t" \
  "s_add_u32 m0, s60, 32768\n\t" \
  "s_nop 0\n\t" \
  "buffer_load_dwordx4 v80, s[36:39], s88 offen  lds\n\t" \
  "v_mfma_f32_16x16x128_f8f6f4 v[200:203], v[96:103], a[16:23], 0\n\t" \
  "s_add_u32 m0, s60, 33792\n\t" \
  "s_nop 0\n\t" \
  "buffer_load_dwordx4 v81, s[36:39], s88 offen  lds\n\t" \
  "v_mfma_f32_16x16x128_f8f6f4 v[204:207], v[96:103], a[24:31], 0\n\t" \
  "s_waitcnt lgkmcnt(4)\n\t" \
  "v_mfma_f32_16x16x128_f8f6f4 v[208:211], v[104:111], a[0:7], 0\n\t" \
  "ds_read_b64_tr_b8 v[120:121], v71 offset:0\n\t" \
  "ds_read_b64_tr_b8 v[122:123], v71 offset:1024\n\t" \
  "v_mfma_f32_16x16x128_f8f6f4 v[212:215], v[104:111], a[8:15], 0\n\t" \
  "ds_read_b64_tr_b8 v[124:125], v71 offset:8192\n\t" \
  "ds_read_b64_tr_b8 v[126:127], v71 offset:9216\n\t" \
  "s_add_u32 m0, s60, 34816\n\t" \
  "s_nop 0\n\t" \
  "buffer_load_dwordx4 v82, s[36:39], s88 offen  lds\n\t" \
  "v_mfma_f32_16x16x128_f8f6f4 v[216:219], v[104:111], a[16:23], 0\n\t" \
  "s_add_u32 m0, s60, 35840\n\t" \
  "s_nop 0\n\t" \
  "buffer_load_dwordx4 v83, s[36:39], s88 offen  lds\n\t" \
  "v_mfma_f32_16x16x128_f8f6f4 v[220:223], v[104:111], a[24:31], 0\n\t" \
  "s_waitcnt vmcnt(4)\n\t" \
  "s_barrier\n\t" \
  "s_waitcnt lgkmcnt(4)\n\t" \
  "v_mfma_f32_16x16x128_f8f6f4 v[224:227], v[112:119], a[0:7], 0\n\t" \
  "ds_read_b64_tr_b8 v[96:97], v64 offset:16384\n\t" \
  "ds_read_b64_tr_b8 v[98:99], v64 offset:17408\n\t" \
  "v_mfma_f32_16x16x128_f8f6f4 v[228:231], v[112:119], a[8:15], 0\n\t" \
  "ds_read_b64_tr_b8 v[100:101], v64 offset:24576\n\t" \
  "ds_read_b64_tr_b8 v[102:103], v64 offset:25600\n\t" \
  "s_add_u32 s84, s61, 256\n\t" \
  "s_add_u32 m0, s34, 16384\n\t" \
  "s_nop 0\n\t" \
  "buffer_load_dwordx4 v24, s[40:43], s84 offen lds\n\t" \
  "v_mfma_f32_16x16x128_f8f6f4 v[232:235], v[112:119], a[16:23], 0\n\t" \
  "s_add_u32 s84, s61, 4352\n\t" \
  "s_add_u32 m0, s34, 17408\n\t" \
  "s_nop 0\n\t" \
  "buffer_load_dwordx4 v25, s[40:43], s84 offen lds\n\t" \
  "ds_read_b128 a[32:35], v26 offset:8192\n\t" \
  "ds_read_b128 a[36:39], v27 offset:8192\n\t" \
  "v_mfma_f32_16x16x128_f8f6f4 v[236:239], v[112:119], a[24:31], 0\n\t" \
  "ds_read_b128 a[40:43], v26 offset:10240\n\t" \
  "ds_read_b128 a[44:47], v27 offset:10240\n\t" \
  "s_waitcnt lgkmcnt(8)\n\t" \
  "v_mfma_f32_16x16x128_f8f6f4 v[240:243], v[120:127], a[0:7], 0\n\t" \
  "ds_read_b64_tr_b8 v[104:105], v65 offset:16384\n\t" \
  "ds_read_b64_tr_b8 v[106:107], v65 offset:17408\n\t" \
  "v_mfma_f32_16x16x128_f8f6f4 v[244:247], v[120:127], a[8:15], 0\n\t" \
  "ds_read_b64_tr_b8 v[108:109], v65 offset:24576\n\t" \
  "ds_read_b64_tr_b8 v[110:111], v65 offset:25600\n\t" \
  "s_add_u32 s84, s61, 8448\n\t" \
  "s_add_u32 m0, s34, 18432\n\t" \
  "s_nop 0\n\t" \
  "buffer_load_dwordx4 v24, s[40:43], s84 offen lds\n\t" \
  "v_mfma_f32_16x16x128_f8f6f4 v[248:251], v[120:127], a[16:23], 0\n\t" \
  "s_add_u32 s84, s61, 12544\n\t" \
  "s_add_u32 m0, s34, 19456\n\t" \
  "s_nop 0\n\t" \
  "buffer_load_dwordx4 v25, s[40:43], s84 offen lds\n\t" \
  "ds_read_b128 a[48:51], v26 offset:12288\n\t" \
  "ds_read_b128 a[52:55], v27 offset:12288\n\t" \
  "v_mfma_f32_16x16x128_f8f6f4 v[252:255], v[120:127], a[24:31], 0\n\t" \
  "ds_read_b128 a[56:59], v26 offset:14336\n\t" \
  "ds_read_b128 a[60:63], v27 offset:14336\n\t" \
  "s_waitcnt lgkmcnt(0)\n\t" \
  "v_mfma_f32_16x16x128_f8f6f4 v[128:131], v[96:103], a[32:39], v[128:131]\n\t" \
  "ds_read_b64_tr_b8 v[112:113], v66 offset:16384\n\t" \
  "ds_read_b64_tr_b8 v[114:115], v66 offset:17408\n\t" \
  "v_mfma_f32_16x16x128_f8f6f4 v[132:135], v[96:103], a[40:47], v[132:135]\n\t" \
  "ds_read_b64_tr_b8 v[116:117], v66 offset:24576\n\t" \
  "ds_read_b64_tr_b8 v[118:119], v66 offset:25600\n\t" \
  "s_add_u32 s84, s61, 16640\n\t" \
  "s_add_u32 m0, s34, 20480\n\t" \
  "s_nop 0\n\t" \
  "buffer_load_dwordx4 v24, s[40:43], s84 offen lds\n\t" \
  "v_mfma_f32_16x16x128_f8f6f4 v[136:139], v[96:103], a[48:55], v[136:139]\n\t" \
  "s_add_u32 s84, s61, 20736\n\t" \
  "s_add_u32 m0, s34, 21504\n\t" \
  "s_nop 0\n\t" \
  "buffer_load_dwordx4 v25, s[40:43], s84 offen lds\n\t" \
  "v_mfma_f32_16x16x128_f8f6f4 v[140:143], v[96:103], a[56:63], v[140:143]\n\t" \
  "s_waitcnt lgkmcnt(8)\n\t" \
  "v_mfma_f32_16x16x128_f8f6f4 v[144:147], v[104:111], a[32:39], v[144:147]\n\t" \
  "ds_read_b64_tr_b8 v[120:121], v67 offset:16384\n\t" \
  "ds_read_b64_tr_b8 v[122:123], v67 offset:17408\n\t" \
  "v_mfma_f32_16x16x128_f8f6f4 v[148:151], v[104:111], a[40:47], v[148:151]\n\t" \
  "ds_read_b64_tr_b8 v[124:125], v67 offset:24576\n\t" \
  "ds_read_b64_tr_b8 v[126:127], v67 offset:25600\n\t" \
  "s_add_u32 s84, s61, 24832\n\t" \
  "s_add_u32 m0, s34, 22528\n\t" \
  "s_nop 0\n\t" \
  "buffer_load_dwordx4 v24, s[40:43], s84 offen lds\n\t" \
  "v_mfma_f32_16x16x128_f8f6f4 v[152:155], v[104:111], a[48:55], v[152:155]\n\t" \
  "s_add_u32 s84, s61, 28928\n\t" \
  "s_add_u32 m0, s34, 23552\n\t" \
  "s_nop 0\n\t" \
  "buffer_load_dwordx4 v25, s[40:43], s84 offen lds\n\t" \
  "v_mfma_f32_16x16x128_f8f6f4 v[156:159], v[104:111], a[56:63], v[156:159]\n\t" \
  "s_waitcnt lgkmcnt(4)\n\t" \
  "v_mfma_f32_16x16x128_f8f6f4 v[160:163], v[112:119], a[32:39], v[160:163]\n\t" \
  "ds_read_b64_tr_b8 v[96:97], v68 offset:16384\n\t" \
  "ds_read_b64_tr_b8 v[98:99], v68 offset:17408\n\t" \
  "v_mfma_f32_16x16x128_f8f6f4 v[164:167], v[112:119], a[40:47], v[164:167]\n\t" \
  "ds_read_b64_tr_b8 v[100:101], v68 offset:24576\n\t" \
  "ds_read_b64_tr_b8 v[102:103], v68 offset:25600\n\t" \
  "s_add_u32 s88, s59, 6291456\n\t" \
  "s_add_u32 m0, s60, 49152\n\t" \
  "s_nop 0\n\t" \
  "buffer_load_dwordx4 v80, s[36:39], s88 offen  lds\n\t" \
  "v_mfma_f32_16x16x128_f8f6f4 v[168:171], v[112:119], a[48:55], v[168:171]\n\t" \
  "s_add_u32 m0, s60, 50176\n\t" \
  "s_nop 0\n\t" \
  "buffer_load_dwordx4 v81, s[36:39], s88 offen  lds\n\t" \
  "v_mfma_f32_16x16x128_f8f6f4 v[172:175], v[112:119], a[56:63], v[172:175]\n\t" \
  "s_waitcnt lgkmcnt(4)\n\t" \
  "v_mfma_f32_16x16x128_f8f6f4 v[176:179], v[120:127], a[32:39], v[176:179]\n\t" \
  "ds_read_b64_tr_b8 v[104:105], v69 offset:16384\n\t" \
  "ds_read_b64_tr_b8 v[106:107], v69 offset:17408\n\t" \
  "v_mfma_f32_16x16x128_f8f6f4 v[180:183], v[120:127], a[40:47], v[180:183]\n\t" \
  "ds_read_b64_tr_b8 v[108:109], v69 offset:24576\n\t" \
  "ds_read_b64_tr_b8 v[110:111], v69 offset:25600\n\t" \
  "s_add_u32 m0, s60, 51200\n\t" \
  "s_nop 0\n\t" \
  "buffer_load_dwordx4 v82, s[36:39], s88 offen  lds\n\t" \
  "v_mfma_f32_16x16x128_f8f6f4 v[184:187], v[120:127], a[48:55], v[184:187]\n\t" \
  "s_add_u32 m0, s60, 52224\n\t" \
  "s_nop 0\n\t" \
  "buffer_load_dwordx4 v83, s[36:39], s88 offen  lds\n\t" \
  "v_mfma_f32_16x16x128_f8f6f4 v[188:191], v[120:127], a[56:63], v[188:191]\n\t" \
  "s_waitcnt lgkmcnt(4)\n\t" \
  "v_mfma_f32_16x16x128_f8f6f4 v[192:195], v[96:103], a[32:39], v[192:195]\n\t" \
  "ds_read_b64_tr_b8 v[112:113], v70 offset:16384\n\t" \
  "ds_read_b64_tr_b8 v[114:115], v70 offset:17408\n\t" \
  "v_mfma_f32_16x16x128_f8f6f4 v[196:199], v[96:103], a[40:47], v[196:199]\n\t" \
  "ds_read_b64_tr_b8 v[116:117], v70 offset:24576\n\t" \
  "ds_read_b64_tr_b8 v[118:119], v70 offset:25600\n\t" \
  "s_add_u32 s84, s61, 384\n\t" \
  "s_add_u32 m0, s34, 0\n\t" \
  "s_nop 0\n\t" \
  "buffer_load_dwordx4 v24, s[40:43], s84 offen lds\n\t" \
  "s_add_u32 s84, s61, 4480\n\t" \
  "s_add_u32 m0, s34, 1024\n\t" \
  "s_nop 0\n\t" \
  "buffer_load_dwordx4 v25, s[40:43], s84 offen lds\n\t" \
  "v_mfma_f32_16x16x128_f8f6f4 v[200:203], v[96:103], a[48:55], v[200:203]\n\t" \
  "s_add_u32 s84, s61, 8576\n\t" \
  "s_add_u32 m0, s34, 2048\n\t" \
  "s_nop 0\n\t" \
  "buffer_load_dwordx4 v24, s[40:43], s84 offen lds\n\t" \
  "s_add_u32 s84, s61, 12672\n\t" \
  "s_add_u32 m0, s34, 3072\n\t" \
  "s_nop 0\n\t" \
  "buffer_load_dwordx4 v25, s[40:43], s84 offen lds\n\t" \
  "v_mfma_f32_16x16x128_f8f6f4 v[204:207], v[96:103], a[56:63], v[204:207]\n\t" \
  "s_waitcnt lgkmcnt(4)\n\t" \
  "v_mfma_f32_16x16x128_f8f6f4 v[208:211], v[104:111], a[32:39], v[208:211]\n\t" \
  "ds_read_b64_tr_b8 v[120:121], v71 offset:16384\n\t" \
  "ds_read_b64_tr_b8 v[122:123], v71 offset:17408\n\t" \
  "v_mfma_f32_16x16x128_f8f6f4 v[212:215], v[104:111], a[40:47], v[212:215]\n\t" \
  "ds_read_b64_tr_b8 v[124:125], v71 offset:24576\n\t" \
  "ds_read_b64_tr_b8 v[126:127], v71 offset:25600\n\t" \
  "s_add_u32 s84, s61, 16768\n\t" \
  "s_add_u32 m0, s34, 4096\n\t" \
  "s_nop 0\n\t" \
  "buffer_load_dwordx4 v24, s[40:43], s84 offen lds\n\t" \
  "s_add_u32 s84, s61, 20864\n\t" \
  "s_add_u32 m0, s34, 5120\n\t" \
  "s_nop 0\n\t" \
  "buffer_load_dwordx4 v25, s[40:43], s84 offen lds\n\t" \
  "v_mfma_f32_16x16x128_f8f6f4 v[216:219], v[104:111], a[48:55], v[216:219]\n\t" \
  "s_add_u32 s84, s61, 24960\n\t" \
  "s_add_u32 m0, s34, 6144\n\t" \
  "s_nop 0\n\t" \
  "buffer_load_dwordx4 v24, s[40:43], s84 offen lds\n\t" \
  "s_add_u32 s84, s61, 29056\n\t" \
  "s_add_u32 m0, s34, 7168\n\t" \
  "s_nop 0\n\t" \
  "buffer_load_dwordx4 v25, s[40:43], s84 offen lds\n\t" \
  "v_mfma_f32_16x16x128_f8f6f4 v[220:223], v[104:111], a[56:63], v[220:223]\n\t" \
  "s_waitcnt vmcnt(12)\n\t" \
  "s_barrier\n\t" \
  "s_waitcnt lgkmcnt(4)\n\t" \
  "v_mfma_f32_16x16x128_f8f6f4 v[224:227], v[112:119], a[32:39], v[224:227]\n\t" \
  "ds_read_b64_tr_b8 v[96:97], v64 offset:32768\n\t" \
  "ds_read_b64_tr_b8 v[98:99], v64 offset:33792\n\t" \
  "v_mfma_f32_16x16x128_f8f6f4 v[228:231], v[112:119], a[40:47], v[228:231]\n\t" \
  "ds_read_b64_tr_b8 v[100:101], v64 offset:40960\n\t" \
  "ds_read_b64_tr_b8 v[102:103], v64 offset:41984\n\t" \
  "s_add_u32 s88, s52, 0\n\t" \
  "s_add_u32 m0, s60, 0\n\t" \
  "s_nop 0\n\t" \
  "buffer_load_dwordx4 v80, s[36:39], s88 offen  lds\n\t" \
  "v_mfma_f32_16x16x128_f8f6f4 v[232:235], v[112:119], a[48:55], v[232:235]\n\t" \
  "s_add_u32 m0, s60, 1024\n\t" \
  "s_nop 0\n\t" \
  "buffer_load_dwordx4 v81, s[36:39], s88 offen  lds\n\t" \
  "ds_read_b128 a[64:67], v26 offset:16384\n\t" \
  "ds_read_b128 a[68:71], v27 offset:16384\n\t" \
  "v_mfma_f32_16x16x128_f8f6f4 v[236:239], v[112:119], a[56:63], v[236:239]\n\t" \
  "ds_read_b128 a[72:75], v26 offset:18432\n\t" \
  "ds_read_b128 a[76:79], v27 offset:18432\n\t" \
  "s_waitcnt lgkmcnt(8)\n\t" \
  "v_mfma_f32_16x16x128_f8f6f4 v[240:243], v[120:127], a[32:39], v[240:243]\n\t" \
  "ds_read_b64_tr_b8 v[104:105], v65 offset:32768\n\t" \
  "ds_read_b64_tr_b8 v[106:107], v65 offset:33792\n\t" \
  "v_mfma_f32_16x16x128_f8f6f4 v[244:247], v[120:127], a[40:47], v[244:247]\n\t" \
  "ds_read_b64_tr_b8 v[108:109], v65 offset:40960\n\t" \
  "ds_read_b64_tr_b8 v[110:111], v65 offset:41984\n\t" \
  "s_add_u32 m0, s60, 2048\n\t" \
  "s_nop 0\n\t" \
  "buffer_load_dwordx4 v82, s[36:39], s88 offen  lds\n\t" \
  "v_mfma_f32_16x16x128_f8f6f4 v[248:251], v[120:127], a[48:55], v[248:251]\n\t" \
  "s_add_u32 m0, s60, 3072\n\t" \
  "s_nop 0\n\t" \
  "buffer_load_dwordx4 v83, s[36:39], s88 offen  lds\n\t" \
  "ds_read_b128 a[80:83], v26 offset:20480\n\t" \
  "ds_read_b128 a[84:87], v27 offset:20480\n\t" \
  "v_mfma_f32_16x16x128_f8f6f4 v[252:255], v[120:127], a[56:63], v[252:255]\n\t" \
  "ds_read_b128 a[88:91], v26 offset:22528\n\t" \
  "ds_read_b128 a[92:95], v27 offset:22528\n\t" \
  "s_cmp_lg_u32 s63, 1\n\t" \
  "s_cbranch_scc1 PS14_%=\n\t" \
  "global_store_dwordx4 v84, v[32:35], s[72:73] offset:0 sc0 sc1\n\t" \
  "global_store_dwordx4 v84, v[36:39], s[72:73] offset:1024 sc0 sc1\n\t" \
  "global_store_dwordx4 v84, v[40:43], s[72:73] offset:2048 sc0 sc1\n\t" \
  "global_store_dwordx4 v84, v[44:47], s[72:73] offset:3072 sc0 sc1\n\t" \
  "global_store_dwordx4 v84, v[48:51], s[74:75] offset:0 sc0 sc1\n\t" \
  "global_store_dwordx4 v84, v[52:55], s[74:75] offset:1024 sc0 sc1\n\t" \
  "global_store_dwordx4 v84, v[56:59], s[74:75] offset:2048 sc0 sc1\n\t" \
  "global_store_dwordx4 v84, v[60:63], s[74:75] offset:3072 sc0 sc1\n\t" \
  "s_nop 1\n\t" \
  "PS14_%=:\n\t" \
  "s_waitcnt lgkmcnt(0)\n\t" \
  "v_mfma_f32_16x16x128_f8f6f4 v[128:131], v[96:103], a[64:71], v[128:131]\n\t" \
  "ds_read_b64_tr_b8 v[112:113], v66 offset:32768\n\t" \
  "ds_read_b64_tr_b8 v[114:115], v66 offset:33792\n\t" \
  "v_mfma_f32_16x16x128_f8f6f4 v[132:135], v[96:103], a[72:79], v[132:135]\n\t" \
  "ds_read_b64_tr_b8 v[116:117], v66 offset:40960\n\t" \
  "ds_read_b64_tr_b8 v[118:119], v66 offset:41984\n\t" \
  "v_mfma_f32_16x16x128_f8f6f4 v[136:139], v[96:103], a[80:87], v[136:139]\n\t" \
  "v_mfma_f32_16x16x128_f8f6f4 v[140:143], v[96:103], a[88:95], v[140:143]\n\t" \
  "s_waitcnt lgkmcnt(8)\n\t" \
  "v_mfma_f32_16x16x128_f8f6f4 v[144:147], v[104:111], a[64:71], v[144:147]\n\t" \
  "ds_read_b64_tr_b8 v[120:121], v67 offset:32768\n\t" \
  "ds_read_b64_tr_b8 v[122:123], v67 offset:33792\n\t" \
  "v_mfma_f32_16x16x128_f8f6f4 v[148:151], v[104:111], a[72:79], v[148:151]\n\t" \
  "ds_read_b64_tr_b8 v[124:125], v67 offset:40960\n\t" \
  "ds_read_b64_tr_b8 v[126:127], v67 offset:41984\n\t" \
  "v_mfma_f32_16x16x128_f8f6f4 v[152:155], v[104:111], a[80:87], v[152:155]\n\t" \
  "v_mfma_f32_16x16x128_f8f6f4 v[156:159], v[104:111], a[88:95], v[156:159]\n\t" \
  "s_sub_u32 s84, s63, 2\n\t" \
  "s_cmp_gt_u32 s84, 1\n\t" \
  "s_cbranch_scc1 PL15_%=\n\t" \
  "s_add_u32 s96, s91, 0\n\t" \
  "s_mov_b32 m0, 147456\n\t" \
  "s_nop 0\n\t" \
  "buffer_load_dwordx4 v24, s[40:43], s96 offen lds\n\t" \
  "s_add_u32 s96, s91, 4096\n\t" \
  "s_mov_b32 m0, 148480\n\t" \
  "s_nop 0\n\t" \
  "buffer_load_dwordx4 v25, s[40:43], s96 offen lds\n\t" \
  "s_add_u32 s96, s91, 8192\n\t" \
  "s_mov_b32 m0, 149504\n\t" \
  "s_nop 0\n\t" \
  "buffer_load_dwordx4 v24, s[40:43], s96 offen lds\n\t" \
  "s_add_u32 s96, s91, 12288\n\t" \
  "s_mov_b32 m0, 150528\n\t" \
  "s_nop 0\n\t" \
  "buffer_load_dwordx4 v25, s[40:43], s96 offen lds\n\t" \
  "s_add_u32 s96, s91, 16384\n\t" \
  "s_mov_b32 m0, 151552\n\t" \
  "s_nop 0\n\t" \
  "buffer_load_dwordx4 v24, s[40:43], s96 offen lds\n\t" \
  "s_add_u32 s96, s91, 20480\n\t" \
  "s_mov_b32 m0, 152576\n\t" \
  "s_nop 0\n\t" \
  "buffer_load_dwordx4 v25, s[40:43], s96 offen lds\n\t" \
  "s_add_u32 s96, s91, 24576\n\t" \
  "s_mov_b32 m0, 153600\n\t" \
  "s_nop 0\n\t" \
  "buffer_load_dwordx4 v24, s[40:43], s96 offen lds\n\t" \
  "s_add_u32 s96, s91, 28672\n\t" \
  "s_mov_b32 m0, 154624\n\t" \
  "s_nop 0\n\t" \
  "buffer_load_dwordx4 v25, s[40:43], s96 offen lds\n\t" \
  "PL15_%=:\n\t" \
  "s_waitcnt lgkmcnt(4)\n\t" \
  "v_mfma_f32_16x16x128_f8f6f4 v[160:163], v[112:119], a[64:71], v[160:163]\n\t" \
  "ds_read_b64_tr_b8 v[96:97], v68 offset:32768\n\t" \
  "ds_read_b64_tr_b8 v[98:99], v68 offset:33792\n\t" \
  "v_mfma_f32_16x16x128_f8f6f4 v[164:167], v[112:119], a[72:79], v[164:167]\n\t" \
  "ds_read_b64_tr_b8 v[100:101], v68 offset:40960\n\t" \
  "ds_read_b64_tr_b8 v[102:103], v68 offset:41984\n\t" \
  "v_mfma_f32_16x16x128_f8f6f4 v[168:171], v[112:119], a[80:87], v[168:171]\n\t" \
  "v_mfma_f32_16x16x128_f8f6f4 v[172:175], v[112:119], a[88:95], v[172:175]\n\t" \
  "s_waitcnt lgkmcnt(4)\n\t" \
  "v_mfma_f32_16x16x128_f8f6f4 v[176:179], v[120:127], a[64:71], v[176:179]\n\t" \
  "ds_read_b64_tr_b8 v[104:105], v69 offset:32768\n\t" \
  "ds_read_b64_tr_b8 v[106:107], v69 offset:33792\n\t" \
  "v_mfma_f32_16x16x128_f8f6f4 v[180:183], v[120:127], a[72:79], v[180:183]\n\t" \
  "ds_read_b64_tr_b8 v[108:109], v69 offset:40960\n\t" \
  "ds_read_b64_tr_b8 v[110:111], v69 offset:41984\n\t" \
  "v_mfma_f32_16x16x128_f8f6f4 v[184:187], v[120:127], a[80:87], v[184:187]\n\t" \
  "v_mfma_f32_16x16x128_f8f6f4 v[188:191], v[120:127], a[88:95], v[188:191]\n\t" \
  "s_waitcnt lgkmcnt(4)\n\t" \
  "v_mfma_f32_16x16x128_f8f6f4 v[192:195], v[96:103], a[64:71], v[192:195]\n\t" \
  "ds_read_b64_tr_b8 v[112:113], v70 offset:32768\n\t" \
  "ds_read_b64_tr_b8 v[114:115], v70 offset:33792\n\t" \
  "v_mfma_f32_16x16x128_f8f6f4 v[196:199], v[96:103], a[72:79], v[196:199]\n\t" \
  "ds_read_b64_tr_b8 v[116:117], v70 offset:40960\n\t" \
  "ds_read_b64_tr_b8 v[118:119], v70 offset:41984\n\t" \
  "v_mfma_f32_16x16x128_f8f6f4 v[200:203], v[96:103], a[80:87], v[200:203]\n\t" \
  "v_mfma_f32_16x16x128_f8f6f4 v[204:207], v[96:103], a[88:95], v[204:207]\n\t" \
  "s_waitcnt lgkmcnt(4)\n\t" \
  "v_mfma_f32_16x16x128_f8f6f4 v[208:211], v[104:111], a[64:71], v[208:211]\n\t" \
  "ds_read_b64_tr_b8 v[120:121], v71 offset:32768\n\t" \
  "ds_read_b64_tr_b8 v[122:123], v71 offset:33792\n\t" \
  "v_mfma_f32_16x16x128_f8f6f4 v[212:215], v[104:111], a[72:79], v[212:215]\n\t" \
  "ds_read_b64_tr_b8 v[124:125], v71 offset:40960\n\t" \
  "ds_read_b64_tr_b8 v[126:127], v71 offset:41984\n\t" \
  "v_mfma_f32_16x16x128_f8f6f4 v[216:219], v[104:111], a[80:87], v[216:219]\n\t" \
  "v_mfma_f32_16x16x128_f8f6f4 v[220:223], v[104:111], a[88:95], v[220:223]\n\t" \
  "s_cmp_lg_u32 s63, 1\n\t" \
  "s_cbranch_scc1 PS16_%=\n\t" \
  "s_waitcnt vmcnt(0)\n\t" \
  "v_mov_b32 v89, s64\n\t" \
  "s_mov_b64 exec, 1\n\t" \
  "global_store_dword v88, v89, s[76:77] sc0 sc1\n\t" \
  "s_mov_b64 exec, -1\n\t" \
  "PS16_%=:\n\t" \
  "s_waitcnt vmcnt(4)\n\t" \
  "s_barrier\n\t" \
  "s_waitcnt lgkmcnt(4)\n\t" \
  "v_mfma_f32_16x16x128_f8f6f4 v[224:227], v[112:119], a[64:71], v[224:227]\n\t" \
  "ds_read_b64_tr_b8 v[96:97], v64 offset:49152\n\t" \
  "ds_read_b64_tr_b8 v[98:99], v64 offset:50176\n\t" \
  "v_mfma_f32_16x16x128_f8f6f4 v[228:231], v[112:119], a[72:79], v[228:231]\n\t" \
  "ds_read_b64_tr_b8 v[100:101], v64 offset:57344\n\t" \
  "ds_read_b64_tr_b8 v[102:103], v64 offset:58368\n\t" \
  "s_add_u32 s88, s53, 0\n\t" \
  "s_add_u32 m0, s60, 16384\n\t" \
  "s_nop 0\n\t" \
  "buffer_load_dwordx4 v80, s[36:39], s88 offen  lds\n\t" \
  "v_mfma_f32_16x16x128_f8f6f4 v[232:235], v[112:119], a[80:87], v[232:235]\n\t" \
  "s_add_u32 m0, s60, 17408\n\t" \
  "s_nop 0\n\t" \
  "buffer_load_dwordx4 v81, s[36:39], s88 offen  lds\n\t" \
  "ds_read_b128 a[96:99], v26 offset:0\n\t" \
  "ds_read_b128 a[100:103], v27 offset:0\n\t" \
  "v_mfma_f32_16x16x128_f8f6f4 v[236:239], v[112:119], a[88:95], v[236:239]\n\t" \
  "ds_read_b128 a[104:107], v26 offset:2048\n\t" \
  "ds_read_b128 a[108:111], v27 offset:2048\n\t" \
  "s_waitcnt lgkmcnt(8)\n\t" \
  "v_mfma_f32_16x16x128_f8f6f4 v[240:243], v[120:127], a[64:71], v[240:243]\n\t" \
  "ds_read_b64_tr_b8 v[104:105], v65 offset:49152\n\t" \
  "ds_read_b64_tr_b8 v[106:107], v65 offset:50176\n\t" \
  "v_mfma_f32_16x16x128_f8f6f4 v[244:247], v[120:127], a[72:79], v[244:247]\n\t" \
  "ds_read_b64_tr_b8 v[108:109], v65 offset:57344\n\t" \
  "ds_read_b64_tr_b8 v[110:111], v65 offset:58368\n\t" \
  "s_add_u32 m0, s60, 18432\n\t" \
  "s_nop 0\n\t" \
  "buffer_load_dwordx4 v82, s[36:39], s88 offen  lds\n\t" \
  "v_mfma_f32_16x16x128_f8f6f4 v[248:251], v[120:127], a[80:87], v[248:251]\n\t" \
  "s_add_u32 m0, s60, 19456\n\t" \
  "s_nop 0\n\t" \
  "buffer_load_dwordx4 v83, s[36:39], s88 offen  lds\n\t" \
  "ds_read_b128 a[112:115], v26 offset:4096\n\t" \
  "ds_read_b128 a[116:119], v27 offset:4096\n\t" \
  "v_mfma_f32_16x16x128_f8f6f4 v[252:255], v[120:127], a[88:95], v[252:255]\n\t" \
  "ds_read_b128 a[120:123], v26 offset:6144\n\t" \
  "ds_read_b128 a[124:127], v27 offset:6144\n\t" \
  "s_waitcnt lgkmcnt(0)\n\t" \
  "v_mfma_f32_16x16x128_f8f6f4 v[128:131], v[96:103], a[96:103], v[128:131]\n\t" \
  "ds_read_b64_tr_b8 v[112:113], v66 offset:49152\n\t" \
  "ds_read_b64_tr_b8 v[114:115], v66 offset:50176\n\t" \
  "v_mfma_f32_16x16x128_f8f6f4 v[132:135], v[96:103], a[104:111], v[132:135]\n\t" \
  "ds_read_b64_tr_b8 v[116:117], v66 offset:57344\n\t" \
  "ds_read_b64_tr_b8 v[118:119], v66 offset:58368\n\t" \
  "v_mfma_f32_16x16x128_f8f6f4 v[136:139], v[96:103], a[112:119], v[136:139]\n\t" \
  "v_mfma_f32_16x16x128_f8f6f4 v[140:143], v[96:103], a[120:127], v[140:143]\n\t" \
  "s_waitcnt lgkmcnt(8)\n\t" \
  "v_mfma_f32_16x16x128_f8f6f4 v[144:147], v[104:111], a[96:103], v[144:147]\n\t" \
  "ds_read_b64_tr_b8 v[120:121], v67 offset:49152\n\t" \
  "ds_read_b64_tr_b8 v[122:123], v67 offset:50176\n\t" \
  "v_mfma_f32_16x16x128_f8f6f4 v[148:151], v[104:111], a[104:111], v[148:151]\n\t" \
  "ds_read_b64_tr_b8 v[124:125], v67 offset:57344\n\t" \
  "ds_read_b64_tr_b8 v[126:127], v67 offset:58368\n\t" \
  "v_mfma_f32_16x16x128_f8f6f4 v[152:155], v[104:111], a[112:119], v[152:155]\n\t" \
  "v_mfma_f32_16x16x128_f8f6f4 v[156:159], v[104:111], a[120:127], v[156:159]\n\t" \
  "s_waitcnt lgkmcnt(4)\n\t" \
  "v_mfma_f32_16x16x128_f8f6f4 v[160:163], v[112:119], a[96:103], v[160:163]\n\t" \
  "ds_read_b64_tr_b8 v[96:97], v68 offset:49152\n\t" \
  "ds_read_b64_tr_b8 v[98:99], v68 offset:50176\n\t" \
  "v_mfma_f32_16x16x128_f8f6f4 v[164:167], v[112:119], a[104:111], v[164:167]\n\t" \
  "ds_read_b64_tr_b8 v[100:101], v68 offset:57344\n\t" \
  "ds_read_b64_tr_b8 v[102:103], v68 offset:58368\n\t" \
  "v_mfma_f32_16x16x128_f8f6f4 v[168:171], v[112:119], a[112:119], v[168:171]\n\t" \
  "v_mfma_f32_16x16x128_f8f6f4 v[172:175], v[112:119], a[120:127], v[172:175]\n\t" \
  "s_waitcnt lgkmcnt(4)\n\t" \
  "v_mfma_f32_16x16x128_f8f6f4 v[176:179], v[120:127], a[96:103], v[176:179]\n\t" \
  "ds_read_b64_tr_b8 v[104:105], v69 offset:49152\n\t" \
  "ds_read_b64_tr_b8 v[106:107], v69 offset:50176\n\t" \
  "v_mfma_f32_16x16x128_f8f6f4 v[180:183], v[120:127], a[104:111], v[180:183]\n\t" \
  "ds_read_b64_tr_b8 v[108:109], v69 offset:57344\n\t" \
  "ds_read_b64_tr_b8 v[110:111], v69 offset:58368\n\t" \
  "v_mfma_f32_16x16x128_f8f6f4 v[184:187], v[120:127], a[112:119], v[184:187]\n\t" \
  "v_mfma_f32_16x16x128_f8f6f4 v[188:191], v[120:127], a[120:127], v[188:191]\n\t" \
  "s_sub_u32 s84, s63, 2\n\t" \
  "s_cmp_gt_u32 s84, 1\n\t" \
  "s_cbranch_scc1 PR17_%=\n\t" \
  "s_waitcnt vmcnt(4)\n\t" \
  "ds_read_b128 v[32:35], v28 offset:0\n\t" \
  "ds_read_b128 v[36:39], v29 offset:0\n\t" \
  "ds_read_b128 v[40:43], v28 offset:2048\n\t" \
  "ds_read_b128 v[44:47], v29 offset:2048\n\t" \
  "ds_read_b128 v[48:51], v28 offset:4096\n\t" \
  "ds_read_b128 v[52:55], v29 offset:4096\n\t" \
  "ds_read_b128 v[56:59], v28 offset:6144\n\t" \
  "ds_read_b128 v[60:63], v29 offset:6144\n\t" \
  "s_waitcnt lgkmcnt(0)\n\t" \
  "PR17_%=:\n\t" \
  "s_waitcnt lgkmcnt(4)\n\t" \
  "v_mfma_f32_16x16x128_f8f6f4 v[192:195], v[96:103], a[96:103], v[192:195]\n\t" \
  "ds_read_b64_tr_b8 v[112:113], v70 offset:49152\n\t" \
  "ds_read_b64_tr_b8 v[114:115], v70 offset:50176\n\t" \
  "v_mfma_f32_16x16x128_f8f6f4 v[196:199], v[96:103], a[104:111], v[196:199]\n\t" \
  "ds_read_b64_tr_b8 v[116:117], v70 offset:57344\n\t" \
  "ds_read_b64_tr_b8 v[118:119], v70 offset:58368\n\t" \
  "v_mfma_f32_16x16x128_f8f6f4 v[200:203], v[96:103], a[112:119], v[200:203]\n\t" \
  "v_mfma_f32_16x16x128_f8f6f4 v[204:207], v[96:103], a[120:127], v[204:207]\n\t" \
  "s_waitcnt lgkmcnt(4)\n\t" \
  "v_mfma_f32_16x16x128_f8f6f4 v[208:211], v[104:111], a[96:103], v[208:211]\n\t" \
  "ds_read_b64_tr_b8 v[120:121], v71 offset:49152\n\t" \
  "ds_read_b64_tr_b8 v[122:123], v71 offset:50176\n\t" \
  "v_mfma_f32_16x16x128_f8f6f4 v[212:215], v[104:111], a[104:111], v[212:215]\n\t" \
  "ds_read_b64_tr_b8 v[124:125], v71 offset:57344\n\t" \
  "ds_read_b64_tr_b8 v[126:127], v71 offset:58368\n\t" \
  "v_mfma_f32_16x16x128_f8f6f4 v[216:219], v[104:111], a[112:119], v[216:219]\n\t" \
  "v_mfma_f32_16x16x128_f8f6f4 v[220:223], v[104:111], a[120:127], v[220:223]\n\t" \
  "s_waitcnt vmcnt(4)\n\t" \
  "s_barrier\n\t" \
  "s_waitcnt lgkmcnt(4)\n\t" \
  "v_mfma_f32_16x16x128_f8f6f4 v[224:227], v[112:119], a[96:103], v[224:227]\n\t" \
  "ds_read_b64_tr_b8 v[96:97], v64 offset:0\n\t" \
  "ds_read_b64_tr_b8 v[98:99], v64 offset:1024\n\t" \
  "v_mfma_f32_16x16x128_f8f6f4 v[228:231], v[112:119], a[104:111], v[228:231]\n\t" \
  "ds_read_b64_tr_b8 v[100:101], v64 offset:8192\n\t" \
  "ds_read_b64_tr_b8 v[102:103], v64 offset:9216\n\t" \
  "s_add_u32 s88, s54, 0\n\t" \
  "s_add_u32 m0, s60, 32768\n\t" \
  "s_nop 0\n\t" \
  "buffer_load_dwordx4 v80, s[36:39], s88 offen  lds\n\t" \
  "v_mfma_f32_16x16x128_f8f6f4 v[232:235], v[112:119], a[112:119], v[232:235]\n\t" \
  "s_add_u32 m0, s60, 33792\n\t" \
  "s_nop 0\n\t" \
  "buffer_load_dwordx4 v81, s[36:39], s88 offen  lds\n\t" \
  "v_mfma_f32_16x16x128_f8f6f4 v[236:239], v[112:119], a[120:127], v[236:239]\n\t" \
  "s_waitcnt lgkmcnt(4)\n\t" \
  "v_mfma_f32_16x16x128_f8f6f4 v[240:243], v[120:127], a[96:103], v[240:243]\n\t" \
  "ds_read_b64_tr_b8 v[104:105], v65 offset:0\n\t" \
  "ds_read_b64_tr_b8 v[106:107], v65 offset:1024\n\t" \
  "v_mfma_f32_16x16x128_f8f6f4 v[244:247], v[120:127], a[104:111], v[244:247]\n\t" \
  "ds_read_b64_tr_b8 v[108:109], v65 offset:8192\n\t" \
  "ds_read_b64_tr_b8 v[110:111], v65 offset:9216\n\t" \
  "s_add_u32 m0, s60, 34816\n\t" \
  "s_nop 0\n\t" \
  "buffer_load_dwordx4 v82, s[36:39], s88 offen  lds\n\t" \
  "v_mfma_f32_16x16x128_f8f6f4 v[248:251], v[120:127], a[112:119], v[248:251]\n\t" \
  "s_add_u32 m0, s60, 35840\n\t" \
  "s_nop 0\n\t" \
  "buffer_load_dwordx4 v83, s[36:39], s88 offen  lds\n\t" \
  "v_mfma_f32_16x16x128_f8f6f4 v[252:255], v[120:127], a[120:127], v[252:255]\n\t" \
  "LPE9_%=:\n\t" \
  "s_cmp_lg_u32 s10, 0\n\t" \
  "s_cbranch_scc0 VER0_%=\n\t" \
  "s_branch VER1_%=\n\t" \
  "VER0_%=:\n\t" \
  "s_bitcmp1_b32 s65, 4\n\t" \
  "s_cbranch_scc1 OK18_%=\n\t" \
  "s_mov_b32 s67, 4\n\t" \
  "s_call_b64 s[68:69], POLL_%=\n\t" \
  "OK18_%=:\n\t" \
  "s_add_u32 s89, s62, 262144\n\t" \
  "s_add_u32 s90, s62, 266240\n\t" \
  "buffer_load_dwordx4 a[128:131], v84, s[44:47], s89 offen offset:0 sc0 sc1\n\t" \
  "buffer_load_dwordx4 a[132:135], v84, s[44:47], s89 offen offset:1024 sc0 sc1\n\t" \
  "buffer_load_dwordx4 a[136:139], v84, s[44:47], s89 offen offset:2048 sc0 sc1\n\t" \
  "buffer_load_dwordx4 a[140:143], v84, s[44:47], s89 offen offset:3072 sc0 sc1\n\t" \
  "buffer_load_dwordx4 a[144:147], v84, s[44:47], s90 offen offset:0 sc0 sc1\n\t" \
  "buffer_load_dwordx4 a[148:151], v84, s[44:47], s90 offen offset:1024 sc0 sc1\n\t" \
  "buffer_load_dwordx4 a[152:155], v84, s[44:47], s90 offen offset:2048 sc0 sc1\n\t" \
  "buffer_load_dwordx4 a[156:159], v84, s[44:47], s90 offen offset:3072 sc0 sc1\n\t" \
  "s_bitcmp1_b32 s65, 5\n\t" \
  "s_cbranch_scc1 OK19_%=\n\t" \
  "s_mov_b32 s67, 5\n\t" \
  "s_call_b64 s[68:69], POLL_%=\n\t" \
  "OK19_%=:\n\t" \
  "s_add_u32 s89, s62, 327680\n\t" \
  "s_add_u32 s90, s62, 331776\n\t" \
  "buffer_load_dwordx4 a[160:163], v84, s[44:47], s89 offen offset:0 sc0 sc1\n\t" \
  "buffer_load_dwordx4 a[164:167], v84, s[44:47], s89 offen offset:1024 sc0 sc1\n\t" \
  "buffer_load_dwordx4 a[168:171], v84, s[44:47], s89 offen offset:2048 sc0 sc1\n\t" \
  "buffer_load_dwordx4 a[172:175], v84, s[44:47], s89 offen offset:3072 sc0 sc1\n\t" \
  "buffer_load_dwordx4 a[176:179], v84, s[44:47], s90 offen offset:0 sc0 sc1\n\t" \
  "buffer_load_dwordx4 a[180:183], v84, s[44:47], s90 offen offset:1024 sc0 sc1\n\t" \
  "buffer_load_dwordx4 a[184:187], v84, s[44:47], s90 offen offset:2048 sc0 sc1\n\t" \
  "buffer_load_dwordx4 a[188:191], v84, s[44:47], s90 offen offset:3072 sc0 sc1\n\t" \
  "s_bitcmp1_b32 s65, 6\n\t" \
  "s_cbranch_scc1 OK20_%=\n\t" \
  "s_mov_b32 s67, 6\n\t" \
  "s_call_b64 s[68:69], POLL_%=\n\t" \
  "OK20_%=:\n\t" \
  "s_add_u32 s89, s62, 393216\n\t" \
  "s_add_u32 s90, s62, 397312\n\t" \
  "buffer_load_dwordx4 a[192:195], v84, s[44:47], s89 offen offset:0 sc0 sc1\n\t" \
  "buffer_load_dwordx4 a[196:199], v84, s[44:47], s89 offen offset:1024 sc0 sc1\n\t" \
  "buffer_load_dwordx4 a[200:203], v84, s[44:47], s89 offen offset:2048 sc0 sc1\n\t" \
  "buffer_load_dwordx4 a[204:207], v84, s[44:47], s89 offen offset:3072 sc0 sc1\n\t" \
  "buffer_load_dwordx4 a[208:211], v84, s[44:47], s90 offen offset:0 sc0 sc1\n\t" \
  "buffer_load_dwordx4 a[212:215], v84, s[44:47], s90 offen offset:1024 sc0 sc1\n\t" \
  "buffer_load_dwordx4 a[216:219], v84, s[44:47], s90 offen offset:2048 sc0 sc1\n\t" \
  "buffer_load_dwordx4 a[220:223], v84, s[44:47], s90 offen offset:3072 sc0 sc1\n\t" \
  "s_waitcnt vmcnt(16)\n\t" \
  "s_bitcmp1_b32 s65, 7\n\t" \
  "s_cbranch_scc1 OK21_%=\n\t" \
  "s_mov_b32 s67, 7\n\t" \
  "s_call_b64 s[68:69], POLL_%=\n\t" \
  "OK21_%=:\n\t" \
  "s_bitcmp1_b32 s65, 8\n\t" \
  "s_cbranch_scc1 OK22_%=\n\t" \
  "s_mov_b32 s67, 8\n\t" \
  "s_call_b64 s[68:69], POLL_%=\n\t" \
  "OK22_%=:\n\t" \
  "s_waitcnt lgkmcnt(4)\n\t" \
  "v_mfma_f32_16x16x128_f8f6f4 v[128:131], v[96:103], a[128:135], v[128:131]\n\t" \
  "ds_read_b64_tr_b8 v[112:113], v66 offset:0\n\t" \
  "ds_read_b64_tr_b8 v[114:115], v66 offset:1024\n\t" \
  "v_mfma_f32_16x16x128_f8f6f4 v[132:135], v[96:103], a[136:143], v[132:135]\n\t" \
  "ds_read_b64_tr_b8 v[116:117], v66 offset:8192\n\t" \
  "ds_read_b64_tr_b8 v[118:119], v66 offset:9216\n\t" \
  "s_add_u32 s88, s55, 0\n\t" \
  "s_add_u32 m0, s60, 65536\n\t" \
  "s_nop 0\n\t" \
  "buffer_load_dwordx4 v80, s[36:39], s88 offen  lds\n\t" \
  "v_mfma_f32_16x16x128_f8f6f4 v[136:139], v[96:103], a[144:151], v[136:139]\n\t" \
  "s_add_u32 m0, s60, 66560\n\t" \
  "s_nop 0\n\t" \
  "buffer_load_dwordx4 v81, s[36:39], s88 offen  lds\n\t" \
  "v_mfma_f32_16x16x128_f8f6f4 v[140:143], v[96:103], a[152:159], v[140:143]\n\t" \
  "s_waitcnt lgkmcnt(4)\n\t" \
  "v_mfma_f32_16x16x128_f8f6f4 v[144:147], v[104:111], a[128:135], v[144:147]\n\t" \
  "ds_read_b64_tr_b8 v[120:121], v67 offset:0\n\t" \
  "ds_read_b64_tr_b8 v[122:123], v67 offset:1024\n\t" \
  "v_mfma_f32_16x16x128_f8f6f4 v[148:151], v[104:111], a[136:143], v[148:151]\n\t" \
  "ds_read_b64_tr_b8 v[124:125], v67 offset:8192\n\t" \
  "ds_read_b64_tr_b8 v[126:127], v67 offset:9216\n\t" \
  "s_add_u32 m0, s60, 67584\n\t" \
  "s_nop 0\n\t" \
  "buffer_load_dwordx4 v82, s[36:39], s88 offen  lds\n\t" \
  "v_mfma_f32_16x16x128_f8f6f4 v[152:155], v[104:111], a[144:151], v[152:155]\n\t" \
  "s_add_u32 m0, s60, 68608\n\t" \
  "s_nop 0\n\t" \
  "buffer_load_dwordx4 v83, s[36:39], s88 offen  lds\n\t" \
  "v_mfma_f32_16x16x128_f8f6f4 v[156:159], v[104:111], a[152:159], v[156:159]\n\t" \
  "s_waitcnt lgkmcnt(4)\n\t" \
  "v_mfma_f32_16x16x128_f8f6f4 v[160:163], v[112:119], a[128:135], v[160:163]\n\t" \
  "ds_read_b64_tr_b8 v[96:97], v68 offset:0\n\t" \
  "ds_read_b64_tr_b8 v[98:99], v68 offset:1024\n\t" \
  "v_mfma_f32_16x16x128_f8f6f4 v[164:167], v[112:119], a[136:143], v[164:167]\n\t" \
  "ds_read_b64_tr_b8 v[100:101], v68 offset:8192\n\t" \
  "ds_read_b64_tr_b8 v[102:103], v68 offset:9216\n\t" \
  "s_add_u32 s88, s56, 0\n\t" \
  "s_add_u32 m0, s60, 81920\n\t" \
  "s_nop 0\n\t" \
  "buffer_load_dwordx4 v80, s[36:39], s88 offen  lds\n\t" \
  "v_mfma_f32_16x16x128_f8f6f4 v[168:171], v[112:119], a[144:151], v[168:171]\n\t" \
  "s_add_u32 m0, s60, 82944\n\t" \
  "s_nop 0\n\t" \
  "buffer_load_dwordx4 v81, s[36:39], s88 offen  lds\n\t" \
  "v_mfma_f32_16x16x128_f8f6f4 v[172:175], v[112:119], a[152:159], v[172:175]\n\t" \
  "s_waitcnt lgkmcnt(4)\n\t" \
  "v_mfma_f32_16x16x128_f8f6f4 v[176:179], v[120:127], a[128:135], v[176:179]\n\t" \
  "ds_read_b64_tr_b8 v[104:105], v69 offset:0\n\t" \
  "ds_read_b64_tr_b8 v[106:107], v69 offset:1024\n\t" \
  "v_mfma_f32_16x16x128_f8f6f4 v[180:183], v[120:127], a[136:143], v[180:183]\n\t" \
  "ds_read_b64_tr_b8 v[108:109], v69 offset:8192\n\t" \
  "ds_read_b64_tr_b8 v[110:111], v69 offset:9216\n\t" \
  "s_add_u32 m0, s60, 83968\n\t" \
  "s_nop 0\n\t" \
  "buffer_load_dwordx4 v82, s[36:39], s88 offen  lds\n\t" \
  "v_mfma_f32_16x16x128_f8f6f4 v[184:187], v[120:127], a[144:151], v[184:187]\n\t" \
  "s_add_u32 m0, s60, 84992\n\t" \
  "s_nop 0\n\t" \
  "buffer_load_dwordx4 v83, s[36:39], s88 offen  lds\n\t" \
  "v_mfma_f32_16x16x128_f8f6f4 v[188:191], v[120:127], a[152:159], v[188:191]\n\t" \
  "s_waitcnt lgkmcnt(4)\n\t" \
  "v_mfma_f32_16x16x128_f8f6f4 v[192:195], v[96:103], a[128:135], v[192:195]\n\t" \
  "ds_read_b64_tr_b8 v[112:113], v70 offset:0\n\t" \
  "ds_read_b64_tr_b8 v[114:115], v70 offset:1024\n\t" \
  "v_mfma_f32_16x16x128_f8f6f4 v[196:199], v[96:103], a[136:143], v[196:199]\n\t" \
  "ds_read_b64_tr_b8 v[116:117], v70 offset:8192\n\t" \
  "ds_read_b64_tr_b8 v[118:119], v70 offset:9216\n\t" \
  "s_add_u32 s89, s62, 458752\n\t" \
  "s_add_u32 s90, s62, 462848\n\t" \
  "buffer_load_dwordx4 a[224:227], v84, s[44:47], s89 offen offset:0 sc0 sc1\n\t" \
  "v_mfma_f32_16x16x128_f8f6f4 v[200:203], v[96:103], a[144:151], v[200:203]\n\t" \
  "buffer_load_dwordx4 a[228:231], v84, s[44:47], s89 offen offset:1024 sc0 sc1\n\t" \
  "v_mfma_f32_16x16x128_f8f6f4 v[204:207], v[96:103], a[152:159], v[204:207]\n\t" \
  "s_waitcnt lgkmcnt(4)\n\t" \
  "v_mfma_f32_16x16x128_f8f6f4 v[208:211], v[104:111], a[128:135], v[208:211]\n\t" \
  "ds_read_b64_tr_b8 v[120:121], v71 offset:0\n\t" \
  "ds_read_b64_tr_b8 v[122:123], v71 offset:1024\n\t" \
  "v_mfma_f32_16x16x128_f8f6f4 v[212:215], v[104:111], a[136:143], v[212:215]\n\t" \
  "ds_read_b64_tr_b8 v[124:125], v71 offset:8192\n\t" \
  "ds_read_b64_tr_b8 v[126:127], v71 offset:9216\n\t" \
  "buffer_load_dwordx4 a[232:235], v84, s[44:47], s89 offen offset:2048 sc0 sc1\n\t" \
  "v_mfma_f32_16x16x128_f8f6f4 v[216:219], v[104:111], a[144:151], v[216:219]\n\t" \
  "buffer_load_dwordx4 a[236:239], v84, s[44:47], s89 offen offset:3072 sc0 sc1\n\t" \
  "v_mfma_f32_16x16x128_f8f6f4 v[220:223], v[104:111], a[152:159], v[220:223]\n\t" \
  "s_waitcnt vmcnt(40)\n\t" \
  "s_barrier\n\t" \
  "s_waitcnt lgkmcnt(4)\n\t" \
  "v_mfma_f32_16x16x128_f8f6f4 v[224:227], v[112:119], a[128:135], v[224:227]\n\t" \
  "ds_read_b64_tr_b8 v[96:97], v64 offset:16384\n\t" \
  "ds_read_b64_tr_b8 v[98:99], v64 offset:17408\n\t" \
  "v_mfma_f32_16x16x128_f8f6f4 v[228:231], v[112:119], a[136:143], v[228:231]\n\t" \
  "ds_read_b64_tr_b8 v[100:101], v64 offset:24576\n\t" \
  "ds_read_b64_tr_b8 v[102:103], v64 offset:25600\n\t" \
  "buffer_load_dwordx4 a[240:243], v84, s[44:47], s90 offen offset:0 sc0 sc1\n\t" \
  "v_mfma_f32_16x16x128_f8f6f4 v[232:235], v[112:119], a[144:151], v[232:235]\n\t" \
  "buffer_load_dwordx4 a[244:247], v84, s[44:47], s90 offen offset:1024 sc0 sc1\n\t" \
  "v_mfma_f32_16x16x128_f8f6f4 v[236:239], v[112:119], a[152:159], v[236:239]\n\t" \
  "s_cmp_lg_u32 s63, 2\n\t" \
  "s_cbranch_scc1 PS23_%=\n\t" \
  "global_store_dwordx4 v84, v[32:35], s[72:73] offset:0 sc0 sc1\n\t" \
  "global_store_dwordx4 v84, v[36:39], s[72:73] offset:1024 sc0 sc1\n\t" \
  "global_store_dwordx4 v84, v[40:43], s[72:73] offset:2048 sc0 sc1\n\t" \
  "global_store_dwordx4 v84, v[44:47], s[72:73] offset:3072 sc0 sc1\n\t" \
  "global_store_dwordx4 v84, v[48:51], s[74:75] offset:0 sc0 sc1\n\t" \
  "global_store_dwordx4 v84, v[52:55], s[74:75] offset:1024 sc0 sc1\n\t" \
  "global_store_dwordx4 v84, v[56:59], s[74:75] offset:2048 sc0 sc1\n\t" \
  "global_store_dwordx4 v84, v[60:63], s[74:75] offset:3072 sc0 sc1\n\t" \
  "s_nop 1\n\t" \
  "PS23_%=:\n\t" \
  "s_waitcnt lgkmcnt(4)\n\t" \
  "v_mfma_f32_16x16x128_f8f6f4 v[240:243], v[120:127], a[128:135], v[240:243]\n\t" \
  "ds_read_b64_tr_b8 v[104:105], v65 offset:16384\n\t" \
  "ds_read_b64_tr_b8 v[106:107], v65 offset:17408\n\t" \
  "v_mfma_f32_16x16x128_f8f6f4 v[244:247], v[120:127], a[136:143], v[244:247]\n\t" \
  "ds_read_b64_tr_b8 v[108:109], v65 offset:24576\n\t" \
  "ds_read_b64_tr_b8 v[110:111], v65 offset:25600\n\t" \
  "buffer_load_dwordx4 a[248:251], v84, s[44:47], s90 offen offset:2048 sc0 sc1\n\t" \
  "v_mfma_f32_16x16x128_f8f6f4 v[248:251], v[120:127], a[144:151], v[248:251]\n\t" \
  "buffer_load_dwordx4 a[252:255], v84, s[44:47], s90 offen offset:3072 sc0 sc1\n\t" \
  "v_mfma_f32_16x16x128_f8f6f4 v[252:255], v[120:127], a[152:159], v[252:255]\n\t" \
  "s_waitcnt vmcnt(24)\n\t" \
  "s_bitcmp1_b32 s65, 9\n\t" \
  "s_cbranch_scc1 OK24_%=\n\t" \
  "s_mov_b32 s67, 9\n\t" \
  "s_call_b64 s[68:69], POLL_%=\n\t" \
  "OK24_%=:\n\t" \
  "s_waitcnt lgkmcnt(4)\n\t" \
  "v_mfma_f32_16x16x128_f8f6f4 v[128:131], v[96:103], a[160:167], v[128:131]\n\t" \
  "ds_read_b64_tr_b8 v[112:113], v66 offset:16384\n\t" \
  "ds_read_b64_tr_b8 v[114:115], v66 offset:17408\n\t" \
  "v_mfma_f32_16x16x128_f8f6f4 v[132:135], v[96:103], a[168:175], v[132:135]\n\t" \
  "ds_read_b64_tr_b8 v[116:117], v66 offset:24576\n\t" \
  "ds_read_b64_tr_b8 v[118:119], v66 offset:25600\n\t" \
  "s_add_u32 s89, s62, 524288\n\t" \
  "s_add_u32 s90, s62, 528384\n\t" \
  "buffer_load_dwordx4 a[0:3], v84, s[44:47], s89 offen offset:0 sc0 sc1\n\t" \
  "v_mfma_f32_16x16x128_f8f6f4 v[136:139], v[96:103], a[176:183], v[136:139]\n\t" \
  "buffer_load_dwordx4 a[4:7], v84, s[44:47], s89 offen offset:1024 sc0 sc1\n\t" \
  "v_mfma_f32_16x16x128_f8f6f4 v[140:143], v[96:103], a[184:191], v[140:143]\n\t" \
  "s_waitcnt lgkmcnt(4)\n\t" \
  "v_mfma_f32_16x16x128_f8f6f4 v[144:147], v[104:111], a[160:167], v[144:147]\n\t" \
  "ds_read_b64_tr_b8 v[120:121], v67 offset:16384\n\t" \
  "ds_read_b64_tr_b8 v[122:123], v67 offset:17408\n\t" \
  "v_mfma_f32_16x16x128_f8f6f4 v[148:151], v[104:111], a[168:175], v[148:151]\n\t" \
  "ds_read_b64_tr_b8 v[124:125], v67 offset:24576\n\t" \
  "ds_read_b64_tr_b8 v[126:127], v67 offset:25600\n\t" \
  "buffer_load_dwordx4 a[8:11], v84, s[44:47], s89 offen offset:2048 sc0 sc1\n\t" \
  "v_mfma_f32_16x16x128_f8f6f4 v[152:155], v[104:111], a[176:183], v[152:155]\n\t" \
  "buffer_load_dwordx4 a[12:15], v84, s[44:47], s89 offen offset:3072 sc0 sc1\n\t" \
  "v_mfma_f32_16x16x128_f8f6f4 v[156:159], v[104:111], a[184:191], v[156:159]\n\t" \
  "s_waitcnt lgkmcnt(4)\n\t" \
  "v_mfma_f32_16x16x128_f8f6f4 v[160:163], v[112:119], a[160:167], v[160:163]\n\t" \
  "ds_read_b64_tr_b8 v[96:97], v68 offset:16384\n\t" \
  "ds_read_b64_tr_b8 v[98:99], v68 offset:17408\n\t" \
  "v_mfma_f32_16x16x128_f8f6f4 v[164:167], v[112:119], a[168:175], v[164:167]\n\t" \
  "ds_read_b64_tr_b8 v[100:101], v68 offset:24576\n\t" \
  "ds_read_b64_tr_b8 v[102:103], v68 offset:25600\n\t" \
  "buffer_load_dwordx4 a[16:19], v84, s[44:47], s90 offen offset:0 sc0 sc1\n\t" \
  "v_mfma_f32_16x16x128_f8f6f4 v[168:171], v[112:119], a[176:183], v[168:171]\n\t" \
  "buffer_load_dwordx4 a[20:23], v84, s[44:47], s90 offen offset:1024 sc0 sc1\n\t" \
  "v_mfma_f32_16x16x128_f8f6f4 v[172:175], v[112:119], a[184:191], v[172:175]\n\t" \
  "s_waitcnt lgkmcnt(4)\n\t" \
  "v_mfma_f32_16x16x128_f8f6f4 v[176:179], v[120:127], a[160:167], v[176:179]\n\t" \
  "ds_read_b64_tr_b8 v[104:105], v69 offset:16384\n\t" \
  "ds_read_b64_tr_b8 v[106:107], v69 offset:17408\n\t" \
  "v_mfma_f32_16x16x128_f8f6f4 v[180:183], v[120:127], a[168:175], v[180:183]\n\t" \
  "ds_read_b64_tr_b8 v[108:109], v69 offset:24576\n\t" \
  "ds_read_b64_tr_b8 v[110:111], v69 offset:25600\n\t" \
  "buffer_load_dwordx4 a[24:27], v84, s[44:47], s90 offen offset:2048 sc0 sc1\n\t" \
  "v_mfma_f32_16x16x128_f8f6f4 v[184:187], v[120:127], a[176:183], v[184:187]\n\t" \
  "buffer_load_dwordx4 a[28:31], v84, s[44:47], s90 offen offset:3072 sc0 sc1\n\t" \
  "v_mfma_f32_16x16x128_f8f6f4 v[188:191], v[120:127], a[184:191], v[188:191]\n\t" \
  "s_waitcnt lgkmcnt(4)\n\t" \
  "v_mfma_f32_16x16x128_f8f6f4 v[192:195], v[96:103], a[160:167], v[192:195]\n\t" \
  "ds_read_b64_tr_b8 v[112:113], v70 offset:16384\n\t" \
  "ds_read_b64_tr_b8 v[114:115], v70 offset:17408\n\t" \
  "v_mfma_f32_16x16x128_f8f6f4 v[196:199], v[96:103], a[168:175], v[196:199]\n\t" \
  "ds_read_b64_tr_b8 v[116:117], v70 offset:24576\n\t" \
  "ds_read_b64_tr_b8 v[118:119], v70 offset:25600\n\t" \
  "s_add_u32 s89, s62, 589824\n\t" \
  "s_add_u32 s90, s62, 593920\n\t" \
  "buffer_load_dwordx4 a[32:35], v84, s[44:47], s89 offen offset:0 sc0 sc1\n\t" \
  "v_mfma_f32_16x16x128_f8f6f4 v[200:203], v[96:103], a[176:183], v[200:203]\n\t" \
  "buffer_load_dwordx4 a[36:39], v84, s[44:47], s89 offen offset:1024 sc0 sc1\n\t" \
  "v_mfma_f32_16x16x128_f8f6f4 v[204:207], v[96:103], a[184:191], v[204:207]\n\t" \
  "s_waitcnt lgkmcnt(4)\n\t" \
  "v_mfma_f32_16x16x128_f8f6f4 v[208:211], v[104:111], a[160:167], v[208:211]\n\t" \
  "ds_read_b64_tr_b8 v[120:121], v71 offset:16384\n\t" \
  "ds_read_b64_tr_b8 v[122:123], v71 offset:17408\n\t" \
  "v_mfma_f32_16x16x128_f8f6f4 v[212:215], v[104:111], a[168:175], v[212:215]\n\t" \
  "ds_read_b64_tr_b8 v[124:125], v71 offset:24576\n\t" \
  "ds_read_b64_tr_b8 v[126:127], v71 offset:25600\n\t" \
  "buffer_load_dwordx4 a[40:43], v84, s[44:47], s89 offen offset:2048 sc0 sc1\n\t" \
  "v_mfma_f32_16x16x128_f8f6f4 v[216:219], v[104:111], a[176:183], v[216:219]\n\t" \
  "buffer_load_dwordx4 a[44:47], v84, s[44:47], s89 offen offset:3072 sc0 sc1\n\t" \
  "v_mfma_f32_16x16x128_f8f6f4 v[220:223], v[104:111], a[184:191], v[220:223]\n\t" \
  "s_waitcnt vmcnt(52)\n\t" \
  "s_barrier\n\t" \
  "s_waitcnt lgkmcnt(4)\n\t" \
  "v_mfma_f32_16x16x128_f8f6f4 v[224:227], v[112:119], a[160:167], v[224:227]\n\t" \
  "ds_read_b64_tr_b8 v[96:97], v64 offset:32768\n\t" \
  "ds_read_b64_tr_b8 v[98:99], v64 offset:33792\n\t" \
  "v_mfma_f32_16x16x128_f8f6f4 v[228:231], v[112:119], a[168:175], v[228:231]\n\t" \
  "ds_read_b64_tr_b8 v[100:101], v64 offset:40960\n\t" \
  "ds_read_b64_tr_b8 v[102:103], v64 offset:41984\n\t" \
  "buffer_load_dwordx4 a[48:51], v84, s[44:47], s90 offen offset:0 sc0 sc1\n\t" \
  "v_mfma_f32_16x16x128_f8f6f4 v[232:235], v[112:119], a[176:183], v[232:235]\n\t" \
  "buffer_load_dwordx4 a[52:55], v84, s[44:47], s90 offen offset:1024 sc0 sc1\n\t" \
  "v_mfma_f32_16x16x128_f8f6f4 v[236:239], v[112:119], a[184:191], v[236:239]\n\t" \
  "s_waitcnt lgkmcnt(4)\n\t" \
  "v_mfma_f32_16x16x128_f8f6f4 v[240:243], v[120:127], a[160:167], v[240:243]\n\t" \
  "ds_read_b64_tr_b8 v[104:105], v65 offset:32768\n\t" \
  "ds_read_b64_tr_b8 v[106:107], v65 offset:33792\n\t" \
  "v_mfma_f32_16x16x128_f8f6f4 v[244:247], v[120:127], a[168:175], v[244:247]\n\t" \
  "ds_read_b64_tr_b8 v[108:109], v65 offset:40960\n\t" \
  "ds_read_b64_tr_b8 v[110:111], v65 offset:41984\n\t" \
  "buffer_load_dwordx4 a[56:59], v84, s[44:47], s90 offen offset:2048 sc0 sc1\n\t" \
  "v_mfma_f32_16x16x128_f8f6f4 v[248:251], v[120:127], a[176:183], v[248:251]\n\t" \
  "buffer_load_dwordx4 a[60:63], v84, s[44:47], s90 offen offset:3072 sc0 sc1\n\t" \
  "v_mfma_f32_16x16x128_f8f6f4 v[252:255], v[120:127], a[184:191], v[252:255]\n\t" \
  "s_waitcnt vmcnt(32)\n\t" \
  "global_load_dword v92, v87, s[70:71] sc0 sc1\n\t" \
  "s_bitcmp1_b32 s65, 10\n\t" \
  "s_cbranch_scc1 OK25_%=\n\t" \
  "s_mov_b32 s67, 10\n\t" \
  "s_call_b64 s[68:69], POLL_%=\n\t" \
  "OK25_%=:\n\t" \
  "s_waitcnt lgkmcnt(4)\n\t" \
  "v_mfma_f32_16x16x128_f8f6f4 v[128:131], v[96:103], a[192:199], v[128:131]\n\t" \
  "ds_read_b64_tr_b8 v[112:113], v66 offset:32768\n\t" \
  "ds_read_b64_tr_b8 v[114:115], v66 offset:33792\n\t" \
  "v_mfma_f32_16x16x128_f8f6f4 v[132:135], v[96:103], a[200:207], v[132:135]\n\t" \
  "ds_read_b64_tr_b8 v[116:117], v66 offset:40960\n\t" \
  "ds_read_b64_tr_b8 v[118:119], v66 offset:41984\n\t" \
  "s_add_u32 s89, s62, 655360\n\t" \
  "s_add_u32 s90, s62, 659456\n\t" \
  "buffer_load_dwordx4 a[64:67], v84, s[44:47], s89 offen offset:0 sc0 sc1\n\t" \
  "v_mfma_f32_16x16x128_f8f6f4 v[136:139], v[96:103], a[208:215], v[136:139]\n\t" \
  "buffer_load_dwordx4 a[68:71], v84, s[44:47], s89 offen offset:1024 sc0 sc1\n\t" \
  "v_mfma_f32_16x16x128_f8f6f4 v[140:143], v[96:103], a[216:223], v[140:143]\n\t" \
  "s_waitcnt lgkmcnt(4)\n\t" \
  "v_mfma_f32_16x16x128_f8f6f4 v[144:147], v[104:111], a[192:199], v[144:147]\n\t" \
  "ds_read_b64_tr_b8 v[120:121], v67 offset:32768\n\t" \
  "ds_read_b64_tr_b8 v[122:123], v67 offset:33792\n\t" \
  "v_mfma_f32_16x16x128_f8f6f4 v[148:151], v[104:111], a[200:207], v[148:151]\n\t" \
  "ds_read_b64_tr_b8 v[124:125], v67 offset:40960\n\t" \
  "ds_read_b64_tr_b8 v[126:127], v67 offset:41984\n\t" \
  "buffer_load_dwordx4 a[72:75], v84, s[44:47], s89 offen offset:2048 sc0 sc1\n\t" \
  "v_mfma_f32_16x16x128_f8f6f4 v[152:155], v[104:111], a[208:215], v[152:155]\n\t" \
  "buffer_load_dwordx4 a[76:79], v84, s[44:47], s89 offen offset:3072 sc0 sc1\n\t" \
  "v_mfma_f32_16x16x128_f8f6f4 v[156:159], v[104:111], a[216:223], v[156:159]\n\t" \
  "s_waitcnt lgkmcnt(4)\n\t" \
  "v_mfma_f32_16x16x128_f8f6f4 v[160:163], v[112:119], a[192:199], v[160:163]\n\t" \
  "ds_read_b64_tr_b8 v[96:97], v68 offset:32768\n\t" \
  "ds_read_b64_tr_b8 v[98:99], v68 offset:33792\n\t" \
  "v_mfma_f32_16x16x128_f8f6f4 v[164:167], v[112:119], a[200:207], v[164:167]\n\t" \
  "ds_read_b64_tr_b8 v[100:101], v68 offset:40960\n\t" \
  "ds_read_b64_tr_b8 v[102:103], v68 offset:41984\n\t" \
  "buffer_load_dwordx4 a[80:83], v84, s[44:47], s90 offen offset:0 sc0 sc1\n\t" \
  "v_mfma_f32_16x16x128_f8f6f4 v[168:171], v[112:119], a[208:215], v[168:171]\n\t" \
  "buffer_load_dwordx4 a[84:87], v84, s[44:47], s90 offen offset:1024 sc0 sc1\n\t" \
  "v_mfma_f32_16x16x128_f8f6f4 v[172:175], v[112:119], a[216:223], v[172:175]\n\t" \
  "s_waitcnt lgkmcnt(4)\n\t" \
  "v_mfma_f32_16x16x128_f8f6f4 v[176:179], v[120:127], a[192:199], v[176:179]\n\t" \
  "ds_read_b64_tr_b8 v[104:105], v69 offset:32768\n\t" \
  "ds_read_b64_tr_b8 v[106:107], v69 offset:33792\n\t" \
  "v_mfma_f32_16x16x128_f8f6f4 v[180:183], v[120:127], a[200:207], v[180:183]\n\t" \
  "ds_read_b64_tr_b8 v[108:109], v69 offset:40960\n\t" \
  "ds_read_b64_tr_b8 v[110:111], v69 offset:41984\n\t" \
  "buffer_load_dwordx4 a[88:91], v84, s[44:47], s90 offen offset:2048 sc0 sc1\n\t" \
  "v_mfma_f32_16x16x128_f8f6f4 v[184:187], v[120:127], a[208:215], v[184:187]\n\t" \
  "buffer_load_dwordx4 a[92:95], v84, s[44:47], s90 offen offset:3072 sc0 sc1\n\t" \
  "v_mfma_f32_16x16x128_f8f6f4 v[188:191], v[120:127], a[216:223], v[188:191]\n\t" \
  "s_waitcnt lgkmcnt(4)\n\t" \
  "v_mfma_f32_16x16x128_f8f6f4 v[192:195], v[96:103], a[192:199], v[192:195]\n\t" \
  "ds_read_b64_tr_b8 v[112:113], v70 offset:32768\n\t" \
  "ds_read_b64_tr_b8 v[114:115], v70 offset:33792\n\t" \
  "v_mfma_f32_16x16x128_f8f6f4 v[196:199], v[96:103], a[200:207], v[196:199]\n\t" \
  "ds_read_b64_tr_b8 v[116:117], v70 offset:40960\n\t" \
  "ds_read_b64_tr_b8 v[118:119], v70 offset:41984\n\t" \
  "v_mfma_f32_16x16x128_f8f6f4 v[200:203], v[96:103], a[208:215], v[200:203]\n\t" \
  "v_mfma_f32_16x16x128_f8f6f4 v[204:207], v[96:103], a[216:223], v[204:207]\n\t" \
  "s_waitcnt lgkmcnt(4)\n\t" \
  "v_mfma_f32_16x16x128_f8f6f4 v[208:211], v[104:111], a[192:199], v[208:211]\n\t" \
  "ds_read_b64_tr_b8 v[120:121], v71 offset:32768\n\t" \
  "ds_read_b64_tr_b8 v[122:123], v71 offset:33792\n\t" \
  "v_mfma_f32_16x16x128_f8f6f4 v[212:215], v[104:111], a[200:207], v[212:215]\n\t" \
  "ds_read_b64_tr_b8 v[124:125], v71 offset:40960\n\t" \
  "ds_read_b64_tr_b8 v[126:127], v71 offset:41984\n\t" \
  "v_mfma_f32_16x16x128_f8f6f4 v[216:219], v[104:111], a[208:215], v[216:219]\n\t" \
  "v_mfma_f32_16x16x128_f8f6f4 v[220:223], v[104:111], a[216:223], v[220:223]\n\t" \
  "s_waitcnt vmcnt(37)\n\t" \
  "s_barrier\n\t" \
  "s_waitcnt lgkmcnt(4)\n\t" \
  "v_mfma_f32_16x16x128_f8f6f4 v[224:227], v[112:119], a[192:199], v[224:227]\n\t" \
  "ds_read_b64_tr_b8 v[96:97], v72 offset:0\n\t" \
  "ds_read_b64_tr_b8 v[98:99], v72 offset:1024\n\t" \
  "v_mfma_f32_16x16x128_f8f6f4 v[228:231], v[112:119], a[200:207], v[228:231]\n\t" \
  "ds_read_b64_tr_b8 v[100:101], v72 offset:8192\n\t" \
  "ds_read_b64_tr_b8 v[102:103], v72 offset:9216\n\t" \
  "v_mfma_f32_16x16x128_f8f6f4 v[232:235], v[112:119], a[208:215], v[232:235]\n\t" \
  "v_mfma_f32_16x16x128_f8f6f4 v[236:239], v[112:119], a[216:223], v[236:239]\n\t" \
  "s_waitcnt lgkmcnt(4)\n\t" \
  "v_mfma_f32_16x16x128_f8f6f4 v[240:243], v[120:127], a[192:199], v[240:243]\n\t" \
  "ds_read_b64_tr_b8 v[104:105], v73 offset:0\n\t" \
  "ds_read_b64_tr_b8 v[106:107], v73 offset:1024\n\t" \
  "v_mfma_f32_16x16x128_f8f6f4 v[244:247], v[120:127], a[200:207], v[244:247]\n\t" \
  "ds_read_b64_tr_b8 v[108:109], v73 offset:8192\n\t" \
  "ds_read_b64_tr_b8 v[110:111], v73 offset:9216\n\t" \
  "v_mfma_f32_16x16x128_f8f6f4 v[248:251], v[120:127], a[208:215], v[248:251]\n\t" \
  "v_mfma_f32_16x16x128_f8f6f4 v[252:255], v[120:127], a[216:223], v[252:255]\n\t" \
  "s_waitcnt vmcnt(8)\n\t" \
  "v_cmp_le_u32 vcc, s64, v92\n\t" \
  "s_nop 1\n\t" \
  "s_or_b32 s65, s65, vcc_lo\n\t" \
  "s_bitcmp1_b32 s65, 11\n\t" \
  "s_cbranch_scc1 OK26_%=\n\t" \
  "s_mov_b32 s67, 11\n\t" \
  "s_call_b64 s[68:69], POLL_%=\n\t" \
  "OK26_%=:\n\t" \
  "s_waitcnt lgkmcnt(4)\n\t" \
  "v_mfma_f32_16x16x128_f8f6f4 v[128:131], v[96:103], a[224:231], v[128:131]\n\t" \
  "ds_read_b64_tr_b8 v[112:113], v74 offset:0\n\t" \
  "ds_read_b64_tr_b8 v[114:115], v74 offset:1024\n\t" \
  "v_mfma_f32_16x16x128_f8f6f4 v[132:135], v[96:103], a[232:239], v[132:135]\n\t" \
  "ds_read_b64_tr_b8 v[116:117], v74 offset:8192\n\t" \
  "ds_read_b64_tr_b8 v[118:119], v74 offset:9216\n\t" \
  "s_add_u32 s89, s62, 720896\n\t" \
  "s_add_u32 s90, s62, 724992\n\t" \
  "buffer_load_dwordx4 a[96:99], v84, s[44:47], s89 offen offset:0 sc0 sc1\n\t" \
  "v_mfma_f32_16x16x128_f8f6f4 v[136:139], v[96:103], a[240:247], v[136:139]\n\t" \
  "buffer_load_dwordx4 a[100:103], v84, s[44:47], s89 offen offset:1024 sc0 sc1\n\t" \
  "v_mfma_f32_16x16x128_f8f6f4 v[140:143], v[96:103], a[248:255], v[140:143]\n\t" \
  "s_waitcnt lgkmcnt(4)\n\t" \
  "v_mfma_f32_16x16x128_f8f6f4 v[144:147], v[104:111], a[224:231], v[144:147]\n\t" \
  "ds_read_b64_tr_b8 v[120:121], v75 offset:0\n\t" \
  "ds_read_b64_tr_b8 v[122:123], v75 offset:1024\n\t" \
  "v_mfma_f32_16x16x128_f8f6f4 v[148:151], v[104:111], a[232:239], v[148:151]\n\t" \
  "ds_read_b64_tr_b8 v[124:125], v75 offset:8192\n\t" \
  "ds_read_b64_tr_b8 v[126:127], v75 offset:9216\n\t" \
  "buffer_load_dwordx4 a[104:107], v84, s[44:47], s89 offen offset:2048 sc0 sc1\n\t" \
  "v_mfma_f32_16x16x128_f8f6f4 v[152:155], v[104:111], a[240:247], v[152:155]\n\t" \
  "buffer_load_dwordx4 a[108:111], v84, s[44:47], s89 offen offset:3072 sc0 sc1\n\t" \
  "v_mfma_f32_16x16x128_f8f6f4 v[156:159], v[104:111], a[248:255], v[156:159]\n\t" \
  "s_waitcnt lgkmcnt(4)\n\t" \
  "v_mfma_f32_16x16x128_f8f6f4 v[160:163], v[112:119], a[224:231], v[160:163]\n\t" \
  "ds_read_b64_tr_b8 v[96:97], v76 offset:0\n\t" \
  "ds_read_b64_tr_b8 v[98:99], v76 offset:1024\n\t" \
  "v_mfma_f32_16x16x128_f8f6f4 v[164:167], v[112:119], a[232:239], v[164:167]\n\t" \
  "ds_read_b64_tr_b8 v[100:101], v76 offset:8192\n\t" \
  "ds_read_b64_tr_b8 v[102:103], v76 offset:9216\n\t" \
  "buffer_load_dwordx4 a[112:115], v84, s[44:47], s90 offen offset:0 sc0 sc1\n\t" \
  "v_mfma_f32_16x16x128_f8f6f4 v[168:171], v[112:119], a[240:247], v[168:171]\n\t" \
  "buffer_load_dwordx4 a[116:119], v84, s[44:47], s90 offen offset:1024 sc0 sc1\n\t" \
  "v_mfma_f32_16x16x128_f8f6f4 v[172:175], v[112:119], a[248:255], v[172:175]\n\t" \
  "s_waitcnt lgkmcnt(4)\n\t" \
  "v_mfma_f32_16x16x128_f8f6f4 v[176:179], v[120:127], a[224:231], v[176:179]\n\t" \
  "ds_read_b64_tr_b8 v[104:105], v77 offset:0\n\t" \
  "ds_read_b64_tr_b8 v[106:107], v77 offset:1024\n\t" \
  "v_mfma_f32_16x16x128_f8f6f4 v[180:183], v[120:127], a[232:239], v[180:183]\n\t" \
  "ds_read_b64_tr_b8 v[108:109], v77 offset:8192\n\t" \
  "ds_read_b64_tr_b8 v[110:111], v77 offset:9216\n\t" \
  "buffer_load_dwordx4 a[120:123], v84, s[44:47], s90 offen offset:2048 sc0 sc1\n\t" \
  "v_mfma_f32_16x16x128_f8f6f4 v[184:187], v[120:127], a[240:247], v[184:187]\n\t" \
  "buffer_load_dwordx4 a[124:127], v84, s[44:47], s90 offen offset:3072 sc0 sc1\n\t" \
  "v_mfma_f32_16x16x128_f8f6f4 v[188:191], v[120:127], a[248:255], v[188:191]\n\t" \
  "s_waitcnt lgkmcnt(4)\n\t" \
  "v_mfma_f32_16x16x128_f8f6f4 v[192:195], v[96:103], a[224:231], v[192:195]\n\t" \
  "ds_read_b64_tr_b8 v[112:113], v78 offset:0\n\t" \
  "ds_read_b64_tr_b8 v[114:115], v78 offset:1024\n\t" \
  "v_mfma_f32_16x16x128_f8f6f4 v[196:199], v[96:103], a[232:239], v[196:199]\n\t" \
  "ds_read_b64_tr_b8 v[116:117], v78 offset:8192\n\t" \
  "ds_read_b64_tr_b8 v[118:119], v78 offset:9216\n\t" \
  "s_add_u32 s88, s57, 0\n\t" \
  "s_add_u32 m0, s60, 98304\n\t" \
  "s_nop 0\n\t" \
  "buffer_load_dwordx4 v80, s[36:39], s88 offen  lds\n\t" \
  "v_mfma_f32_16x16x128_f8f6f4 v[200:203], v[96:103], a[240:247], v[200:203]\n\t" \
  "s_add_u32 m0, s60, 99328\n\t" \
  "s_nop 0\n\t" \
  "buffer_load_dwordx4 v81, s[36:39], s88 offen  lds\n\t" \
  "v_mfma_f32_16x16x128_f8f6f4 v[204:207], v[96:103], a[248:255], v[204:207]\n\t" \
  "s_waitcnt lgkmcnt(4)\n\t" \
  "v_mfma_f32_16x16x128_f8f6f4 v[208:211], v[104:111], a[224:231], v[208:211]\n\t" \
  "ds_read_b64_tr_b8 v[120:121], v79 offset:0\n\t" \
  "ds_read_b64_tr_b8 v[122:123], v79 offset:1024\n\t" \
  "v_mfma_f32_16x16x128_f8f6f4 v[212:215], v[104:111], a[232:239], v[212:215]\n\t" \
  "ds_read_b64_tr_b8 v[124:125], v79 offset:8192\n\t" \
  "ds_read_b64_tr_b8 v[126:127], v79 offset:9216\n\t" \
  "s_add_u32 m0, s60, 100352\n\t" \
  "s_nop 0\n\t" \
  "buffer_load_dwordx4 v82, s[36:39], s88 offen  lds\n\t" \
  "v_mfma_f32_16x16x128_f8f6f4 v[216:219], v[104:111], a[240:247], v[216:219]\n\t" \
  "s_add_u32 m0, s60, 101376\n\t" \
  "s_nop 0\n\t" \
  "buffer_load_dwordx4 v83, s[36:39], s88 offen  lds\n\t" \
  "v_mfma_f32_16x16x128_f8f6f4 v[220:223], v[104:111], a[248:255], v[220:223]\n\t" \
  "s_waitcnt vmcnt(45)\n\t" \
  "s_barrier\n\t" \
  "s_waitcnt lgkmcnt(4)\n\t" \
  "v_mfma_f32_16x16x128_f8f6f4 v[224:227], v[112:119], a[224:231], v[224:227]\n\t" \
  "ds_read_b64_tr_b8 v[96:97], v72 offset:16384\n\t" \
  "ds_read_b64_tr_b8 v[98:99], v72 offset:17408\n\t" \
  "v_mfma_f32_16x16x128_f8f6f4 v[228:231], v[112:119], a[232:239], v[228:231]\n\t" \
  "ds_read_b64_tr_b8 v[100:101], v72 offset:24576\n\t" \
  "ds_read_b64_tr_b8 v[102:103], v72 offset:25600\n\t" \
  "s_add_u32 s88, s58, 0\n\t" \
  "s_add_u32 m0, s60, 114688\n\t" \
  "s_nop 0\n\t" \
  "buffer_load_dwordx4 v80, s[36:39], s88 offen  lds\n\t" \
  "v_mfma_f32_16x16x128_f8f6f4 v[232:235], v[112:119], a[240:247], v[232:235]\n\t" \
  "s_add_u32 m0, s60, 115712\n\t" \
  "s_nop 0\n\t" \
  "buffer_load_dwordx4 v81, s[36:39], s88 offen  lds\n\t" \
  "v_mfma_f32_16x16x128_f8f6f4 v[236:239], v[112:119], a[248:255], v[236:239]\n\t" \
  "s_waitcnt lgkmcnt(4)\n\t" \
  "v_mfma_f32_16x16x128_f8f6f4 v[240:243], v[120:127], a[224:231], v[240:243]\n\t" \
  "ds_read_b64_tr_b8 v[104:105], v73 offset:16384\n\t" \
  "ds_read_b64_tr_b8 v[106:107], v73 offset:17408\n\t" \
  "v_mfma_f32_16x16x128_f8f6f4 v[244:247], v[120:127], a[232:239], v[244:247]\n\t" \
  "ds_read_b64_tr_b8 v[108:109], v73 offset:24576\n\t" \
  "ds_read_b64_tr_b8 v[110:111], v73 offset:25600\n\t" \
  "s_add_u32 m0, s60, 116736\n\t" \
  "s_nop 0\n\t" \
  "buffer_load_dwordx4 v82, s[36:39], s88 offen  lds\n\t" \
  "v_mfma_f32_16x16x128_f8f6f4 v[248:251], v[120:127], a[240:247], v[248:251]\n\t" \
  "s_add_u32 m0, s60, 117760\n\t" \
  "s_nop 0\n\t" \
  "buffer_load_dwordx4 v83, s[36:39], s88 offen  lds\n\t" \
  "v_mfma_f32_16x16x128_f8f6f4 v[252:255], v[120:127], a[248:255], v[252:255]\n\t" \
  "s_waitcnt vmcnt(33)\n\t" \
  "s_bitcmp1_b32 s65, 12\n\t" \
  "s_cbranch_scc1 OK27_%=\n\t" \
  "s_mov_b32 s67, 12\n\t" \
  "s_call_b64 s[68:69], POLL_%=\n\t" \
  "OK27_%=:\n\t" \
  "s_waitcnt lgkmcnt(4)\n\t" \
  "v_mfma_f32_16x16x128_f8f6f4 v[128:131], v[96:103], a[0:7], v[128:131]\n\t" \
  "ds_read_b64_tr_b8 v[112:113], v74 offset:16384\n\t" \
  "ds_read_b64_tr_b8 v[114:115], v74 offset:17408\n\t" \
  "s_bfe_u32 s96, s65, 262157\n\t" \
  "s_cmp_lg_u32 s96, 15\n\t" \
  "s_cbranch_scc1 XWS28_%=\n\t" \
  "s_add_u32 s96, s62, 851968\n\t" \
  "buffer_load_dword v8, v13, s[44:47], s96 offen sc0 sc1\n\t" \
  "XWS28_%=:\n\t" \
  "v_mfma_f32_16x16x128_f8f6f4 v[132:135], v[96:103], a[8:15], v[132:135]\n\t" \
  "ds_read_b64_tr_b8 v[116:117], v74 offset:24576\n\t" \
  "ds_read_b64_tr_b8 v[118:119], v74 offset:25600\n\t" \
  "s_add_u32 s88, s52, 2097152\n\t" \
  "s_add_u32 m0, s60, 131072\n\t" \
  "s_nop 0\n\t" \
  "buffer_load_dwordx4 v80, s[36:39], s88 offen  lds\n\t" \
  "v_mfma_f32_16x16x128_f8f6f4 v[136:139], v[96:103], a[16:23], v[136:139]\n\t" \
  "s_add_u32 m0, s60, 132096\n\t" \
  "s_nop 0\n\t" \
  "buffer_load_dwordx4 v81, s[36:39], s88 offen  lds\n\t" \
  "v_mfma_f32_16x16x128_f8f6f4 v[140:143], v[96:103], a[24:31], v[140:143]\n\t" \
  "s_waitcnt lgkmcnt(4)\n\t" \
  "v_mfma_f32_16x16x128_f8f6f4 v[144:147], v[104:111], a[0:7], v[144:147]\n\t" \
  "ds_read_b64_tr_b8 v[120:121], v75 offset:16384\n\t" \
  "ds_read_b64_tr_b8 v[122:123], v75 offset:17408\n\t" \
  "v_mfma_f32_16x16x128_f8f6f4 v[148:151], v[104:111], a[8:15], v[148:151]\n\t" \
  "ds_read_b64_tr_b8 v[124:125], v75 offset:24576\n\t" \
  "ds_read_b64_tr_b8 v[126:127], v75 offset:25600\n\t" \
  "s_add_u32 m0, s60, 133120\n\t" \
  "s_nop 0\n\t" \
  "buffer_load_dwordx4 v82, s[36:39], s88 offen  lds\n\t" \
  "v_mfma_f32_16x16x128_f8f6f4 v[152:155], v[104:111], a[16:23], v[152:155]\n\t" \
  "s_add_u32 m0, s60, 134144\n\t" \
  "s_nop 0\n\t" \
  "buffer_load_dwordx4 v83, s[36:39], s88 offen  lds\n\t" \
  "v_mfma_f32_16x16x128_f8f6f4 v[156:159], v[104:111], a[24:31], v[156:159]\n\t" \
  "s_waitcnt lgkmcnt(4)\n\t" \
  "v_mfma_f32_16x16x128_f8f6f4 v[160:163], v[112:119], a[0:7], v[160:163]\n\t" \
  "ds_read_b64_tr_b8 v[96:97], v76 offset:16384\n\t" \
  "ds_read_b64_tr_b8 v[98:99], v76 offset:17408\n\t" \
  "v_mfma_f32_16x16x128_f8f6f4 v[164:167], v[112:119], a[8:15], v[164:167]\n\t" \
  "ds_read_b64_tr_b8 v[100:101], v76 offset:24576\n\t" \
  "ds_read_b64_tr_b8 v[102:103], v76 offset:25600\n\t" \
  "s_add_u32 s88, s53, 2097152\n\t" \
  "s_add_u32 m0, s60, 49152\n\t" \
  "s_nop 0\n\t" \
  "buffer_load_dwordx4 v80, s[36:39], s88 offen  lds\n\t" \
  "v_mfma_f32_16x16x128_f8f6f4 v[168:171], v[112:119], a[16:23], v[168:171]\n\t" \
  "s_add_u32 m0, s60, 50176\n\t" \
  "s_nop 0\n\t" \
  "buffer_load_dwordx4 v81, s[36:39], s88 offen  lds\n\t" \
  "v_mfma_f32_16x16x128_f8f6f4 v[172:175], v[112:119], a[24:31], v[172:175]\n\t" \
  "s_waitcnt lgkmcnt(4)\n\t" \
  "v_mfma_f32_16x16x128_f8f6f4 v[176:179], v[120:127], a[0:7], v[176:179]\n\t" \
  "ds_read_b64_tr_b8 v[104:105], v77 offset:16384\n\t" \
  "ds_read_b64_tr_b8 v[106:107], v77 offset:17408\n\t" \
  "v_mfma_f32_16x16x128_f8f6f4 v[180:183], v[120:127], a[8:15], v[180:183]\n\t" \
  "ds_read_b64_tr_b8 v[108:109], v77 offset:24576\n\t" \
  "ds_read_b64_tr_b8 v[110:111], v77 offset:25600\n\t" \
  "s_add_u32 m0, s60, 51200\n\t" \
  "s_nop 0\n\t" \
  "buffer_load_dwordx4 v82, s[36:39], s88 offen  lds\n\t" \
  "v_mfma_f32_16x16x128_f8f6f4 v[184:187], v[120:127], a[16:23], v[184:187]\n\t" \
  "s_add_u32 m0, s60, 52224\n\t" \
  "s_nop 0\n\t" \
  "buffer_load_dwordx4 v83, s[36:39], s88 offen  lds\n\t" \
  "v_mfma_f32_16x16x128_f8f6f4 v[188:191], v[120:127], a[24:31], v[188:191]\n\t" \
  "s_waitcnt lgkmcnt(4)\n\t" \
  "v_mfma_f32_16x16x128_f8f6f4 v[192:195], v[96:103], a[0:7], v[192:195]\n\t" \
  "ds_read_b64_tr_b8 v[112:113], v78 offset:16384\n\t" \
  "ds_read_b64_tr_b8 v[114:115], v78 offset:17408\n\t" \
  "v_mfma_f32_16x16x128_f8f6f4 v[196:199], v[96:103], a[8:15], v[196:199]\n\t" \
  "ds_read_b64_tr_b8 v[116:117], v78 offset:24576\n\t" \
  "ds_read_b64_tr_b8 v[118:119], v78 offset:25600\n\t" \
  "s_add_u32 s89, s62, 786432\n\t" \
  "s_add_u32 s90, s62, 790528\n\t" \
  "buffer_load_dwordx4 a[128:131], v84, s[44:47], s89 offen offset:0 sc0 sc1\n\t" \
  "v_mfma_f32_16x16x128_f8f6f4 v[200:203], v[96:103], a[16:23], v[200:203]\n\t" \
  "buffer_load_dwordx4 a[132:135], v84, s[44:47], s89 offen offset:1024 sc0 sc1\n\t" \
  "v_mfma_f32_16x16x128_f8f6f4 v[204:207], v[96:103], a[24:31], v[204:207]\n\t" \
  "s_waitcnt lgkmcnt(4)\n\t" \
  "v_mfma_f32_16x16x128_f8f6f4 v[208:211], v[104:111], a[0:7], v[208:211]\n\t" \
  "ds_read_b64_tr_b8 v[120:121], v79 offset:16384\n\t" \
  "ds_read_b64_tr_b8 v[122:123], v79 offset:17408\n\t" \
  "v_mfma_f32_16x16x128_f8f6f4 v[212:215], v[104:111], a[8:15], v[212:215]\n\t" \
  "ds_read_b64_tr_b8 v[124:125], v79 offset:24576\n\t" \
  "ds_read_b64_tr_b8 v[126:127], v79 offset:25600\n\t" \
  "buffer_load_dwordx4 a[136:139], v84, s[44:47], s89 offen offset:2048 sc0 sc1\n\t" \
  "v_mfma_f32_16x16x128_f8f6f4 v[216:219], v[104:111], a[16:23], v[216:219]\n\t" \
  "v_mfma_f32_16x16x128_f8f6f4 v[220:223], v[104:111], a[24:31], v[220:223]\n\t" \
  "s_waitcnt vmcnt(15)\n\t" \
  "s_barrier\n\t" \
  "s_waitcnt lgkmcnt(4)\n\t" \
  "v_mfma_f32_16x16x128_f8f6f4 v[224:227], v[112:119], a[0:7], v[224:227]\n\t" \
  "ds_read_b64_tr_b8 v[96:97], v72 offset:32768\n\t" \
  "ds_read_b64_tr_b8 v[98:99], v72 offset:33792\n\t" \
  "v_mfma_f32_16x16x128_f8f6f4 v[228:231], v[112:119], a[8:15], v[228:231]\n\t" \
  "ds_read_b64_tr_b8 v[100:101], v72 offset:40960\n\t" \
  "ds_read_b64_tr_b8 v[102:103], v72 offset:41984\n\t" \
  "v_mfma_f32_16x16x128_f8f6f4 v[232:235], v[112:119], a[16:23], v[232:235]\n\t" \
  "v_mfma_f32_16x16x128_f8f6f4 v[236:239], v[112:119], a[24:31], v[236:239]\n\t" \
  "s_cmp_lg_u32 s63, 2\n\t" \
  "s_cbranch_scc1 PS29_%=\n\t" \
  "s_waitcnt vmcnt(54)\n\t" \
  "v_mov_b32 v89, s64\n\t" \
  "s_mov_b64 exec, 1\n\t" \
  "global_store_dword v88, v89, s[76:77] sc0 sc1\n\t" \
  "s_mov_b64 exec, -1\n\t" \
  "PS29_%=:\n\t" \
  "s_cmp_lg_u32 s63, 3\n\t" \
  "s_cbranch_scc1 PS30_%=\n\t" \
  "global_store_dwordx4 v84, v[32:35], s[72:73] offset:0 sc0 sc1\n\t" \
  "global_store_dwordx4 v84, v[36:39], s[72:73] offset:1024 sc0 sc1\n\t" \
  "global_store_dwordx4 v84, v[40:43], s[72:73] offset:2048 sc0 sc1\n\t" \
  "global_store_dwordx4 v84, v[44:47], s[72:73] offset:3072 sc0 sc1\n\t" \
  "global_store_dwordx4 v84, v[48:51], s[74:75] offset:0 sc0 sc1\n\t" \
  "global_store_dwordx4 v84, v[52:55], s[74:75] offset:1024 sc0 sc1\n\t" \
  "global_store_dwordx4 v84, v[56:59], s[74:75] offset:2048 sc0 sc1\n\t" \
  "global_store_dwordx4 v84, v[60:63], s[74:75] offset:3072 sc0 sc1\n\t" \
  "s_nop 1\n\t" \
  "PS30_%=:\n\t" \
  "s_waitcnt lgkmcnt(4)\n\t" \
  "v_mfma_f32_16x16x128_f8f6f4 v[240:243], v[120:127], a[0:7], v[240:243]\n\t" \
  "ds_read_b64_tr_b8 v[104:105], v73 offset:32768\n\t" \
  "ds_read_b64_tr_b8 v[106:107], v73 offset:33792\n\t" \
  "v_mfma_f32_16x16x128_f8f6f4 v[244:247], v[120:127], a[8:15], v[244:247]\n\t" \
  "ds_read_b64_tr_b8 v[108:109], v73 offset:40960\n\t" \
  "ds_read_b64_tr_b8 v[110:111], v73 offset:41984\n\t" \
  "v_mfma_f32_16x16x128_f8f6f4 v[248:251], v[120:127], a[16:23], v[248:251]\n\t" \
  "v_mfma_f32_16x16x128_f8f6f4 v[252:255], v[120:127], a[24:31], v[252:255]\n\t" \
  "s_waitcnt vmcnt(36)\n\t" \
  "s_bitcmp1_b32 s65, 13\n\t" \
  "s_cbranch_scc1 OK31_%=\n\t" \
  "s_mov_b32 s67, 13\n\t" \
  "s_call_b64 s[68:69], POLL_%=\n\t" \
  "OK31_%=:\n\t" \
  "s_waitcnt lgkmcnt(4)\n\t" \
  "v_mfma_f32_16x16x128_f8f6f4 v[128:131], v[96:103], a[32:39], v[128:131]\n\t" \
  "ds_read_b64_tr_b8 v[112:113], v74 offset:32768\n\t" \
  "ds_read_b64_tr_b8 v[114:115], v74 offset:33792\n\t" \
  "v_mfma_f32_16x16x128_f8f6f4 v[132:135], v[96:103], a[40:47], v[132:135]\n\t" \
  "ds_read_b64_tr_b8 v[116:117], v74 offset:40960\n\t" \
  "ds_read_b64_tr_b8 v[118:119], v74 offset:41984\n\t" \
  "v_mfma_f32_16x16x128_f8f6f4 v[136:139], v[96:103], a[48:55], v[136:139]\n\t" \
  "v_mfma_f32_16x16x128_f8f6f4 v[140:143], v[96:103], a[56:63], v[140:143]\n\t" \
  "s_waitcnt lgkmcnt(4)\n\t" \
  "v_mfma_f32_16x16x128_f8f6f4 v[144:147], v[104:111], a[32:39], v[144:147]\n\t" \
  "ds_read_b64_tr_b8 v[120:121], v75 offset:32768\n\t" \
  "ds_read_b64_tr_b8 v[122:123], v75 offset:33792\n\t" \
  "v_mfma_f32_16x16x128_f8f6f4 v[148:151], v[104:111], a[40:47], v[148:151]\n\t" \
  "ds_read_b64_tr_b8 v[124:125], v75 offset:40960\n\t" \
  "ds_read_b64_tr_b8 v[126:127], v75 offset:41984\n\t" \
  "v_mfma_f32_16x16x128_f8f6f4 v[152:155], v[104:111], a[48:55], v[152:155]\n\t" \
  "v_mfma_f32_16x16x128_f8f6f4 v[156:159], v[104:111], a[56:63], v[156:159]\n\t" \
  "s_waitcnt lgkmcnt(4)\n\t" \
  "v_mfma_f32_16x16x128_f8f6f4 v[160:163], v[112:119], a[32:39], v[160:163]\n\t" \
  "ds_read_b64_tr_b8 v[96:97], v76 offset:32768\n\t" \
  "ds_read_b64_tr_b8 v[98:99], v76 offset:33792\n\t" \
  "v_mfma_f32_16x16x128_f8f6f4 v[164:167], v[112:119], a[40:47], v[164:167]\n\t" \
  "ds_read_b64_tr_b8 v[100:101], v76 offset:40960\n\t" \
  "ds_read_b64_tr_b8 v[102:103], v76 offset:41984\n\t" \
  "v_mfma_f32_16x16x128_f8f6f4 v[168:171], v[112:119], a[48:55], v[168:171]\n\t" \
  "v_mfma_f32_16x16x128_f8f6f4 v[172:175], v[112:119], a[56:63], v[172:175]\n\t" \
  "s_waitcnt lgkmcnt(4)\n\t" \
  "v_mfma_f32_16x16x128_f8f6f4 v[176:179], v[120:127], a[32:39], v[176:179]\n\t" \
  "ds_read_b64_tr_b8 v[104:105], v77 offset:32768\n\t" \
  "ds_read_b64_tr_b8 v[106:107], v77 offset:33792\n\t" \
  "v_mfma_f32_16x16x128_f8f6f4 v[180:183], v[120:127], a[40:47], v[180:183]\n\t" \
  "ds_read_b64_tr_b8 v[108:109], v77 offset:40960\n\t" \
  "ds_read_b64_tr_b8 v[110:111], v77 offset:41984\n\t" \
  "v_mfma_f32_16x16x128_f8f6f4 v[184:187], v[120:127], a[48:55], v[184:187]\n\t" \
  "v_mfma_f32_16x16x128_f8f6f4 v[188:191], v[120:127], a[56:63], v[188:191]\n\t" \
  "s_waitcnt lgkmcnt(4)\n\t" \
  "v_mfma_f32_16x16x128_f8f6f4 v[192:195], v[96:103], a[32:39], v[192:195]\n\t" \
  "ds_read_b64_tr_b8 v[112:113], v78 offset:32768\n\t" \
  "ds_read_b64_tr_b8 v[114:115], v78 offset:33792\n\t" \
  "v_mfma_f32_16x16x128_f8f6f4 v[196:199], v[96:103], a[40:47], v[196:199]\n\t" \
  "ds_read_b64_tr_b8 v[116:117], v78 offset:40960\n\t" \
  "ds_read_b64_tr_b8 v[118:119], v78 offset:41984\n\t" \
  "v_mfma_f32_16x16x128_f8f6f4 v[200:203], v[96:103], a[48:55], v[200:203]\n\t" \
  "v_mfma_f32_16x16x128_f8f6f4 v[204:207], v[96:103], a[56:63], v[204:207]\n\t" \
  "s_waitcnt lgkmcnt(4)\n\t" \
  "v_mfma_f32_16x16x128_f8f6f4 v[208:211], v[104:111], a[32:39], v[208:211]\n\t" \
  "ds_read_b64_tr_b8 v[120:121], v79 offset:32768\n\t" \
  "ds_read_b64_tr_b8 v[122:123], v79 offset:33792\n\t" \
  "v_mfma_f32_16x16x128_f8f6f4 v[212:215], v[104:111], a[40:47], v[212:215]\n\t" \
  "ds_read_b64_tr_b8 v[124:125], v79 offset:40960\n\t" \
  "ds_read_b64_tr_b8 v[126:127], v79 offset:41984\n\t" \
  "v_mfma_f32_16x16x128_f8f6f4 v[216:219], v[104:111], a[48:55], v[216:219]\n\t" \
  "v_mfma_f32_16x16x128_f8f6f4 v[220:223], v[104:111], a[56:63], v[220:223]\n\t" \
  "s_waitcnt vmcnt(11)\n\t" \
  "s_barrier\n\t" \
  "s_waitcnt lgkmcnt(4)\n\t" \
  "v_mfma_f32_16x16x128_f8f6f4 v[224:227], v[112:119], a[32:39], v[224:227]\n\t" \
  "ds_read_b64_tr_b8 v[96:97], v72 offset:49152\n\t" \
  "ds_read_b64_tr_b8 v[98:99], v72 offset:50176\n\t" \
  "v_mfma_f32_16x16x128_f8f6f4 v[228:231], v[112:119], a[40:47], v[228:231]\n\t" \
  "ds_read_b64_tr_b8 v[100:101], v72 offset:57344\n\t" \
  "ds_read_b64_tr_b8 v[102:103], v72 offset:58368\n\t" \
  "buffer_load_dwordx4 a[140:143], v84, s[44:47], s89 offen offset:3072 sc0 sc1\n\t" \
  "v_mfma_f32_16x16x128_f8f6f4 v[232:235], v[112:119], a[48:55], v[232:235]\n\t" \
  "buffer_load_dwordx4 a[144:147], v84, s[44:47], s90 offen offset:0 sc0 sc1\n\t" \
  "v_mfma_f32_16x16x128_f8f6f4 v[236:239], v[112:119], a[56:63], v[236:239]\n\t" \
  "s_waitcnt lgkmcnt(4)\n\t" \
  "v_mfma_f32_16x16x128_f8f6f4 v[240:243], v[120:127], a[32:39], v[240:243]\n\t" \
  "ds_read_b64_tr_b8 v[104:105], v73 offset:49152\n\t" \
  "ds_read_b64_tr_b8 v[106:107], v73 offset:50176\n\t" \
  "v_mfma_f32_16x16x128_f8f6f4 v[244:247], v[120:127], a[40:47], v[244:247]\n\t" \
  "ds_read_b64_tr_b8 v[108:109], v73 offset:57344\n\t" \
  "ds_read_b64_tr_b8 v[110:111], v73 offset:58368\n\t" \
  "buffer_load_dwordx4 a[148:151], v84, s[44:47], s90 offen offset:1024 sc0 sc1\n\t" \
  "v_mfma_f32_16x16x128_f8f6f4 v[248:251], v[120:127], a[48:55], v[248:251]\n\t" \
  "buffer_load_dwordx4 a[152:155], v84, s[44:47], s90 offen offset:2048 sc0 sc1\n\t" \
  "v_mfma_f32_16x16x128_f8f6f4 v[252:255], v[120:127], a[56:63], v[252:255]\n\t" \
  "s_waitcnt vmcnt(31)\n\t" \
  "s_bitcmp1_b32 s65, 14\n\t" \
  "s_cbranch_scc1 OK32_%=\n\t" \
  "s_mov_b32 s67, 14\n\t" \
  "s_call_b64 s[68:69], POLL_%=\n\t" \
  "OK32_%=:\n\t" \
  "s_waitcnt lgkmcnt(4)\n\t" \
  "v_mfma_f32_16x16x128_f8f6f4 v[128:131], v[96:103], a[64:71], v[128:131]\n\t" \
  "ds_read_b64_tr_b8 v[112:113], v74 offset:49152\n\t" \
  "ds_read_b64_tr_b8 v[114:115], v74 offset:50176\n\t" \
  "v_mfma_f32_16x16x128_f8f6f4 v[132:135], v[96:103], a[72:79], v[132:135]\n\t" \
  "ds_read_b64_tr_b8 v[116:117], v74 offset:57344\n\t" \
  "ds_read_b64_tr_b8 v[118:119], v74 offset:58368\n\t" \
  "buffer_load_dwordx4 a[156:159], v84, s[44:47], s90 offen offset:3072 sc0 sc1\n\t" \
  "v_mfma_f32_16x16x128_f8f6f4 v[136:139], v[96:103], a[80:87], v[136:139]\n\t" \
  "s_add_u32 s89, s62, 851968\n\t" \
  "s_add_u32 s90, s62, 856064\n\t" \
  "buffer_load_dwordx4 a[160:163], v84, s[44:47], s89 offen offset:0 sc0 sc1\n\t" \
  "v_mfma_f32_16x16x128_f8f6f4 v[140:143], v[96:103], a[88:95], v[140:143]\n\t" \
  "s_waitcnt lgkmcnt(4)\n\t" \
  "v_mfma_f32_16x16x128_f8f6f4 v[144:147], v[104:111], a[64:71], v[144:147]\n\t" \
  "ds_read_b64_tr_b8 v[120:121], v75 offset:49152\n\t" \
  "ds_read_b64_tr_b8 v[122:123], v75 offset:50176\n\t" \
  "v_mfma_f32_16x16x128_f8f6f4 v[148:151], v[104:111], a[72:79], v[148:151]\n\t" \
  "ds_read_b64_tr_b8 v[124:125], v75 offset:57344\n\t" \
  "ds_read_b64_tr_b8 v[126:127], v75 offset:58368\n\t" \
  "buffer_load_dwordx4 a[164:167], v84, s[44:47], s89 offen offset:1024 sc0 sc1\n\t" \
  "v_mfma_f32_16x16x128_f8f6f4 v[152:155], v[104:111], a[80:87], v[152:155]\n\t" \
  "buffer_load_dwordx4 a[168:171], v84, s[44:47], s89 offen offset:2048 sc0 sc1\n\t" \
  "v_mfma_f32_16x16x128_f8f6f4 v[156:159], v[104:111], a[88:95], v[156:159]\n\t" \
  "s_waitcnt lgkmcnt(4)\n\t" \
  "v_mfma_f32_16x16x128_f8f6f4 v[160:163], v[112:119], a[64:71], v[160:163]\n\t" \
  "ds_read_b64_tr_b8 v[96:97], v76 offset:49152\n\t" \
  "ds_read_b64_tr_b8 v[98:99], v76 offset:50176\n\t" \
  "v_mfma_f32_16x16x128_f8f6f4 v[164:167], v[112:119], a[72:79], v[164:167]\n\t" \
  "ds_read_b64_tr_b8 v[100:101], v76 offset:57344\n\t" \
  "ds_read_b64_tr_b8 v[102:103], v76 offset:58368\n\t" \
  "buffer_load_dwordx4 a[172:175], v84, s[44:47], s89 offen offset:3072 sc0 sc1\n\t" \
  "v_mfma_f32_16x16x128_f8f6f4 v[168:171], v[112:119], a[80:87], v[168:171]\n\t" \
  "buffer_load_dwordx4 a[176:179], v84, s[44:47], s90 offen offset:0 sc0 sc1\n\t" \
  "v_mfma_f32_16x16x128_f8f6f4 v[172:175], v[112:119], a[88:95], v[172:175]\n\t" \
  "s_waitcnt lgkmcnt(4)\n\t" \
  "v_mfma_f32_16x16x128_f8f6f4 v[176:179], v[120:127], a[64:71], v[176:179]\n\t" \
  "ds_read_b64_tr_b8 v[104:105], v77 offset:49152\n\t" \
  "ds_read_b64_tr_b8 v[106:107], v77 offset:50176\n\t" \
  "v_mfma_f32_16x16x128_f8f6f4 v[180:183], v[120:127], a[72:79], v[180:183]\n\t" \
  "ds_read_b64_tr_b8 v[108:109], v77 offset:57344\n\t" \
  "ds_read_b64_tr_b8 v[110:111], v77 offset:58368\n\t" \
  "buffer_load_dwordx4 a[180:183], v84, s[44:47], s90 offen offset:1024 sc0 sc1\n\t" \
  "v_mfma_f32_16x16x128_f8f6f4 v[184:187], v[120:127], a[80:87], v[184:187]\n\t" \
  "buffer_load_dwordx4 a[184:187], v84, s[44:47], s90 offen offset:2048 sc0 sc1\n\t" \
  "v_mfma_f32_16x16x128_f8f6f4 v[188:191], v[120:127], a[88:95], v[188:191]\n\t" \
  "s_waitcnt lgkmcnt(4)\n\t" \
  "v_mfma_f32_16x16x128_f8f6f4 v[192:195], v[96:103], a[64:71], v[192:195]\n\t" \
  "ds_read_b64_tr_b8 v[112:113], v78 offset:49152\n\t" \
  "ds_read_b64_tr_b8 v[114:115], v78 offset:50176\n\t" \
  "v_mfma_f32_16x16x128_f8f6f4 v[196:199], v[96:103], a[72:79], v[196:199]\n\t" \
  "ds_read_b64_tr_b8 v[116:117], v78 offset:57344\n\t" \
  "ds_read_b64_tr_b8 v[118:119], v78 offset:58368\n\t" \
  "buffer_load_dwordx4 a[188:191], v84, s[44:47], s90 offen offset:3072 sc0 sc1\n\t" \
  "v_mfma_f32_16x16x128_f8f6f4 v[200:203], v[96:103], a[80:87], v[200:203]\n\t" \
  "s_add_u32 s89, s62, 917504\n\t" \
  "s_add_u32 s90, s62, 921600\n\t" \
  "buffer_load_dwordx4 a[192:195], v84, s[44:47], s89 offen offset:0 sc0 sc1\n\t" \
  "v_mfma_f32_16x16x128_f8f6f4 v[204:207], v[96:103], a[88:95], v[204:207]\n\t" \
  "s_waitcnt lgkmcnt(4)\n\t" \
  "v_mfma_f32_16x16x128_f8f6f4 v[208:211], v[104:111], a[64:71], v[208:211]\n\t" \
  "ds_read_b64_tr_b8 v[120:121], v79 offset:49152\n\t" \
  "ds_read_b64_tr_b8 v[122:123], v79 offset:50176\n\t" \
  "v_mfma_f32_16x16x128_f8f6f4 v[212:215], v[104:111], a[72:79], v[212:215]\n\t" \
  "ds_read_b64_tr_b8 v[124:125], v79 offset:57344\n\t" \
  "ds_read_b64_tr_b8 v[126:127], v79 offset:58368\n\t" \
  "buffer_load_dwordx4 a[196:199], v84, s[44:47], s89 offen offset:1024 sc0 sc1\n\t" \
  "v_mfma_f32_16x16x128_f8f6f4 v[216:219], v[104:111], a[80:87], v[216:219]\n\t" \
  "buffer_load_dwordx4 a[200:203], v84, s[44:47], s89 offen offset:2048 sc0 sc1\n\t" \
  "v_mfma_f32_16x16x128_f8f6f4 v[220:223], v[104:111], a[88:95], v[220:223]\n\t" \
  "s_waitcnt vmcnt(23)\n\t" \
  "s_barrier\n\t" \
  "s_waitcnt lgkmcnt(4)\n\t" \
  "v_mfma_f32_16x16x128_f8f6f4 v[224:227], v[112:119], a[64:71], v[224:227]\n\t" \
  "ds_read_b64_tr_b8 v[96:97], v16 offset:0\n\t" \
  "ds_read_b64_tr_b8 v[98:99], v16 offset:1024\n\t" \
  "v_mfma_f32_16x16x128_f8f6f4 v[228:231], v[112:119], a[72:79], v[228:231]\n\t" \
  "ds_read_b64_tr_b8 v[100:101], v16 offset:8192\n\t" \
  "ds_read_b64_tr_b8 v[102:103], v16 offset:9216\n\t" \
  "buffer_load_dwordx4 a[204:207], v84, s[44:47], s89 offen offset:3072 sc0 sc1\n\t" \
  "v_mfma_f32_16x16x128_f8f6f4 v[232:235], v[112:119], a[80:87], v[232:235]\n\t" \
  "buffer_load_dwordx4 a[208:211], v84, s[44:47], s90 offen offset:0 sc0 sc1\n\t" \
  "v_mfma_f32_16x16x128_f8f6f4 v[236:239], v[112:119], a[88:95], v[236:239]\n\t" \
  "s_waitcnt lgkmcnt(4)\n\t" \
  "v_mfma_f32_16x16x128_f8f6f4 v[240:243], v[120:127], a[64:71], v[240:243]\n\t" \
  "ds_read_b64_tr_b8 v[104:105], v17 offset:0\n\t" \
  "ds_read_b64_tr_b8 v[106:107], v17 offset:1024\n\t" \
  "v_mfma_f32_16x16x128_f8f6f4 v[244:247], v[120:127], a[72:79], v[244:247]\n\t" \
  "ds_read_b64_tr_b8 v[108:109], v17 offset:8192\n\t" \
  "ds_read_b64_tr_b8 v[110:111], v17 offset:9216\n\t" \
  "buffer_load_dwordx4 a[212:215], v84, s[44:47], s90 offen offset:1024 sc0 sc1\n\t" \
  "v_mfma_f32_16x16x128_f8f6f4 v[248:251], v[120:127], a[80:87], v[248:251]\n\t" \
  "buffer_load_dwordx4 a[216:219], v84, s[44:47], s90 offen offset:2048 sc0 sc1\n\t" \
  "v_mfma_f32_16x16x128_f8f6f4 v[252:255], v[120:127], a[88:95], v[252:255]\n\t" \
  "s_waitcnt vmcnt(39)\n\t" \
  "s_bitcmp1_b32 s65, 15\n\t" \
  "s_cbranch_scc1 OK33_%=\n\t" \
  "s_mov_b32 s67, 15\n\t" \
  "s_call_b64 s[68:69], POLL_%=\n\t" \
  "OK33_%=:\n\t" \
  "s_waitcnt lgkmcnt(4)\n\t" \
  "v_mfma_f32_16x16x128_f8f6f4 v[128:131], v[96:103], a[96:103], v[128:131]\n\t" \
  "ds_read_b64_tr_b8 v[112:113], v18 offset:0\n\t" \
  "ds_read_b64_tr_b8 v[114:115], v18 offset:1024\n\t" \
  "v_mfma_f32_16x16x128_f8f6f4 v[132:135], v[96:103], a[104:111], v[132:135]\n\t" \
  "ds_read_b64_tr_b8 v[116:117], v18 offset:8192\n\t" \
  "ds_read_b64_tr_b8 v[118:119], v18 offset:9216\n\t" \
  "buffer_load_dwordx4 a[220:223], v84, s[44:47], s90 offen offset:3072 sc0 sc1\n\t" \
  "v_mfma_f32_16x16x128_f8f6f4 v[136:139], v[96:103], a[112:119], v[136:139]\n\t" \
  "s_add_u32 s89, s62, 983040\n\t" \
  "s_add_u32 s90, s62, 987136\n\t" \
  "buffer_load_dwordx4 a[224:227], v84, s[44:47], s89 offen offset:0 sc0 sc1\n\t" \
  "v_mfma_f32_16x16x128_f8f6f4 v[140:143], v[96:103], a[120:127], v[140:143]\n\t" \
  "s_waitcnt lgkmcnt(4)\n\t" \
  "v_mfma_f32_16x16x128_f8f6f4 v[144:147], v[104:111], a[96:103], v[144:147]\n\t" \
  "ds_read_b64_tr_b8 v[120:121], v19 offset:0\n\t" \
  "ds_read_b64_tr_b8 v[122:123], v19 offset:1024\n\t" \
  "v_mfma_f32_16x16x128_f8f6f4 v[148:151], v[104:111], a[104:111], v[148:151]\n\t" \
  "ds_read_b64_tr_b8 v[124:125], v19 offset:8192\n\t" \
  "ds_read_b64_tr_b8 v[126:127], v19 offset:9216\n\t" \
  "buffer_load_dwordx4 a[228:231], v84, s[44:47], s89 offen offset:1024 sc0 sc1\n\t" \
  "v_mfma_f32_16x16x128_f8f6f4 v[152:155], v[104:111], a[112:119], v[152:155]\n\t" \
  "buffer_load_dwordx4 a[232:235], v84, s[44:47], s89 offen offset:2048 sc0 sc1\n\t" \
  "v_mfma_f32_16x16x128_f8f6f4 v[156:159], v[104:111], a[120:127], v[156:159]\n\t" \
  "s_waitcnt lgkmcnt(4)\n\t" \
  "v_mfma_f32_16x16x128_f8f6f4 v[160:163], v[112:119], a[96:103], v[160:163]\n\t" \
  "ds_read_b64_tr_b8 v[96:97], v20 offset:0\n\t" \
  "ds_read_b64_tr_b8 v[98:99], v20 offset:1024\n\t" \
  "v_mfma_f32_16x16x128_f8f6f4 v[164:167], v[112:119], a[104:111], v[164:167]\n\t" \
  "ds_read_b64_tr_b8 v[100:101], v20 offset:8192\n\t" \
  "ds_read_b64_tr_b8 v[102:103], v20 offset:9216\n\t" \
  "buffer_load_dwordx4 a[236:239], v84, s[44:47], s89 offen offset:3072 sc0 sc1\n\t" \
  "v_mfma_f32_16x16x128_f8f6f4 v[168:171], v[112:119], a[112:119], v[168:171]\n\t" \
  "buffer_load_dwordx4 a[240:243], v84, s[44:47], s90 offen offset:0 sc0 sc1\n\t" \
  "v_mfma_f32_16x16x128_f8f6f4 v[172:175], v[112:119], a[120:127], v[172:175]\n\t" \
  "s_waitcnt lgkmcnt(4)\n\t" \
  "v_mfma_f32_16x16x128_f8f6f4 v[176:179], v[120:127], a[96:103], v[176:179]\n\t" \
  "ds_read_b64_tr_b8 v[104:105], v21 offset:0\n\t" \
  "ds_read_b64_tr_b8 v[106:107], v21 offset:1024\n\t" \
  "v_mfma_f32_16x16x128_f8f6f4 v[180:183], v[120:127], a[104:111], v[180:183]\n\t" \
  "ds_read_b64_tr_b8 v[108:109], v21 offset:8192\n\t" \
  "ds_read_b64_tr_b8 v[110:111], v21 offset:9216\n\t" \
  "buffer_load_dwordx4 a[244:247], v84, s[44:47], s90 offen offset:1024 sc0 sc1\n\t" \
  "v_mfma_f32_16x16x128_f8f6f4 v[184:187], v[120:127], a[112:119], v[184:187]\n\t" \
  "buffer_load_dwordx4 a[248:251], v84, s[44:47], s90 offen offset:2048 sc0 sc1\n\t" \
  "v_mfma_f32_16x16x128_f8f6f4 v[188:191], v[120:127], a[120:127], v[188:191]\n\t" \
  "s_waitcnt lgkmcnt(4)\n\t" \
  "v_mfma_f32_16x16x128_f8f6f4 v[192:195], v[96:103], a[96:103], v[192:195]\n\t" \
  "ds_read_b64_tr_b8 v[112:113], v22 offset:0\n\t" \
  "ds_read_b64_tr_b8 v[114:115], v22 offset:1024\n\t" \
  "v_mfma_f32_16x16x128_f8f6f4 v[196:199], v[96:103], a[104:111], v[196:199]\n\t" \
  "ds_read_b64_tr_b8 v[116:117], v22 offset:8192\n\t" \
  "ds_read_b64_tr_b8 v[118:119], v22 offset:9216\n\t" \
  "s_add_u32 s88, s54, 2097152\n\t" \
  "s_add_u32 m0, s60, 0\n\t" \
  "s_nop 0\n\t" \
  "buffer_load_dwordx4 v80, s[36:39], s88 offen  lds\n\t" \
  "v_mfma_f32_16x16x128_f8f6f4 v[200:203], v[96:103], a[112:119], v[200:203]\n\t" \
  "s_add_u32 m0, s60, 1024\n\t" \
  "s_nop 0\n\t" \
  "buffer_load_dwordx4 v81, s[36:39], s88 offen  lds\n\t" \
  "v_mfma_f32_16x16x128_f8f6f4 v[204:207], v[96:103], a[120:127], v[204:207]\n\t" \
  "s_waitcnt lgkmcnt(4)\n\t" \
  "v_mfma_f32_16x16x128_f8f6f4 v[208:211], v[104:111], a[96:103], v[208:211]\n\t" \
  "ds_read_b64_tr_b8 v[120:121], v23 offset:0\n\t" \
  "ds_read_b64_tr_b8 v[122:123], v23 offset:1024\n\t" \
  "v_mfma_f32_16x16x128_f8f6f4 v[212:215], v[104:111], a[104:111], v[212:215]\n\t" \
  "ds_read_b64_tr_b8 v[124:125], v23 offset:8192\n\t" \
  "ds_read_b64_tr_b8 v[126:127], v23 offset:9216\n\t" \
  "s_add_u32 m0, s60, 2048\n\t" \
  "s_nop 0\n\t" \
  "buffer_load_dwordx4 v82, s[36:39], s88 offen  lds\n\t" \
  "v_mfma_f32_16x16x128_f8f6f4 v[216:219], v[104:111], a[112:119], v[216:219]\n\t" \
  "s_add_u32 m0, s60, 3072\n\t" \
  "s_nop 0\n\t" \
  "buffer_load_dwordx4 v83, s[36:39], s88 offen  lds\n\t" \
  "v_mfma_f32_16x16x128_f8f6f4 v[220:223], v[104:111], a[120:127], v[220:223]\n\t" \
  "s_waitcnt vmcnt(35)\n\t" \
  "s_barrier\n\t" \
  "s_waitcnt lgkmcnt(4)\n\t" \
  "v_mfma_f32_16x16x128_f8f6f4 v[224:227], v[112:119], a[96:103], v[224:227]\n\t" \
  "ds_read_b64_tr_b8 v[96:97], v64 offset:49152\n\t" \
  "ds_read_b64_tr_b8 v[98:99], v64 offset:50176\n\t" \
  "v_mfma_f32_16x16x128_f8f6f4 v[228:231], v[112:119], a[104:111], v[228:231]\n\t" \
  "ds_read_b64_tr_b8 v[100:101], v64 offset:57344\n\t" \
  "ds_read_b64_tr_b8 v[102:103], v64 offset:58368\n\t" \
  "s_add_u32 s88, s55, 2097152\n\t" \
  "s_add_u32 m0, s60, 16384\n\t" \
  "s_nop 0\n\t" \
  "buffer_load_dwordx4 v80, s[36:39], s88 offen  lds\n\t" \
  "v_mfma_f32_16x16x128_f8f6f4 v[232:235], v[112:119], a[112:119], v[232:235]\n\t" \
  "s_add_u32 m0, s60, 17408\n\t" \
  "s_nop 0\n\t" \
  "buffer_load_dwordx4 v81, s[36:39], s88 offen  lds\n\t" \
  "v_mfma_f32_16x16x128_f8f6f4 v[236:239], v[112:119], a[120:127], v[236:239]\n\t" \
  "s_waitcnt lgkmcnt(4)\n\t" \
  "v_mfma_f32_16x16x128_f8f6f4 v[240:243], v[120:127], a[96:103], v[240:243]\n\t" \
  "ds_read_b64_tr_b8 v[104:105], v65 offset:49152\n\t" \
  "ds_read_b64_tr_b8 v[106:107], v65 offset:50176\n\t" \
  "v_mfma_f32_16x16x128_f8f6f4 v[244:247], v[120:127], a[104:111], v[244:247]\n\t" \
  "ds_read_b64_tr_b8 v[108:109], v65 offset:57344\n\t" \
  "ds_read_b64_tr_b8 v[110:111], v65 offset:58368\n\t" \
  "s_add_u32 m0, s60, 18432\n\t" \
  "s_nop 0\n\t" \
  "buffer_load_dwordx4 v82, s[36:39], s88 offen  lds\n\t" \
  "v_mfma_f32_16x16x128_f8f6f4 v[248:251], v[120:127], a[112:119], v[248:251]\n\t" \
  "s_add_u32 m0, s60, 19456\n\t" \
  "s_nop 0\n\t" \
  "buffer_load_dwordx4 v83, s[36:39], s88 offen  lds\n\t" \
  "v_mfma_f32_16x16x128_f8f6f4 v[252:255], v[120:127], a[120:127], v[252:255]\n\t" \
  "s_waitcnt vmcnt(31)\n\t" \
  "s_bitcmp1_b32 s65, 16\n\t" \
  "s_cbranch_scc1 OK34_%=\n\t" \
  "s_mov_b32 s67, 16\n\t" \
  "s_call_b64 s[68:69], POLL_%=\n\t" \
  "OK34_%=:\n\t" \
  "s_waitcnt lgkmcnt(4)\n\t" \
  "v_mfma_f32_16x16x128_f8f6f4 v[128:131], v[96:103], a[128:135], v[128:131]\n\t" \
  "ds_read_b64_tr_b8 v[112:113], v66 offset:49152\n\t" \
  "ds_read_b64_tr_b8 v[114:115], v66 offset:50176\n\t" \
  "s_bfe_u32 s96, s65, 262161\n\t" \
  "s_cmp_lg_u32 s96, 15\n\t" \
  "s_cbranch_scc1 XWS35_%=\n\t" \
  "s_add_u32 s96, s62, 1114112\n\t" \
  "buffer_load_dword v8, v13, s[44:47], s96 offen sc0 sc1\n\t" \
  "XWS35_%=:\n\t" \
  "v_mfma_f32_16x16x128_f8f6f4 v[132:135], v[96:103], a[136:143], v[132:135]\n\t" \
  "ds_read_b64_tr_b8 v[116:117], v66 offset:57344\n\t" \
  "ds_read_b64_tr_b8 v[118:119], v66 offset:58368\n\t" \
  "s_add_u32 s88, s56, 2097152\n\t" \
  "s_add_u32 m0, s60, 32768\n\t" \
  "s_nop 0\n\t" \
  "buffer_load_dwordx4 v80, s[36:39], s88 offen  lds\n\t" \
  "v_mfma_f32_16x16x128_f8f6f4 v[136:139], v[96:103], a[144:151], v[136:139]\n\t" \
  "s_add_u32 m0, s60, 33792\n\t" \
  "s_nop 0\n\t" \
  "buffer_load_dwordx4 v81, s[36:39], s88 offen  lds\n\t" \
  "v_mfma_f32_16x16x128_f8f6f4 v[140:143], v[96:103], a[152:159], v[140:143]\n\t" \
  "s_waitcnt lgkmcnt(4)\n\t" \
  "v_mfma_f32_16x16x128_f8f6f4 v[144:147], v[104:111], a[128:135], v[144:147]\n\t" \
  "ds_read_b64_tr_b8 v[120:121], v67 offset:49152\n\t" \
  "ds_read_b64_tr_b8 v[122:123], v67 offset:50176\n\t" \
  "v_mfma_f32_16x16x128_f8f6f4 v[148:151], v[104:111], a[136:143], v[148:151]\n\t" \
  "ds_read_b64_tr_b8 v[124:125], v67 offset:57344\n\t" \
  "ds_read_b64_tr_b8 v[126:127], v67 offset:58368\n\t" \
  "s_add_u32 m0, s60, 34816\n\t" \
  "s_nop 0\n\t" \
  "buffer_load_dwordx4 v82, s[36:39], s88 offen  lds\n\t" \
  "v_mfma_f32_16x16x128_f8f6f4 v[152:155], v[104:111], a[144:151], v[152:155]\n\t" \
  "s_add_u32 m0, s60, 35840\n\t" \
  "s_nop 0\n\t" \
  "buffer_load_dwordx4 v83, s[36:39], s88 offen  lds\n\t" \
  "v_mfma_f32_16x16x128_f8f6f4 v[156:159], v[104:111], a[152:159], v[156:159]\n\t" \
  "s_waitcnt lgkmcnt(4)\n\t" \
  "v_mfma_f32_16x16x128_f8f6f4 v[160:163], v[112:119], a[128:135], v[160:163]\n\t" \
  "ds_read_b64_tr_b8 v[96:97], v68 offset:49152\n\t" \
  "ds_read_b64_tr_b8 v[98:99], v68 offset:50176\n\t" \
  "v_mfma_f32_16x16x128_f8f6f4 v[164:167], v[112:119], a[136:143], v[164:167]\n\t" \
  "ds_read_b64_tr_b8 v[100:101], v68 offset:57344\n\t" \
  "ds_read_b64_tr_b8 v[102:103], v68 offset:58368\n\t" \
  "s_add_u32 s88, s57, 2097152\n\t" \
  "s_add_u32 m0, s60, 65536\n\t" \
  "s_nop 0\n\t" \
  "buffer_load_dwordx4 v80, s[36:39], s88 offen  lds\n\t" \
  "v_mfma_f32_16x16x128_f8f6f4 v[168:171], v[112:119], a[144:151], v[168:171]\n\t" \
  "s_add_u32 m0, s60, 66560\n\t" \
  "s_nop 0\n\t" \
  "buffer_load_dwordx4 v81, s[36:39], s88 offen  lds\n\t" \
  "v_mfma_f32_16x16x128_f8f6f4 v[172:175], v[112:119], a[152:159], v[172:175]\n\t" \
  "s_waitcnt lgkmcnt(4)\n\t" \
  "v_mfma_f32_16x16x128_f8f6f4 v[176:179], v[120:127], a[128:135], v[176:179]\n\t" \
  "ds_read_b64_tr_b8 v[104:105], v69 offset:49152\n\t" \
  "ds_read_b64_tr_b8 v[106:107], v69 offset:50176\n\t" \
  "v_mfma_f32_16x16x128_f8f6f4 v[180:183], v[120:127], a[136:143], v[180:183]\n\t" \
  "ds_read_b64_tr_b8 v[108:109], v69 offset:57344\n\t" \
  "ds_read_b64_tr_b8 v[110:111], v69 offset:58368\n\t" \
  "s_add_u32 m0, s60, 67584\n\t" \
  "s_nop 0\n\t" \
  "buffer_load_dwordx4 v82, s[36:39], s88 offen  lds\n\t" \
  "v_mfma_f32_16x16x128_f8f6f4 v[184:187], v[120:127], a[144:151], v[184:187]\n\t" \
  "s_add_u32 m0, s60, 68608\n\t" \
  "s_nop 0\n\t" \
  "buffer_load_dwordx4 v83, s[36:39], s88 offen  lds\n\t" \
  "v_mfma_f32_16x16x128_f8f6f4 v[188:191], v[120:127], a[152:159], v[188:191]\n\t" \
  "s_waitcnt lgkmcnt(4)\n\t" \
  "v_mfma_f32_16x16x128_f8f6f4 v[192:195], v[96:103], a[128:135], v[192:195]\n\t" \
  "ds_read_b64_tr_b8 v[112:113], v70 offset:49152\n\t" \
  "ds_read_b64_tr_b8 v[114:115], v70 offset:50176\n\t" \
  "v_mfma_f32_16x16x128_f8f6f4 v[196:199], v[96:103], a[136:143], v[196:199]\n\t" \
  "ds_read_b64_tr_b8 v[116:117], v70 offset:57344\n\t" \
  "ds_read_b64_tr_b8 v[118:119], v70 offset:58368\n\t" \
  "buffer_load_dwordx4 a[252:255], v84, s[44:47], s90 offen offset:3072 sc0 sc1\n\t" \
  "v_mfma_f32_16x16x128_f8f6f4 v[200:203], v[96:103], a[144:151], v[200:203]\n\t" \
  "s_add_u32 s89, s62, 1048576\n\t" \
  "s_add_u32 s90, s62, 1052672\n\t" \
  "buffer_load_dwordx4 a[0:3], v84, s[44:47], s89 offen offset:0 sc0 sc1\n\t" \
  "v_mfma_f32_16x16x128_f8f6f4 v[204:207], v[96:103], a[152:159], v[204:207]\n\t" \
  "s_waitcnt lgkmcnt(4)\n\t" \
  "v_mfma_f32_16x16x128_f8f6f4 v[208:211], v[104:111], a[128:135], v[208:211]\n\t" \
  "ds_read_b64_tr_b8 v[120:121], v71 offset:49152\n\t" \
  "ds_read_b64_tr_b8 v[122:123], v71 offset:50176\n\t" \
  "v_mfma_f32_16x16x128_f8f6f4 v[212:215], v[104:111], a[136:143], v[212:215]\n\t" \
  "ds_read_b64_tr_b8 v[124:125], v71 offset:57344\n\t" \
  "ds_read_b64_tr_b8 v[126:127], v71 offset:58368\n\t" \
  "buffer_load_dwordx4 a[4:7], v84, s[44:47], s89 offen offset:1024 sc0 sc1\n\t" \
  "v_mfma_f32_16x16x128_f8f6f4 v[216:219], v[104:111], a[144:151], v[216:219]\n\t" \
  "v_mfma_f32_16x16x128_f8f6f4 v[220:223], v[104:111], a[152:159], v[220:223]\n\t" \
  "s_waitcnt vmcnt(15)\n\t" \
  "s_barrier\n\t" \
  "s_waitcnt lgkmcnt(4)\n\t" \
  "v_mfma_f32_16x16x128_f8f6f4 v[224:227], v[112:119], a[128:135], v[224:227]\n\t" \
  "ds_read_b64_tr_b8 v[96:97], v64 offset:0\n\t" \
  "ds_read_b64_tr_b8 v[98:99], v64 offset:1024\n\t" \
  "v_mfma_f32_16x16x128_f8f6f4 v[228:231], v[112:119], a[136:143], v[228:231]\n\t" \
  "ds_read_b64_tr_b8 v[100:101], v64 offset:8192\n\t" \
  "ds_read_b64_tr_b8 v[102:103], v64 offset:9216\n\t" \
  "v_mfma_f32_16x16x128_f8f6f4 v[232:235], v[112:119], a[144:151], v[232:235]\n\t" \
  "v_mfma_f32_16x16x128_f8f6f4 v[236:239], v[112:119], a[152:159], v[236:239]\n\t" \
  "s_cmp_lg_u32 s63, 3\n\t" \
  "s_cbranch_scc1 PS36_%=\n\t" \
  "s_waitcnt vmcnt(47)\n\t" \
  "v_mov_b32 v89, s64\n\t" \
  "s_mov_b64 exec, 1\n\t" \
  "global_store_dword v88, v89, s[76:77] sc0 sc1\n\t" \
  "s_mov_b64 exec, -1\n\t" \
  "PS36_%=:\n\t" \
  "s_waitcnt lgkmcnt(4)\n\t" \
  "v_mfma_f32_16x16x128_f8f6f4 v[240:243], v[120:127], a[128:135], v[240:243]\n\t" \
  "ds_read_b64_tr_b8 v[104:105], v65 offset:0\n\t" \
  "ds_read_b64_tr_b8 v[106:107], v65 offset:1024\n\t" \
  "v_mfma_f32_16x16x128_f8f6f4 v[244:247], v[120:127], a[136:143], v[244:247]\n\t" \
  "ds_read_b64_tr_b8 v[108:109], v65 offset:8192\n\t" \
  "ds_read_b64_tr_b8 v[110:111], v65 offset:9216\n\t" \
  "v_mfma_f32_16x16x128_f8f6f4 v[248:251], v[120:127], a[144:151], v[248:251]\n\t" \
  "v_mfma_f32_16x16x128_f8f6f4 v[252:255], v[120:127], a[152:159], v[252:255]\n\t" \
  "s_waitcnt vmcnt(34)\n\t" \
  "global_load_dword v91, v87, s[70:71] sc0 sc1\n\t" \
  "s_bitcmp1_b32 s65, 17\n\t" \
  "s_cbranch_scc1 OK37_%=\n\t" \
  "s_mov_b32 s67, 17\n\t" \
  "s_call_b64 s[68:69], POLL_%=\n\t" \
  "OK37_%=:\n\t" \
  "s_waitcnt lgkmcnt(4)\n\t" \
  "v_mfma_f32_16x16x128_f8f6f4 v[128:131], v[96:103], a[160:167], v[128:131]\n\t" \
  "ds_read_b64_tr_b8 v[112:113], v66 offset:0\n\t" \
  "ds_read_b64_tr_b8 v[114:115], v66 offset:1024\n\t" \
  "v_mfma_f32_16x16x128_f8f6f4 v[132:135], v[96:103], a[168:175], v[132:135]\n\t" \
  "ds_read_b64_tr_b8 v[116:117], v66 offset:8192\n\t" \
  "ds_read_b64_tr_b8 v[118:119], v66 offset:9216\n\t" \
  "v_mfma_f32_16x16x128_f8f6f4 v[136:139], v[96:103], a[176:183], v[136:139]\n\t" \
  "v_mfma_f32_16x16x128_f8f6f4 v[140:143], v[96:103], a[184:191], v[140:143]\n\t" \
  "s_waitcnt lgkmcnt(4)\n\t" \
  "v_mfma_f32_16x16x128_f8f6f4 v[144:147], v[104:111], a[160:167], v[144:147]\n\t" \
  "ds_read_b64_tr_b8 v[120:121], v67 offset:0\n\t" \
  "ds_read_b64_tr_b8 v[122:123], v67 offset:1024\n\t" \
  "v_mfma_f32_16x16x128_f8f6f4 v[148:151], v[104:111], a[168:175], v[148:151]\n\t" \
  "ds_read_b64_tr_b8 v[124:125], v67 offset:8192\n\t" \
  "ds_read_b64_tr_b8 v[126:127], v67 offset:9216\n\t" \
  "v_mfma_f32_16x16x128_f8f6f4 v[152:155], v[104:111], a[176:183], v[152:155]\n\t" \
  "v_mfma_f32_16x16x128_f8f6f4 v[156:159], v[104:111], a[184:191], v[156:159]\n\t" \
  "s_waitcnt lgkmcnt(4)\n\t" \
  "v_mfma_f32_16x16x128_f8f6f4 v[160:163], v[112:119], a[160:167], v[160:163]\n\t" \
  "ds_read_b64_tr_b8 v[96:97], v68 offset:0\n\t" \
  "ds_read_b64_tr_b8 v[98:99], v68 offset:1024\n\t" \
  "v_mfma_f32_16x16x128_f8f6f4 v[164:167], v[112:119], a[168:175], v[164:167]\n\t" \
  "ds_read_b64_tr_b8 v[100:101], v68 offset:8192\n\t" \
  "ds_read_b64_tr_b8 v[102:103], v68 offset:9216\n\t" \
  "v_mfma_f32_16x16x128_f8f6f4 v[168:171], v[112:119], a[176:183], v[168:171]\n\t" \
  "v_mfma_f32_16x16x128_f8f6f4 v[172:175], v[112:119], a[184:191], v[172:175]\n\t" \
  "s_waitcnt lgkmcnt(4)\n\t" \
  "v_mfma_f32_16x16x128_f8f6f4 v[176:179], v[120:127], a[160:167], v[176:179]\n\t" \
  "ds_read_b64_tr_b8 v[104:105], v69 offset:0\n\t" \
  "ds_read_b64_tr_b8 v[106:107], v69 offset:1024\n\t" \
  "v_mfma_f32_16x16x128_f8f6f4 v[180:183], v[120:127], a[168:175], v[180:183]\n\t" \
  "ds_read_b64_tr_b8 v[108:109], v69 offset:8192\n\t" \
  "ds_read_b64_tr_b8 v[110:111], v69 offset:9216\n\t" \
  "v_mfma_f32_16x16x128_f8f6f4 v[184:187], v[120:127], a[176:183], v[184:187]\n\t" \
  "v_mfma_f32_16x16x128_f8f6f4 v[188:191], v[120:127], a[184:191], v[188:191]\n\t" \
  "s_waitcnt lgkmcnt(4)\n\t" \
  "v_mfma_f32_16x16x128_f8f6f4 v[192:195], v[96:103], a[160:167], v[192:195]\n\t" \
  "ds_read_b64_tr_b8 v[112:113], v70 offset:0\n\t" \
  "ds_read_b64_tr_b8 v[114:115], v70 offset:1024\n\t" \
  "v_mfma_f32_16x16x128_f8f6f4 v[196:199], v[96:103], a[168:175], v[196:199]\n\t" \
  "ds_read_b64_tr_b8 v[116:117], v70 offset:8192\n\t" \
  "ds_read_b64_tr_b8 v[118:119], v70 offset:9216\n\t" \
  "v_mfma_f32_16x16x128_f8f6f4 v[200:203], v[96:103], a[176:183], v[200:203]\n\t" \
  "v_mfma_f32_16x16x128_f8f6f4 v[204:207], v[96:103], a[184:191], v[204:207]\n\t" \
  "s_waitcnt lgkmcnt(4)\n\t" \
  "v_mfma_f32_16x16x128_f8f6f4 v[208:211], v[104:111], a[160:167], v[208:211]\n\t" \
  "ds_read_b64_tr_b8 v[120:121], v71 offset:0\n\t" \
  "ds_read_b64_tr_b8 v[122:123], v71 offset:1024\n\t" \
  "v_mfma_f32_16x16x128_f8f6f4 v[212:215], v[104:111], a[168:175], v[212:215]\n\t" \
  "ds_read_b64_tr_b8 v[124:125], v71 offset:8192\n\t" \
  "ds_read_b64_tr_b8 v[126:127], v71 offset:9216\n\t" \
  "v_mfma_f32_16x16x128_f8f6f4 v[216:219], v[104:111], a[176:183], v[216:219]\n\t" \
  "v_mfma_f32_16x16x128_f8f6f4 v[220:223], v[104:111], a[184:191], v[220:223]\n\t" \
  "s_waitcnt vmcnt(12)\n\t" \
  "s_barrier\n\t" \
  "s_waitcnt lgkmcnt(4)\n\t" \
  "v_mfma_f32_16x16x128_f8f6f4 v[224:227], v[112:119], a[160:167], v[224:227]\n\t" \
  "ds_read_b64_tr_b8 v[96:97], v64 offset:16384\n\t" \
  "ds_read_b64_tr_b8 v[98:99], v64 offset:17408\n\t" \
  "v_mfma_f32_16x16x128_f8f6f4 v[228:231], v[112:119], a[168:175], v[228:231]\n\t" \
  "ds_read_b64_tr_b8 v[100:101], v64 offset:24576\n\t" \
  "ds_read_b64_tr_b8 v[102:103], v64 offset:25600\n\t" \
  "buffer_load_dwordx4 a[8:11], v84, s[44:47], s89 offen offset:2048 sc0 sc1\n\t" \
  "v_mfma_f32_16x16x128_f8f6f4 v[232:235], v[112:119], a[176:183], v[232:235]\n\t" \
  "buffer_load_dwordx4 a[12:15], v84, s[44:47], s89 offen offset:3072 sc0 sc1\n\t" \
  "v_mfma_f32_16x16x128_f8f6f4 v[236:239], v[112:119], a[184:191], v[236:239]\n\t" \
  "s_waitcnt lgkmcnt(4)\n\t" \
  "v_mfma_f32_16x16x128_f8f6f4 v[240:243], v[120:127], a[160:167], v[240:243]\n\t" \
  "ds_read_b64_tr_b8 v[104:105], v65 offset:16384\n\t" \
  "ds_read_b64_tr_b8 v[106:107], v65 offset:17408\n\t" \
  "v_mfma_f32_16x16x128_f8f6f4 v[244:247], v[120:127], a[168:175], v[244:247]\n\t" \
  "ds_read_b64_tr_b8 v[108:109], v65 offset:24576\n\t" \
  "ds_read_b64_tr_b8 v[110:111], v65 offset:25600\n\t" \
  "buffer_load_dwordx4 a[16:19], v84, s[44:47], s90 offen offset:0 sc0 sc1\n\t" \
  "v_mfma_f32_16x16x128_f8f6f4 v[248:251], v[120:127], a[176:183], v[248:251]\n\t" \
  "buffer_load_dwordx4 a[20:23], v84, s[44:47], s90 offen offset:1024 sc0 sc1\n\t" \
  "v_mfma_f32_16x16x128_f8f6f4 v[252:255], v[120:127], a[184:191], v[252:255]\n\t" \
  "s_waitcnt vmcnt(4)\n\t" \
  "v_cmp_le_u32 vcc, s64, v91\n\t" \
  "s_nop 1\n\t" \
  "s_or_b32 s65, s65, vcc_lo\n\t" \
  "s_bitcmp1_b32 s65, 18\n\t" \
  "s_cbranch_scc1 OK38_%=\n\t" \
  "s_mov_b32 s67, 18\n\t" \
  "s_call_b64 s[68:69], POLL_%=\n\t" \
  "OK38_%=:\n\t" \
  "s_waitcnt lgkmcnt(4)\n\t" \
  "v_mfma_f32_16x16x128_f8f6f4 v[128:131], v[96:103], a[192:199], v[128:131]\n\t" \
  "ds_read_b64_tr_b8 v[112:113], v66 offset:16384\n\t" \
  "ds_read_b64_tr_b8 v[114:115], v66 offset:17408\n\t" \
  "v_mfma_f32_16x16x128_f8f6f4 v[132:135], v[96:103], a[200:207], v[132:135]\n\t" \
  "ds_read_b64_tr_b8 v[116:117], v66 offset:24576\n\t" \
  "ds_read_b64_tr_b8 v[118:119], v66 offset:25600\n\t" \
  "buffer_load_dwordx4 a[24:27], v84, s[44:47], s90 offen offset:2048 sc0 sc1\n\t" \
  "v_mfma_f32_16x16x128_f8f6f4 v[136:139], v[96:103], a[208:215], v[136:139]\n\t" \
  "buffer_load_dwordx4 a[28:31], v84, s[44:47], s90 offen offset:3072 sc0 sc1\n\t" \
  "v_mfma_f32_16x16x128_f8f6f4 v[140:143], v[96:103], a[216:223], v[140:143]\n\t" \
  "s_waitcnt lgkmcnt(4)\n\t" \
  "v_mfma_f32_16x16x128_f8f6f4 v[144:147], v[104:111], a[192:199], v[144:147]\n\t" \
  "ds_read_b64_tr_b8 v[120:121], v67 offset:16384\n\t" \
  "ds_read_b64_tr_b8 v[122:123], v67 offset:17408\n\t" \
  "v_mfma_f32_16x16x128_f8f6f4 v[148:151], v[104:111], a[200:207], v[148:151]\n\t" \
  "ds_read_b64_tr_b8 v[124:125], v67 offset:24576\n\t" \
  "ds_read_b64_tr_b8 v[126:127], v67 offset:25600\n\t" \
  "s_add_u32 s89, s62, 1114112\n\t" \
  "s_add_u32 s90, s62, 1118208\n\t" \
  "buffer_load_dwordx4 a[32:35], v84, s[44:47], s89 offen offset:0 sc0 sc1\n\t" \
  "v_mfma_f32_16x16x128_f8f6f4 v[152:155], v[104:111], a[208:215], v[152:155]\n\t" \
  "buffer_load_dwordx4 a[36:39], v84, s[44:47], s89 offen offset:1024 sc0 sc1\n\t" \
  "v_mfma_f32_16x16x128_f8f6f4 v[156:159], v[104:111], a[216:223], v[156:159]\n\t" \
  "s_waitcnt lgkmcnt(4)\n\t" \
  "v_mfma_f32_16x16x128_f8f6f4 v[160:163], v[112:119], a[192:199], v[160:163]\n\t" \
  "ds_read_b64_tr_b8 v[96:97], v68 offset:16384\n\t" \
  "ds_read_b64_tr_b8 v[98:99], v68 offset:17408\n\t" \
  "v_mfma_f32_16x16x128_f8f6f4 v[164:167], v[112:119], a[200:207], v[164:167]\n\t" \
  "ds_read_b64_tr_b8 v[100:101], v68 offset:24576\n\t" \
  "ds_read_b64_tr_b8 v[102:103], v68 offset:25600\n\t" \
  "buffer_load_dwordx4 a[40:43], v84, s[44:47], s89 offen offset:2048 sc0 sc1\n\t" \
  "v_mfma_f32_16x16x128_f8f6f4 v[168:171], v[112:119], a[208:215], v[168:171]\n\t" \
  "buffer_load_dwordx4 a[44:47], v84, s[44:47], s89 offen offset:3072 sc0 sc1\n\t" \
  "v_mfma_f32_16x16x128_f8f6f4 v[172:175], v[112:119], a[216:223], v[172:175]\n\t" \
  "s_waitcnt lgkmcnt(4)\n\t" \
  "v_mfma_f32_16x16x128_f8f6f4 v[176:179], v[120:127], a[192:199], v[176:179]\n\t" \
  "ds_read_b64_tr_b8 v[104:105], v69 offset:16384\n\t" \
  "ds_read_b64_tr_b8 v[106:107], v69 offset:17408\n\t" \
  "v_mfma_f32_16x16x128_f8f6f4 v[180:183], v[120:127], a[200:207], v[180:183]\n\t" \
  "ds_read_b64_tr_b8 v[108:109], v69 offset:24576\n\t" \
  "ds_read_b64_tr_b8 v[110:111], v69 offset:25600\n\t" \
  "buffer_load_dwordx4 a[48:51], v84, s[44:47], s90 offen offset:0 sc0 sc1\n\t" \
  "v_mfma_f32_16x16x128_f8f6f4 v[184:187], v[120:127], a[208:215], v[184:187]\n\t" \
  "buffer_load_dwordx4 a[52:55], v84, s[44:47], s90 offen offset:1024 sc0 sc1\n\t" \
  "v_mfma_f32_16x16x128_f8f6f4 v[188:191], v[120:127], a[216:223], v[188:191]\n\t" \
  "s_waitcnt lgkmcnt(4)\n\t" \
  "v_mfma_f32_16x16x128_f8f6f4 v[192:195], v[96:103], a[192:199], v[192:195]\n\t" \
  "ds_read_b64_tr_b8 v[112:113], v70 offset:16384\n\t" \
  "ds_read_b64_tr_b8 v[114:115], v70 offset:17408\n\t" \
  "v_mfma_f32_16x16x128_f8f6f4 v[196:199], v[96:103], a[200:207], v[196:199]\n\t" \
  "ds_read_b64_tr_b8 v[116:117], v70 offset:24576\n\t" \
  "ds_read_b64_tr_b8 v[118:119], v70 offset:25600\n\t" \
  "buffer_load_dwordx4 a[56:59], v84, s[44:47], s90 offen offset:2048 sc0 sc1\n\t" \
  "v_mfma_f32_16x16x128_f8f6f4 v[200:203], v[96:103], a[208:215], v[200:203]\n\t" \
  "buffer_load_dwordx4 a[60:63], v84, s[44:47], s90 offen offset:3072 sc0 sc1\n\t" \
  "v_mfma_f32_16x16x128_f8f6f4 v[204:207], v[96:103], a[216:223], v[204:207]\n\t" \
  "s_waitcnt lgkmcnt(4)\n\t" \
  "v_mfma_f32_16x16x128_f8f6f4 v[208:211], v[104:111], a[192:199], v[208:211]\n\t" \
  "ds_read_b64_tr_b8 v[120:121], v71 offset:16384\n\t" \
  "ds_read_b64_tr_b8 v[122:123], v71 offset:17408\n\t" \
  "v_mfma_f32_16x16x128_f8f6f4 v[212:215], v[104:111], a[200:207], v[212:215]\n\t" \
  "ds_read_b64_tr_b8 v[124:125], v71 offset:24576\n\t" \
  "ds_read_b64_tr_b8 v[126:127], v71 offset:25600\n\t" \
  "s_add_u32 s89, s62, 1179648\n\t" \
  "s_add_u32 s90, s62, 1183744\n\t" \
  "buffer_load_dwordx4 a[64:67], v84, s[44:47], s89 offen offset:0 sc0 sc1\n\t" \
  "v_mfma_f32_16x16x128_f8f6f4 v[216:219], v[104:111], a[208:215], v[216:219]\n\t" \
  "buffer_load_dwordx4 a[68:71], v84, s[44:47], s89 offen offset:1024 sc0 sc1\n\t" \
  "v_mfma_f32_16x16x128_f8f6f4 v[220:223], v[104:111], a[216:223], v[220:223]\n\t" \
  "s_waitcnt vmcnt(24)\n\t" \
  "s_barrier\n\t" \
  "s_waitcnt lgkmcnt(4)\n\t" \
  "v_mfma_f32_16x16x128_f8f6f4 v[224:227], v[112:119], a[192:199], v[224:227]\n\t" \
  "ds_read_b64_tr_b8 v[96:97], v64 offset:32768\n\t" \
  "ds_read_b64_tr_b8 v[98:99], v64 offset:33792\n\t" \
  "v_mfma_f32_16x16x128_f8f6f4 v[228:231], v[112:119], a[200:207], v[228:231]\n\t" \
  "ds_read_b64_tr_b8 v[100:101], v64 offset:40960\n\t" \
  "ds_read_b64_tr_b8 v[102:103], v64 offset:41984\n\t" \
  "buffer_load_dwordx4 a[72:75], v84, s[44:47], s89 offen offset:2048 sc0 sc1\n\t" \
  "v_mfma_f32_16x16x128_f8f6f4 v[232:235], v[112:119], a[208:215], v[232:235]\n\t" \
  "buffer_load_dwordx4 a[76:79], v84, s[44:47], s89 offen offset:3072 sc0 sc1\n\t" \
  "v_mfma_f32_16x16x128_f8f6f4 v[236:239], v[112:119], a[216:223], v[236:239]\n\t" \
  "s_waitcnt lgkmcnt(4)\n\t" \
  "v_mfma_f32_16x16x128_f8f6f4 v[240:243], v[120:127], a[192:199], v[240:243]\n\t" \
  "ds_read_b64_tr_b8 v[104:105], v65 offset:32768\n\t" \
  "ds_read_b64_tr_b8 v[106:107], v65 offset:33792\n\t" \
  "v_mfma_f32_16x16x128_f8f6f4 v[244:247], v[120:127], a[200:207], v[244:247]\n\t" \
  "ds_read_b64_tr_b8 v[108:109], v65 offset:40960\n\t" \
  "ds_read_b64_tr_b8 v[110:111], v65 offset:41984\n\t" \
  "buffer_load_dwordx4 a[80:83], v84, s[44:47], s90 offen offset:0 sc0 sc1\n\t" \
  "v_mfma_f32_16x16x128_f8f6f4 v[248:251], v[120:127], a[208:215], v[248:251]\n\t" \
  "buffer_load_dwordx4 a[84:87], v84, s[44:47], s90 offen offset:1024 sc0 sc1\n\t" \
  "v_mfma_f32_16x16x128_f8f6f4 v[252:255], v[120:127], a[216:223], v[252:255]\n\t" \
  "s_waitcnt vmcnt(23)\n\t" \
  "s_bitcmp1_b32 s65, 19\n\t" \
  "s_cbranch_scc1 OK39_%=\n\t" \
  "s_mov_b32 s67, 19\n\t" \
  "s_call_b64 s[68:69], POLL_%=\n\t" \
  "OK39_%=:\n\t" \
  "s_waitcnt lgkmcnt(4)\n\t" \
  "v_mfma_f32_16x16x128_f8f6f4 v[128:131], v[96:103], a[224:231], v[128:131]\n\t" \
  "ds_read_b64_tr_b8 v[112:113], v66 offset:32768\n\t" \
  "ds_read_b64_tr_b8 v[114:115], v66 offset:33792\n\t" \
  "v_mfma_f32_16x16x128_f8f6f4 v[132:135], v[96:103], a[232:239], v[132:135]\n\t" \
  "ds_read_b64_tr_b8 v[116:117], v66 offset:40960\n\t" \
  "ds_read_b64_tr_b8 v[118:119], v66 offset:41984\n\t" \
  "buffer_load_dwordx4 a[88:91], v84, s[44:47], s90 offen offset:2048 sc0 sc1\n\t" \
  "v_mfma_f32_16x16x128_f8f6f4 v[136:139], v[96:103], a[240:247], v[136:139]\n\t" \
  "buffer_load_dwordx4 a[92:95], v84, s[44:47], s90 offen offset:3072 sc0 sc1\n\t" \
  "v_mfma_f32_16x16x128_f8f6f4 v[140:143], v[96:103], a[248:255], v[140:143]\n\t" \
  "s_waitcnt lgkmcnt(4)\n\t" \
  "v_mfma_f32_16x16x128_f8f6f4 v[144:147], v[104:111], a[224:231], v[144:147]\n\t" \
  "ds_read_b64_tr_b8 v[120:121], v67 offset:32768\n\t" \
  "ds_read_b64_tr_b8 v[122:123], v67 offset:33792\n\t" \
  "v_mfma_f32_16x16x128_f8f6f4 v[148:151], v[104:111], a[232:239], v[148:151]\n\t" \
  "ds_read_b64_tr_b8 v[124:125], v67 offset:40960\n\t" \
  "ds_read_b64_tr_b8 v[126:127], v67 offset:41984\n\t" \
  "s_add_u32 s89, s62, 1245184\n\t" \
  "s_add_u32 s90, s62, 1249280\n\t" \
  "buffer_load_dwordx4 a[96:99], v84, s[44:47], s89 offen offset:0 sc0 sc1\n\t" \
  "v_mfma_f32_16x16x128_f8f6f4 v[152:155], v[104:111], a[240:247], v[152:155]\n\t" \
  "buffer_load_dwordx4 a[100:103], v84, s[44:47], s89 offen offset:1024 sc0 sc1\n\t" \
  "v_mfma_f32_16x16x128_f8f6f4 v[156:159], v[104:111], a[248:255], v[156:159]\n\t" \
  "s_waitcnt lgkmcnt(4)\n\t" \
  "v_mfma_f32_16x16x128_f8f6f4 v[160:163], v[112:119], a[224:231], v[160:163]\n\t" \
  "ds_read_b64_tr_b8 v[96:97], v68 offset:32768\n\t" \
  "ds_read_b64_tr_b8 v[98:99], v68 offset:33792\n\t" \
  "v_mfma_f32_16x16x128_f8f6f4 v[164:167], v[112:119], a[232:239], v[164:167]\n\t" \
  "ds_read_b64_tr_b8 v[100:101], v68 offset:40960\n\t" \
  "ds_read_b64_tr_b8 v[102:103], v68 offset:41984\n\t" \
  "buffer_load_dwordx4 a[104:107], v84, s[44:47], s89 offen offset:2048 sc0 sc1\n\t" \
  "v_mfma_f32_16x16x128_f8f6f4 v[168:171], v[112:119], a[240:247], v[168:171]\n\t" \
  "buffer_load_dwordx4 a[108:111], v84, s[44:47], s89 offen offset:3072 sc0 sc1\n\t" \
  "v_mfma_f32_16x16x128_f8f6f4 v[172:175], v[112:119], a[248:255], v[172:175]\n\t" \
  "s_waitcnt lgkmcnt(4)\n\t" \
  "v_mfma_f32_16x16x128_f8f6f4 v[176:179], v[120:127], a[224:231], v[176:179]\n\t" \
  "ds_read_b64_tr_b8 v[104:105], v69 offset:32768\n\t" \
  "ds_read_b64_tr_b8 v[106:107], v69 offset:33792\n\t" \
  "v_mfma_f32_16x16x128_f8f6f4 v[180:183], v[120:127], a[232:239], v[180:183]\n\t" \
  "ds_read_b64_tr_b8 v[108:109], v69 offset:40960\n\t" \
  "ds_read_b64_tr_b8 v[110:111], v69 offset:41984\n\t" \
  "buffer_load_dwordx4 a[112:115], v84, s[44:47], s90 offen offset:0 sc0 sc1\n\t" \
  "v_mfma_f32_16x16x128_f8f6f4 v[184:187], v[120:127], a[240:247], v[184:187]\n\t" \
  "buffer_load_dwordx4 a[116:119], v84, s[44:47], s90 offen offset:1024 sc0 sc1\n\t" \
  "v_mfma_f32_16x16x128_f8f6f4 v[188:191], v[120:127], a[248:255], v[188:191]\n\t" \
  "s_waitcnt lgkmcnt(4)\n\t" \
  "v_mfma_f32_16x16x128_f8f6f4 v[192:195], v[96:103], a[224:231], v[192:195]\n\t" \
  "ds_read_b64_tr_b8 v[112:113], v70 offset:32768\n\t" \
  "ds_read_b64_tr_b8 v[114:115], v70 offset:33792\n\t" \
  "v_mfma_f32_16x16x128_f8f6f4 v[196:199], v[96:103], a[232:239], v[196:199]\n\t" \
  "ds_read_b64_tr_b8 v[116:117], v70 offset:40960\n\t" \
  "ds_read_b64_tr_b8 v[118:119], v70 offset:41984\n\t" \
  "s_add_u32 s88, s58, 2097152\n\t" \
  "s_add_u32 m0, s60, 81920\n\t" \
  "s_nop 0\n\t" \
  "buffer_load_dwordx4 v80, s[36:39], s88 offen  lds\n\t" \
  "v_mfma_f32_16x16x128_f8f6f4 v[200:203], v[96:103], a[240:247], v[200:203]\n\t" \
  "s_add_u32 m0, s60, 82944\n\t" \
  "s_nop 0\n\t" \
  "buffer_load_dwordx4 v81, s[36:39], s88 offen  lds\n\t" \
  "v_mfma_f32_16x16x128_f8f6f4 v[204:207], v[96:103], a[248:255], v[204:207]\n\t" \
  "s_waitcnt lgkmcnt(4)\n\t" \
  "v_mfma_f32_16x16x128_f8f6f4 v[208:211], v[104:111], a[224:231], v[208:211]\n\t" \
  "ds_read_b64_tr_b8 v[120:121], v71 offset:32768\n\t" \
  "ds_read_b64_tr_b8 v[122:123], v71 offset:33792\n\t" \
  "v_mfma_f32_16x16x128_f8f6f4 v[212:215], v[104:111], a[232:239], v[212:215]\n\t" \
  "ds_read_b64_tr_b8 v[124:125], v71 offset:40960\n\t" \
  "ds_read_b64_tr_b8 v[126:127], v71 offset:41984\n\t" \
  "s_add_u32 m0, s60, 83968\n\t" \
  "s_nop 0\n\t" \
  "buffer_load_dwordx4 v82, s[36:39], s88 offen  lds\n\t" \
  "v_mfma_f32_16x16x128_f8f6f4 v[216:219], v[104:111], a[240:247], v[216:219]\n\t" \
  "s_add_u32 m0, s60, 84992\n\t" \
  "s_nop 0\n\t" \
  "buffer_load_dwordx4 v83, s[36:39], s88 offen  lds\n\t" \
  "v_mfma_f32_16x16x128_f8f6f4 v[220:223], v[104:111], a[248:255], v[220:223]\n\t" \
  "s_waitcnt vmcnt(36)\n\t" \
  "s_barrier\n\t" \
  "s_waitcnt lgkmcnt(4)\n\t" \
  "v_mfma_f32_16x16x128_f8f6f4 v[224:227], v[112:119], a[224:231], v[224:227]\n\t" \
  "ds_read_b64_tr_b8 v[96:97], v72 offset:0\n\t" \
  "ds_read_b64_tr_b8 v[98:99], v72 offset:1024\n\t" \
  "v_mfma_f32_16x16x128_f8f6f4 v[228:231], v[112:119], a[232:239], v[228:231]\n\t" \
  "ds_read_b64_tr_b8 v[100:101], v72 offset:8192\n\t" \
  "ds_read_b64_tr_b8 v[102:103], v72 offset:9216\n\t" \
  "s_add_u32 s88, s52, 4194304\n\t" \
  "s_add_u32 m0, s60, 98304\n\t" \
  "s_nop 0\n\t" \
  "buffer_load_dwordx4 v80, s[36:39], s88 offen  lds\n\t" \
  "v_mfma_f32_16x16x128_f8f6f4 v[232:235], v[112:119], a[240:247], v[232:235]\n\t" \
  "s_add_u32 m0, s60, 99328\n\t" \
  "s_nop 0\n\t" \
  "buffer_load_dwordx4 v81, s[36:39], s88 offen  lds\n\t" \
  "v_mfma_f32_16x16x128_f8f6f4 v[236:239], v[112:119], a[248:255], v[236:239]\n\t" \
  "s_waitcnt lgkmcnt(4)\n\t" \
  "v_mfma_f32_16x16x128_f8f6f4 v[240:243], v[120:127], a[224:231], v[240:243]\n\t" \
  "ds_read_b64_tr_b8 v[104:105], v73 offset:0\n\t" \
  "ds_read_b64_tr_b8 v[106:107], v73 offset:1024\n\t" \
  "v_mfma_f32_16x16x128_f8f6f4 v[244:247], v[120:127], a[232:239], v[244:247]\n\t" \
  "ds_read_b64_tr_b8 v[108:109], v73 offset:8192\n\t" \
  "ds_read_b64_tr_b8 v[110:111], v73 offset:9216\n\t" \
  "s_add_u32 m0, s60, 100352\n\t" \
  "s_nop 0\n\t" \
  "buffer_load_dwordx4 v82, s[36:39], s88 offen  lds\n\t" \
  "v_mfma_f32_16x16x128_f8f6f4 v[248:251], v[120:127], a[240:247], v[248:251]\n\t" \
  "s_add_u32 m0, s60, 101376\n\t" \
  "s_nop 0\n\t" \
  "buffer_load_dwordx4 v83, s[36:39], s88 offen  lds\n\t" \
  "v_mfma_f32_16x16x128_f8f6f4 v[252:255], v[120:127], a[248:255], v[252:255]\n\t" \
  "s_waitcnt vmcnt(30)\n\t" \
  "s_bitcmp1_b32 s65, 20\n\t" \
  "s_cbranch_scc1 OK40_%=\n\t" \
  "s_mov_b32 s67, 20\n\t" \
  "s_call_b64 s[68:69], POLL_%=\n\t" \
  "OK40_%=:\n\t" \
  "s_waitcnt lgkmcnt(4)\n\t" \
  "v_mfma_f32_16x16x128_f8f6f4 v[128:131], v[96:103], a[0:7], v[128:131]\n\t" \
  "ds_read_b64_tr_b8 v[112:113], v74 offset:0\n\t" \
  "ds_read_b64_tr_b8 v[114:115], v74 offset:1024\n\t" \
  "s_bfe_u32 s96, s65, 262165\n\t" \
  "s_cmp_lg_u32 s96, 15\n\t" \
  "s_cbranch_scc1 XWS41_%=\n\t" \
  "s_add_u32 s96, s62, 1376256\n\t" \
  "buffer_load_dword v8, v13, s[44:47], s96 offen sc0 sc1\n\t" \
  "XWS41_%=:\n\t" \
  "v_mfma_f32_16x16x128_f8f6f4 v[132:135], v[96:103], a[8:15], v[132:135]\n\t" \
  "ds_read_b64_tr_b8 v[116:117], v74 offset:8192\n\t" \
  "ds_read_b64_tr_b8 v[118:119], v74 offset:9216\n\t" \
  "s_add_u32 s88, s53, 4194304\n\t" \
  "s_add_u32 m0, s60, 114688\n\t" \
  "s_nop 0\n\t" \
  "buffer_load_dwordx4 v80, s[36:39], s88 offen  lds\n\t" \
  "v_mfma_f32_16x16x128_f8f6f4 v[136:139], v[96:103], a[16:23], v[136:139]\n\t" \
  "s_add_u32 m0, s60, 115712\n\t" \
  "s_nop 0\n\t" \
  "buffer_load_dwordx4 v81, s[36:39], s88 offen  lds\n\t" \
  "v_mfma_f32_16x16x128_f8f6f4 v[140:143], v[96:103], a[24:31], v[140:143]\n\t" \
  "s_add_u32 m0, s60, 116736\n\t" \
  "s_nop 0\n\t" \
  "buffer_load_dwordx4 v82, s[36:39], s88 offen  lds\n\t" \
  "s_waitcnt lgkmcnt(4)\n\t" \
  "v_mfma_f32_16x16x128_f8f6f4 v[144:147], v[104:111], a[0:7], v[144:147]\n\t" \
  "ds_read_b64_tr_b8 v[120:121], v75 offset:0\n\t" \
  "ds_read_b64_tr_b8 v[122:123], v75 offset:1024\n\t" \
  "v_mfma_f32_16x16x128_f8f6f4 v[148:151], v[104:111], a[8:15], v[148:151]\n\t" \
  "ds_read_b64_tr_b8 v[124:125], v75 offset:8192\n\t" \
  "ds_read_b64_tr_b8 v[126:127], v75 offset:9216\n\t" \
  "s_add_u32 m0, s60, 117760\n\t" \
  "s_nop 0\n\t" \
  "buffer_load_dwordx4 v83, s[36:39], s88 offen  lds\n\t" \
  "v_mfma_f32_16x16x128_f8f6f4 v[152:155], v[104:111], a[16:23], v[152:155]\n\t" \
  "s_add_u32 s88, s54, 4194304\n\t" \
  "s_add_u32 m0, s60, 131072\n\t" \
  "s_nop 0\n\t" \
  "buffer_load_dwordx4 v80, s[36:39], s88 offen  lds\n\t" \
  "v_mfma_f32_16x16x128_f8f6f4 v[156:159], v[104:111], a[24:31], v[156:159]\n\t" \
  "s_add_u32 m0, s60, 132096\n\t" \
  "s_nop 0\n\t" \
  "buffer_load_dwordx4 v81, s[36:39], s88 offen  lds\n\t" \
  "s_waitcnt lgkmcnt(4)\n\t" \
  "v_mfma_f32_16x16x128_f8f6f4 v[160:163], v[112:119], a[0:7], v[160:163]\n\t" \
  "ds_read_b64_tr_b8 v[96:97], v76 offset:0\n\t" \
  "ds_read_b64_tr_b8 v[98:99], v76 offset:1024\n\t" \
  "v_mfma_f32_16x16x128_f8f6f4 v[164:167], v[112:119], a[8:15], v[164:167]\n\t" \
  "ds_read_b64_tr_b8 v[100:101], v76 offset:8192\n\t" \
  "ds_read_b64_tr_b8 v[102:103], v76 offset:9216\n\t" \
  "s_add_u32 m0, s60, 133120\n\t" \
  "s_nop 0\n\t" \
  "buffer_load_dwordx4 v82, s[36:39], s88 offen  lds\n\t" \
  "v_mfma_f32_16x16x128_f8f6f4 v[168:171], v[112:119], a[16:23], v[168:171]\n\t" \
  "s_add_u32 m0, s60, 134144\n\t" \
  "s_nop 0\n\t" \
  "buffer_load_dwordx4 v83, s[36:39], s88 offen  lds\n\t" \
  "v_mfma_f32_16x16x128_f8f6f4 v[172:175], v[112:119], a[24:31], v[172:175]\n\t" \
  "s_add_u32 s88, s55, 4194304\n\t" \
  "s_add_u32 m0, s60, 49152\n\t" \
  "s_nop 0\n\t" \
  "buffer_load_dwordx4 v80, s[36:39], s88 offen  lds\n\t" \
  "s_waitcnt lgkmcnt(4)\n\t" \
  "v_mfma_f32_16x16x128_f8f6f4 v[176:179], v[120:127], a[0:7], v[176:179]\n\t" \
  "ds_read_b64_tr_b8 v[104:105], v77 offset:0\n\t" \
  "ds_read_b64_tr_b8 v[106:107], v77 offset:1024\n\t" \
  "v_mfma_f32_16x16x128_f8f6f4 v[180:183], v[120:127], a[8:15], v[180:183]\n\t" \
  "ds_read_b64_tr_b8 v[108:109], v77 offset:8192\n\t" \
  "ds_read_b64_tr_b8 v[110:111], v77 offset:9216\n\t" \
  "s_add_u32 m0, s60, 50176\n\t" \
  "s_nop 0\n\t" \
  "buffer_load_dwordx4 v81, s[36:39], s88 offen  lds\n\t" \
  "v_mfma_f32_16x16x128_f8f6f4 v[184:187], v[120:127], a[16:23], v[184:187]\n\t" \
  "s_add_u32 m0, s60, 51200\n\t" \
  "s_nop 0\n\t" \
  "buffer_load_dwordx4 v82, s[36:39], s88 offen  lds\n\t" \
  "v_mfma_f32_16x16x128_f8f6f4 v[188:191], v[120:127], a[24:31], v[188:191]\n\t" \
  "s_add_u32 m0, s60, 52224\n\t" \
  "s_nop 0\n\t" \
  "buffer_load_dwordx4 v83, s[36:39], s88 offen  lds\n\t" \
  "s_waitcnt lgkmcnt(4)\n\t" \
  "v_mfma_f32_16x16x128_f8f6f4 v[192:195], v[96:103], a[0:7], v[192:195]\n\t" \
  "ds_read_b64_tr_b8 v[112:113], v78 offset:0\n\t" \
  "ds_read_b64_tr_b8 v[114:115], v78 offset:1024\n\t" \
  "v_mfma_f32_16x16x128_f8f6f4 v[196:199], v[96:103], a[8:15], v[196:199]\n\t" \
  "ds_read_b64_tr_b8 v[116:117], v78 offset:8192\n\t" \
  "ds_read_b64_tr_b8 v[118:119], v78 offset:9216\n\t" \
  "buffer_load_dwordx4 a[120:123], v84, s[44:47], s90 offen offset:2048 sc0 sc1\n\t" \
  "v_mfma_f32_16x16x128_f8f6f4 v[200:203], v[96:103], a[16:23], v[200:203]\n\t" \
  "buffer_load_dwordx4 a[124:127], v84, s[44:47], s90 offen offset:3072 sc0 sc1\n\t" \
  "v_mfma_f32_16x16x128_f8f6f4 v[204:207], v[96:103], a[24:31], v[204:207]\n\t" \
  "s_waitcnt lgkmcnt(4)\n\t" \
  "v_mfma_f32_16x16x128_f8f6f4 v[208:211], v[104:111], a[0:7], v[208:211]\n\t" \
  "ds_read_b64_tr_b8 v[120:121], v79 offset:0\n\t" \
  "ds_read_b64_tr_b8 v[122:123], v79 offset:1024\n\t" \
  "v_mfma_f32_16x16x128_f8f6f4 v[212:215], v[104:111], a[8:15], v[212:215]\n\t" \
  "ds_read_b64_tr_b8 v[124:125], v79 offset:8192\n\t" \
  "ds_read_b64_tr_b8 v[126:127], v79 offset:9216\n\t" \
  "s_add_u32 s89, s62, 1310720\n\t" \
  "s_add_u32 s90, s62, 1314816\n\t" \
  "buffer_load_dwordx4 a[128:131], v84, s[44:47], s89 offen offset:0 sc0 sc1\n\t" \
  "v_mfma_f32_16x16x128_f8f6f4 v[216:219], v[104:111], a[16:23], v[216:219]\n\t" \
  "v_mfma_f32_16x16x128_f8f6f4 v[220:223], v[104:111], a[24:31], v[220:223]\n\t" \
  "s_waitcnt vmcnt(19)\n\t" \
  "s_barrier\n\t" \
  "s_waitcnt lgkmcnt(4)\n\t" \
  "v_mfma_f32_16x16x128_f8f6f4 v[224:227], v[112:119], a[0:7], v[224:227]\n\t" \
  "ds_read_b64_tr_b8 v[96:97], v72 offset:16384\n\t" \
  "ds_read_b64_tr_b8 v[98:99], v72 offset:17408\n\t" \
  "v_mfma_f32_16x16x128_f8f6f4 v[228:231], v[112:119], a[8:15], v[228:231]\n\t" \
  "ds_read_b64_tr_b8 v[100:101], v72 offset:24576\n\t" \
  "ds_read_b64_tr_b8 v[102:103], v72 offset:25600\n\t" \
  "v_mfma_f32_16x16x128_f8f6f4 v[232:235], v[112:119], a[16:23], v[232:235]\n\t" \
  "v_mfma_f32_16x16x128_f8f6f4 v[236:239], v[112:119], a[24:31], v[236:239]\n\t" \
  "s_waitcnt lgkmcnt(4)\n\t" \
  "v_mfma_f32_16x16x128_f8f6f4 v[240:243], v[120:127], a[0:7], v[240:243]\n\t" \
  "ds_read_b64_tr_b8 v[104:105], v73 offset:16384\n\t" \
  "ds_read_b64_tr_b8 v[106:107], v73 offset:17408\n\t" \
  "v_mfma_f32_16x16x128_f8f6f4 v[244:247], v[120:127], a[8:15], v[244:247]\n\t" \
  "ds_read_b64_tr_b8 v[108:109], v73 offset:24576\n\t" \
  "ds_read_b64_tr_b8 v[110:111], v73 offset:25600\n\t" \
  "v_mfma_f32_16x16x128_f8f6f4 v[248:251], v[120:127], a[16:23], v[248:251]\n\t" \
  "v_mfma_f32_16x16x128_f8f6f4 v[252:255], v[120:127], a[24:31], v[252:255]\n\t" \
  "s_waitcnt vmcnt(37)\n\t" \
  "s_bitcmp1_b32 s65, 21\n\t" \
  "s_cbranch_scc1 OK42_%=\n\t" \
  "s_mov_b32 s67, 21\n\t" \
  "s_call_b64 s[68:69], POLL_%=\n\t" \
  "OK42_%=:\n\t" \
  "s_waitcnt lgkmcnt(4)\n\t" \
  "v_mfma_f32_16x16x128_f8f6f4 v[128:131], v[96:103], a[32:39], v[128:131]\n\t" \
  "ds_read_b64_tr_b8 v[112:113], v74 offset:16384\n\t" \
  "ds_read_b64_tr_b8 v[114:115], v74 offset:17408\n\t" \
  "v_mfma_f32_16x16x128_f8f6f4 v[132:135], v[96:103], a[40:47], v[132:135]\n\t" \
  "ds_read_b64_tr_b8 v[116:117], v74 offset:24576\n\t" \
  "ds_read_b64_tr_b8 v[118:119], v74 offset:25600\n\t" \
  "v_mfma_f32_16x16x128_f8f6f4 v[136:139], v[96:103], a[48:55], v[136:139]\n\t" \
  "v_mfma_f32_16x16x128_f8f6f4 v[140:143], v[96:103], a[56:63], v[140:143]\n\t" \
  "s_waitcnt lgkmcnt(4)\n\t" \
  "v_mfma_f32_16x16x128_f8f6f4 v[144:147], v[104:111], a[32:39], v[144:147]\n\t" \
  "ds_read_b64_tr_b8 v[120:121], v75 offset:16384\n\t" \
  "ds_read_b64_tr_b8 v[122:123], v75 offset:17408\n\t" \
  "v_mfma_f32_16x16x128_f8f6f4 v[148:151], v[104:111], a[40:47], v[148:151]\n\t" \
  "ds_read_b64_tr_b8 v[124:125], v75 offset:24576\n\t" \
  "ds_read_b64_tr_b8 v[126:127], v75 offset:25600\n\t" \
  "v_mfma_f32_16x16x128_f8f6f4 v[152:155], v[104:111], a[48:55], v[152:155]\n\t" \
  "v_mfma_f32_16x16x128_f8f6f4 v[156:159], v[104:111], a[56:63], v[156:159]\n\t" \
  "s_waitcnt lgkmcnt(4)\n\t" \
  "v_mfma_f32_16x16x128_f8f6f4 v[160:163], v[112:119], a[32:39], v[160:163]\n\t" \
  "ds_read_b64_tr_b8 v[96:97], v76 offset:16384\n\t" \
  "ds_read_b64_tr_b8 v[98:99], v76 offset:17408\n\t" \
  "v_mfma_f32_16x16x128_f8f6f4 v[164:167], v[112:119], a[40:47], v[164:167]\n\t" \
  "ds_read_b64_tr_b8 v[100:101], v76 offset:24576\n\t" \
  "ds_read_b64_tr_b8 v[102:103], v76 offset:25600\n\t" \
  "v_mfma_f32_16x16x128_f8f6f4 v[168:171], v[112:119], a[48:55], v[168:171]\n\t" \
  "v_mfma_f32_16x16x128_f8f6f4 v[172:175], v[112:119], a[56:63], v[172:175]\n\t" \
  "s_waitcnt lgkmcnt(4)\n\t" \
  "v_mfma_f32_16x16x128_f8f6f4 v[176:179], v[120:127], a[32:39], v[176:179]\n\t" \
  "ds_read_b64_tr_b8 v[104:105], v77 offset:16384\n\t" \
  "ds_read_b64_tr_b8 v[106:107], v77 offset:17408\n\t" \
  "v_mfma_f32_16x16x128_f8f6f4 v[180:183], v[120:127], a[40:47], v[180:183]\n\t" \
  "ds_read_b64_tr_b8 v[108:109], v77 offset:24576\n\t" \
  "ds_read_b64_tr_b8 v[110:111], v77 offset:25600\n\t" \
  "v_mfma_f32_16x16x128_f8f6f4 v[184:187], v[120:127], a[48:55], v[184:187]\n\t" \
  "v_mfma_f32_16x16x128_f8f6f4 v[188:191], v[120:127], a[56:63], v[188:191]\n\t" \
  "s_waitcnt lgkmcnt(4)\n\t" \
  "v_mfma_f32_16x16x128_f8f6f4 v[192:195], v[96:103], a[32:39], v[192:195]\n\t" \
  "ds_read_b64_tr_b8 v[112:113], v78 offset:16384\n\t" \
  "ds_read_b64_tr_b8 v[114:115], v78 offset:17408\n\t" \
  "v_mfma_f32_16x16x128_f8f6f4 v[196:199], v[96:103], a[40:47], v[196:199]\n\t" \
  "ds_read_b64_tr_b8 v[116:117], v78 offset:24576\n\t" \
  "ds_read_b64_tr_b8 v[118:119], v78 offset:25600\n\t" \
  "v_mfma_f32_16x16x128_f8f6f4 v[200:203], v[96:103], a[48:55], v[200:203]\n\t" \
  "v_mfma_f32_16x16x128_f8f6f4 v[204:207], v[96:103], a[56:63], v[204:207]\n\t" \
  "s_waitcnt lgkmcnt(4)\n\t" \
  "v_mfma_f32_16x16x128_f8f6f4 v[208:211], v[104:111], a[32:39], v[208:211]\n\t" \
  "ds_read_b64_tr_b8 v[120:121], v79 offset:16384\n\t" \
  "ds_read_b64_tr_b8 v[122:123], v79 offset:17408\n\t" \
  "v_mfma_f32_16x16x128_f8f6f4 v[212:215], v[104:111], a[40:47], v[212:215]\n\t" \
  "ds_read_b64_tr_b8 v[124:125], v79 offset:24576\n\t" \
  "ds_read_b64_tr_b8 v[126:127], v79 offset:25600\n\t" \
  "v_mfma_f32_16x16x128_f8f6f4 v[216:219], v[104:111], a[48:55], v[216:219]\n\t" \
  "v_mfma_f32_16x16x128_f8f6f4 v[220:223], v[104:111], a[56:63], v[220:223]\n\t" \
  "s_waitcnt vmcnt(15)\n\t" \
  "s_barrier\n\t" \
  "s_waitcnt lgkmcnt(4)\n\t" \
  "v_mfma_f32_16x16x128_f8f6f4 v[224:227], v[112:119], a[32:39], v[224:227]\n\t" \
  "ds_read_b64_tr_b8 v[96:97], v72 offset:32768\n\t" \
  "ds_read_b64_tr_b8 v[98:99], v72 offset:33792\n\t" \
  "v_mfma_f32_16x16x128_f8f6f4 v[228:231], v[112:119], a[40:47], v[228:231]\n\t" \
  "ds_read_b64_tr_b8 v[100:101], v72 offset:40960\n\t" \
  "ds_read_b64_tr_b8 v[102:103], v72 offset:41984\n\t" \
  "buffer_load_dwordx4 a[132:135], v84, s[44:47], s89 offen offset:1024 sc0 sc1\n\t" \
  "v_mfma_f32_16x16x128_f8f6f4 v[232:235], v[112:119], a[48:55], v[232:235]\n\t" \
  "buffer_load_dwordx4 a[136:139], v84, s[44:47], s89 offen offset:2048 sc0 sc1\n\t" \
  "v_mfma_f32_16x16x128_f8f6f4 v[236:239], v[112:119], a[56:63], v[236:239]\n\t" \
  "s_waitcnt lgkmcnt(4)\n\t" \
  "v_mfma_f32_16x16x128_f8f6f4 v[240:243], v[120:127], a[32:39], v[240:243]\n\t" \
  "ds_read_b64_tr_b8 v[104:105], v73 offset:32768\n\t" \
  "ds_read_b64_tr_b8 v[106:107], v73 offset:33792\n\t" \
  "v_mfma_f32_16x16x128_f8f6f4 v[244:247], v[120:127], a[40:47], v[244:247]\n\t" \
  "ds_read_b64_tr_b8 v[108:109], v73 offset:40960\n\t" \
  "ds_read_b64_tr_b8 v[110:111], v73 offset:41984\n\t" \
  "buffer_load_dwordx4 a[140:143], v84, s[44:47], s89 offen offset:3072 sc0 sc1\n\t" \
  "v_mfma_f32_16x16x128_f8f6f4 v[248:251], v[120:127], a[48:55], v[248:251]\n\t" \
  "buffer_load_dwordx4 a[144:147], v84, s[44:47], s90 offen offset:0 sc0 sc1\n\t" \
  "v_mfma_f32_16x16x128_f8f6f4 v[252:255], v[120:127], a[56:63], v[252:255]\n\t" \
  "s_waitcnt vmcnt(33)\n\t" \
  "global_load_dword v92, v87, s[70:71] sc0 sc1\n\t" \
  "s_bitcmp1_b32 s65, 22\n\t" \
  "s_cbranch_scc1 OK43_%=\n\t" \
  "s_mov_b32 s67, 22\n\t" \
  "s_call_b64 s[68:69], POLL_%=\n\t" \
  "OK43_%=:\n\t" \
  "s_waitcnt lgkmcnt(4)\n\t" \
  "v_mfma_f32_16x16x128_f8f6f4 v[128:131], v[96:103], a[64:71], v[128:131]\n\t" \
  "ds_read_b64_tr_b8 v[112:113], v74 offset:32768\n\t" \
  "ds_read_b64_tr_b8 v[114:115], v74 offset:33792\n\t" \
  "v_mfma_f32_16x16x128_f8f6f4 v[132:135], v[96:103], a[72:79], v[132:135]\n\t" \
  "ds_read_b64_tr_b8 v[116:117], v74 offset:40960\n\t" \
  "ds_read_b64_tr_b8 v[118:119], v74 offset:41984\n\t" \
  "buffer_load_dwordx4 a[148:151], v84, s[44:47], s90 offen offset:1024 sc0 sc1\n\t" \
  "v_mfma_f32_16x16x128_f8f6f4 v[136:139], v[96:103], a[80:87], v[136:139]\n\t" \
  "buffer_load_dwordx4 a[152:155], v84, s[44:47], s90 offen offset:2048 sc0 sc1\n\t" \
  "v_mfma_f32_16x16x128_f8f6f4 v[140:143], v[96:103], a[88:95], v[140:143]\n\t" \
  "s_waitcnt lgkmcnt(4)\n\t" \
  "v_mfma_f32_16x16x128_f8f6f4 v[144:147], v[104:111], a[64:71], v[144:147]\n\t" \
  "ds_read_b64_tr_b8 v[120:121], v75 offset:32768\n\t" \
  "ds_read_b64_tr_b8 v[122:123], v75 offset:33792\n\t" \
  "v_mfma_f32_16x16x128_f8f6f4 v[148:151], v[104:111], a[72:79], v[148:151]\n\t" \
  "ds_read_b64_tr_b8 v[124:125], v75 offset:40960\n\t" \
  "ds_read_b64_tr_b8 v[126:127], v75 offset:41984\n\t" \
  "buffer_load_dwordx4 a[156:159], v84, s[44:47], s90 offen offset:3072 sc0 sc1\n\t" \
  "v_mfma_f32_16x16x128_f8f6f4 v[152:155], v[104:111], a[80:87], v[152:155]\n\t" \
  "s_add_u32 s89, s62, 1376256\n\t" \
  "s_add_u32 s90, s62, 1380352\n\t" \
  "buffer_load_dwordx4 a[160:163], v84, s[44:47], s89 offen offset:0 sc0 sc1\n\t" \
  "v_mfma_f32_16x16x128_f8f6f4 v[156:159], v[104:111], a[88:95], v[156:159]\n\t" \
  "s_waitcnt lgkmcnt(4)\n\t" \
  "v_mfma_f32_16x16x128_f8f6f4 v[160:163], v[112:119], a[64:71], v[160:163]\n\t" \
  "ds_read_b64_tr_b8 v[96:97], v76 offset:32768\n\t" \
  "ds_read_b64_tr_b8 v[98:99], v76 offset:33792\n\t" \
  "v_mfma_f32_16x16x128_f8f6f4 v[164:167], v[112:119], a[72:79], v[164:167]\n\t" \
  "ds_read_b64_tr_b8 v[100:101], v76 offset:40960\n\t" \
  "ds_read_b64_tr_b8 v[102:103], v76 offset:41984\n\t" \
  "buffer_load_dwordx4 a[164:167], v84, s[44:47], s89 offen offset:1024 sc0 sc1\n\t" \
  "v_mfma_f32_16x16x128_f8f6f4 v[168:171], v[112:119], a[80:87], v[168:171]\n\t" \
  "buffer_load_dwordx4 a[168:171], v84, s[44:47], s89 offen offset:2048 sc0 sc1\n\t" \
  "v_mfma_f32_16x16x128_f8f6f4 v[172:175], v[112:119], a[88:95], v[172:175]\n\t" \
  "s_waitcnt lgkmcnt(4)\n\t" \
  "v_mfma_f32_16x16x128_f8f6f4 v[176:179], v[120:127], a[64:71], v[176:179]\n\t" \
  "ds_read_b64_tr_b8 v[104:105], v77 offset:32768\n\t" \
  "ds_read_b64_tr_b8 v[106:107], v77 offset:33792\n\t" \
  "v_mfma_f32_16x16x128_f8f6f4 v[180:183], v[120:127], a[72:79], v[180:183]\n\t" \
  "ds_read_b64_tr_b8 v[108:109], v77 offset:40960\n\t" \
  "ds_read_b64_tr_b8 v[110:111], v77 offset:41984\n\t" \
  "buffer_load_dwordx4 a[172:175], v84, s[44:47], s89 offen offset:3072 sc0 sc1\n\t" \
  "v_mfma_f32_16x16x128_f8f6f4 v[184:187], v[120:127], a[80:87], v[184:187]\n\t" \
  "buffer_load_dwordx4 a[176:179], v84, s[44:47], s90 offen offset:0 sc0 sc1\n\t" \
  "v_mfma_f32_16x16x128_f8f6f4 v[188:191], v[120:127], a[88:95], v[188:191]\n\t" \
  "s_waitcnt lgkmcnt(4)\n\t" \
  "v_mfma_f32_16x16x128_f8f6f4 v[192:195], v[96:103], a[64:71], v[192:195]\n\t" \
  "ds_read_b64_tr_b8 v[112:113], v78 offset:32768\n\t" \
  "ds_read_b64_tr_b8 v[114:115], v78 offset:33792\n\t" \
  "v_mfma_f32_16x16x128_f8f6f4 v[196:199], v[96:103], a[72:79], v[196:199]\n\t" \
  "ds_read_b64_tr_b8 v[116:117], v78 offset:40960\n\t" \
  "ds_read_b64_tr_b8 v[118:119], v78 offset:41984\n\t" \
  "buffer_load_dwordx4 a[180:183], v84, s[44:47], s90 offen offset:1024 sc0 sc1\n\t" \
  "v_mfma_f32_16x16x128_f8f6f4 v[200:203], v[96:103], a[80:87], v[200:203]\n\t" \
  "buffer_load_dwordx4 a[184:187], v84, s[44:47], s90 offen offset:2048 sc0 sc1\n\t" \
  "v_mfma_f32_16x16x128_f8f6f4 v[204:207], v[96:103], a[88:95], v[204:207]\n\t" \
  "s_waitcnt lgkmcnt(4)\n\t" \
  "v_mfma_f32_16x16x128_f8f6f4 v[208:211], v[104:111], a[64:71], v[208:211]\n\t" \
  "ds_read_b64_tr_b8 v[120:121], v79 offset:32768\n\t" \
  "ds_read_b64_tr_b8 v[122:123], v79 offset:33792\n\t" \
  "v_mfma_f32_16x16x128_f8f6f4 v[212:215], v[104:111], a[72:79], v[212:215]\n\t" \
  "ds_read_b64_tr_b8 v[124:125], v79 offset:40960\n\t" \
  "ds_read_b64_tr_b8 v[126:127], v79 offset:41984\n\t" \
  "buffer_load_dwordx4 a[188:191], v84, s[44:47], s90 offen offset:3072 sc0 sc1\n\t" \
  "v_mfma_f32_16x16x128_f8f6f4 v[216:219], v[104:111], a[80:87], v[216:219]\n\t" \
  "s_add_u32 s89, s62, 1441792\n\t" \
  "s_add_u32 s90, s62, 1445888\n\t" \
  "buffer_load_dwordx4 a[192:195], v84, s[44:47], s89 offen offset:0 sc0 sc1\n\t" \
  "v_mfma_f32_16x16x128_f8f6f4 v[220:223], v[104:111], a[88:95], v[220:223]\n\t" \
  "s_waitcnt vmcnt(28)\n\t" \
  "s_barrier\n\t" \
  "s_waitcnt lgkmcnt(4)\n\t" \
  "v_mfma_f32_16x16x128_f8f6f4 v[224:227], v[112:119], a[64:71], v[224:227]\n\t" \
  "ds_read_b64_tr_b8 v[96:97], v72 offset:49152\n\t" \
  "ds_read_b64_tr_b8 v[98:99], v72 offset:50176\n\t" \
  "v_mfma_f32_16x16x128_f8f6f4 v[228:231], v[112:119], a[72:79], v[228:231]\n\t" \
  "ds_read_b64_tr_b8 v[100:101], v72 offset:57344\n\t" \
  "ds_read_b64_tr_b8 v[102:103], v72 offset:58368\n\t" \
  "buffer_load_dwordx4 a[196:199], v84, s[44:47], s89 offen offset:1024 sc0 sc1\n\t" \
  "v_mfma_f32_16x16x128_f8f6f4 v[232:235], v[112:119], a[80:87], v[232:235]\n\t" \
  "buffer_load_dwordx4 a[200:203], v84, s[44:47], s89 offen offset:2048 sc0 sc1\n\t" \
  "v_mfma_f32_16x16x128_f8f6f4 v[236:239], v[112:119], a[88:95], v[236:239]\n\t" \
  "s_waitcnt lgkmcnt(4)\n\t" \
  "v_mfma_f32_16x16x128_f8f6f4 v[240:243], v[120:127], a[64:71], v[240:243]\n\t" \
  "ds_read_b64_tr_b8 v[104:105], v73 offset:49152\n\t" \
  "ds_read_b64_tr_b8 v[106:107], v73 offset:50176\n\t" \
  "v_mfma_f32_16x16x128_f8f6f4 v[244:247], v[120:127], a[72:79], v[244:247]\n\t" \
  "ds_read_b64_tr_b8 v[108:109], v73 offset:57344\n\t" \
  "ds_read_b64_tr_b8 v[110:111], v73 offset:58368\n\t" \
  "buffer_load_dwordx4 a[204:207], v84, s[44:47], s89 offen offset:3072 sc0 sc1\n\t" \
  "v_mfma_f32_16x16x128_f8f6f4 v[248:251], v[120:127], a[80:87], v[248:251]\n\t" \
  "buffer_load_dwordx4 a[208:211], v84, s[44:47], s90 offen offset:0 sc0 sc1\n\t" \
  "v_mfma_f32_16x16x128_f8f6f4 v[252:255], v[120:127], a[88:95], v[252:255]\n\t" \
  "s_waitcnt vmcnt(16)\n\t" \
  "v_cmp_le_u32 vcc, s64, v92\n\t" \
  "s_nop 1\n\t" \
  "s_or_b32 s65, s65, vcc_lo\n\t" \
  "s_bitcmp1_b32 s65, 23\n\t" \
  "s_cbranch_scc1 OK44_%=\n\t" \
  "s_mov_b32 s67, 23\n\t" \
  "s_call_b64 s[68:69], POLL_%=\n\t" \
  "OK44_%=:\n\t" \
  "s_waitcnt lgkmcnt(4)\n\t" \
  "v_mfma_f32_16x16x128_f8f6f4 v[128:131], v[96:103], a[96:103], v[128:131]\n\t" \
  "ds_read_b64_tr_b8 v[112:113], v74 offset:49152\n\t" \
  "ds_read_b64_tr_b8 v[114:115], v74 offset:50176\n\t" \
  "v_mfma_f32_16x16x128_f8f6f4 v[132:135], v[96:103], a[104:111], v[132:135]\n\t" \
  "ds_read_b64_tr_b8 v[116:117], v74 offset:57344\n\t" \
  "ds_read_b64_tr_b8 v[118:119], v74 offset:58368\n\t" \
  "buffer_load_dwordx4 a[212:215], v84, s[44:47], s90 offen offset:1024 sc0 sc1\n\t" \
  "v_mfma_f32_16x16x128_f8f6f4 v[136:139], v[96:103], a[112:119], v[136:139]\n\t" \
  "buffer_load_dwordx4 a[216:219], v84, s[44:47], s90 offen offset:2048 sc0 sc1\n\t" \
  "v_mfma_f32_16x16x128_f8f6f4 v[140:143], v[96:103], a[120:127], v[140:143]\n\t" \
  "s_waitcnt lgkmcnt(4)\n\t" \
  "v_mfma_f32_16x16x128_f8f6f4 v[144:147], v[104:111], a[96:103], v[144:147]\n\t" \
  "ds_read_b64_tr_b8 v[120:121], v75 offset:49152\n\t" \
  "ds_read_b64_tr_b8 v[122:123], v75 offset:50176\n\t" \
  "v_mfma_f32_16x16x128_f8f6f4 v[148:151], v[104:111], a[104:111], v[148:151]\n\t" \
  "ds_read_b64_tr_b8 v[124:125], v75 offset:57344\n\t" \
  "ds_read_b64_tr_b8 v[126:127], v75 offset:58368\n\t" \
  "buffer_load_dwordx4 a[220:223], v84, s[44:47], s90 offen offset:3072 sc0 sc1\n\t" \
  "v_mfma_f32_16x16x128_f8f6f4 v[152:155], v[104:111], a[112:119], v[152:155]\n\t" \
  "s_add_u32 s89, s62, 1507328\n\t" \
  "s_add_u32 s90, s62, 1511424\n\t" \
  "buffer_load_dwordx4 a[224:227], v84, s[44:47], s89 offen offset:0 sc0 sc1\n\t" \
  "v_mfma_f32_16x16x128_f8f6f4 v[156:159], v[104:111], a[120:127], v[156:159]\n\t" \
  "s_waitcnt lgkmcnt(4)\n\t" \
  "v_mfma_f32_16x16x128_f8f6f4 v[160:163], v[112:119], a[96:103], v[160:163]\n\t" \
  "ds_read_b64_tr_b8 v[96:97], v76 offset:49152\n\t" \
  "ds_read_b64_tr_b8 v[98:99], v76 offset:50176\n\t" \
  "v_mfma_f32_16x16x128_f8f6f4 v[164:167], v[112:119], a[104:111], v[164:167]\n\t" \
  "ds_read_b64_tr_b8 v[100:101], v76 offset:57344\n\t" \
  "ds_read_b64_tr_b8 v[102:103], v76 offset:58368\n\t" \
  "buffer_load_dwordx4 a[228:231], v84, s[44:47], s89 offen offset:1024 sc0 sc1\n\t" \
  "v_mfma_f32_16x16x128_f8f6f4 v[168:171], v[112:119], a[112:119], v[168:171]\n\t" \
  "buffer_load_dwordx4 a[232:235], v84, s[44:47], s89 offen offset:2048 sc0 sc1\n\t" \
  "v_mfma_f32_16x16x128_f8f6f4 v[172:175], v[112:119], a[120:127], v[172:175]\n\t" \
  "s_waitcnt lgkmcnt(4)\n\t" \
  "v_mfma_f32_16x16x128_f8f6f4 v[176:179], v[120:127], a[96:103], v[176:179]\n\t" \
  "ds_read_b64_tr_b8 v[104:105], v77 offset:49152\n\t" \
  "ds_read_b64_tr_b8 v[106:107], v77 offset:50176\n\t" \
  "v_mfma_f32_16x16x128_f8f6f4 v[180:183], v[120:127], a[104:111], v[180:183]\n\t" \
  "ds_read_b64_tr_b8 v[108:109], v77 offset:57344\n\t" \
  "ds_read_b64_tr_b8 v[110:111], v77 offset:58368\n\t" \
  "buffer_load_dwordx4 a[236:239], v84, s[44:47], s89 offen offset:3072 sc0 sc1\n\t" \
  "v_mfma_f32_16x16x128_f8f6f4 v[184:187], v[120:127], a[112:119], v[184:187]\n\t" \
  "buffer_load_dwordx4 a[240:243], v84, s[44:47], s90 offen offset:0 sc0 sc1\n\t" \
  "v_mfma_f32_16x16x128_f8f6f4 v[188:191], v[120:127], a[120:127], v[188:191]\n\t" \
  "s_waitcnt lgkmcnt(4)\n\t" \
  "v_mfma_f32_16x16x128_f8f6f4 v[192:195], v[96:103], a[96:103], v[192:195]\n\t" \
  "ds_read_b64_tr_b8 v[112:113], v78 offset:49152\n\t" \
  "ds_read_b64_tr_b8 v[114:115], v78 offset:50176\n\t" \
  "v_mfma_f32_16x16x128_f8f6f4 v[196:199], v[96:103], a[104:111], v[196:199]\n\t" \
  "ds_read_b64_tr_b8 v[116:117], v78 offset:57344\n\t" \
  "ds_read_b64_tr_b8 v[118:119], v78 offset:58368\n\t" \
  "s_add_u32 s88, s56, 4194304\n\t" \
  "s_add_u32 m0, s60, 0\n\t" \
  "s_nop 0\n\t" \
  "buffer_load_dwordx4 v80, s[36:39], s88 offen  lds\n\t" \
  "v_mfma_f32_16x16x128_f8f6f4 v[200:203], v[96:103], a[112:119], v[200:203]\n\t" \
  "s_add_u32 m0, s60, 1024\n\t" \
  "s_nop 0\n\t" \
  "buffer_load_dwordx4 v81, s[36:39], s88 offen  lds\n\t" \
  "v_mfma_f32_16x16x128_f8f6f4 v[204:207], v[96:103], a[120:127], v[204:207]\n\t" \
  "s_waitcnt lgkmcnt(4)\n\t" \
  "v_mfma_f32_16x16x128_f8f6f4 v[208:211], v[104:111], a[96:103], v[208:211]\n\t" \
  "ds_read_b64_tr_b8 v[120:121], v79 offset:49152\n\t" \
  "ds_read_b64_tr_b8 v[122:123], v79 offset:50176\n\t" \
  "v_mfma_f32_16x16x128_f8f6f4 v[212:215], v[104:111], a[104:111], v[212:215]\n\t" \
  "ds_read_b64_tr_b8 v[124:125], v79 offset:57344\n\t" \
  "ds_read_b64_tr_b8 v[126:127], v79 offset:58368\n\t" \
  "s_add_u32 m0, s60, 2048\n\t" \
  "s_nop 0\n\t" \
  "buffer_load_dwordx4 v82, s[36:39], s88 offen  lds\n\t" \
  "v_mfma_f32_16x16x128_f8f6f4 v[216:219], v[104:111], a[112:119], v[216:219]\n\t" \
  "s_add_u32 m0, s60, 3072\n\t" \
  "s_nop 0\n\t" \
  "buffer_load_dwordx4 v83, s[36:39], s88 offen  lds\n\t" \
  "v_mfma_f32_16x16x128_f8f6f4 v[220:223], v[104:111], a[120:127], v[220:223]\n\t" \
  "s_waitcnt vmcnt(40)\n\t" \
  "s_barrier\n\t" \
  "s_waitcnt lgkmcnt(4)\n\t" \
  "v_mfma_f32_16x16x128_f8f6f4 v[224:227], v[112:119], a[96:103], v[224:227]\n\t" \
  "ds_read_b64_tr_b8 v[96:97], v16 offset:0\n\t" \
  "ds_read_b64_tr_b8 v[98:99], v16 offset:1024\n\t" \
  "v_mfma_f32_16x16x128_f8f6f4 v[228:231], v[112:119], a[104:111], v[228:231]\n\t" \
  "ds_read_b64_tr_b8 v[100:101], v16 offset:8192\n\t" \
  "ds_read_b64_tr_b8 v[102:103], v16 offset:9216\n\t" \
  "s_add_u32 s88, s57, 4194304\n\t" \
  "s_add_u32 m0, s60, 16384\n\t" \
  "s_nop 0\n\t" \
  "buffer_load_dwordx4 v80, s[36:39], s88 offen  lds\n\t" \
  "v_mfma_f32_16x16x128_f8f6f4 v[232:235], v[112:119], a[112:119], v[232:235]\n\t" \
  "s_add_u32 m0, s60, 17408\n\t" \
  "s_nop 0\n\t" \
  "buffer_load_dwordx4 v81, s[36:39], s88 offen  lds\n\t" \
  "v_mfma_f32_16x16x128_f8f6f4 v[236:239], v[112:119], a[120:127], v[236:239]\n\t" \
  "s_waitcnt lgkmcnt(4)\n\t" \
  "v_mfma_f32_16x16x128_f8f6f4 v[240:243], v[120:127], a[96:103], v[240:243]\n\t" \
  "ds_read_b64_tr_b8 v[104:105], v17 offset:0\n\t" \
  "ds_read_b64_tr_b8 v[106:107], v17 offset:1024\n\t" \
  "v_mfma_f32_16x16x128_f8f6f4 v[244:247], v[120:127], a[104:111], v[244:247]\n\t" \
  "ds_read_b64_tr_b8 v[108:109], v17 offset:8192\n\t" \
  "ds_read_b64_tr_b8 v[110:111], v17 offset:9216\n\t" \
  "s_add_u32 m0, s60, 18432\n\t" \
  "s_nop 0\n\t" \
  "buffer_load_dwordx4 v82, s[36:39], s88 offen  lds\n\t" \
  "v_mfma_f32_16x16x128_f8f6f4 v[248:251], v[120:127], a[112:119], v[248:251]\n\t" \
  "s_add_u32 m0, s60, 19456\n\t" \
  "s_nop 0\n\t" \
  "buffer_load_dwordx4 v83, s[36:39], s88 offen  lds\n\t" \
  "v_mfma_f32_16x16x128_f8f6f4 v[252:255], v[120:127], a[120:127], v[252:255]\n\t" \
  "s_waitcnt vmcnt(29)\n\t" \
  "s_bitcmp1_b32 s65, 24\n\t" \
  "s_cbranch_scc1 OK45_%=\n\t" \
  "s_mov_b32 s67, 24\n\t" \
  "s_call_b64 s[68:69], POLL_%=\n\t" \
  "OK45_%=:\n\t" \
  "s_waitcnt lgkmcnt(4)\n\t" \
  "v_mfma_f32_16x16x128_f8f6f4 v[128:131], v[96:103], a[128:135], v[128:131]\n\t" \
  "ds_read_b64_tr_b8 v[112:113], v18 offset:0\n\t" \
  "ds_read_b64_tr_b8 v[114:115], v18 offset:1024\n\t" \
  "s_bfe_u32 s96, s65, 262169\n\t" \
  "s_cmp_lg_u32 s96, 15\n\t" \
  "s_cbranch_scc1 XWS46_%=\n\t" \
  "s_add_u32 s96, s62, 1638400\n\t" \
  "buffer_load_dword v8, v13, s[44:47], s96 offen sc0 sc1\n\t" \
  "XWS46_%=:\n\t" \
  "v_mfma_f32_16x16x128_f8f6f4 v[132:135], v[96:103], a[136:143], v[132:135]\n\t" \
  "ds_read_b64_tr_b8 v[116:117], v18 offset:8192\n\t" \
  "ds_read_b64_tr_b8 v[118:119], v18 offset:9216\n\t" \
  "s_add_u32 s88, s58, 4194304\n\t" \
  "s_add_u32 m0, s60, 32768\n\t" \
  "s_nop 0\n\t" \
  "buffer_load_dwordx4 v80, s[36:39], s88 offen  lds\n\t" \
  "v_mfma_f32_16x16x128_f8f6f4 v[136:139], v[96:103], a[144:151], v[136:139]\n\t" \
  "s_add_u32 m0, s60, 33792\n\t" \
  "s_nop 0\n\t" \
  "buffer_load_dwordx4 v81, s[36:39], s88 offen  lds\n\t" \
  "v_mfma_f32_16x16x128_f8f6f4 v[140:143], v[96:103], a[152:159], v[140:143]\n\t" \
  "s_add_u32 m0, s60, 34816\n\t" \
  "s_nop 0\n\t" \
  "buffer_load_dwordx4 v82, s[36:39], s88 offen  lds\n\t" \
  "s_waitcnt lgkmcnt(4)\n\t" \
  "v_mfma_f32_16x16x128_f8f6f4 v[144:147], v[104:111], a[128:135], v[144:147]\n\t" \
  "ds_read_b64_tr_b8 v[120:121], v19 offset:0\n\t" \
  "ds_read_b64_tr_b8 v[122:123], v19 offset:1024\n\t" \
  "v_mfma_f32_16x16x128_f8f6f4 v[148:151], v[104:111], a[136:143], v[148:151]\n\t" \
  "ds_read_b64_tr_b8 v[124:125], v19 offset:8192\n\t" \
  "ds_read_b64_tr_b8 v[126:127], v19 offset:9216\n\t" \
  "s_add_u32 m0, s60, 35840\n\t" \
  "s_nop 0\n\t" \
  "buffer_load_dwordx4 v83, s[36:39], s88 offen  lds\n\t" \
  "v_mfma_f32_16x16x128_f8f6f4 v[152:155], v[104:111], a[144:151], v[152:155]\n\t" \
  "s_add_u32 s88, s52, 6291456\n\t" \
  "s_add_u32 m0, s60, 65536\n\t" \
  "s_nop 0\n\t" \
  "buffer_load_dwordx4 v80, s[36:39], s88 offen  lds\n\t" \
  "v_mfma_f32_16x16x128_f8f6f4 v[156:159], v[104:111], a[152:159], v[156:159]\n\t" \
  "s_add_u32 m0, s60, 66560\n\t" \
  "s_nop 0\n\t" \
  "buffer_load_dwordx4 v81, s[36:39], s88 offen  lds\n\t" \
  "s_waitcnt lgkmcnt(4)\n\t" \
  "v_mfma_f32_16x16x128_f8f6f4 v[160:163], v[112:119], a[128:135], v[160:163]\n\t" \
  "ds_read_b64_tr_b8 v[96:97], v20 offset:0\n\t" \
  "ds_read_b64_tr_b8 v[98:99], v20 offset:1024\n\t" \
  "v_mfma_f32_16x16x128_f8f6f4 v[164:167], v[112:119], a[136:143], v[164:167]\n\t" \
  "ds_read_b64_tr_b8 v[100:101], v20 offset:8192\n\t" \
  "ds_read_b64_tr_b8 v[102:103], v20 offset:9216\n\t" \
  "s_add_u32 m0, s60, 67584\n\t" \
  "s_nop 0\n\t" \
  "buffer_load_dwordx4 v82, s[36:39], s88 offen  lds\n\t" \
  "v_mfma_f32_16x16x128_f8f6f4 v[168:171], v[112:119], a[144:151], v[168:171]\n\t" \
  "s_add_u32 m0, s60, 68608\n\t" \
  "s_nop 0\n\t" \
  "buffer_load_dwordx4 v83, s[36:39], s88 offen  lds\n\t" \
  "v_mfma_f32_16x16x128_f8f6f4 v[172:175], v[112:119], a[152:159], v[172:175]\n\t" \
  "s_add_u32 s88, s53, 6291456\n\t" \
  "s_add_u32 m0, s60, 81920\n\t" \
  "s_nop 0\n\t" \
  "buffer_load_dwordx4 v80, s[36:39], s88 offen  lds\n\t" \
  "s_waitcnt lgkmcnt(4)\n\t" \
  "v_mfma_f32_16x16x128_f8f6f4 v[176:179], v[120:127], a[128:135], v[176:179]\n\t" \
  "ds_read_b64_tr_b8 v[104:105], v21 offset:0\n\t" \
  "ds_read_b64_tr_b8 v[106:107], v21 offset:1024\n\t" \
  "v_mfma_f32_16x16x128_f8f6f4 v[180:183], v[120:127], a[136:143], v[180:183]\n\t" \
  "ds_read_b64_tr_b8 v[108:109], v21 offset:8192\n\t" \
  "ds_read_b64_tr_b8 v[110:111], v21 offset:9216\n\t" \
  "s_add_u32 m0, s60, 82944\n\t" \
  "s_nop 0\n\t" \
  "buffer_load_dwordx4 v81, s[36:39], s88 offen  lds\n\t" \
  "v_mfma_f32_16x16x128_f8f6f4 v[184:187], v[120:127], a[144:151], v[184:187]\n\t" \
  "s_add_u32 m0, s60, 83968\n\t" \
  "s_nop 0\n\t" \
  "buffer_load_dwordx4 v82, s[36:39], s88 offen  lds\n\t" \
  "v_mfma_f32_16x16x128_f8f6f4 v[188:191], v[120:127], a[152:159], v[188:191]\n\t" \
  "s_add_u32 m0, s60, 84992\n\t" \
  "s_nop 0\n\t" \
  "buffer_load_dwordx4 v83, s[36:39], s88 offen  lds\n\t" \
  "s_waitcnt lgkmcnt(4)\n\t" \
  "v_mfma_f32_16x16x128_f8f6f4 v[192:195], v[96:103], a[128:135], v[192:195]\n\t" \
  "ds_read_b64_tr_b8 v[112:113], v22 offset:0\n\t" \
  "ds_read_b64_tr_b8 v[114:115], v22 offset:1024\n\t" \
  "v_mfma_f32_16x16x128_f8f6f4 v[196:199], v[96:103], a[136:143], v[196:199]\n\t" \
  "ds_read_b64_tr_b8 v[116:117], v22 offset:8192\n\t" \
  "ds_read_b64_tr_b8 v[118:119], v22 offset:9216\n\t" \
  "s_add_u32 s88, s54, 6291456\n\t" \
  "s_add_u32 m0, s60, 98304\n\t" \
  "s_nop 0\n\t" \
  "buffer_load_dwordx4 v80, s[36:39], s88 offen  lds\n\t" \
  "v_mfma_f32_16x16x128_f8f6f4 v[200:203], v[96:103], a[144:151], v[200:203]\n\t" \
  "s_add_u32 m0, s60, 99328\n\t" \
  "s_nop 0\n\t" \
  "buffer_load_dwordx4 v81, s[36:39], s88 offen  lds\n\t" \
  "v_mfma_f32_16x16x128_f8f6f4 v[204:207], v[96:103], a[152:159], v[204:207]\n\t" \
  "s_add_u32 m0, s60, 100352\n\t" \
  "s_nop 0\n\t" \
  "buffer_load_dwordx4 v82, s[36:39], s88 offen  lds\n\t" \
  "s_waitcnt lgkmcnt(4)\n\t" \
  "v_mfma_f32_16x16x128_f8f6f4 v[208:211], v[104:111], a[128:135], v[208:211]\n\t" \
  "ds_read_b64_tr_b8 v[120:121], v23 offset:0\n\t" \
  "ds_read_b64_tr_b8 v[122:123], v23 offset:1024\n\t" \
  "v_mfma_f32_16x16x128_f8f6f4 v[212:215], v[104:111], a[136:143], v[212:215]\n\t" \
  "ds_read_b64_tr_b8 v[124:125], v23 offset:8192\n\t" \
  "ds_read_b64_tr_b8 v[126:127], v23 offset:9216\n\t" \
  "s_add_u32 m0, s60, 101376\n\t" \
  "s_nop 0\n\t" \
  "buffer_load_dwordx4 v83, s[36:39], s88 offen  lds\n\t" \
  "v_mfma_f32_16x16x128_f8f6f4 v[216:219], v[104:111], a[144:151], v[216:219]\n\t" \
  "v_mfma_f32_16x16x128_f8f6f4 v[220:223], v[104:111], a[152:159], v[220:223]\n\t" \
  "s_waitcnt vmcnt(56)\n\t" \
  "s_barrier\n\t" \
  "s_waitcnt lgkmcnt(4)\n\t" \
  "v_mfma_f32_16x16x128_f8f6f4 v[224:227], v[112:119], a[128:135], v[224:227]\n\t" \
  "ds_read_b64_tr_b8 v[96:97], v64 offset:49152\n\t" \
  "ds_read_b64_tr_b8 v[98:99], v64 offset:50176\n\t" \
  "v_mfma_f32_16x16x128_f8f6f4 v[228:231], v[112:119], a[136:143], v[228:231]\n\t" \
  "ds_read_b64_tr_b8 v[100:101], v64 offset:57344\n\t" \
  "ds_read_b64_tr_b8 v[102:103], v64 offset:58368\n\t" \
  "v_mfma_f32_16x16x128_f8f6f4 v[232:235], v[112:119], a[144:151], v[232:235]\n\t" \
  "v_mfma_f32_16x16x128_f8f6f4 v[236:239], v[112:119], a[152:159], v[236:239]\n\t" \
  "s_waitcnt lgkmcnt(4)\n\t" \
  "v_mfma_f32_16x16x128_f8f6f4 v[240:243], v[120:127], a[128:135], v[240:243]\n\t" \
  "ds_read_b64_tr_b8 v[104:105], v65 offset:49152\n\t" \
  "ds_read_b64_tr_b8 v[106:107], v65 offset:50176\n\t" \
  "v_mfma_f32_16x16x128_f8f6f4 v[244:247], v[120:127], a[136:143], v[244:247]\n\t" \
  "ds_read_b64_tr_b8 v[108:109], v65 offset:57344\n\t" \
  "ds_read_b64_tr_b8 v[110:111], v65 offset:58368\n\t" \
  "v_mfma_f32_16x16x128_f8f6f4 v[248:251], v[120:127], a[144:151], v[248:251]\n\t" \
  "v_mfma_f32_16x16x128_f8f6f4 v[252:255], v[120:127], a[152:159], v[252:255]\n\t" \
  "s_waitcnt vmcnt(37)\n\t" \
  "s_bitcmp1_b32 s65, 25\n\t" \
  "s_cbranch_scc1 OK47_%=\n\t" \
  "s_mov_b32 s67, 25\n\t" \
  "s_call_b64 s[68:69], POLL_%=\n\t" \
  "OK47_%=:\n\t" \
  "s_waitcnt lgkmcnt(4)\n\t" \
  "v_mfma_f32_16x16x128_f8f6f4 v[128:131], v[96:103], a[160:167], v[128:131]\n\t" \
  "ds_read_b64_tr_b8 v[112:113], v66 offset:49152\n\t" \
  "ds_read_b64_tr_b8 v[114:115], v66 offset:50176\n\t" \
  "v_mfma_f32_16x16x128_f8f6f4 v[132:135], v[96:103], a[168:175], v[132:135]\n\t" \
  "ds_read_b64_tr_b8 v[116:117], v66 offset:57344\n\t" \
  "ds_read_b64_tr_b8 v[118:119], v66 offset:58368\n\t" \
  "v_mfma_f32_16x16x128_f8f6f4 v[136:139], v[96:103], a[176:183], v[136:139]\n\t" \
  "v_mfma_f32_16x16x128_f8f6f4 v[140:143], v[96:103], a[184:191], v[140:143]\n\t" \
  "s_waitcnt lgkmcnt(4)\n\t" \
  "v_mfma_f32_16x16x128_f8f6f4 v[144:147], v[104:111], a[160:167], v[144:147]\n\t" \
  "ds_read_b64_tr_b8 v[120:121], v67 offset:49152\n\t" \
  "ds_read_b64_tr_b8 v[122:123], v67 offset:50176\n\t" \
  "v_mfma_f32_16x16x128_f8f6f4 v[148:151], v[104:111], a[168:175], v[148:151]\n\t" \
  "ds_read_b64_tr_b8 v[124:125], v67 offset:57344\n\t" \
  "ds_read_b64_tr_b8 v[126:127], v67 offset:58368\n\t" \
  "v_mfma_f32_16x16x128_f8f6f4 v[152:155], v[104:111], a[176:183], v[152:155]\n\t" \
  "v_mfma_f32_16x16x128_f8f6f4 v[156:159], v[104:111], a[184:191], v[156:159]\n\t" \
  "s_waitcnt lgkmcnt(4)\n\t" \
  "v_mfma_f32_16x16x128_f8f6f4 v[160:163], v[112:119], a[160:167], v[160:163]\n\t" \
  "ds_read_b64_tr_b8 v[96:97], v68 offset:49152\n\t" \
  "ds_read_b64_tr_b8 v[98:99], v68 offset:50176\n\t" \
  "v_mfma_f32_16x16x128_f8f6f4 v[164:167], v[112:119], a[168:175], v[164:167]\n\t" \
  "ds_read_b64_tr_b8 v[100:101], v68 offset:57344\n\t" \
  "ds_read_b64_tr_b8 v[102:103], v68 offset:58368\n\t" \
  "v_mfma_f32_16x16x128_f8f6f4 v[168:171], v[112:119], a[176:183], v[168:171]\n\t" \
  "v_mfma_f32_16x16x128_f8f6f4 v[172:175], v[112:119], a[184:191], v[172:175]\n\t" \
  "s_waitcnt lgkmcnt(4)\n\t" \
  "v_mfma_f32_16x16x128_f8f6f4 v[176:179], v[120:127], a[160:167], v[176:179]\n\t" \
  "ds_read_b64_tr_b8 v[104:105], v69 offset:49152\n\t" \
  "ds_read_b64_tr_b8 v[106:107], v69 offset:50176\n\t" \
  "v_mfma_f32_16x16x128_f8f6f4 v[180:183], v[120:127], a[168:175], v[180:183]\n\t" \
  "ds_read_b64_tr_b8 v[108:109], v69 offset:57344\n\t" \
  "ds_read_b64_tr_b8 v[110:111], v69 offset:58368\n\t" \
  "v_mfma_f32_16x16x128_f8f6f4 v[184:187], v[120:127], a[176:183], v[184:187]\n\t" \
  "v_mfma_f32_16x16x128_f8f6f4 v[188:191], v[120:127], a[184:191], v[188:191]\n\t" \
  "s_waitcnt lgkmcnt(4)\n\t" \
  "v_mfma_f32_16x16x128_f8f6f4 v[192:195], v[96:103], a[160:167], v[192:195]\n\t" \
  "ds_read_b64_tr_b8 v[112:113], v70 offset:49152\n\t" \
  "ds_read_b64_tr_b8 v[114:115], v70 offset:50176\n\t" \
  "v_mfma_f32_16x16x128_f8f6f4 v[196:199], v[96:103], a[168:175], v[196:199]\n\t" \
  "ds_read_b64_tr_b8 v[116:117], v70 offset:57344\n\t" \
  "ds_read_b64_tr_b8 v[118:119], v70 offset:58368\n\t" \
  "v_mfma_f32_16x16x128_f8f6f4 v[200:203], v[96:103], a[176:183], v[200:203]\n\t" \
  "v_mfma_f32_16x16x128_f8f6f4 v[204:207], v[96:103], a[184:191], v[204:207]\n\t" \
  "s_waitcnt lgkmcnt(4)\n\t" \
  "v_mfma_f32_16x16x128_f8f6f4 v[208:211], v[104:111], a[160:167], v[208:211]\n\t" \
  "ds_read_b64_tr_b8 v[120:121], v71 offset:49152\n\t" \
  "ds_read_b64_tr_b8 v[122:123], v71 offset:50176\n\t" \
  "v_mfma_f32_16x16x128_f8f6f4 v[212:215], v[104:111], a[168:175], v[212:215]\n\t" \
  "ds_read_b64_tr_b8 v[124:125], v71 offset:57344\n\t" \
  "ds_read_b64_tr_b8 v[126:127], v71 offset:58368\n\t" \
  "v_mfma_f32_16x16x128_f8f6f4 v[216:219], v[104:111], a[176:183], v[216:219]\n\t" \
  "v_mfma_f32_16x16x128_f8f6f4 v[220:223], v[104:111], a[184:191], v[220:223]\n\t" \
  "s_waitcnt vmcnt(20)\n\t" \
  "s_barrier\n\t" \
  "s_waitcnt lgkmcnt(4)\n\t" \
  "v_mfma_f32_16x16x128_f8f6f4 v[224:227], v[112:119], a[160:167], v[224:227]\n\t" \
  "ds_read_b64_tr_b8 v[96:97], v64 offset:0\n\t" \
  "ds_read_b64_tr_b8 v[98:99], v64 offset:1024\n\t" \
  "v_mfma_f32_16x16x128_f8f6f4 v[228:231], v[112:119], a[168:175], v[228:231]\n\t" \
  "ds_read_b64_tr_b8 v[100:101], v64 offset:8192\n\t" \
  "ds_read_b64_tr_b8 v[102:103], v64 offset:9216\n\t" \
  "buffer_load_dwordx4 a[244:247], v84, s[44:47], s90 offen offset:1024 sc0 sc1\n\t" \
  "v_mfma_f32_16x16x128_f8f6f4 v[232:235], v[112:119], a[176:183], v[232:235]\n\t" \
  "buffer_load_dwordx4 a[248:251], v84, s[44:47], s90 offen offset:2048 sc0 sc1\n\t" \
  "v_mfma_f32_16x16x128_f8f6f4 v[236:239], v[112:119], a[184:191], v[236:239]\n\t" \
  "buffer_load_dwordx4 a[252:255], v84, s[44:47], s90 offen offset:3072 sc0 sc1\n\t" \
  "s_waitcnt lgkmcnt(4)\n\t" \
  "v_mfma_f32_16x16x128_f8f6f4 v[240:243], v[120:127], a[160:167], v[240:243]\n\t" \
  "ds_read_b64_tr_b8 v[104:105], v65 offset:0\n\t" \
  "ds_read_b64_tr_b8 v[106:107], v65 offset:1024\n\t" \
  "v_mfma_f32_16x16x128_f8f6f4 v[244:247], v[120:127], a[168:175], v[244:247]\n\t" \
  "ds_read_b64_tr_b8 v[108:109], v65 offset:8192\n\t" \
  "ds_read_b64_tr_b8 v[110:111], v65 offset:9216\n\t" \
  "s_add_u32 s89, s62, 1572864\n\t" \
  "s_add_u32 s90, s62, 1576960\n\t" \
  "buffer_load_dwordx4 a[0:3], v84, s[44:47], s89 offen offset:0 sc0 sc1\n\t" \
  "v_mfma_f32_16x16x128_f8f6f4 v[248:251], v[120:127], a[176:183], v[248:251]\n\t" \
  "buffer_load_dwordx4 a[4:7], v84, s[44:47], s89 offen offset:1024 sc0 sc1\n\t" \
  "v_mfma_f32_16x16x128_f8f6f4 v[252:255], v[120:127], a[184:191], v[252:255]\n\t" \
  "s_waitcnt vmcnt(34)\n\t" \
  "s_bitcmp1_b32 s65, 26\n\t" \
  "s_cbranch_scc1 OK48_%=\n\t" \
  "s_mov_b32 s67, 26\n\t" \
  "s_call_b64 s[68:69], POLL_%=\n\t" \
  "OK48_%=:\n\t" \
  "s_waitcnt lgkmcnt(4)\n\t" \
  "v_mfma_f32_16x16x128_f8f6f4 v[128:131], v[96:103], a[192:199], v[128:131]\n\t" \
  "ds_read_b64_tr_b8 v[112:113], v66 offset:0\n\t" \
  "ds_read_b64_tr_b8 v[114:115], v66 offset:1024\n\t" \
  "v_mfma_f32_16x16x128_f8f6f4 v[132:135], v[96:103], a[200:207], v[132:135]\n\t" \
  "ds_read_b64_tr_b8 v[116:117], v66 offset:8192\n\t" \
  "ds_read_b64_tr_b8 v[118:119], v66 offset:9216\n\t" \
  "buffer_load_dwordx4 a[8:11], v84, s[44:47], s89 offen offset:2048 sc0 sc1\n\t" \
  "v_mfma_f32_16x16x128_f8f6f4 v[136:139], v[96:103], a[208:215], v[136:139]\n\t" \
  "buffer_load_dwordx4 a[12:15], v84, s[44:47], s89 offen offset:3072 sc0 sc1\n\t" \
  "v_mfma_f32_16x16x128_f8f6f4 v[140:143], v[96:103], a[216:223], v[140:143]\n\t" \
  "s_waitcnt lgkmcnt(4)\n\t" \
  "v_mfma_f32_16x16x128_f8f6f4 v[144:147], v[104:111], a[192:199], v[144:147]\n\t" \
  "ds_read_b64_tr_b8 v[120:121], v67 offset:0\n\t" \
  "ds_read_b64_tr_b8 v[122:123], v67 offset:1024\n\t" \
  "v_mfma_f32_16x16x128_f8f6f4 v[148:151], v[104:111], a[200:207], v[148:151]\n\t" \
  "ds_read_b64_tr_b8 v[124:125], v67 offset:8192\n\t" \
  "ds_read_b64_tr_b8 v[126:127], v67 offset:9216\n\t" \
  "buffer_load_dwordx4 a[16:19], v84, s[44:47], s90 offen offset:0 sc0 sc1\n\t" \
  "v_mfma_f32_16x16x128_f8f6f4 v[152:155], v[104:111], a[208:215], v[152:155]\n\t" \
  "buffer_load_dwordx4 a[20:23], v84, s[44:47], s90 offen offset:1024 sc0 sc1\n\t" \
  "v_mfma_f32_16x16x128_f8f6f4 v[156:159], v[104:111], a[216:223], v[156:159]\n\t" \
  "s_waitcnt lgkmcnt(4)\n\t" \
  "v_mfma_f32_16x16x128_f8f6f4 v[160:163], v[112:119], a[192:199], v[160:163]\n\t" \
  "ds_read_b64_tr_b8 v[96:97], v68 offset:0\n\t" \
  "ds_read_b64_tr_b8 v[98:99], v68 offset:1024\n\t" \
  "v_mfma_f32_16x16x128_f8f6f4 v[164:167], v[112:119], a[200:207], v[164:167]\n\t" \
  "ds_read_b64_tr_b8 v[100:101], v68 offset:8192\n\t" \
  "ds_read_b64_tr_b8 v[102:103], v68 offset:9216\n\t" \
  "buffer_load_dwordx4 a[24:27], v84, s[44:47], s90 offen offset:2048 sc0 sc1\n\t" \
  "v_mfma_f32_16x16x128_f8f6f4 v[168:171], v[112:119], a[208:215], v[168:171]\n\t" \
  "buffer_load_dwordx4 a[28:31], v84, s[44:47], s90 offen offset:3072 sc0 sc1\n\t" \
  "v_mfma_f32_16x16x128_f8f6f4 v[172:175], v[112:119], a[216:223], v[172:175]\n\t" \
  "s_waitcnt lgkmcnt(4)\n\t" \
  "v_mfma_f32_16x16x128_f8f6f4 v[176:179], v[120:127], a[192:199], v[176:179]\n\t" \
  "ds_read_b64_tr_b8 v[104:105], v69 offset:0\n\t" \
  "ds_read_b64_tr_b8 v[106:107], v69 offset:1024\n\t" \
  "v_mfma_f32_16x16x128_f8f6f4 v[180:183], v[120:127], a[200:207], v[180:183]\n\t" \
  "ds_read_b64_tr_b8 v[108:109], v69 offset:8192\n\t" \
  "ds_read_b64_tr_b8 v[110:111], v69 offset:9216\n\t" \
  "s_add_u32 s89, s62, 1638400\n\t" \
  "s_add_u32 s90, s62, 1642496\n\t" \
  "buffer_load_dwordx4 a[32:35], v84, s[44:47], s89 offen offset:0 sc0 sc1\n\t" \
  "v_mfma_f32_16x16x128_f8f6f4 v[184:187], v[120:127], a[208:215], v[184:187]\n\t" \
  "buffer_load_dwordx4 a[36:39], v84, s[44:47], s89 offen offset:1024 sc0 sc1\n\t" \
  "v_mfma_f32_16x16x128_f8f6f4 v[188:191], v[120:127], a[216:223], v[188:191]\n\t" \
  "s_waitcnt lgkmcnt(4)\n\t" \
  "v_mfma_f32_16x16x128_f8f6f4 v[192:195], v[96:103], a[192:199], v[192:195]\n\t" \
  "ds_read_b64_tr_b8 v[112:113], v70 offset:0\n\t" \
  "ds_read_b64_tr_b8 v[114:115], v70 offset:1024\n\t" \
  "v_mfma_f32_16x16x128_f8f6f4 v[196:199], v[96:103], a[200:207], v[196:199]\n\t" \
  "ds_read_b64_tr_b8 v[116:117], v70 offset:8192\n\t" \
  "ds_read_b64_tr_b8 v[118:119], v70 offset:9216\n\t" \
  "buffer_load_dwordx4 a[40:43], v84, s[44:47], s89 offen offset:2048 sc0 sc1\n\t" \
  "v_mfma_f32_16x16x128_f8f6f4 v[200:203], v[96:103], a[208:215], v[200:203]\n\t" \
  "buffer_load_dwordx4 a[44:47], v84, s[44:47], s89 offen offset:3072 sc0 sc1\n\t" \
  "v_mfma_f32_16x16x128_f8f6f4 v[204:207], v[96:103], a[216:223], v[204:207]\n\t" \
  "s_waitcnt lgkmcnt(4)\n\t" \
  "v_mfma_f32_16x16x128_f8f6f4 v[208:211], v[104:111], a[192:199], v[208:211]\n\t" \
  "ds_read_b64_tr_b8 v[120:121], v71 offset:0\n\t" \
  "ds_read_b64_tr_b8 v[122:123], v71 offset:1024\n\t" \
  "v_mfma_f32_16x16x128_f8f6f4 v[212:215], v[104:111], a[200:207], v[212:215]\n\t" \
  "ds_read_b64_tr_b8 v[124:125], v71 offset:8192\n\t" \
  "ds_read_b64_tr_b8 v[126:127], v71 offset:9216\n\t" \
  "buffer_load_dwordx4 a[48:51], v84, s[44:47], s90 offen offset:0 sc0 sc1\n\t" \
  "v_mfma_f32_16x16x128_f8f6f4 v[216:219], v[104:111], a[208:215], v[216:219]\n\t" \
  "buffer_load_dwordx4 a[52:55], v84, s[44:47], s90 offen offset:1024 sc0 sc1\n\t" \
  "v_mfma_f32_16x16x128_f8f6f4 v[220:223], v[104:111], a[216:223], v[220:223]\n\t" \
  "s_waitcnt vmcnt(33)\n\t" \
  "s_barrier\n\t" \
  "s_waitcnt lgkmcnt(4)\n\t" \
  "v_mfma_f32_16x16x128_f8f6f4 v[224:227], v[112:119], a[192:199], v[224:227]\n\t" \
  "ds_read_b64_tr_b8 v[96:97], v64 offset:16384\n\t" \
  "ds_read_b64_tr_b8 v[98:99], v64 offset:17408\n\t" \
  "v_mfma_f32_16x16x128_f8f6f4 v[228:231], v[112:119], a[200:207], v[228:231]\n\t" \
  "ds_read_b64_tr_b8 v[100:101], v64 offset:24576\n\t" \
  "ds_read_b64_tr_b8 v[102:103], v64 offset:25600\n\t" \
  "buffer_load_dwordx4 a[56:59], v84, s[44:47], s90 offen offset:2048 sc0 sc1\n\t" \
  "v_mfma_f32_16x16x128_f8f6f4 v[232:235], v[112:119], a[208:215], v[232:235]\n\t" \
  "buffer_load_dwordx4 a[60:63], v84, s[44:47], s90 offen offset:3072 sc0 sc1\n\t" \
  "v_mfma_f32_16x16x128_f8f6f4 v[236:239], v[112:119], a[216:223], v[236:239]\n\t" \
  "s_waitcnt lgkmcnt(4)\n\t" \
  "v_mfma_f32_16x16x128_f8f6f4 v[240:243], v[120:127], a[192:199], v[240:243]\n\t" \
  "ds_read_b64_tr_b8 v[104:105], v65 offset:16384\n\t" \
  "ds_read_b64_tr_b8 v[106:107], v65 offset:17408\n\t" \
  "v_mfma_f32_16x16x128_f8f6f4 v[244:247], v[120:127], a[200:207], v[244:247]\n\t" \
  "ds_read_b64_tr_b8 v[108:109], v65 offset:24576\n\t" \
  "ds_read_b64_tr_b8 v[110:111], v65 offset:25600\n\t" \
  "s_add_u32 s89, s62, 1703936\n\t" \
  "s_add_u32 s90, s62, 1708032\n\t" \
  "buffer_load_dwordx4 a[64:67], v84, s[44:47], s89 offen offset:0 sc0 sc1\n\t" \
  "v_mfma_f32_16x16x128_f8f6f4 v[248:251], v[120:127], a[208:215], v[248:251]\n\t" \
  "buffer_load_dwordx4 a[68:71], v84, s[44:47], s89 offen offset:1024 sc0 sc1\n\t" \
  "v_mfma_f32_16x16x128_f8f6f4 v[252:255], v[120:127], a[216:223], v[252:255]\n\t" \
  "s_waitcnt vmcnt(18)\n\t" \
  "s_bitcmp1_b32 s65, 27\n\t" \
  "s_cbranch_scc1 OK49_%=\n\t" \
  "s_mov_b32 s67, 27\n\t" \
  "s_call_b64 s[68:69], POLL_%=\n\t" \
  "OK49_%=:\n\t" \
  "s_waitcnt lgkmcnt(4)\n\t" \
  "v_mfma_f32_16x16x128_f8f6f4 v[128:131], v[96:103], a[224:231], v[128:131]\n\t" \
  "ds_read_b64_tr_b8 v[112:113], v66 offset:16384\n\t" \
  "ds_read_b64_tr_b8 v[114:115], v66 offset:17408\n\t" \
  "v_mfma_f32_16x16x128_f8f6f4 v[132:135], v[96:103], a[232:239], v[132:135]\n\t" \
  "ds_read_b64_tr_b8 v[116:117], v66 offset:24576\n\t" \
  "ds_read_b64_tr_b8 v[118:119], v66 offset:25600\n\t" \
  "buffer_load_dwordx4 a[72:75], v84, s[44:47], s89 offen offset:2048 sc0 sc1\n\t" \
  "v_mfma_f32_16x16x128_f8f6f4 v[136:139], v[96:103], a[240:247], v[136:139]\n\t" \
  "buffer_load_dwordx4 a[76:79], v84, s[44:47], s89 offen offset:3072 sc0 sc1\n\t" \
  "v_mfma_f32_16x16x128_f8f6f4 v[140:143], v[96:103], a[248:255], v[140:143]\n\t" \
  "s_waitcnt lgkmcnt(4)\n\t" \
  "v_mfma_f32_16x16x128_f8f6f4 v[144:147], v[104:111], a[224:231], v[144:147]\n\t" \
  "ds_read_b64_tr_b8 v[120:121], v67 offset:16384\n\t" \
  "ds_read_b64_tr_b8 v[122:123], v67 offset:17408\n\t" \
  "v_mfma_f32_16x16x128_f8f6f4 v[148:151], v[104:111], a[232:239], v[148:151]\n\t" \
  "ds_read_b64_tr_b8 v[124:125], v67 offset:24576\n\t" \
  "ds_read_b64_tr_b8 v[126:127], v67 offset:25600\n\t" \
  "buffer_load_dwordx4 a[80:83], v84, s[44:47], s90 offen offset:0 sc0 sc1\n\t" \
  "v_mfma_f32_16x16x128_f8f6f4 v[152:155], v[104:111], a[240:247], v[152:155]\n\t" \
  "buffer_load_dwordx4 a[84:87], v84, s[44:47], s90 offen offset:1024 sc0 sc1\n\t" \
  "v_mfma_f32_16x16x128_f8f6f4 v[156:159], v[104:111], a[248:255], v[156:159]\n\t" \
  "s_waitcnt lgkmcnt(4)\n\t" \
  "v_mfma_f32_16x16x128_f8f6f4 v[160:163], v[112:119], a[224:231], v[160:163]\n\t" \
  "ds_read_b64_tr_b8 v[96:97], v68 offset:16384\n\t" \
  "ds_read_b64_tr_b8 v[98:99], v68 offset:17408\n\t" \
  "v_mfma_f32_16x16x128_f8f6f4 v[164:167], v[112:119], a[232:239], v[164:167]\n\t" \
  "ds_read_b64_tr_b8 v[100:101], v68 offset:24576\n\t" \
  "ds_read_b64_tr_b8 v[102:103], v68 offset:25600\n\t" \
  "buffer_load_dwordx4 a[88:91], v84, s[44:47], s90 offen offset:2048 sc0 sc1\n\t" \
  "v_mfma_f32_16x16x128_f8f6f4 v[168:171], v[112:119], a[240:247], v[168:171]\n\t" \
  "buffer_load_dwordx4 a[92:95], v84, s[44:47], s90 offen offset:3072 sc0 sc1\n\t" \
  "v_mfma_f32_16x16x128_f8f6f4 v[172:175], v[112:119], a[248:255], v[172:175]\n\t" \
  "s_waitcnt lgkmcnt(4)\n\t" \
  "v_mfma_f32_16x16x128_f8f6f4 v[176:179], v[120:127], a[224:231], v[176:179]\n\t" \
  "ds_read_b64_tr_b8 v[104:105], v69 offset:16384\n\t" \
  "ds_read_b64_tr_b8 v[106:107], v69 offset:17408\n\t" \
  "v_mfma_f32_16x16x128_f8f6f4 v[180:183], v[120:127], a[232:239], v[180:183]\n\t" \
  "ds_read_b64_tr_b8 v[108:109], v69 offset:24576\n\t" \
  "ds_read_b64_tr_b8 v[110:111], v69 offset:25600\n\t" \
  "s_add_u32 s89, s62, 1769472\n\t" \
  "s_add_u32 s90, s62, 1773568\n\t" \
  "buffer_load_dwordx4 a[96:99], v84, s[44:47], s89 offen offset:0 sc0 sc1\n\t" \
  "v_mfma_f32_16x16x128_f8f6f4 v[184:187], v[120:127], a[240:247], v[184:187]\n\t" \
  "buffer_load_dwordx4 a[100:103], v84, s[44:47], s89 offen offset:1024 sc0 sc1\n\t" \
  "v_mfma_f32_16x16x128_f8f6f4 v[188:191], v[120:127], a[248:255], v[188:191]\n\t" \
  "s_waitcnt lgkmcnt(4)\n\t" \
  "v_mfma_f32_16x16x128_f8f6f4 v[192:195], v[96:103], a[224:231], v[192:195]\n\t" \
  "ds_read_b64_tr_b8 v[112:113], v70 offset:16384\n\t" \
  "ds_read_b64_tr_b8 v[114:115], v70 offset:17408\n\t" \
  "v_mfma_f32_16x16x128_f8f6f4 v[196:199], v[96:103], a[232:239], v[196:199]\n\t" \
  "ds_read_b64_tr_b8 v[116:117], v70 offset:24576\n\t" \
  "ds_read_b64_tr_b8 v[118:119], v70 offset:25600\n\t" \
  "s_add_u32 s88, s55, 6291456\n\t" \
  "s_add_u32 m0, s60, 114688\n\t" \
  "s_nop 0\n\t" \
  "buffer_load_dwordx4 v80, s[36:39], s88 offen  lds\n\t" \
  "v_mfma_f32_16x16x128_f8f6f4 v[200:203], v[96:103], a[240:247], v[200:203]\n\t" \
  "s_add_u32 m0, s60, 115712\n\t" \
  "s_nop 0\n\t" \
  "buffer_load_dwordx4 v81, s[36:39], s88 offen  lds\n\t" \
  "v_mfma_f32_16x16x128_f8f6f4 v[204:207], v[96:103], a[248:255], v[204:207]\n\t" \
  "s_waitcnt lgkmcnt(4)\n\t" \
  "v_mfma_f32_16x16x128_f8f6f4 v[208:211], v[104:111], a[224:231], v[208:211]\n\t" \
  "ds_read_b64_tr_b8 v[120:121], v71 offset:16384\n\t" \
  "ds_read_b64_tr_b8 v[122:123], v71 offset:17408\n\t" \
  "v_mfma_f32_16x16x128_f8f6f4 v[212:215], v[104:111], a[232:239], v[212:215]\n\t" \
  "ds_read_b64_tr_b8 v[124:125], v71 offset:24576\n\t" \
  "ds_read_b64_tr_b8 v[126:127], v71 offset:25600\n\t" \
  "s_add_u32 m0, s60, 116736\n\t" \
  "s_nop 0\n\t" \
  "buffer_load_dwordx4 v82, s[36:39], s88 offen  lds\n\t" \
  "v_mfma_f32_16x16x128_f8f6f4 v[216:219], v[104:111], a[240:247], v[216:219]\n\t" \
  "s_add_u32 m0, s60, 117760\n\t" \
  "s_nop 0\n\t" \
  "buffer_load_dwordx4 v83, s[36:39], s88 offen  lds\n\t" \
  "v_mfma_f32_16x16x128_f8f6f4 v[220:223], v[104:111], a[248:255], v[220:223]\n\t" \
  "s_waitcnt vmcnt(45)\n\t" \
  "s_barrier\n\t" \
  "s_waitcnt lgkmcnt(4)\n\t" \
  "v_mfma_f32_16x16x128_f8f6f4 v[224:227], v[112:119], a[224:231], v[224:227]\n\t" \
  "ds_read_b64_tr_b8 v[96:97], v64 offset:32768\n\t" \
  "ds_read_b64_tr_b8 v[98:99], v64 offset:33792\n\t" \
  "v_mfma_f32_16x16x128_f8f6f4 v[228:231], v[112:119], a[232:239], v[228:231]\n\t" \
  "ds_read_b64_tr_b8 v[100:101], v64 offset:40960\n\t" \
  "ds_read_b64_tr_b8 v[102:103], v64 offset:41984\n\t" \
  "s_add_u32 s88, s56, 6291456\n\t" \
  "s_add_u32 m0, s60, 131072\n\t" \
  "s_nop 0\n\t" \
  "buffer_load_dwordx4 v80, s[36:39], s88 offen  lds\n\t" \
  "v_mfma_f32_16x16x128_f8f6f4 v[232:235], v[112:119], a[240:247], v[232:235]\n\t" \
  "s_add_u32 m0, s60, 132096\n\t" \
  "s_nop 0\n\t" \
  "buffer_load_dwordx4 v81, s[36:39], s88 offen  lds\n\t" \
  "v_mfma_f32_16x16x128_f8f6f4 v[236:239], v[112:119], a[248:255], v[236:239]\n\t" \
  "s_waitcnt lgkmcnt(4)\n\t" \
  "v_mfma_f32_16x16x128_f8f6f4 v[240:243], v[120:127], a[224:231], v[240:243]\n\t" \
  "ds_read_b64_tr_b8 v[104:105], v65 offset:32768\n\t" \
  "ds_read_b64_tr_b8 v[106:107], v65 offset:33792\n\t" \
  "v_mfma_f32_16x16x128_f8f6f4 v[244:247], v[120:127], a[232:239], v[244:247]\n\t" \
  "ds_read_b64_tr_b8 v[108:109], v65 offset:40960\n\t" \
  "ds_read_b64_tr_b8 v[110:111], v65 offset:41984\n\t" \
  "s_add_u32 m0, s60, 133120\n\t" \
  "s_nop 0\n\t" \
  "buffer_load_dwordx4 v82, s[36:39], s88 offen  lds\n\t" \
  "v_mfma_f32_16x16x128_f8f6f4 v[248:251], v[120:127], a[240:247], v[248:251]\n\t" \
  "s_add_u32 m0, s60, 134144\n\t" \
  "s_nop 0\n\t" \
  "buffer_load_dwordx4 v83, s[36:39], s88 offen  lds\n\t" \
  "v_mfma_f32_16x16x128_f8f6f4 v[252:255], v[120:127], a[248:255], v[252:255]\n\t" \
  "s_waitcnt vmcnt(26)\n\t" \
  "s_bitcmp1_b32 s65, 28\n\t" \
  "s_cbranch_scc1 OK50_%=\n\t" \
  "s_mov_b32 s67, 28\n\t" \
  "s_call_b64 s[68:69], POLL_%=\n\t" \
  "OK50_%=:\n\t" \
  "s_bitcmp1_b32 s65, 29\n\t" \
  "s_cbranch_scc1 OK51_%=\n\t" \
  "s_mov_b32 s67, 29\n\t" \
  "s_call_b64 s[68:69], POLL_%=\n\t" \
  "OK51_%=:\n\t" \
  "s_waitcnt lgkmcnt(4)\n\t" \
  "v_mfma_f32_16x16x128_f8f6f4 v[128:131], v[96:103], a[0:7], v[128:131]\n\t" \
  "ds_read_b64_tr_b8 v[112:113], v66 offset:32768\n\t" \
  "ds_read_b64_tr_b8 v[114:115], v66 offset:33792\n\t" \
  "s_bfe_u32 s96, s65, 196637\n\t" \
  "s_cmp_lg_u32 s96, 7\n\t" \
  "s_cbranch_scc1 XWS52_%=\n\t" \
  "s_add_u32 s96, s62, 1900544\n\t" \
  "buffer_load_dword v8, v9, s[44:47], s96 offen sc0 sc1\n\t" \
  "XWS52_%=:\n\t" \
  "v_mfma_f32_16x16x128_f8f6f4 v[132:135], v[96:103], a[8:15], v[132:135]\n\t" \
  "ds_read_b64_tr_b8 v[116:117], v66 offset:40960\n\t" \
  "ds_read_b64_tr_b8 v[118:119], v66 offset:41984\n\t" \
  "s_add_u32 s88, s57, 6291456\n\t" \
  "s_add_u32 m0, s60, 49152\n\t" \
  "s_nop 0\n\t" \
  "buffer_load_dwordx4 v80, s[36:39], s88 offen  lds\n\t" \
  "v_mfma_f32_16x16x128_f8f6f4 v[136:139], v[96:103], a[16:23], v[136:139]\n\t" \
  "s_add_u32 m0, s60, 50176\n\t" \
  "s_nop 0\n\t" \
  "buffer_load_dwordx4 v81, s[36:39], s88 offen  lds\n\t" \
  "v_mfma_f32_16x16x128_f8f6f4 v[140:143], v[96:103], a[24:31], v[140:143]\n\t" \
  "s_waitcnt lgkmcnt(4)\n\t" \
  "v_mfma_f32_16x16x128_f8f6f4 v[144:147], v[104:111], a[0:7], v[144:147]\n\t" \
  "ds_read_b64_tr_b8 v[120:121], v67 offset:32768\n\t" \
  "ds_read_b64_tr_b8 v[122:123], v67 offset:33792\n\t" \
  "v_mfma_f32_16x16x128_f8f6f4 v[148:151], v[104:111], a[8:15], v[148:151]\n\t" \
  "ds_read_b64_tr_b8 v[124:125], v67 offset:40960\n\t" \
  "ds_read_b64_tr_b8 v[126:127], v67 offset:41984\n\t" \
  "s_add_u32 m0, s60, 51200\n\t" \
  "s_nop 0\n\t" \
  "buffer_load_dwordx4 v82, s[36:39], s88 offen  lds\n\t" \
  "v_mfma_f32_16x16x128_f8f6f4 v[152:155], v[104:111], a[16:23], v[152:155]\n\t" \
  "s_add_u32 m0, s60, 52224\n\t" \
  "s_nop 0\n\t" \
  "buffer_load_dwordx4 v83, s[36:39], s88 offen  lds\n\t" \
  "v_mfma_f32_16x16x128_f8f6f4 v[156:159], v[104:111], a[24:31], v[156:159]\n\t" \
  "s_waitcnt lgkmcnt(4)\n\t" \
  "v_mfma_f32_16x16x128_f8f6f4 v[160:163], v[112:119], a[0:7], v[160:163]\n\t" \
  "ds_read_b64_tr_b8 v[96:97], v68 offset:32768\n\t" \
  "ds_read_b64_tr_b8 v[98:99], v68 offset:33792\n\t" \
  "v_mfma_f32_16x16x128_f8f6f4 v[164:167], v[112:119], a[8:15], v[164:167]\n\t" \
  "ds_read_b64_tr_b8 v[100:101], v68 offset:40960\n\t" \
  "ds_read_b64_tr_b8 v[102:103], v68 offset:41984\n\t" \
  "s_add_u32 s88, s58, 6291456\n\t" \
  "s_add_u32 m0, s60, 0\n\t" \
  "s_nop 0\n\t" \
  "buffer_load_dwordx4 v80, s[36:39], s88 offen  lds\n\t" \
  "v_mfma_f32_16x16x128_f8f6f4 v[168:171], v[112:119], a[16:23], v[168:171]\n\t" \
  "s_add_u32 m0, s60, 1024\n\t" \
  "s_nop 0\n\t" \
  "buffer_load_dwordx4 v81, s[36:39], s88 offen  lds\n\t" \
  "v_mfma_f32_16x16x128_f8f6f4 v[172:175], v[112:119], a[24:31], v[172:175]\n\t" \
  "s_waitcnt lgkmcnt(4)\n\t" \
  "v_mfma_f32_16x16x128_f8f6f4 v[176:179], v[120:127], a[0:7], v[176:179]\n\t" \
  "ds_read_b64_tr_b8 v[104:105], v69 offset:32768\n\t" \
  "ds_read_b64_tr_b8 v[106:107], v69 offset:33792\n\t" \
  "v_mfma_f32_16x16x128_f8f6f4 v[180:183], v[120:127], a[8:15], v[180:183]\n\t" \
  "ds_read_b64_tr_b8 v[108:109], v69 offset:40960\n\t" \
  "ds_read_b64_tr_b8 v[110:111], v69 offset:41984\n\t" \
  "s_add_u32 m0, s60, 2048\n\t" \
  "s_nop 0\n\t" \
  "buffer_load_dwordx4 v82, s[36:39], s88 offen  lds\n\t" \
  "v_mfma_f32_16x16x128_f8f6f4 v[184:187], v[120:127], a[16:23], v[184:187]\n\t" \
  "s_add_u32 m0, s60, 3072\n\t" \
  "s_nop 0\n\t" \
  "buffer_load_dwordx4 v83, s[36:39], s88 offen  lds\n\t" \
  "v_mfma_f32_16x16x128_f8f6f4 v[188:191], v[120:127], a[24:31], v[188:191]\n\t" \
  "s_waitcnt lgkmcnt(4)\n\t" \
  "v_mfma_f32_16x16x128_f8f6f4 v[192:195], v[96:103], a[0:7], v[192:195]\n\t" \
  "ds_read_b64_tr_b8 v[112:113], v70 offset:32768\n\t" \
  "ds_read_b64_tr_b8 v[114:115], v70 offset:33792\n\t" \
  "v_mfma_f32_16x16x128_f8f6f4 v[196:199], v[96:103], a[8:15], v[196:199]\n\t" \
  "ds_read_b64_tr_b8 v[116:117], v70 offset:40960\n\t" \
  "ds_read_b64_tr_b8 v[118:119], v70 offset:41984\n\t" \
  "buffer_load_dwordx4 a[104:107], v84, s[44:47], s89 offen offset:2048 sc0 sc1\n\t" \
  "v_mfma_f32_16x16x128_f8f6f4 v[200:203], v[96:103], a[16:23], v[200:203]\n\t" \
  "buffer_load_dwordx4 a[108:111], v84, s[44:47], s89 offen offset:3072 sc0 sc1\n\t" \
  "v_mfma_f32_16x16x128_f8f6f4 v[204:207], v[96:103], a[24:31], v[204:207]\n\t" \
  "buffer_load_dwordx4 a[112:115], v84, s[44:47], s90 offen offset:0 sc0 sc1\n\t" \
  "s_waitcnt lgkmcnt(4)\n\t" \
  "v_mfma_f32_16x16x128_f8f6f4 v[208:211], v[104:111], a[0:7], v[208:211]\n\t" \
  "ds_read_b64_tr_b8 v[120:121], v71 offset:32768\n\t" \
  "ds_read_b64_tr_b8 v[122:123], v71 offset:33792\n\t" \
  "v_mfma_f32_16x16x128_f8f6f4 v[212:215], v[104:111], a[8:15], v[212:215]\n\t" \
  "ds_read_b64_tr_b8 v[124:125], v71 offset:40960\n\t" \
  "ds_read_b64_tr_b8 v[126:127], v71 offset:41984\n\t" \
  "buffer_load_dwordx4 a[116:119], v84, s[44:47], s90 offen offset:1024 sc0 sc1\n\t" \
  "v_mfma_f32_16x16x128_f8f6f4 v[216:219], v[104:111], a[16:23], v[216:219]\n\t" \
  "v_mfma_f32_16x16x128_f8f6f4 v[220:223], v[104:111], a[24:31], v[220:223]\n\t" \
  "s_waitcnt vmcnt(57)\n\t" \
  "s_barrier\n\t" \
  "s_waitcnt lgkmcnt(4)\n\t" \
  "v_mfma_f32_16x16x128_f8f6f4 v[224:227], v[112:119], a[0:7], v[224:227]\n\t" \
  "ds_read_b64_tr_b8 v[96:97], v72 offset:0\n\t" \
  "ds_read_b64_tr_b8 v[98:99], v72 offset:1024\n\t" \
  "v_mfma_f32_16x16x128_f8f6f4 v[228:231], v[112:119], a[8:15], v[228:231]\n\t" \
  "ds_read_b64_tr_b8 v[100:101], v72 offset:8192\n\t" \
  "ds_read_b64_tr_b8 v[102:103], v72 offset:9216\n\t" \
  "v_mfma_f32_16x16x128_f8f6f4 v[232:235], v[112:119], a[16:23], v[232:235]\n\t" \
  "v_mfma_f32_16x16x128_f8f6f4 v[236:239], v[112:119], a[24:31], v[236:239]\n\t" \
  "s_waitcnt lgkmcnt(4)\n\t" \
  "v_mfma_f32_16x16x128_f8f6f4 v[240:243], v[120:127], a[0:7], v[240:243]\n\t" \
  "ds_read_b64_tr_b8 v[104:105], v73 offset:0\n\t" \
  "ds_read_b64_tr_b8 v[106:107], v73 offset:1024\n\t" \
  "v_mfma_f32_16x16x128_f8f6f4 v[244:247], v[120:127], a[8:15], v[244:247]\n\t" \
  "ds_read_b64_tr_b8 v[108:109], v73 offset:8192\n\t" \
  "ds_read_b64_tr_b8 v[110:111], v73 offset:9216\n\t" \
  "v_mfma_f32_16x16x128_f8f6f4 v[248:251], v[120:127], a[16:23], v[248:251]\n\t" \
  "v_mfma_f32_16x16x128_f8f6f4 v[252:255], v[120:127], a[24:31], v[252:255]\n\t" \
  "s_waitcnt vmcnt(30)\n\t" \
  "s_bitcmp1_b32 s65, 30\n\t" \
  "s_cbranch_scc1 OK53_%=\n\t" \
  "s_mov_b32 s67, 30\n\t" \
  "s_call_b64 s[68:69], POLL_%=\n\t" \
  "OK53_%=:\n\t" \
  "s_waitcnt lgkmcnt(4)\n\t" \
  "v_mfma_f32_16x16x128_f8f6f4 v[128:131], v[96:103], a[32:39], v[128:131]\n\t" \
  "ds_read_b64_tr_b8 v[112:113], v74 offset:0\n\t" \
  "ds_read_b64_tr_b8 v[114:115], v74 offset:1024\n\t" \
  "v_mfma_f32_16x16x128_f8f6f4 v[132:135], v[96:103], a[40:47], v[132:135]\n\t" \
  "ds_read_b64_tr_b8 v[116:117], v74 offset:8192\n\t" \
  "ds_read_b64_tr_b8 v[118:119], v74 offset:9216\n\t" \
  "v_mfma_f32_16x16x128_f8f6f4 v[136:139], v[96:103], a[48:55], v[136:139]\n\t" \
  "v_mfma_f32_16x16x128_f8f6f4 v[140:143], v[96:103], a[56:63], v[140:143]\n\t" \
  "s_waitcnt lgkmcnt(4)\n\t" \
  "v_mfma_f32_16x16x128_f8f6f4 v[144:147], v[104:111], a[32:39], v[144:147]\n\t" \
  "ds_read_b64_tr_b8 v[120:121], v75 offset:0\n\t" \
  "ds_read_b64_tr_b8 v[122:123], v75 offset:1024\n\t" \
  "v_mfma_f32_16x16x128_f8f6f4 v[148:151], v[104:111], a[40:47], v[148:151]\n\t" \
  "ds_read_b64_tr_b8 v[124:125], v75 offset:8192\n\t" \
  "ds_read_b64_tr_b8 v[126:127], v75 offset:9216\n\t" \
  "v_mfma_f32_16x16x128_f8f6f4 v[152:155], v[104:111], a[48:55], v[152:155]\n\t" \
  "v_mfma_f32_16x16x128_f8f6f4 v[156:159], v[104:111], a[56:63], v[156:159]\n\t" \
  "s_waitcnt lgkmcnt(4)\n\t" \
  "v_mfma_f32_16x16x128_f8f6f4 v[160:163], v[112:119], a[32:39], v[160:163]\n\t" \
  "ds_read_b64_tr_b8 v[96:97], v76 offset:0\n\t" \
  "ds_read_b64_tr_b8 v[98:99], v76 offset:1024\n\t" \
  "v_mfma_f32_16x16x128_f8f6f4 v[164:167], v[112:119], a[40:47], v[164:167]\n\t" \
  "ds_read_b64_tr_b8 v[100:101], v76 offset:8192\n\t" \
  "ds_read_b64_tr_b8 v[102:103], v76 offset:9216\n\t" \
  "v_mfma_f32_16x16x128_f8f6f4 v[168:171], v[112:119], a[48:55], v[168:171]\n\t" \
  "v_mfma_f32_16x16x128_f8f6f4 v[172:175], v[112:119], a[56:63], v[172:175]\n\t" \
  "s_waitcnt lgkmcnt(4)\n\t" \
  "v_mfma_f32_16x16x128_f8f6f4 v[176:179], v[120:127], a[32:39], v[176:179]\n\t" \
  "ds_read_b64_tr_b8 v[104:105], v77 offset:0\n\t" \
  "ds_read_b64_tr_b8 v[106:107], v77 offset:1024\n\t" \
  "v_mfma_f32_16x16x128_f8f6f4 v[180:183], v[120:127], a[40:47], v[180:183]\n\t" \
  "ds_read_b64_tr_b8 v[108:109], v77 offset:8192\n\t" \
  "ds_read_b64_tr_b8 v[110:111], v77 offset:9216\n\t" \
  "v_mfma_f32_16x16x128_f8f6f4 v[184:187], v[120:127], a[48:55], v[184:187]\n\t" \
  "v_mfma_f32_16x16x128_f8f6f4 v[188:191], v[120:127], a[56:63], v[188:191]\n\t" \
  "s_waitcnt lgkmcnt(4)\n\t" \
  "v_mfma_f32_16x16x128_f8f6f4 v[192:195], v[96:103], a[32:39], v[192:195]\n\t" \
  "ds_read_b64_tr_b8 v[112:113], v78 offset:0\n\t" \
  "ds_read_b64_tr_b8 v[114:115], v78 offset:1024\n\t" \
  "v_mfma_f32_16x16x128_f8f6f4 v[196:199], v[96:103], a[40:47], v[196:199]\n\t" \
  "ds_read_b64_tr_b8 v[116:117], v78 offset:8192\n\t" \
  "ds_read_b64_tr_b8 v[118:119], v78 offset:9216\n\t" \
  "v_mfma_f32_16x16x128_f8f6f4 v[200:203], v[96:103], a[48:55], v[200:203]\n\t" \
  "v_mfma_f32_16x16x128_f8f6f4 v[204:207], v[96:103], a[56:63], v[204:207]\n\t" \
  "s_waitcnt lgkmcnt(4)\n\t" \
  "v_mfma_f32_16x16x128_f8f6f4 v[208:211], v[104:111], a[32:39], v[208:211]\n\t" \
  "ds_read_b64_tr_b8 v[120:121], v79 offset:0\n\t" \
  "ds_read_b64_tr_b8 v[122:123], v79 offset:1024\n\t" \
  "v_mfma_f32_16x16x128_f8f6f4 v[212:215], v[104:111], a[40:47], v[212:215]\n\t" \
  "ds_read_b64_tr_b8 v[124:125], v79 offset:8192\n\t" \
  "ds_read_b64_tr_b8 v[126:127], v79 offset:9216\n\t" \
  "buffer_load_dwordx4 a[120:123], v84, s[44:47], s90 offen offset:2048 sc0 sc1\n\t" \
  "v_mfma_f32_16x16x128_f8f6f4 v[216:219], v[104:111], a[48:55], v[216:219]\n\t" \
  "buffer_load_dwordx4 a[124:127], v84, s[44:47], s90 offen offset:3072 sc0 sc1\n\t" \
  "v_mfma_f32_16x16x128_f8f6f4 v[220:223], v[104:111], a[56:63], v[220:223]\n\t" \
  "s_add_u32 s89, s62, 1835008\n\t" \
  "s_add_u32 s90, s62, 1839104\n\t" \
  "buffer_load_dwordx4 a[128:131], v84, s[44:47], s89 offen offset:0 sc0 sc1\n\t" \
  "s_waitcnt vmcnt(56)\n\t" \
  "s_barrier\n\t" \
  "s_waitcnt lgkmcnt(4)\n\t" \
  "v_mfma_f32_16x16x128_f8f6f4 v[224:227], v[112:119], a[32:39], v[224:227]\n\t" \
  "ds_read_b64_tr_b8 v[96:97], v72 offset:16384\n\t" \
  "ds_read_b64_tr_b8 v[98:99], v72 offset:17408\n\t" \
  "v_mfma_f32_16x16x128_f8f6f4 v[228:231], v[112:119], a[40:47], v[228:231]\n\t" \
  "ds_read_b64_tr_b8 v[100:101], v72 offset:24576\n\t" \
  "ds_read_b64_tr_b8 v[102:103], v72 offset:25600\n\t" \
  "buffer_load_dwordx4 a[132:135], v84, s[44:47], s89 offen offset:1024 sc0 sc1\n\t" \
  "v_mfma_f32_16x16x128_f8f6f4 v[232:235], v[112:119], a[48:55], v[232:235]\n\t" \
  "buffer_load_dwordx4 a[136:139], v84, s[44:47], s89 offen offset:2048 sc0 sc1\n\t" \
  "v_mfma_f32_16x16x128_f8f6f4 v[236:239], v[112:119], a[56:63], v[236:239]\n\t" \
  "buffer_load_dwordx4 a[140:143], v84, s[44:47], s89 offen offset:3072 sc0 sc1\n\t" \
  "s_waitcnt lgkmcnt(4)\n\t" \
  "v_mfma_f32_16x16x128_f8f6f4 v[240:243], v[120:127], a[32:39], v[240:243]\n\t" \
  "ds_read_b64_tr_b8 v[104:105], v73 offset:16384\n\t" \
  "ds_read_b64_tr_b8 v[106:107], v73 offset:17408\n\t" \
  "v_mfma_f32_16x16x128_f8f6f4 v[244:247], v[120:127], a[40:47], v[244:247]\n\t" \
  "ds_read_b64_tr_b8 v[108:109], v73 offset:24576\n\t" \
  "ds_read_b64_tr_b8 v[110:111], v73 offset:25600\n\t" \
  "buffer_load_dwordx4 a[144:147], v84, s[44:47], s90 offen offset:0 sc0 sc1\n\t" \
  "v_mfma_f32_16x16x128_f8f6f4 v[248:251], v[120:127], a[48:55], v[248:251]\n\t" \
  "buffer_load_dwordx4 a[148:151], v84, s[44:47], s90 offen offset:1024 sc0 sc1\n\t" \
  "v_mfma_f32_16x16x128_f8f6f4 v[252:255], v[120:127], a[56:63], v[252:255]\n\t" \
  "buffer_load_dwordx4 a[152:155], v84, s[44:47], s90 offen offset:2048 sc0 sc1\n\t" \
  "s_waitcnt vmcnt(31)\n\t" \
  "s_bitcmp1_b32 s65, 31\n\t" \
  "s_cbranch_scc1 OK54_%=\n\t" \
  "s_mov_b32 s67, 31\n\t" \
  "s_call_b64 s[68:69], POLL_%=\n\t" \
  "OK54_%=:\n\t" \
  "global_load_dword v94, v88, s[98:99] sc1\n\t" \
  "s_waitcnt lgkmcnt(4)\n\t" \
  "v_mfma_f32_16x16x128_f8f6f4 v[128:131], v[96:103], a[64:71], v[128:131]\n\t" \
  "ds_read_b64_tr_b8 v[112:113], v74 offset:16384\n\t" \
  "ds_read_b64_tr_b8 v[114:115], v74 offset:17408\n\t" \
  "v_mfma_f32_16x16x128_f8f6f4 v[132:135], v[96:103], a[72:79], v[132:135]\n\t" \
  "ds_read_b64_tr_b8 v[116:117], v74 offset:24576\n\t" \
  "ds_read_b64_tr_b8 v[118:119], v74 offset:25600\n\t" \
  "buffer_load_dwordx4 a[156:159], v84, s[44:47], s90 offen offset:3072 sc0 sc1\n\t" \
  "v_mfma_f32_16x16x128_f8f6f4 v[136:139], v[96:103], a[80:87], v[136:139]\n\t" \
  "s_add_u32 s89, s62, 1900544\n\t" \
  "s_add_u32 s90, s62, 1904640\n\t" \
  "buffer_load_dwordx4 a[160:163], v84, s[44:47], s89 offen offset:0 sc0 sc1\n\t" \
  "v_mfma_f32_16x16x128_f8f6f4 v[140:143], v[96:103], a[88:95], v[140:143]\n\t" \
  "buffer_load_dwordx4 a[164:167], v84, s[44:47], s89 offen offset:1024 sc0 sc1\n\t" \
  "s_waitcnt lgkmcnt(4)\n\t" \
  "v_mfma_f32_16x16x128_f8f6f4 v[144:147], v[104:111], a[64:71], v[144:147]\n\t" \
  "ds_read_b64_tr_b8 v[120:121], v75 offset:16384\n\t" \
  "ds_read_b64_tr_b8 v[122:123], v75 offset:17408\n\t" \
  "v_mfma_f32_16x16x128_f8f6f4 v[148:151], v[104:111], a[72:79], v[148:151]\n\t" \
  "ds_read_b64_tr_b8 v[124:125], v75 offset:24576\n\t" \
  "ds_read_b64_tr_b8 v[126:127], v75 offset:25600\n\t" \
  "buffer_load_dwordx4 a[168:171], v84, s[44:47], s89 offen offset:2048 sc0 sc1\n\t" \
  "v_mfma_f32_16x16x128_f8f6f4 v[152:155], v[104:111], a[80:87], v[152:155]\n\t" \
  "buffer_load_dwordx4 a[172:175], v84, s[44:47], s89 offen offset:3072 sc0 sc1\n\t" \
  "v_mfma_f32_16x16x128_f8f6f4 v[156:159], v[104:111], a[88:95], v[156:159]\n\t" \
  "buffer_load_dwordx4 a[176:179], v84, s[44:47], s90 offen offset:0 sc0 sc1\n\t" \
  "s_waitcnt lgkmcnt(4)\n\t" \
  "v_mfma_f32_16x16x128_f8f6f4 v[160:163], v[112:119], a[64:71], v[160:163]\n\t" \
  "ds_read_b64_tr_b8 v[96:97], v76 offset:16384\n\t" \
  "ds_read_b64_tr_b8 v[98:99], v76 offset:17408\n\t" \
  "v_mfma_f32_16x16x128_f8f6f4 v[164:167], v[112:119], a[72:79], v[164:167]\n\t" \
  "ds_read_b64_tr_b8 v[100:101], v76 offset:24576\n\t" \
  "ds_read_b64_tr_b8 v[102:103], v76 offset:25600\n\t" \
  "buffer_load_dwordx4 a[180:183], v84, s[44:47], s90 offen offset:1024 sc0 sc1\n\t" \
  "v_mfma_f32_16x16x128_f8f6f4 v[168:171], v[112:119], a[80:87], v[168:171]\n\t" \
  "buffer_load_dwordx4 a[184:187], v84, s[44:47], s90 offen offset:2048 sc0 sc1\n\t" \
  "v_mfma_f32_16x16x128_f8f6f4 v[172:175], v[112:119], a[88:95], v[172:175]\n\t" \
  "buffer_load_dwordx4 a[188:191], v84, s[44:47], s90 offen offset:3072 sc0 sc1\n\t" \
  "s_waitcnt lgkmcnt(4)\n\t" \
  "v_mfma_f32_16x16x128_f8f6f4 v[176:179], v[120:127], a[64:71], v[176:179]\n\t" \
  "ds_read_b64_tr_b8 v[104:105], v77 offset:16384\n\t" \
  "ds_read_b64_tr_b8 v[106:107], v77 offset:17408\n\t" \
  "v_mfma_f32_16x16x128_f8f6f4 v[180:183], v[120:127], a[72:79], v[180:183]\n\t" \
  "ds_read_b64_tr_b8 v[108:109], v77 offset:24576\n\t" \
  "ds_read_b64_tr_b8 v[110:111], v77 offset:25600\n\t" \
  "s_add_u32 s89, s62, 1966080\n\t" \
  "s_add_u32 s90, s62, 1970176\n\t" \
  "buffer_load_dwordx4 a[192:195], v84, s[44:47], s89 offen offset:0 sc0 sc1\n\t" \
  "v_mfma_f32_16x16x128_f8f6f4 v[184:187], v[120:127], a[80:87], v[184:187]\n\t" \
  "buffer_load_dwordx4 a[196:199], v84, s[44:47], s89 offen offset:1024 sc0 sc1\n\t" \
  "v_mfma_f32_16x16x128_f8f6f4 v[188:191], v[120:127], a[88:95], v[188:191]\n\t" \
  "buffer_load_dwordx4 a[200:203], v84, s[44:47], s89 offen offset:2048 sc0 sc1\n\t" \
  "s_waitcnt lgkmcnt(4)\n\t" \
  "v_mfma_f32_16x16x128_f8f6f4 v[192:195], v[96:103], a[64:71], v[192:195]\n\t" \
  "ds_read_b64_tr_b8 v[112:113], v78 offset:16384\n\t" \
  "ds_read_b64_tr_b8 v[114:115], v78 offset:17408\n\t" \
  "v_mfma_f32_16x16x128_f8f6f4 v[196:199], v[96:103], a[72:79], v[196:199]\n\t" \
  "ds_read_b64_tr_b8 v[116:117], v78 offset:24576\n\t" \
  "ds_read_b64_tr_b8 v[118:119], v78 offset:25600\n\t" \
  "buffer_load_dwordx4 a[204:207], v84, s[44:47], s89 offen offset:3072 sc0 sc1\n\t" \
  "v_mfma_f32_16x16x128_f8f6f4 v[200:203], v[96:103], a[80:87], v[200:203]\n\t" \
  "buffer_load_dwordx4 a[208:211], v84, s[44:47], s90 offen offset:0 sc0 sc1\n\t" \
  "v_mfma_f32_16x16x128_f8f6f4 v[204:207], v[96:103], a[88:95], v[204:207]\n\t" \
  "buffer_load_dwordx4 a[212:215], v84, s[44:47], s90 offen offset:1024 sc0 sc1\n\t" \
  "s_waitcnt lgkmcnt(4)\n\t" \
  "v_mfma_f32_16x16x128_f8f6f4 v[208:211], v[104:111], a[64:71], v[208:211]\n\t" \
  "ds_read_b64_tr_b8 v[120:121], v79 offset:16384\n\t" \
  "ds_read_b64_tr_b8 v[122:123], v79 offset:17408\n\t" \
  "v_mfma_f32_16x16x128_f8f6f4 v[212:215], v[104:111], a[72:79], v[212:215]\n\t" \
  "ds_read_b64_tr_b8 v[124:125], v79 offset:24576\n\t" \
  "ds_read_b64_tr_b8 v[126:127], v79 offset:25600\n\t" \
  "buffer_load_dwordx4 a[216:219], v84, s[44:47], s90 offen offset:2048 sc0 sc1\n\t" \
  "v_mfma_f32_16x16x128_f8f6f4 v[216:219], v[104:111], a[80:87], v[216:219]\n\t" \
  "buffer_load_dwordx4 a[220:223], v84, s[44:47], s90 offen offset:3072 sc0 sc1\n\t" \
  "v_mfma_f32_16x16x128_f8f6f4 v[220:223], v[104:111], a[88:95], v[220:223]\n\t" \
  "s_add_u32 s89, s62, 2031616\n\t" \
  "s_add_u32 s90, s62, 2035712\n\t" \
  "buffer_load_dwordx4 a[224:227], v84, s[44:47], s89 offen offset:0 sc0 sc1\n\t" \
  "s_waitcnt vmcnt(63)\n\t" \
  "s_barrier\n\t" \
  "s_waitcnt lgkmcnt(4)\n\t" \
  "v_mfma_f32_16x16x128_f8f6f4 v[224:227], v[112:119], a[64:71], v[224:227]\n\t" \
  "ds_read_b64_tr_b8 v[96:97], v72 offset:32768\n\t" \
  "ds_read_b64_tr_b8 v[98:99], v72 offset:33792\n\t" \
  "v_mfma_f32_16x16x128_f8f6f4 v[228:231], v[112:119], a[72:79], v[228:231]\n\t" \
  "ds_read_b64_tr_b8 v[100:101], v72 offset:40960\n\t" \
  "ds_read_b64_tr_b8 v[102:103], v72 offset:41984\n\t" \
  "buffer_load_dwordx4 a[228:231], v84, s[44:47], s89 offen offset:1024 sc0 sc1\n\t" \
  "v_mfma_f32_16x16x128_f8f6f4 v[232:235], v[112:119], a[80:87], v[232:235]\n\t" \
  "buffer_load_dwordx4 a[232:235], v84, s[44:47], s89 offen offset:2048 sc0 sc1\n\t" \
  "v_mfma_f32_16x16x128_f8f6f4 v[236:239], v[112:119], a[88:95], v[236:239]\n\t" \
  "buffer_load_dwordx4 a[236:239], v84, s[44:47], s89 offen offset:3072 sc0 sc1\n\t" \
  "s_waitcnt lgkmcnt(4)\n\t" \
  "v_mfma_f32_16x16x128_f8f6f4 v[240:243], v[120:127], a[64:71], v[240:243]\n\t" \
  "ds_read_b64_tr_b8 v[104:105], v73 offset:32768\n\t" \
  "ds_read_b64_tr_b8 v[106:107], v73 offset:33792\n\t" \
  "v_mfma_f32_16x16x128_f8f6f4 v[244:247], v[120:127], a[72:79], v[244:247]\n\t" \
  "ds_read_b64_tr_b8 v[108:109], v73 offset:40960\n\t" \
  "ds_read_b64_tr_b8 v[110:111], v73 offset:41984\n\t" \
  "buffer_load_dwordx4 a[240:243], v84, s[44:47], s90 offen offset:0 sc0 sc1\n\t" \
  "v_mfma_f32_16x16x128_f8f6f4 v[248:251], v[120:127], a[80:87], v[248:251]\n\t" \
  "buffer_load_dwordx4 a[244:247], v84, s[44:47], s90 offen offset:1024 sc0 sc1\n\t" \
  "v_mfma_f32_16x16x128_f8f6f4 v[252:255], v[120:127], a[88:95], v[252:255]\n\t" \
  "buffer_load_dwordx4 a[248:251], v84, s[44:47], s90 offen offset:2048 sc0 sc1\n\t" \
  "s_waitcnt vmcnt(32)\n\t" \
  "s_waitcnt lgkmcnt(4)\n\t" \
  "v_mfma_f32_16x16x128_f8f6f4 v[128:131], v[96:103], a[96:103], v[128:131]\n\t" \
  "ds_read_b64_tr_b8 v[112:113], v74 offset:32768\n\t" \
  "ds_read_b64_tr_b8 v[114:115], v74 offset:33792\n\t" \
  "v_mfma_f32_16x16x128_f8f6f4 v[132:135], v[96:103], a[104:111], v[132:135]\n\t" \
  "ds_read_b64_tr_b8 v[116:117], v74 offset:40960\n\t" \
  "ds_read_b64_tr_b8 v[118:119], v74 offset:41984\n\t" \
  "buffer_load_dwordx4 a[252:255], v84, s[44:47], s90 offen offset:3072 sc0 sc1\n\t" \
  "v_mfma_f32_16x16x128_f8f6f4 v[136:139], v[96:103], a[112:119], v[136:139]\n\t" \
  "v_mfma_f32_16x16x128_f8f6f4 v[140:143], v[96:103], a[120:127], v[140:143]\n\t" \
  "s_waitcnt lgkmcnt(4)\n\t" \
  "v_mfma_f32_16x16x128_f8f6f4 v[144:147], v[104:111], a[96:103], v[144:147]\n\t" \
  "ds_read_b64_tr_b8 v[120:121], v75 offset:32768\n\t" \
  "ds_read_b64_tr_b8 v[122:123], v75 offset:33792\n\t" \
  "v_mfma_f32_16x16x128_f8f6f4 v[148:151], v[104:111], a[104:111], v[148:151]\n\t" \
  "ds_read_b64_tr_b8 v[124:125], v75 offset:40960\n\t" \
  "ds_read_b64_tr_b8 v[126:127], v75 offset:41984\n\t" \
  "v_mfma_f32_16x16x128_f8f6f4 v[152:155], v[104:111], a[112:119], v[152:155]\n\t" \
  "v_mfma_f32_16x16x128_f8f6f4 v[156:159], v[104:111], a[120:127], v[156:159]\n\t" \
  "s_waitcnt lgkmcnt(4)\n\t" \
  "v_mfma_f32_16x16x128_f8f6f4 v[160:163], v[112:119], a[96:103], v[160:163]\n\t" \
  "ds_read_b64_tr_b8 v[96:97], v76 offset:32768\n\t" \
  "ds_read_b64_tr_b8 v[98:99], v76 offset:33792\n\t" \
  "v_mfma_f32_16x16x128_f8f6f4 v[164:167], v[112:119], a[104:111], v[164:167]\n\t" \
  "ds_read_b64_tr_b8 v[100:101], v76 offset:40960\n\t" \
  "ds_read_b64_tr_b8 v[102:103], v76 offset:41984\n\t" \
  "v_mfma_f32_16x16x128_f8f6f4 v[168:171], v[112:119], a[112:119], v[168:171]\n\t" \
  "v_mfma_f32_16x16x128_f8f6f4 v[172:175], v[112:119], a[120:127], v[172:175]\n\t" \
  "s_waitcnt lgkmcnt(4)\n\t" \
  "v_mfma_f32_16x16x128_f8f6f4 v[176:179], v[120:127], a[96:103], v[176:179]\n\t" \
  "ds_read_b64_tr_b8 v[104:105], v77 offset:32768\n\t" \
  "ds_read_b64_tr_b8 v[106:107], v77 offset:33792\n\t" \
  "v_mfma_f32_16x16x128_f8f6f4 v[180:183], v[120:127], a[104:111], v[180:183]\n\t" \
  "ds_read_b64_tr_b8 v[108:109], v77 offset:40960\n\t" \
  "ds_read_b64_tr_b8 v[110:111], v77 offset:41984\n\t" \
  "v_mfma_f32_16x16x128_f8f6f4 v[184:187], v[120:127], a[112:119], v[184:187]\n\t" \
  "v_mfma_f32_16x16x128_f8f6f4 v[188:191], v[120:127], a[120:127], v[188:191]\n\t" \
  "s_waitcnt lgkmcnt(4)\n\t" \
  "v_mfma_f32_16x16x128_f8f6f4 v[192:195], v[96:103], a[96:103], v[192:195]\n\t" \
  "ds_read_b64_tr_b8 v[112:113], v78 offset:32768\n\t" \
  "ds_read_b64_tr_b8 v[114:115], v78 offset:33792\n\t" \
  "v_mfma_f32_16x16x128_f8f6f4 v[196:199], v[96:103], a[104:111], v[196:199]\n\t" \
  "ds_read_b64_tr_b8 v[116:117], v78 offset:40960\n\t" \
  "ds_read_b64_tr_b8 v[118:119], v78 offset:41984\n\t" \
  "v_mfma_f32_16x16x128_f8f6f4 v[200:203], v[96:103], a[112:119], v[200:203]\n\t" \
  "v_mfma_f32_16x16x128_f8f6f4 v[204:207], v[96:103], a[120:127], v[204:207]\n\t" \
  "s_waitcnt lgkmcnt(4)\n\t" \
  "v_mfma_f32_16x16x128_f8f6f4 v[208:211], v[104:111], a[96:103], v[208:211]\n\t" \
  "ds_read_b64_tr_b8 v[120:121], v79 offset:32768\n\t" \
  "ds_read_b64_tr_b8 v[122:123], v79 offset:33792\n\t" \
  "v_mfma_f32_16x16x128_f8f6f4 v[212:215], v[104:111], a[104:111], v[212:215]\n\t" \
  "ds_read_b64_tr_b8 v[124:125], v79 offset:40960\n\t" \
  "ds_read_b64_tr_b8 v[126:127], v79 offset:41984\n\t" \
  "v_mfma_f32_16x16x128_f8f6f4 v[216:219], v[104:111], a[112:119], v[216:219]\n\t" \
  "v_mfma_f32_16x16x128_f8f6f4 v[220:223], v[104:111], a[120:127], v[220:223]\n\t" \
  "s_waitcnt vmcnt(39)\n\t" \
  "s_barrier\n\t" \
  "s_waitcnt lgkmcnt(4)\n\t" \
  "v_mfma_f32_16x16x128_f8f6f4 v[224:227], v[112:119], a[96:103], v[224:227]\n\t" \
  "ds_read_b64_tr_b8 v[96:97], v72 offset:49152\n\t" \
  "ds_read_b64_tr_b8 v[98:99], v72 offset:50176\n\t" \
  "v_mfma_f32_16x16x128_f8f6f4 v[228:231], v[112:119], a[104:111], v[228:231]\n\t" \
  "ds_read_b64_tr_b8 v[100:101], v72 offset:57344\n\t" \
  "ds_read_b64_tr_b8 v[102:103], v72 offset:58368\n\t" \
  "v_mfma_f32_16x16x128_f8f6f4 v[232:235], v[112:119], a[112:119], v[232:235]\n\t" \
  "v_mfma_f32_16x16x128_f8f6f4 v[236:239], v[112:119], a[120:127], v[236:239]\n\t" \
  "s_waitcnt lgkmcnt(4)\n\t" \
  "v_mfma_f32_16x16x128_f8f6f4 v[240:243], v[120:127], a[96:103], v[240:243]\n\t" \
  "ds_read_b64_tr_b8 v[104:105], v16 offset:0\n\t" \
  "ds_read_b64_tr_b8 v[106:107], v16 offset:1024\n\t" \
  "v_mfma_f32_16x16x128_f8f6f4 v[244:247], v[120:127], a[104:111], v[244:247]\n\t" \
  "ds_read_b64_tr_b8 v[108:109], v16 offset:8192\n\t" \
  "ds_read_b64_tr_b8 v[110:111], v16 offset:9216\n\t" \
  "v_mfma_f32_16x16x128_f8f6f4 v[248:251], v[120:127], a[112:119], v[248:251]\n\t" \
  "v_mfma_f32_16x16x128_f8f6f4 v[252:255], v[120:127], a[120:127], v[252:255]\n\t" \
  "s_waitcnt vmcnt(25)\n\t" \
  "v_readfirstlane_b32 s80, v94\n\t" \
  "s_nop 3\n\t" \
  "s_cmp_eq_u32 s80, s64\n\t" \
  "s_cselect_b32 s80, 1, 0\n\t" \
  "s_or_b32 s66, s66, s80\n\t" \
  "s_cmp_lg_u32 s66, 0\n\t" \
  "s_cbranch_scc0 GOOD55_%=\n\t" \
  "s_branch BADOUT_%=\n\t" \
  "GOOD55_%=:\n\t" \
  "s_mov_b32 s84, 0\n\t" \
  "s_mov_b32 s85, 1048576\n\t" \
  "s_mov_b32 s86, 2097152\n\t" \
  "s_mov_b32 s87, 3145728\n\t" \
  "s_waitcnt vmcnt(24)\n\t" \
  "s_waitcnt lgkmcnt(4)\n\t" \
  "v_mfma_f32_16x16x128_f8f6f4 v[128:131], v[96:103], a[128:135], v[128:131]\n\t" \
  "ds_read_b64_tr_b8 v[112:113], v64 offset:49152\n\t" \
  "ds_read_b64_tr_b8 v[114:115], v64 offset:50176\n\t" \
  "v_mfma_f32_16x16x128_f8f6f4 v[132:135], v[96:103], a[136:143], v[132:135]\n\t" \
  "ds_read_b64_tr_b8 v[116:117], v64 offset:57344\n\t" \
  "ds_read_b64_tr_b8 v[118:119], v64 offset:58368\n\t" \
  "v_mfma_f32_16x16x128_f8f6f4 v[136:139], v[96:103], a[144:151], v[136:139]\n\t" \
  "v_mfma_f32_16x16x128_f8f6f4 v[140:143], v[96:103], a[152:159], v[140:143]\n\t" \
  "s_waitcnt vmcnt(16)\n\t" \
  "s_waitcnt lgkmcnt(4)\n\t" \
  "v_mfma_f32_16x16x128_f8f6f4 v[128:131], v[104:111], a[160:167], v[128:131]\n\t" \
  "ds_read_b64_tr_b8 v[120:121], v64 offset:0\n\t" \
  "ds_read_b64_tr_b8 v[122:123], v64 offset:1024\n\t" \
  "v_mfma_f32_16x16x128_f8f6f4 v[132:135], v[104:111], a[168:175], v[132:135]\n\t" \
  "ds_read_b64_tr_b8 v[124:125], v64 offset:8192\n\t" \
  "ds_read_b64_tr_b8 v[126:127], v64 offset:9216\n\t" \
  "v_mfma_f32_16x16x128_f8f6f4 v[136:139], v[104:111], a[176:183], v[136:139]\n\t" \
  "v_mfma_f32_16x16x128_f8f6f4 v[140:143], v[104:111], a[184:191], v[140:143]\n\t" \
  "s_waitcnt vmcnt(8)\n\t" \
  "s_waitcnt lgkmcnt(4)\n\t" \
  "v_mfma_f32_16x16x128_f8f6f4 v[128:131], v[112:119], a[192:199], v[128:131]\n\t" \
  "ds_read_b64_tr_b8 v[96:97], v73 offset:49152\n\t" \
  "ds_read_b64_tr_b8 v[98:99], v73 offset:50176\n\t" \
  "v_mfma_f32_16x16x128_f8f6f4 v[132:135], v[112:119], a[200:207], v[132:135]\n\t" \
  "ds_read_b64_tr_b8 v[100:101], v73 offset:57344\n\t" \
  "ds_read_b64_tr_b8 v[102:103], v73 offset:58368\n\t" \
  "v_mfma_f32_16x16x128_f8f6f4 v[136:139], v[112:119], a[208:215], v[136:139]\n\t" \
  "v_mfma_f32_16x16x128_f8f6f4 v[140:143], v[112:119], a[216:223], v[140:143]\n\t" \
  "s_waitcnt vmcnt(0)\n\t" \
  "s_waitcnt lgkmcnt(4)\n\t" \
  "v_mfma_f32_16x16x128_f8f6f4 v[128:131], v[120:127], a[224:231], v[128:131]\n\t" \
  "ds_read_b64_tr_b8 v[104:105], v17 offset:0\n\t" \
  "ds_read_b64_tr_b8 v[106:107], v17 offset:1024\n\t" \
  "v_mfma_f32_16x16x128_f8f6f4 v[132:135], v[120:127], a[232:239], v[132:135]\n\t" \
  "ds_read_b64_tr_b8 v[108:109], v17 offset:8192\n\t" \
  "ds_read_b64_tr_b8 v[110:111], v17 offset:9216\n\t" \
  "v_mfma_f32_16x16x128_f8f6f4 v[136:139], v[120:127], a[240:247], v[136:139]\n\t" \
  "v_mfma_f32_16x16x128_f8f6f4 v[140:143], v[120:127], a[248:255], v[140:143]\n\t" \
  "s_waitcnt lgkmcnt(4)\n\t" \
  "v_mfma_f32_16x16x128_f8f6f4 v[144:147], v[96:103], a[128:135], v[144:147]\n\t" \
  "ds_read_b64_tr_b8 v[112:113], v65 offset:49152\n\t" \
  "ds_read_b64_tr_b8 v[114:115], v65 offset:50176\n\t" \
  "v_mfma_f32_16x16x128_f8f6f4 v[148:151], v[96:103], a[136:143], v[148:151]\n\t" \
  "ds_read_b64_tr_b8 v[116:117], v65 offset:57344\n\t" \
  "ds_read_b64_tr_b8 v[118:119], v65 offset:58368\n\t" \
  "v_mfma_f32_16x16x128_f8f6f4 v[152:155], v[96:103], a[144:151], v[152:155]\n\t" \
  "v_mfma_f32_16x16x128_f8f6f4 v[156:159], v[96:103], a[152:159], v[156:159]\n\t" \
  "v_pk_mul_f32 v[128:129], v[128:129], s[78:79]\n\t" \
  "v_pk_mul_f32 v[130:131], v[130:131], s[78:79]\n\t" \
  "v_max_f32 v128, 0, v128\n\t" \
  "s_waitcnt lgkmcnt(4)\n\t" \
  "v_mfma_f32_16x16x128_f8f6f4 v[144:147], v[104:111], a[160:167], v[144:147]\n\t" \
  "ds_read_b64_tr_b8 v[120:121], v65 offset:0\n\t" \
  "ds_read_b64_tr_b8 v[122:123], v65 offset:1024\n\t" \
  "v_max_f32 v129, 0, v129\n\t" \
  "v_max_f32 v130, 0, v130\n\t" \
  "v_max_f32 v131, 0, v131\n\t" \
  "v_mfma_f32_16x16x128_f8f6f4 v[148:151], v[104:111], a[168:175], v[148:151]\n\t" \
  "ds_read_b64_tr_b8 v[124:125], v65 offset:8192\n\t" \
  "ds_read_b64_tr_b8 v[126:127], v65 offset:9216\n\t" \
  "buffer_store_dwordx4 v[128:131], v85, s[48:51], s84 offen offset:0 sc1\n\t" \
  "v_pk_mul_f32 v[132:133], v[132:133], s[78:79]\n\t" \
  "v_mfma_f32_16x16x128_f8f6f4 v[152:155], v[104:111], a[176:183], v[152:155]\n\t" \
  "v_pk_mul_f32 v[134:135], v[134:135], s[78:79]\n\t" \
  "v_max_f32 v132, 0, v132\n\t" \
  "v_mfma_f32_16x16x128_f8f6f4 v[156:159], v[104:111], a[184:191], v[156:159]\n\t" \
  "v_max_f32 v133, 0, v133\n\t" \
  "v_max_f32 v134, 0, v134\n\t" \
  "s_waitcnt lgkmcnt(4)\n\t" \
  "v_mfma_f32_16x16x128_f8f6f4 v[144:147], v[112:119], a[192:199], v[144:147]\n\t" \
  "ds_read_b64_tr_b8 v[96:97], v74 offset:49152\n\t" \
  "ds_read_b64_tr_b8 v[98:99], v74 offset:50176\n\t" \
  "v_max_f32 v135, 0, v135\n\t" \
  "buffer_store_dwordx4 v[132:135], v85, s[48:51], s85 offen offset:0 sc1\n\t" \
  "v_mfma_f32_16x16x128_f8f6f4 v[148:151], v[112:119], a[200:207], v[148:151]\n\t" \
  "ds_read_b64_tr_b8 v[100:101], v74 offset:57344\n\t" \
  "ds_read_b64_tr_b8 v[102:103], v74 offset:58368\n\t" \
  "v_pk_mul_f32 v[136:137], v[136:137], s[78:79]\n\t" \
  "v_pk_mul_f32 v[138:139], v[138:139], s[78:79]\n\t" \
  "v_mfma_f32_16x16x128_f8f6f4 v[152:155], v[112:119], a[208:215], v[152:155]\n\t" \
  "v_max_f32 v136, 0, v136\n\t" \
  "v_max_f32 v137, 0, v137\n\t" \
  "v_mfma_f32_16x16x128_f8f6f4 v[156:159], v[112:119], a[216:223], v[156:159]\n\t" \
  "v_max_f32 v138, 0, v138\n\t" \
  "v_max_f32 v139, 0, v139\n\t" \
  "s_waitcnt lgkmcnt(4)\n\t" \
  "v_mfma_f32_16x16x128_f8f6f4 v[144:147], v[120:127], a[224:231], v[144:147]\n\t" \
  "ds_read_b64_tr_b8 v[104:105], v18 offset:0\n\t" \
  "ds_read_b64_tr_b8 v[106:107], v18 offset:1024\n\t" \
  "buffer_store_dwordx4 v[136:139], v85, s[48:51], s86 offen offset:0 sc1\n\t" \
  "v_pk_mul_f32 v[140:141], v[140:141], s[78:79]\n\t" \
  "v_mfma_f32_16x16x128_f8f6f4 v[148:151], v[120:127], a[232:239], v[148:151]\n\t" \
  "ds_read_b64_tr_b8 v[108:109], v18 offset:8192\n\t" \
  "ds_read_b64_tr_b8 v[110:111], v18 offset:9216\n\t" \
  "v_pk_mul_f32 v[142:143], v[142:143], s[78:79]\n\t" \
  "v_max_f32 v140, 0, v140\n\t" \
  "v_mfma_f32_16x16x128_f8f6f4 v[152:155], v[120:127], a[240:247], v[152:155]\n\t" \
  "v_max_f32 v141, 0, v141\n\t" \
  "v_max_f32 v142, 0, v142\n\t" \
  "v_mfma_f32_16x16x128_f8f6f4 v[156:159], v[120:127], a[248:255], v[156:159]\n\t" \
  "v_max_f32 v143, 0, v143\n\t" \
  "buffer_store_dwordx4 v[140:143], v85, s[48:51], s87 offen offset:0 sc1\n\t" \
  "s_waitcnt lgkmcnt(4)\n\t" \
  "v_mfma_f32_16x16x128_f8f6f4 v[160:163], v[96:103], a[128:135], v[160:163]\n\t" \
  "ds_read_b64_tr_b8 v[112:113], v66 offset:49152\n\t" \
  "ds_read_b64_tr_b8 v[114:115], v66 offset:50176\n\t" \
  "v_mfma_f32_16x16x128_f8f6f4 v[164:167], v[96:103], a[136:143], v[164:167]\n\t" \
  "ds_read_b64_tr_b8 v[116:117], v66 offset:57344\n\t" \
  "ds_read_b64_tr_b8 v[118:119], v66 offset:58368\n\t" \
  "v_mfma_f32_16x16x128_f8f6f4 v[168:171], v[96:103], a[144:151], v[168:171]\n\t" \
  "v_mfma_f32_16x16x128_f8f6f4 v[172:175], v[96:103], a[152:159], v[172:175]\n\t" \
  "v_pk_mul_f32 v[144:145], v[144:145], s[78:79]\n\t" \
  "v_pk_mul_f32 v[146:147], v[146:147], s[78:79]\n\t" \
  "v_max_f32 v144, 0, v144\n\t" \
  "s_waitcnt lgkmcnt(4)\n\t" \
  "v_mfma_f32_16x16x128_f8f6f4 v[160:163], v[104:111], a[160:167], v[160:163]\n\t" \
  "ds_read_b64_tr_b8 v[120:121], v66 offset:0\n\t" \
  "ds_read_b64_tr_b8 v[122:123], v66 offset:1024\n\t" \
  "v_max_f32 v145, 0, v145\n\t" \
  "v_max_f32 v146, 0, v146\n\t" \
  "v_max_f32 v147, 0, v147\n\t" \
  "v_mfma_f32_16x16x128_f8f6f4 v[164:167], v[104:111], a[168:175], v[164:167]\n\t" \
  "ds_read_b64_tr_b8 v[124:125], v66 offset:8192\n\t" \
  "ds_read_b64_tr_b8 v[126:127], v66 offset:9216\n\t" \
  "buffer_store_dwordx4 v[144:147], v85, s[48:51], s84 offen offset:64 sc1\n\t" \
  "v_pk_mul_f32 v[148:149], v[148:149], s[78:79]\n\t" \
  "v_mfma_f32_16x16x128_f8f6f4 v[168:171], v[104:111], a[176:183], v[168:171]\n\t" \
  "v_pk_mul_f32 v[150:151], v[150:151], s[78:79]\n\t" \
  "v_max_f32 v148, 0, v148\n\t" \
  "v_mfma_f32_16x16x128_f8f6f4 v[172:175], v[104:111], a[184:191], v[172:175]\n\t" \
  "v_max_f32 v149, 0, v149\n\t" \
  "v_max_f32 v150, 0, v150\n\t" \
  "s_waitcnt lgkmcnt(4)\n\t" \
  "v_mfma_f32_16x16x128_f8f6f4 v[160:163], v[112:119], a[192:199], v[160:163]\n\t" \
  "ds_read_b64_tr_b8 v[96:97], v75 offset:49152\n\t" \
  "ds_read_b64_tr_b8 v[98:99], v75 offset:50176\n\t" \
  "v_max_f32 v151, 0, v151\n\t" \
  "buffer_store_dwordx4 v[148:151], v85, s[48:51], s85 offen offset:64 sc1\n\t" \
  "v_mfma_f32_16x16x128_f8f6f4 v[164:167], v[112:119], a[200:207], v[164:167]\n\t" \
  "ds_read_b64_tr_b8 v[100:101], v75 offset:57344\n\t" \
  "ds_read_b64_tr_b8 v[102:103], v75 offset:58368\n\t" \
  "v_pk_mul_f32 v[152:153], v[152:153], s[78:79]\n\t" \
  "v_pk_mul_f32 v[154:155], v[154:155], s[78:79]\n\t" \
  "v_mfma_f32_16x16x128_f8f6f4 v[168:171], v[112:119], a[208:215], v[168:171]\n\t" \
  "v_max_f32 v152, 0, v152\n\t" \
  "v_max_f32 v153, 0, v153\n\t" \
  "v_mfma_f32_16x16x128_f8f6f4 v[172:175], v[112:119], a[216:223], v[172:175]\n\t" \
  "v_max_f32 v154, 0, v154\n\t" \
  "v_max_f32 v155, 0, v155\n\t" \
  "s_waitcnt lgkmcnt(4)\n\t" \
  "v_mfma_f32_16x16x128_f8f6f4 v[160:163], v[120:127], a[224:231], v[160:163]\n\t" \
  "ds_read_b64_tr_b8 v[104:105], v19 offset:0\n\t" \
  "ds_read_b64_tr_b8 v[106:107], v19 offset:1024\n\t" \
  "buffer_store_dwordx4 v[152:155], v85, s[48:51], s86 offen offset:64 sc1\n\t" \
  "v_pk_mul_f32 v[156:157], v[156:157], s[78:79]\n\t" \
  "v_mfma_f32_16x16x128_f8f6f4 v[164:167], v[120:127], a[232:239], v[164:167]\n\t" \
  "ds_read_b64_tr_b8 v[108:109], v19 offset:8192\n\t" \
  "ds_read_b64_tr_b8 v[110:111], v19 offset:9216\n\t" \
  "v_pk_mul_f32 v[158:159], v[158:159], s[78:79]\n\t" \
  "v_max_f32 v156, 0, v156\n\t" \
  "v_mfma_f32_16x16x128_f8f6f4 v[168:171], v[120:127], a[240:247], v[168:171]\n\t" \
  "v_max_f32 v157, 0, v157\n\t" \
  "v_max_f32 v158, 0, v158\n\t" \
  "v_mfma_f32_16x16x128_f8f6f4 v[172:175], v[120:127], a[248:255], v[172:175]\n\t" \
  "v_max_f32 v159, 0, v159\n\t" \
  "buffer_store_dwordx4 v[156:159], v85, s[48:51], s87 offen offset:64 sc1\n\t" \
  "s_waitcnt lgkmcnt(4)\n\t" \
  "v_mfma_f32_16x16x128_f8f6f4 v[176:179], v[96:103], a[128:135], v[176:179]\n\t" \
  "ds_read_b64_tr_b8 v[112:113], v67 offset:49152\n\t" \
  "ds_read_b64_tr_b8 v[114:115], v67 offset:50176\n\t" \
  "v_mfma_f32_16x16x128_f8f6f4 v[180:183], v[96:103], a[136:143], v[180:183]\n\t" \
  "ds_read_b64_tr_b8 v[116:117], v67 offset:57344\n\t" \
  "ds_read_b64_tr_b8 v[118:119], v67 offset:58368\n\t" \
  "v_mfma_f32_16x16x128_f8f6f4 v[184:187], v[96:103], a[144:151], v[184:187]\n\t" \
  "v_mfma_f32_16x16x128_f8f6f4 v[188:191], v[96:103], a[152:159], v[188:191]\n\t" \
  "v_pk_mul_f32 v[160:161], v[160:161], s[78:79]\n\t" \
  "v_pk_mul_f32 v[162:163], v[162:163], s[78:79]\n\t" \
  "v_max_f32 v160, 0, v160\n\t" \
  "s_waitcnt lgkmcnt(4)\n\t" \
  "v_mfma_f32_16x16x128_f8f6f4 v[176:179], v[104:111], a[160:167], v[176:179]\n\t" \
  "ds_read_b64_tr_b8 v[120:121], v67 offset:0\n\t" \
  "ds_read_b64_tr_b8 v[122:123], v67 offset:1024\n\t" \
  "v_max_f32 v161, 0, v161\n\t" \
  "v_max_f32 v162, 0, v162\n\t" \
  "v_max_f32 v163, 0, v163\n\t" \
  "v_mfma_f32_16x16x128_f8f6f4 v[180:183], v[104:111], a[168:175], v[180:183]\n\t" \
  "ds_read_b64_tr_b8 v[124:125], v67 offset:8192\n\t" \
  "ds_read_b64_tr_b8 v[126:127], v67 offset:9216\n\t" \
  "buffer_store_dwordx4 v[160:163], v85, s[48:51], s84 offen offset:128 sc1\n\t" \
  "v_pk_mul_f32 v[164:165], v[164:165], s[78:79]\n\t" \
  "v_mfma_f32_16x16x128_f8f6f4 v[184:187], v[104:111], a[176:183], v[184:187]\n\t" \
  "v_pk_mul_f32 v[166:167], v[166:167], s[78:79]\n\t" \
  "v_max_f32 v164, 0, v164\n\t" \
  "v_mfma_f32_16x16x128_f8f6f4 v[188:191], v[104:111], a[184:191], v[188:191]\n\t" \
  "v_max_f32 v165, 0, v165\n\t" \
  "v_max_f32 v166, 0, v166\n\t" \
  "s_waitcnt lgkmcnt(4)\n\t" \
  "v_mfma_f32_16x16x128_f8f6f4 v[176:179], v[112:119], a[192:199], v[176:179]\n\t" \
  "ds_read_b64_tr_b8 v[96:97], v76 offset:49152\n\t" \
  "ds_read_b64_tr_b8 v[98:99], v76 offset:50176\n\t" \
  "v_max_f32 v167, 0, v167\n\t" \
  "buffer_store_dwordx4 v[164:167], v85, s[48:51], s85 offen offset:128 sc1\n\t" \
  "v_mfma_f32_16x16x128_f8f6f4 v[180:183], v[112:119], a[200:207], v[180:183]\n\t" \
  "ds_read_b64_tr_b8 v[100:101], v76 offset:57344\n\t" \
  "ds_read_b64_tr_b8 v[102:103], v76 offset:58368\n\t" \
  "v_pk_mul_f32 v[168:169], v[168:169], s[78:79]\n\t" \
  "v_pk_mul_f32 v[170:171], v[170:171], s[78:79]\n\t" \
  "v_mfma_f32_16x16x128_f8f6f4 v[184:187], v[112:119], a[208:215], v[184:187]\n\t" \
  "v_max_f32 v168, 0, v168\n\t" \
  "v_max_f32 v169, 0, v169\n\t" \
  "v_mfma_f32_16x16x128_f8f6f4 v[188:191], v[112:119], a[216:223], v[188:191]\n\t" \
  "v_max_f32 v170, 0, v170\n\t" \
  "v_max_f32 v171, 0, v171\n\t" \
  "s_waitcnt lgkmcnt(4)\n\t" \
  "v_mfma_f32_16x16x128_f8f6f4 v[176:179], v[120:127], a[224:231], v[176:179]\n\t" \
  "ds_read_b64_tr_b8 v[104:105], v20 offset:0\n\t" \
  "ds_read_b64_tr_b8 v[106:107], v20 offset:1024\n\t" \
  "buffer_store_dwordx4 v[168:171], v85, s[48:51], s86 offen offset:128 sc1\n\t" \
  "v_pk_mul_f32 v[172:173], v[172:173], s[78:79]\n\t" \
  "v_mfma_f32_16x16x128_f8f6f4 v[180:183], v[120:127], a[232:239], v[180:183]\n\t" \
  "ds_read_b64_tr_b8 v[108:109], v20 offset:8192\n\t" \
  "ds_read_b64_tr_b8 v[110:111], v20 offset:9216\n\t" \
  "v_pk_mul_f32 v[174:175], v[174:175], s[78:79]\n\t" \
  "v_max_f32 v172, 0, v172\n\t" \
  "v_mfma_f32_16x16x128_f8f6f4 v[184:187], v[120:127], a[240:247], v[184:187]\n\t" \
  "v_max_f32 v173, 0, v173\n\t" \
  "v_max_f32 v174, 0, v174\n\t" \
  "v_mfma_f32_16x16x128_f8f6f4 v[188:191], v[120:127], a[248:255], v[188:191]\n\t" \
  "v_max_f32 v175, 0, v175\n\t" \
  "buffer_store_dwordx4 v[172:175], v85, s[48:51], s87 offen offset:128 sc1\n\t" \
  "s_waitcnt lgkmcnt(4)\n\t" \
  "v_mfma_f32_16x16x128_f8f6f4 v[192:195], v[96:103], a[128:135], v[192:195]\n\t" \
  "ds_read_b64_tr_b8 v[112:113], v68 offset:49152\n\t" \
  "ds_read_b64_tr_b8 v[114:115], v68 offset:50176\n\t" \
  "v_mfma_f32_16x16x128_f8f6f4 v[196:199], v[96:103], a[136:143], v[196:199]\n\t" \
  "ds_read_b64_tr_b8 v[116:117], v68 offset:57344\n\t" \
  "ds_read_b64_tr_b8 v[118:119], v68 offset:58368\n\t" \
  "v_mfma_f32_16x16x128_f8f6f4 v[200:203], v[96:103], a[144:151], v[200:203]\n\t" \
  "v_mfma_f32_16x16x128_f8f6f4 v[204:207], v[96:103], a[152:159], v[204:207]\n\t" \
  "v_pk_mul_f32 v[176:177], v[176:177], s[78:79]\n\t" \
  "v_pk_mul_f32 v[178:179], v[178:179], s[78:79]\n\t" \
  "v_max_f32 v176, 0, v176\n\t" \
  "s_waitcnt lgkmcnt(4)\n\t" \
  "v_mfma_f32_16x16x128_f8f6f4 v[192:195], v[104:111], a[160:167], v[192:195]\n\t" \
  "ds_read_b64_tr_b8 v[120:121], v68 offset:0\n\t" \
  "ds_read_b64_tr_b8 v[122:123], v68 offset:1024\n\t" \
  "v_max_f32 v177, 0, v177\n\t" \
  "v_max_f32 v178, 0, v178\n\t" \
  "v_max_f32 v179, 0, v179\n\t" \
  "v_mfma_f32_16x16x128_f8f6f4 v[196:199], v[104:111], a[168:175], v[196:199]\n\t" \
  "ds_read_b64_tr_b8 v[124:125], v68 offset:8192\n\t" \
  "ds_read_b64_tr_b8 v[126:127], v68 offset:9216\n\t" \
  "buffer_store_dwordx4 v[176:179], v85, s[48:51], s84 offen offset:192 sc1\n\t" \
  "v_pk_mul_f32 v[180:181], v[180:181], s[78:79]\n\t" \
  "v_mfma_f32_16x16x128_f8f6f4 v[200:203], v[104:111], a[176:183], v[200:203]\n\t" \
  "v_pk_mul_f32 v[182:183], v[182:183], s[78:79]\n\t" \
  "v_max_f32 v180, 0, v180\n\t" \
  "v_mfma_f32_16x16x128_f8f6f4 v[204:207], v[104:111], a[184:191], v[204:207]\n\t" \
  "v_max_f32 v181, 0, v181\n\t" \
  "v_max_f32 v182, 0, v182\n\t" \
  "s_waitcnt lgkmcnt(4)\n\t" \
  "v_mfma_f32_16x16x128_f8f6f4 v[192:195], v[112:119], a[192:199], v[192:195]\n\t" \
  "ds_read_b64_tr_b8 v[96:97], v77 offset:49152\n\t" \
  "ds_read_b64_tr_b8 v[98:99], v77 offset:50176\n\t" \
  "v_max_f32 v183, 0, v183\n\t" \
  "buffer_store_dwordx4 v[180:183], v85, s[48:51], s85 offen offset:192 sc1\n\t" \
  "v_mfma_f32_16x16x128_f8f6f4 v[196:199], v[112:119], a[200:207], v[196:199]\n\t" \
  "ds_read_b64_tr_b8 v[100:101], v77 offset:57344\n\t" \
  "ds_read_b64_tr_b8 v[102:103], v77 offset:58368\n\t" \
  "v_pk_mul_f32 v[184:185], v[184:185], s[78:79]\n\t" \
  "v_pk_mul_f32 v[186:187], v[186:187], s[78:79]\n\t" \
  "v_mfma_f32_16x16x128_f8f6f4 v[200:203], v[112:119], a[208:215], v[200:203]\n\t" \
  "v_max_f32 v184, 0, v184\n\t" \
  "v_max_f32 v185, 0, v185\n\t" \
  "v_mfma_f32_16x16x128_f8f6f4 v[204:207], v[112:119], a[216:223], v[204:207]\n\t" \
  "v_max_f32 v186, 0, v186\n\t" \
  "v_max_f32 v187, 0, v187\n\t" \
  "s_waitcnt lgkmcnt(4)\n\t" \
  "v_mfma_f32_16x16x128_f8f6f4 v[192:195], v[120:127], a[224:231], v[192:195]\n\t" \
  "ds_read_b64_tr_b8 v[104:105], v21 offset:0\n\t" \
  "ds_read_b64_tr_b8 v[106:107], v21 offset:1024\n\t" \
  "buffer_store_dwordx4 v[184:187], v85, s[48:51], s86 offen offset:192 sc1\n\t" \
  "v_pk_mul_f32 v[188:189], v[188:189], s[78:79]\n\t" \
  "v_mfma_f32_16x16x128_f8f6f4 v[196:199], v[120:127], a[232:239], v[196:199]\n\t" \
  "ds_read_b64_tr_b8 v[108:109], v21 offset:8192\n\t" \
  "ds_read_b64_tr_b8 v[110:111], v21 offset:9216\n\t" \
  "v_pk_mul_f32 v[190:191], v[190:191], s[78:79]\n\t" \
  "v_max_f32 v188, 0, v188\n\t" \
  "v_mfma_f32_16x16x128_f8f6f4 v[200:203], v[120:127], a[240:247], v[200:203]\n\t" \
  "v_max_f32 v189, 0, v189\n\t" \
  "v_max_f32 v190, 0, v190\n\t" \
  "v_mfma_f32_16x16x128_f8f6f4 v[204:207], v[120:127], a[248:255], v[204:207]\n\t" \
  "v_max_f32 v191, 0, v191\n\t" \
  "buffer_store_dwordx4 v[188:191], v85, s[48:51], s87 offen offset:192 sc1\n\t" \
  "s_waitcnt lgkmcnt(4)\n\t" \
  "v_mfma_f32_16x16x128_f8f6f4 v[208:211], v[96:103], a[128:135], v[208:211]\n\t" \
  "ds_read_b64_tr_b8 v[112:113], v69 offset:49152\n\t" \
  "ds_read_b64_tr_b8 v[114:115], v69 offset:50176\n\t" \
  "v_mfma_f32_16x16x128_f8f6f4 v[212:215], v[96:103], a[136:143], v[212:215]\n\t" \
  "ds_read_b64_tr_b8 v[116:117], v69 offset:57344\n\t" \
  "ds_read_b64_tr_b8 v[118:119], v69 offset:58368\n\t" \
  "v_mfma_f32_16x16x128_f8f6f4 v[216:219], v[96:103], a[144:151], v[216:219]\n\t" \
  "v_mfma_f32_16x16x128_f8f6f4 v[220:223], v[96:103], a[152:159], v[220:223]\n\t" \
  "v_pk_mul_f32 v[192:193], v[192:193], s[78:79]\n\t" \
  "v_pk_mul_f32 v[194:195], v[194:195], s[78:79]\n\t" \
  "v_max_f32 v192, 0, v192\n\t" \
  "s_waitcnt lgkmcnt(4)\n\t" \
  "v_mfma_f32_16x16x128_f8f6f4 v[208:211], v[104:111], a[160:167], v[208:211]\n\t" \
  "ds_read_b64_tr_b8 v[120:121], v69 offset:0\n\t" \
  "ds_read_b64_tr_b8 v[122:123], v69 offset:1024\n\t" \
  "v_max_f32 v193, 0, v193\n\t" \
  "v_max_f32 v194, 0, v194\n\t" \
  "v_max_f32 v195, 0, v195\n\t" \
  "v_mfma_f32_16x16x128_f8f6f4 v[212:215], v[104:111], a[168:175], v[212:215]\n\t" \
  "ds_read_b64_tr_b8 v[124:125], v69 offset:8192\n\t" \
  "ds_read_b64_tr_b8 v[126:127], v69 offset:9216\n\t" \
  "buffer_store_dwordx4 v[192:195], v85, s[48:51], s84 offen offset:256 sc1\n\t" \
  "v_pk_mul_f32 v[196:197], v[196:197], s[78:79]\n\t" \
  "v_mfma_f32_16x16x128_f8f6f4 v[216:219], v[104:111], a[176:183], v[216:219]\n\t" \
  "v_pk_mul_f32 v[198:199], v[198:199], s[78:79]\n\t" \
  "v_max_f32 v196, 0, v196\n\t" \
  "v_mfma_f32_16x16x128_f8f6f4 v[220:223], v[104:111], a[184:191], v[220:223]\n\t" \
  "v_max_f32 v197, 0, v197\n\t" \
  "v_max_f32 v198, 0, v198\n\t" \
  "s_waitcnt lgkmcnt(4)\n\t" \
  "v_mfma_f32_16x16x128_f8f6f4 v[208:211], v[112:119], a[192:199], v[208:211]\n\t" \
  "ds_read_b64_tr_b8 v[96:97], v78 offset:49152\n\t" \
  "ds_read_b64_tr_b8 v[98:99], v78 offset:50176\n\t" \
  "v_max_f32 v199, 0, v199\n\t" \
  "buffer_store_dwordx4 v[196:199], v85, s[48:51], s85 offen offset:256 sc1\n\t" \
  "v_mfma_f32_16x16x128_f8f6f4 v[212:215], v[112:119], a[200:207], v[212:215]\n\t" \
  "ds_read_b64_tr_b8 v[100:101], v78 offset:57344\n\t" \
  "ds_read_b64_tr_b8 v[102:103], v78 offset:58368\n\t" \
  "v_pk_mul_f32 v[200:201], v[200:201], s[78:79]\n\t" \
  "v_pk_mul_f32 v[202:203], v[202:203], s[78:79]\n\t" \
  "v_mfma_f32_16x16x128_f8f6f4 v[216:219], v[112:119], a[208:215], v[216:219]\n\t" \
  "v_max_f32 v200, 0, v200\n\t" \
  "v_max_f32 v201, 0, v201\n\t" \
  "v_mfma_f32_16x16x128_f8f6f4 v[220:223], v[112:119], a[216:223], v[220:223]\n\t" \
  "v_max_f32 v202, 0, v202\n\t" \
  "v_max_f32 v203, 0, v203\n\t" \
  "s_waitcnt lgkmcnt(4)\n\t" \
  "v_mfma_f32_16x16x128_f8f6f4 v[208:211], v[120:127], a[224:231], v[208:211]\n\t" \
  "ds_read_b64_tr_b8 v[104:105], v22 offset:0\n\t" \
  "ds_read_b64_tr_b8 v[106:107], v22 offset:1024\n\t" \
  "buffer_store_dwordx4 v[200:203], v85, s[48:51], s86 offen offset:256 sc1\n\t" \
  "v_pk_mul_f32 v[204:205], v[204:205], s[78:79]\n\t" \
  "v_mfma_f32_16x16x128_f8f6f4 v[212:215], v[120:127], a[232:239], v[212:215]\n\t" \
  "ds_read_b64_tr_b8 v[108:109], v22 offset:8192\n\t" \
  "ds_read_b64_tr_b8 v[110:111], v22 offset:9216\n\t" \
  "v_pk_mul_f32 v[206:207], v[206:207], s[78:79]\n\t" \
  "v_max_f32 v204, 0, v204\n\t" \
  "v_mfma_f32_16x16x128_f8f6f4 v[216:219], v[120:127], a[240:247], v[216:219]\n\t" \
  "v_max_f32 v205, 0, v205\n\t" \
  "v_max_f32 v206, 0, v206\n\t" \
  "v_mfma_f32_16x16x128_f8f6f4 v[220:223], v[120:127], a[248:255], v[220:223]\n\t" \
  "v_max_f32 v207, 0, v207\n\t" \
  "buffer_store_dwordx4 v[204:207], v85, s[48:51], s87 offen offset:256 sc1\n\t" \
  "s_waitcnt lgkmcnt(4)\n\t" \
  "v_mfma_f32_16x16x128_f8f6f4 v[224:227], v[96:103], a[128:135], v[224:227]\n\t" \
  "ds_read_b64_tr_b8 v[112:113], v70 offset:49152\n\t" \
  "ds_read_b64_tr_b8 v[114:115], v70 offset:50176\n\t" \
  "v_mfma_f32_16x16x128_f8f6f4 v[228:231], v[96:103], a[136:143], v[228:231]\n\t" \
  "ds_read_b64_tr_b8 v[116:117], v70 offset:57344\n\t" \
  "ds_read_b64_tr_b8 v[118:119], v70 offset:58368\n\t" \
  "v_mfma_f32_16x16x128_f8f6f4 v[232:235], v[96:103], a[144:151], v[232:235]\n\t" \
  "v_mfma_f32_16x16x128_f8f6f4 v[236:239], v[96:103], a[152:159], v[236:239]\n\t" \
  "v_pk_mul_f32 v[208:209], v[208:209], s[78:79]\n\t" \
  "v_pk_mul_f32 v[210:211], v[210:211], s[78:79]\n\t" \
  "v_max_f32 v208, 0, v208\n\t" \
  "s_waitcnt lgkmcnt(4)\n\t" \
  "v_mfma_f32_16x16x128_f8f6f4 v[224:227], v[104:111], a[160:167], v[224:227]\n\t" \
  "ds_read_b64_tr_b8 v[120:121], v70 offset:0\n\t" \
  "ds_read_b64_tr_b8 v[122:123], v70 offset:1024\n\t" \
  "v_max_f32 v209, 0, v209\n\t" \
  "v_max_f32 v210, 0, v210\n\t" \
  "v_max_f32 v211, 0, v211\n\t" \
  "v_mfma_f32_16x16x128_f8f6f4 v[228:231], v[104:111], a[168:175], v[228:231]\n\t" \
  "ds_read_b64_tr_b8 v[124:125], v70 offset:8192\n\t" \
  "ds_read_b64_tr_b8 v[126:127], v70 offset:9216\n\t" \
  "buffer_store_dwordx4 v[208:211], v85, s[48:51], s84 offen offset:320 sc1\n\t" \
  "v_pk_mul_f32 v[212:213], v[212:213], s[78:79]\n\t" \
  "v_mfma_f32_16x16x128_f8f6f4 v[232:235], v[104:111], a[176:183], v[232:235]\n\t" \
  "v_pk_mul_f32 v[214:215], v[214:215], s[78:79]\n\t" \
  "v_max_f32 v212, 0, v212\n\t" \
  "v_mfma_f32_16x16x128_f8f6f4 v[236:239], v[104:111], a[184:191], v[236:239]\n\t" \
  "v_max_f32 v213, 0, v213\n\t" \
  "v_max_f32 v214, 0, v214\n\t" \
  "s_waitcnt lgkmcnt(4)\n\t" \
  "v_mfma_f32_16x16x128_f8f6f4 v[224:227], v[112:119], a[192:199], v[224:227]\n\t" \
  "ds_read_b64_tr_b8 v[96:97], v79 offset:49152\n\t" \
  "ds_read_b64_tr_b8 v[98:99], v79 offset:50176\n\t" \
  "v_max_f32 v215, 0, v215\n\t" \
  "buffer_store_dwordx4 v[212:215], v85, s[48:51], s85 offen offset:320 sc1\n\t" \
  "v_mfma_f32_16x16x128_f8f6f4 v[228:231], v[112:119], a[200:207], v[228:231]\n\t" \
  "ds_read_b64_tr_b8 v[100:101], v79 offset:57344\n\t" \
  "ds_read_b64_tr_b8 v[102:103], v79 offset:58368\n\t" \
  "v_pk_mul_f32 v[216:217], v[216:217], s[78:79]\n\t" \
  "v_pk_mul_f32 v[218:219], v[218:219], s[78:79]\n\t" \
  "v_mfma_f32_16x16x128_f8f6f4 v[232:235], v[112:119], a[208:215], v[232:235]\n\t" \
  "v_max_f32 v216, 0, v216\n\t" \
  "v_max_f32 v217, 0, v217\n\t" \
  "v_mfma_f32_16x16x128_f8f6f4 v[236:239], v[112:119], a[216:223], v[236:239]\n\t" \
  "v_max_f32 v218, 0, v218\n\t" \
  "v_max_f32 v219, 0, v219\n\t" \
  "s_waitcnt lgkmcnt(4)\n\t" \
  "v_mfma_f32_16x16x128_f8f6f4 v[224:227], v[120:127], a[224:231], v[224:227]\n\t" \
  "ds_read_b64_tr_b8 v[104:105], v23 offset:0\n\t" \
  "ds_read_b64_tr_b8 v[106:107], v23 offset:1024\n\t" \
  "buffer_store_dwordx4 v[216:219], v85, s[48:51], s86 offen offset:320 sc1\n\t" \
  "v_pk_mul_f32 v[220:221], v[220:221], s[78:79]\n\t" \
  "v_mfma_f32_16x16x128_f8f6f4 v[228:231], v[120:127], a[232:239], v[228:231]\n\t" \
  "ds_read_b64_tr_b8 v[108:109], v23 offset:8192\n\t" \
  "ds_read_b64_tr_b8 v[110:111], v23 offset:9216\n\t" \
  "v_pk_mul_f32 v[222:223], v[222:223], s[78:79]\n\t" \
  "v_max_f32 v220, 0, v220\n\t" \
  "v_mfma_f32_16x16x128_f8f6f4 v[232:235], v[120:127], a[240:247], v[232:235]\n\t" \
  "v_max_f32 v221, 0, v221\n\t" \
  "v_max_f32 v222, 0, v222\n\t" \
  "v_mfma_f32_16x16x128_f8f6f4 v[236:239], v[120:127], a[248:255], v[236:239]\n\t" \
  "v_max_f32 v223, 0, v223\n\t" \
  "buffer_store_dwordx4 v[220:223], v85, s[48:51], s87 offen offset:320 sc1\n\t" \
  "s_waitcnt lgkmcnt(4)\n\t" \
  "v_mfma_f32_16x16x128_f8f6f4 v[240:243], v[96:103], a[128:135], v[240:243]\n\t" \
  "ds_read_b64_tr_b8 v[112:113], v71 offset:49152\n\t" \
  "ds_read_b64_tr_b8 v[114:115], v71 offset:50176\n\t" \
  "v_mfma_f32_16x16x128_f8f6f4 v[244:247], v[96:103], a[136:143], v[244:247]\n\t" \
  "ds_read_b64_tr_b8 v[116:117], v71 offset:57344\n\t" \
  "ds_read_b64_tr_b8 v[118:119], v71 offset:58368\n\t" \
  "v_mfma_f32_16x16x128_f8f6f4 v[248:251], v[96:103], a[144:151], v[248:251]\n\t" \
  "v_mfma_f32_16x16x128_f8f6f4 v[252:255], v[96:103], a[152:159], v[252:255]\n\t" \
  "v_pk_mul_f32 v[224:225], v[224:225], s[78:79]\n\t" \
  "v_pk_mul_f32 v[226:227], v[226:227], s[78:79]\n\t" \
  "v_max_f32 v224, 0, v224\n\t" \
  "s_waitcnt lgkmcnt(4)\n\t" \
  "v_mfma_f32_16x16x128_f8f6f4 v[240:243], v[104:111], a[160:167], v[240:243]\n\t" \
  "ds_read_b64_tr_b8 v[120:121], v71 offset:0\n\t" \
  "ds_read_b64_tr_b8 v[122:123], v71 offset:1024\n\t" \
  "v_max_f32 v225, 0, v225\n\t" \
  "v_max_f32 v226, 0, v226\n\t" \
  "v_max_f32 v227, 0, v227\n\t" \
  "v_mfma_f32_16x16x128_f8f6f4 v[244:247], v[104:111], a[168:175], v[244:247]\n\t" \
  "ds_read_b64_tr_b8 v[124:125], v71 offset:8192\n\t" \
  "ds_read_b64_tr_b8 v[126:127], v71 offset:9216\n\t" \
  "buffer_store_dwordx4 v[224:227], v85, s[48:51], s84 offen offset:384 sc1\n\t" \
  "v_pk_mul_f32 v[228:229], v[228:229], s[78:79]\n\t" \
  "v_mfma_f32_16x16x128_f8f6f4 v[248:251], v[104:111], a[176:183], v[248:251]\n\t" \
  "v_pk_mul_f32 v[230:231], v[230:231], s[78:79]\n\t" \
  "v_max_f32 v228, 0, v228\n\t" \
  "v_mfma_f32_16x16x128_f8f6f4 v[252:255], v[104:111], a[184:191], v[252:255]\n\t" \
  "v_max_f32 v229, 0, v229\n\t" \
  "v_max_f32 v230, 0, v230\n\t" \
  "s_waitcnt lgkmcnt(4)\n\t" \
  "v_mfma_f32_16x16x128_f8f6f4 v[240:243], v[112:119], a[192:199], v[240:243]\n\t" \
  "v_max_f32 v231, 0, v231\n\t" \
  "buffer_store_dwordx4 v[228:231], v85, s[48:51], s85 offen offset:384 sc1\n\t" \
  "v_mfma_f32_16x16x128_f8f6f4 v[244:247], v[112:119], a[200:207], v[244:247]\n\t" \
  "v_pk_mul_f32 v[232:233], v[232:233], s[78:79]\n\t" \
  "v_pk_mul_f32 v[234:235], v[234:235], s[78:79]\n\t" \
  "v_mfma_f32_16x16x128_f8f6f4 v[248:251], v[112:119], a[208:215], v[248:251]\n\t" \
  "v_max_f32 v232, 0, v232\n\t" \
  "v_max_f32 v233, 0, v233\n\t" \
  "v_mfma_f32_16x16x128_f8f6f4 v[252:255], v[112:119], a[216:223], v[252:255]\n\t" \
  "v_max_f32 v234, 0, v234\n\t" \
  "v_max_f32 v235, 0, v235\n\t" \
  "s_waitcnt lgkmcnt(0)\n\t" \
  "v_mfma_f32_16x16x128_f8f6f4 v[240:243], v[120:127], a[224:231], v[240:243]\n\t" \
  "buffer_store_dwordx4 v[232:235], v85, s[48:51], s86 offen offset:384 sc1\n\t" \
  "v_pk_mul_f32 v[236:237], v[236:237], s[78:79]\n\t" \
  "v_mfma_f32_16x16x128_f8f6f4 v[244:247], v[120:127], a[232:239], v[244:247]\n\t" \
  "v_pk_mul_f32 v[238:239], v[238:239], s[78:79]\n\t" \
  "v_max_f32 v236, 0, v236\n\t" \
  "v_mfma_f32_16x16x128_f8f6f4 v[248:251], v[120:127], a[240:247], v[248:251]\n\t" \
  "v_max_f32 v237, 0, v237\n\t" \
  "v_max_f32 v238, 0, v238\n\t" \
  "v_mfma_f32_16x16x128_f8f6f4 v[252:255], v[120:127], a[248:255], v[252:255]\n\t" \
  "v_max_f32 v239, 0, v239\n\t" \
  "buffer_store_dwordx4 v[236:239], v85, s[48:51], s87 offen offset:384 sc1\n\t" \
  "s_nop 15\n\t" \
  "v_pk_mul_f32 v[240:241], v[240:241], s[78:79]\n\t" \
  "v_pk_mul_f32 v[242:243], v[242:243], s[78:79]\n\t" \
  "v_max_f32 v240, 0, v240\n\t" \
  "v_max_f32 v241, 0, v241\n\t" \
  "v_max_f32 v242, 0, v242\n\t" \
  "v_max_f32 v243, 0, v243\n\t" \
  "buffer_store_dwordx4 v[240:243], v85, s[48:51], s84 offen offset:448 sc1\n\t" \
  "v_pk_mul_f32 v[244:245], v[244:245], s[78:79]\n\t" \
  "v_pk_mul_f32 v[246:247], v[246:247], s[78:79]\n\t" \
  "v_max_f32 v244, 0, v244\n\t" \
  "v_max_f32 v245, 0, v245\n\t" \
  "v_max_f32 v246, 0, v246\n\t" \
  "v_max_f32 v247, 0, v247\n\t" \
  "buffer_store_dwordx4 v[244:247], v85, s[48:51], s85 offen offset:448 sc1\n\t" \
  "v_pk_mul_f32 v[248:249], v[248:249], s[78:79]\n\t" \
  "v_pk_mul_f32 v[250:251], v[250:251], s[78:79]\n\t" \
  "v_max_f32 v248, 0, v248\n\t" \
  "v_max_f32 v249, 0, v249\n\t" \
  "v_max_f32 v250, 0, v250\n\t" \
  "v_max_f32 v251, 0, v251\n\t" \
  "buffer_store_dwordx4 v[248:251], v85, s[48:51], s86 offen offset:448 sc1\n\t" \
  "v_pk_mul_f32 v[252:253], v[252:253], s[78:79]\n\t" \
  "v_pk_mul_f32 v[254:255], v[254:255], s[78:79]\n\t" \
  "v_max_f32 v252, 0, v252\n\t" \
  "v_max_f32 v253, 0, v253\n\t" \
  "v_max_f32 v254, 0, v254\n\t" \
  "v_max_f32 v255, 0, v255\n\t" \
  "buffer_store_dwordx4 v[252:255], v85, s[48:51], s87 offen offset:448 sc1\n\t" \
  "s_branch FIN_%=\n\t" \
  "VER1_%=:\n\t" \
  "s_bitcmp1_b32 s65, 4\n\t" \
  "s_cbranch_scc1 OK56_%=\n\t" \
  "s_mov_b32 s67, 4\n\t" \
  "s_call_b64 s[68:69], POLL_%=\n\t" \
  "OK56_%=:\n\t" \
  "s_add_u32 s89, s62, 262144\n\t" \
  "s_add_u32 s90, s62, 266240\n\t" \
  "buffer_load_dwordx4 a[128:131], v84, s[44:47], s89 offen offset:0 sc0 sc1\n\t" \
  "buffer_load_dwordx4 a[132:135], v84, s[44:47], s89 offen offset:1024 sc0 sc1\n\t" \
  "buffer_load_dwordx4 a[136:139], v84, s[44:47], s89 offen offset:2048 sc0 sc1\n\t" \
  "buffer_load_dwordx4 a[140:143], v84, s[44:47], s89 offen offset:3072 sc0 sc1\n\t" \
  "buffer_load_dwordx4 a[144:147], v84, s[44:47], s90 offen offset:0 sc0 sc1\n\t" \
  "buffer_load_dwordx4 a[148:151], v84, s[44:47], s90 offen offset:1024 sc0 sc1\n\t" \
  "buffer_load_dwordx4 a[152:155], v84, s[44:47], s90 offen offset:2048 sc0 sc1\n\t" \
  "buffer_load_dwordx4 a[156:159], v84, s[44:47], s90 offen offset:3072 sc0 sc1\n\t" \
  "s_bitcmp1_b32 s65, 5\n\t" \
  "s_cbranch_scc1 OK57_%=\n\t" \
  "s_mov_b32 s67, 5\n\t" \
  "s_call_b64 s[68:69], POLL_%=\n\t" \
  "OK57_%=:\n\t" \
  "s_add_u32 s89, s62, 327680\n\t" \
  "s_add_u32 s90, s62, 331776\n\t" \
  "buffer_load_dwordx4 a[160:163], v84, s[44:47], s89 offen offset:0 sc0 sc1\n\t" \
  "buffer_load_dwordx4 a[164:167], v84, s[44:47], s89 offen offset:1024 sc0 sc1\n\t" \
  "buffer_load_dwordx4 a[168:171], v84, s[44:47], s89 offen offset:2048 sc0 sc1\n\t" \
  "buffer_load_dwordx4 a[172:175], v84, s[44:47], s89 offen offset:3072 sc0 sc1\n\t" \
  "buffer_load_dwordx4 a[176:179], v84, s[44:47], s90 offen offset:0 sc0 sc1\n\t" \
  "buffer_load_dwordx4 a[180:183], v84, s[44:47], s90 offen offset:1024 sc0 sc1\n\t" \
  "buffer_load_dwordx4 a[184:187], v84, s[44:47], s90 offen offset:2048 sc0 sc1\n\t" \
  "buffer_load_dwordx4 a[188:191], v84, s[44:47], s90 offen offset:3072 sc0 sc1\n\t" \
  "s_bitcmp1_b32 s65, 6\n\t" \
  "s_cbranch_scc1 OK58_%=\n\t" \
  "s_mov_b32 s67, 6\n\t" \
  "s_call_b64 s[68:69], POLL_%=\n\t" \
  "OK58_%=:\n\t" \
  "s_add_u32 s89, s62, 393216\n\t" \
  "s_add_u32 s90, s62, 397312\n\t" \
  "buffer_load_dwordx4 a[192:195], v84, s[44:47], s89 offen offset:0 sc0 sc1\n\t" \
  "buffer_load_dwordx4 a[196:199], v84, s[44:47], s89 offen offset:1024 sc0 sc1\n\t" \
  "buffer_load_dwordx4 a[200:203], v84, s[44:47], s89 offen offset:2048 sc0 sc1\n\t" \
  "buffer_load_dwordx4 a[204:207], v84, s[44:47], s89 offen offset:3072 sc0 sc1\n\t" \
  "buffer_load_dwordx4 a[208:211], v84, s[44:47], s90 offen offset:0 sc0 sc1\n\t" \
  "buffer_load_dwordx4 a[212:215], v84, s[44:47], s90 offen offset:1024 sc0 sc1\n\t" \
  "buffer_load_dwordx4 a[216:219], v84, s[44:47], s90 offen offset:2048 sc0 sc1\n\t" \
  "buffer_load_dwordx4 a[220:223], v84, s[44:47], s90 offen offset:3072 sc0 sc1\n\t" \
  "s_waitcnt vmcnt(16)\n\t" \
  "s_bitcmp1_b32 s65, 7\n\t" \
  "s_cbranch_scc1 OK59_%=\n\t" \
  "s_mov_b32 s67, 7\n\t" \
  "s_call_b64 s[68:69], POLL_%=\n\t" \
  "OK59_%=:\n\t" \
  "s_bitcmp1_b32 s65, 8\n\t" \
  "s_cbranch_scc1 OK60_%=\n\t" \
  "s_mov_b32 s67, 8\n\t" \
  "s_call_b64 s[68:69], POLL_%=\n\t" \
  "OK60_%=:\n\t" \
  "s_waitcnt lgkmcnt(4)\n\t" \
  "v_mfma_f32_16x16x128_f8f6f4 v[128:131], v[96:103], a[128:135], v[128:131]\n\t" \
  "ds_read_b64_tr_b8 v[112:113], v66 offset:0\n\t" \
  "ds_read_b64_tr_b8 v[114:115], v66 offset:1024\n\t" \
  "v_mfma_f32_16x16x128_f8f6f4 v[132:135], v[96:103], a[136:143], v[132:135]\n\t" \
  "ds_read_b64_tr_b8 v[116:117], v66 offset:8192\n\t" \
  "ds_read_b64_tr_b8 v[118:119], v66 offset:9216\n\t" \
  "s_add_u32 s89, s62, 458752\n\t" \
  "s_add_u32 s90, s62, 462848\n\t" \
  "buffer_load_dwordx4 a[224:227], v84, s[44:47], s89 offen offset:0 sc0 sc1\n\t" \
  "v_mfma_f32_16x16x128_f8f6f4 v[136:139], v[96:103], a[144:151], v[136:139]\n\t" \
  "buffer_load_dwordx4 a[228:231], v84, s[44:47], s89 offen offset:1024 sc0 sc1\n\t" \
  "v_mfma_f32_16x16x128_f8f6f4 v[140:143], v[96:103], a[152:159], v[140:143]\n\t" \
  "s_waitcnt lgkmcnt(4)\n\t" \
  "v_mfma_f32_16x16x128_f8f6f4 v[144:147], v[104:111], a[128:135], v[144:147]\n\t" \
  "ds_read_b64_tr_b8 v[120:121], v67 offset:0\n\t" \
  "ds_read_b64_tr_b8 v[122:123], v67 offset:1024\n\t" \
  "v_mfma_f32_16x16x128_f8f6f4 v[148:151], v[104:111], a[136:143], v[148:151]\n\t" \
  "ds_read_b64_tr_b8 v[124:125], v67 offset:8192\n\t" \
  "ds_read_b64_tr_b8 v[126:127], v67 offset:9216\n\t" \
  "buffer_load_dwordx4 a[232:235], v84, s[44:47], s89 offen offset:2048 sc0 sc1\n\t" \
  "v_mfma_f32_16x16x128_f8f6f4 v[152:155], v[104:111], a[144:151], v[152:155]\n\t" \
  "buffer_load_dwordx4 a[236:239], v84, s[44:47], s89 offen offset:3072 sc0 sc1\n\t" \
  "v_mfma_f32_16x16x128_f8f6f4 v[156:159], v[104:111], a[152:159], v[156:159]\n\t" \
  "s_waitcnt lgkmcnt(4)\n\t" \
  "v_mfma_f32_16x16x128_f8f6f4 v[160:163], v[112:119], a[128:135], v[160:163]\n\t" \
  "ds_read_b64_tr_b8 v[96:97], v68 offset:0\n\t" \
  "ds_read_b64_tr_b8 v[98:99], v68 offset:1024\n\t" \
  "v_mfma_f32_16x16x128_f8f6f4 v[164:167], v[112:119], a[136:143], v[164:167]\n\t" \
  "ds_read_b64_tr_b8 v[100:101], v68 offset:8192\n\t" \
  "ds_read_b64_tr_b8 v[102:103], v68 offset:9216\n\t" \
  "buffer_load_dwordx4 a[240:243], v84, s[44:47], s90 offen offset:0 sc0 sc1\n\t" \
  "v_mfma_f32_16x16x128_f8f6f4 v[168:171], v[112:119], a[144:151], v[168:171]\n\t" \
  "buffer_load_dwordx4 a[244:247], v84, s[44:47], s90 offen offset:1024 sc0 sc1\n\t" \
  "v_mfma_f32_16x16x128_f8f6f4 v[172:175], v[112:119], a[152:159], v[172:175]\n\t" \
  "s_waitcnt lgkmcnt(4)\n\t" \
  "v_mfma_f32_16x16x128_f8f6f4 v[176:179], v[120:127], a[128:135], v[176:179]\n\t" \
  "ds_read_b64_tr_b8 v[104:105], v69 offset:0\n\t" \
  "ds_read_b64_tr_b8 v[106:107], v69 offset:1024\n\t" \
  "v_mfma_f32_16x16x128_f8f6f4 v[180:183], v[120:127], a[136:143], v[180:183]\n\t" \
  "ds_read_b64_tr_b8 v[108:109], v69 offset:8192\n\t" \
  "ds_read_b64_tr_b8 v[110:111], v69 offset:9216\n\t" \
  "buffer_load_dwordx4 a[248:251], v84, s[44:47], s90 offen offset:2048 sc0 sc1\n\t" \
  "v_mfma_f32_16x16x128_f8f6f4 v[184:187], v[120:127], a[144:151], v[184:187]\n\t" \
  "buffer_load_dwordx4 a[252:255], v84, s[44:47], s90 offen offset:3072 sc0 sc1\n\t" \
  "v_mfma_f32_16x16x128_f8f6f4 v[188:191], v[120:127], a[152:159], v[188:191]\n\t" \
  "s_waitcnt lgkmcnt(4)\n\t" \
  "v_mfma_f32_16x16x128_f8f6f4 v[192:195], v[96:103], a[128:135], v[192:195]\n\t" \
  "ds_read_b64_tr_b8 v[112:113], v70 offset:0\n\t" \
  "ds_read_b64_tr_b8 v[114:115], v70 offset:1024\n\t" \
  "v_mfma_f32_16x16x128_f8f6f4 v[196:199], v[96:103], a[136:143], v[196:199]\n\t" \
  "ds_read_b64_tr_b8 v[116:117], v70 offset:8192\n\t" \
  "ds_read_b64_tr_b8 v[118:119], v70 offset:9216\n\t" \
  "s_add_u32 s89, s62, 524288\n\t" \
  "s_add_u32 s90, s62, 528384\n\t" \
  "buffer_load_dwordx4 a[0:3], v84, s[44:47], s89 offen offset:0 sc0 sc1\n\t" \
  "v_mfma_f32_16x16x128_f8f6f4 v[200:203], v[96:103], a[144:151], v[200:203]\n\t" \
  "buffer_load_dwordx4 a[4:7], v84, s[44:47], s89 offen offset:1024 sc0 sc1\n\t" \
  "v_mfma_f32_16x16x128_f8f6f4 v[204:207], v[96:103], a[152:159], v[204:207]\n\t" \
  "s_waitcnt lgkmcnt(4)\n\t" \
  "v_mfma_f32_16x16x128_f8f6f4 v[208:211], v[104:111], a[128:135], v[208:211]\n\t" \
  "ds_read_b64_tr_b8 v[120:121], v71 offset:0\n\t" \
  "ds_read_b64_tr_b8 v[122:123], v71 offset:1024\n\t" \
  "v_mfma_f32_16x16x128_f8f6f4 v[212:215], v[104:111], a[136:143], v[212:215]\n\t" \
  "ds_read_b64_tr_b8 v[124:125], v71 offset:8192\n\t" \
  "ds_read_b64_tr_b8 v[126:127], v71 offset:9216\n\t" \
  "buffer_load_dwordx4 a[8:11], v84, s[44:47], s89 offen offset:2048 sc0 sc1\n\t" \
  "v_mfma_f32_16x16x128_f8f6f4 v[216:219], v[104:111], a[144:151], v[216:219]\n\t" \
  "buffer_load_dwordx4 a[12:15], v84, s[44:47], s89 offen offset:3072 sc0 sc1\n\t" \
  "v_mfma_f32_16x16x128_f8f6f4 v[220:223], v[104:111], a[152:159], v[220:223]\n\t" \
  "s_waitcnt vmcnt(40)\n\t" \
  "s_barrier\n\t" \
  "s_waitcnt lgkmcnt(4)\n\t" \
  "v_mfma_f32_16x16x128_f8f6f4 v[224:227], v[112:119], a[128:135], v[224:227]\n\t" \
  "ds_read_b64_tr_b8 v[96:97], v64 offset:16384\n\t" \
  "ds_read_b64_tr_b8 v[98:99], v64 offset:17408\n\t" \
  "v_mfma_f32_16x16x128_f8f6f4 v[228:231], v[112:119], a[136:143], v[228:231]\n\t" \
  "ds_read_b64_tr_b8 v[100:101], v64 offset:24576\n\t" \
  "ds_read_b64_tr_b8 v[102:103], v64 offset:25600\n\t" \
  "buffer_load_dwordx4 a[16:19], v84, s[44:47], s90 offen offset:0 sc0 sc1\n\t" \
  "v_mfma_f32_16x16x128_f8f6f4 v[232:235], v[112:119], a[144:151], v[232:235]\n\t" \
  "buffer_load_dwordx4 a[20:23], v84, s[44:47], s90 offen offset:1024 sc0 sc1\n\t" \
  "v_mfma_f32_16x16x128_f8f6f4 v[236:239], v[112:119], a[152:159], v[236:239]\n\t" \
  "s_waitcnt lgkmcnt(4)\n\t" \
  "v_mfma_f32_16x16x128_f8f6f4 v[240:243], v[120:127], a[128:135], v[240:243]\n\t" \
  "ds_read_b64_tr_b8 v[104:105], v65 offset:16384\n\t" \
  "ds_read_b64_tr_b8 v[106:107], v65 offset:17408\n\t" \
  "v_mfma_f32_16x16x128_f8f6f4 v[244:247], v[120:127], a[136:143], v[244:247]\n\t" \
  "ds_read_b64_tr_b8 v[108:109], v65 offset:24576\n\t" \
  "ds_read_b64_tr_b8 v[110:111], v65 offset:25600\n\t" \
  "buffer_load_dwordx4 a[24:27], v84, s[44:47], s90 offen offset:2048 sc0 sc1\n\t" \
  "v_mfma_f32_16x16x128_f8f6f4 v[248:251], v[120:127], a[144:151], v[248:251]\n\t" \
  "buffer_load_dwordx4 a[28:31], v84, s[44:47], s90 offen offset:3072 sc0 sc1\n\t" \
  "v_mfma_f32_16x16x128_f8f6f4 v[252:255], v[120:127], a[152:159], v[252:255]\n\t" \
  "s_waitcnt vmcnt(24)\n\t" \
  "s_bitcmp1_b32 s65, 9\n\t" \
  "s_cbranch_scc1 OK61_%=\n\t" \
  "s_mov_b32 s67, 9\n\t" \
  "s_call_b64 s[68:69], POLL_%=\n\t" \
  "OK61_%=:\n\t" \
  "s_waitcnt lgkmcnt(4)\n\t" \
  "v_mfma_f32_16x16x128_f8f6f4 v[128:131], v[96:103], a[160:167], v[128:131]\n\t" \
  "ds_read_b64_tr_b8 v[112:113], v66 offset:16384\n\t" \
  "ds_read_b64_tr_b8 v[114:115], v66 offset:17408\n\t" \
  "v_mfma_f32_16x16x128_f8f6f4 v[132:135], v[96:103], a[168:175], v[132:135]\n\t" \
  "ds_read_b64_tr_b8 v[116:117], v66 offset:24576\n\t" \
  "ds_read_b64_tr_b8 v[118:119], v66 offset:25600\n\t" \
  "s_add_u32 s89, s62, 589824\n\t" \
  "s_add_u32 s90, s62, 593920\n\t" \
  "buffer_load_dwordx4 a[32:35], v84, s[44:47], s89 offen offset:0 sc0 sc1\n\t" \
  "v_mfma_f32_16x16x128_f8f6f4 v[136:139], v[96:103], a[176:183], v[136:139]\n\t" \
  "buffer_load_dwordx4 a[36:39], v84, s[44:47], s89 offen offset:1024 sc0 sc1\n\t" \
  "v_mfma_f32_16x16x128_f8f6f4 v[140:143], v[96:103], a[184:191], v[140:143]\n\t" \
  "s_waitcnt lgkmcnt(4)\n\t" \
  "v_mfma_f32_16x16x128_f8f6f4 v[144:147], v[104:111], a[160:167], v[144:147]\n\t" \
  "ds_read_b64_tr_b8 v[120:121], v67 offset:16384\n\t" \
  "ds_read_b64_tr_b8 v[122:123], v67 offset:17408\n\t" \
  "v_mfma_f32_16x16x128_f8f6f4 v[148:151], v[104:111], a[168:175], v[148:151]\n\t" \
  "ds_read_b64_tr_b8 v[124:125], v67 offset:24576\n\t" \
  "ds_read_b64_tr_b8 v[126:127], v67 offset:25600\n\t" \
  "buffer_load_dwordx4 a[40:43], v84, s[44:47], s89 offen offset:2048 sc0 sc1\n\t" \
  "v_mfma_f32_16x16x128_f8f6f4 v[152:155], v[104:111], a[176:183], v[152:155]\n\t" \
  "buffer_load_dwordx4 a[44:47], v84, s[44:47], s89 offen offset:3072 sc0 sc1\n\t" \
  "v_mfma_f32_16x16x128_f8f6f4 v[156:159], v[104:111], a[184:191], v[156:159]\n\t" \
  "s_waitcnt lgkmcnt(4)\n\t" \
  "v_mfma_f32_16x16x128_f8f6f4 v[160:163], v[112:119], a[160:167], v[160:163]\n\t" \
  "ds_read_b64_tr_b8 v[96:97], v68 offset:16384\n\t" \
  "ds_read_b64_tr_b8 v[98:99], v68 offset:17408\n\t" \
  "v_mfma_f32_16x16x128_f8f6f4 v[164:167], v[112:119], a[168:175], v[164:167]\n\t" \
  "ds_read_b64_tr_b8 v[100:101], v68 offset:24576\n\t" \
  "ds_read_b64_tr_b8 v[102:103], v68 offset:25600\n\t" \
  "buffer_load_dwordx4 a[48:51], v84, s[44:47], s90 offen offset:0 sc0 sc1\n\t" \
  "v_mfma_f32_16x16x128_f8f6f4 v[168:171], v[112:119], a[176:183], v[168:171]\n\t" \
  "buffer_load_dwordx4 a[52:55], v84, s[44:47], s90 offen offset:1024 sc0 sc1\n\t" \
  "v_mfma_f32_16x16x128_f8f6f4 v[172:175], v[112:119], a[184:191], v[172:175]\n\t" \
  "s_waitcnt lgkmcnt(4)\n\t" \
  "v_mfma_f32_16x16x128_f8f6f4 v[176:179], v[120:127], a[160:167], v[176:179]\n\t" \
  "ds_read_b64_tr_b8 v[104:105], v69 offset:16384\n\t" \
  "ds_read_b64_tr_b8 v[106:107], v69 offset:17408\n\t" \
  "v_mfma_f32_16x16x128_f8f6f4 v[180:183], v[120:127], a[168:175], v[180:183]\n\t" \
  "ds_read_b64_tr_b8 v[108:109], v69 offset:24576\n\t" \
  "ds_read_b64_tr_b8 v[110:111], v69 offset:25600\n\t" \
  "buffer_load_dwordx4 a[56:59], v84, s[44:47], s90 offen offset:2048 sc0 sc1\n\t" \
  "v_mfma_f32_16x16x128_f8f6f4 v[184:187], v[120:127], a[176:183], v[184:187]\n\t" \
  "buffer_load_dwordx4 a[60:63], v84, s[44:47], s90 offen offset:3072 sc0 sc1\n\t" \
  "v_mfma_f32_16x16x128_f8f6f4 v[188:191], v[120:127], a[184:191], v[188:191]\n\t" \
  "s_waitcnt lgkmcnt(4)\n\t" \
  "v_mfma_f32_16x16x128_f8f6f4 v[192:195], v[96:103], a[160:167], v[192:195]\n\t" \
  "ds_read_b64_tr_b8 v[112:113], v70 offset:16384\n\t" \
  "ds_read_b64_tr_b8 v[114:115], v70 offset:17408\n\t" \
  "v_mfma_f32_16x16x128_f8f6f4 v[196:199], v[96:103], a[168:175], v[196:199]\n\t" \
  "ds_read_b64_tr_b8 v[116:117], v70 offset:24576\n\t" \
  "ds_read_b64_tr_b8 v[118:119], v70 offset:25600\n\t" \
  "s_add_u32 s88, s55, 0\n\t" \
  "s_add_u32 m0, s60, 65536\n\t" \
  "s_nop 0\n\t" \
  "buffer_load_dwordx4 v80, s[36:39], s88 offen  lds\n\t" \
  "v_mfma_f32_16x16x128_f8f6f4 v[200:203], v[96:103], a[176:183], v[200:203]\n\t" \
  "s_add_u32 m0, s60, 66560\n\t" \
  "s_nop 0\n\t" \
  "buffer_load_dwordx4 v81, s[36:39], s88 offen  lds\n\t" \
  "v_mfma_f32_16x16x128_f8f6f4 v[204:207], v[96:103], a[184:191], v[204:207]\n\t" \
  "s_waitcnt lgkmcnt(4)\n\t" \
  "v_mfma_f32_16x16x128_f8f6f4 v[208:211], v[104:111], a[160:167], v[208:211]\n\t" \
  "ds_read_b64_tr_b8 v[120:121], v71 offset:16384\n\t" \
  "ds_read_b64_tr_b8 v[122:123], v71 offset:17408\n\t" \
  "v_mfma_f32_16x16x128_f8f6f4 v[212:215], v[104:111], a[168:175], v[212:215]\n\t" \
  "ds_read_b64_tr_b8 v[124:125], v71 offset:24576\n\t" \
  "ds_read_b64_tr_b8 v[126:127], v71 offset:25600\n\t" \
  "s_add_u32 m0, s60, 67584\n\t" \
  "s_nop 0\n\t" \
  "buffer_load_dwordx4 v82, s[36:39], s88 offen  lds\n\t" \
  "v_mfma_f32_16x16x128_f8f6f4 v[216:219], v[104:111], a[176:183], v[216:219]\n\t" \
  "s_add_u32 m0, s60, 68608\n\t" \
  "s_nop 0\n\t" \
  "buffer_load_dwordx4 v83, s[36:39], s88 offen  lds\n\t" \
  "v_mfma_f32_16x16x128_f8f6f4 v[220:223], v[104:111], a[184:191], v[220:223]\n\t" \
  "s_waitcnt vmcnt(52)\n\t" \
  "s_barrier\n\t" \
  "s_waitcnt lgkmcnt(4)\n\t" \
  "v_mfma_f32_16x16x128_f8f6f4 v[224:227], v[112:119], a[160:167], v[224:227]\n\t" \
  "ds_read_b64_tr_b8 v[96:97], v64 offset:32768\n\t" \
  "ds_read_b64_tr_b8 v[98:99], v64 offset:33792\n\t" \
  "v_mfma_f32_16x16x128_f8f6f4 v[228:231], v[112:119], a[168:175], v[228:231]\n\t" \
  "ds_read_b64_tr_b8 v[100:101], v64 offset:40960\n\t" \
  "ds_read_b64_tr_b8 v[102:103], v64 offset:41984\n\t" \
  "s_add_u32 s88, s56, 0\n\t" \
  "s_add_u32 m0, s60, 81920\n\t" \
  "s_nop 0\n\t" \
  "buffer_load_dwordx4 v80, s[36:39], s88 offen  lds\n\t" \
  "v_mfma_f32_16x16x128_f8f6f4 v[232:235], v[112:119], a[176:183], v[232:235]\n\t" \
  "s_add_u32 m0, s60, 82944\n\t" \
  "s_nop 0\n\t" \
  "buffer_load_dwordx4 v81, s[36:39], s88 offen  lds\n\t" \
  "v_mfma_f32_16x16x128_f8f6f4 v[236:239], v[112:119], a[184:191], v[236:239]\n\t" \
  "s_waitcnt lgkmcnt(4)\n\t" \
  "v_mfma_f32_16x16x128_f8f6f4 v[240:243], v[120:127], a[160:167], v[240:243]\n\t" \
  "ds_read_b64_tr_b8 v[104:105], v65 offset:32768\n\t" \
  "ds_read_b64_tr_b8 v[106:107], v65 offset:33792\n\t" \
  "v_mfma_f32_16x16x128_f8f6f4 v[244:247], v[120:127], a[168:175], v[244:247]\n\t" \
  "ds_read_b64_tr_b8 v[108:109], v65 offset:40960\n\t" \
  "ds_read_b64_tr_b8 v[110:111], v65 offset:41984\n\t" \
  "s_add_u32 m0, s60, 83968\n\t" \
  "s_nop 0\n\t" \
  "buffer_load_dwordx4 v82, s[36:39], s88 offen  lds\n\t" \
  "v_mfma_f32_16x16x128_f8f6f4 v[248:251], v[120:127], a[176:183], v[248:251]\n\t" \
  "s_add_u32 m0, s60, 84992\n\t" \
  "s_nop 0\n\t" \
  "buffer_load_dwordx4 v83, s[36:39], s88 offen  lds\n\t" \
  "v_mfma_f32_16x16x128_f8f6f4 v[252:255], v[120:127], a[184:191], v[252:255]\n\t" \
  "s_waitcnt vmcnt(32)\n\t" \
  "global_load_dword v92, v87, s[70:71] sc0 sc1\n\t" \
  "s_bitcmp1_b32 s65, 10\n\t" \
  "s_cbranch_scc1 OK62_%=\n\t" \
  "s_mov_b32 s67, 10\n\t" \
  "s_call_b64 s[68:69], POLL_%=\n\t" \
  "OK62_%=:\n\t" \
  "s_waitcnt lgkmcnt(4)\n\t" \
  "v_mfma_f32_16x16x128_f8f6f4 v[128:131], v[96:103], a[192:199], v[128:131]\n\t" \
  "ds_read_b64_tr_b8 v[112:113], v66 offset:32768\n\t" \
  "ds_read_b64_tr_b8 v[114:115], v66 offset:33792\n\t" \
  "s_bfe_u32 s96, s65, 262155\n\t" \
  "s_cmp_lg_u32 s96, 15\n\t" \
  "s_cbranch_scc1 XWS63_%=\n\t" \
  "s_add_u32 s96, s62, 720896\n\t" \
  "buffer_load_dword v8, v13, s[44:47], s96 offen sc0 sc1\n\t" \
  "XWS63_%=:\n\t" \
  "v_mfma_f32_16x16x128_f8f6f4 v[132:135], v[96:103], a[200:207], v[132:135]\n\t" \
  "ds_read_b64_tr_b8 v[116:117], v66 offset:40960\n\t" \
  "ds_read_b64_tr_b8 v[118:119], v66 offset:41984\n\t" \
  "s_add_u32 s88, s57, 0\n\t" \
  "s_add_u32 m0, s60, 98304\n\t" \
  "s_nop 0\n\t" \
  "buffer_load_dwordx4 v80, s[36:39], s88 offen  lds\n\t" \
  "v_mfma_f32_16x16x128_f8f6f4 v[136:139], v[96:103], a[208:215], v[136:139]\n\t" \
  "s_add_u32 m0, s60, 99328\n\t" \
  "s_nop 0\n\t" \
  "buffer_load_dwordx4 v81, s[36:39], s88 offen  lds\n\t" \
  "v_mfma_f32_16x16x128_f8f6f4 v[140:143], v[96:103], a[216:223], v[140:143]\n\t" \
  "s_waitcnt lgkmcnt(4)\n\t" \
  "v_mfma_f32_16x16x128_f8f6f4 v[144:147], v[104:111], a[192:199], v[144:147]\n\t" \
  "ds_read_b64_tr_b8 v[120:121], v67 offset:32768\n\t" \
  "ds_read_b64_tr_b8 v[122:123], v67 offset:33792\n\t" \
  "v_mfma_f32_16x16x128_f8f6f4 v[148:151], v[104:111], a[200:207], v[148:151]\n\t" \
  "ds_read_b64_tr_b8 v[124:125], v67 offset:40960\n\t" \
  "ds_read_b64_tr_b8 v[126:127], v67 offset:41984\n\t" \
  "s_add_u32 m0, s60, 100352\n\t" \
  "s_nop 0\n\t" \
  "buffer_load_dwordx4 v82, s[36:39], s88 offen  lds\n\t" \
  "v_mfma_f32_16x16x128_f8f6f4 v[152:155], v[104:111], a[208:215], v[152:155]\n\t" \
  "s_add_u32 m0, s60, 101376\n\t" \
  "s_nop 0\n\t" \
  "buffer_load_dwordx4 v83, s[36:39], s88 offen  lds\n\t" \
  "v_mfma_f32_16x16x128_f8f6f4 v[156:159], v[104:111], a[216:223], v[156:159]\n\t" \
  "s_waitcnt lgkmcnt(4)\n\t" \
  "v_mfma_f32_16x16x128_f8f6f4 v[160:163], v[112:119], a[192:199], v[160:163]\n\t" \
  "ds_read_b64_tr_b8 v[96:97], v68 offset:32768\n\t" \
  "ds_read_b64_tr_b8 v[98:99], v68 offset:33792\n\t" \
  "v_mfma_f32_16x16x128_f8f6f4 v[164:167], v[112:119], a[200:207], v[164:167]\n\t" \
  "ds_read_b64_tr_b8 v[100:101], v68 offset:40960\n\t" \
  "ds_read_b64_tr_b8 v[102:103], v68 offset:41984\n\t" \
  "s_add_u32 s88, s58, 0\n\t" \
  "s_add_u32 m0, s60, 114688\n\t" \
  "s_nop 0\n\t" \
  "buffer_load_dwordx4 v80, s[36:39], s88 offen  lds\n\t" \
  "v_mfma_f32_16x16x128_f8f6f4 v[168:171], v[112:119], a[208:215], v[168:171]\n\t" \
  "s_add_u32 m0, s60, 115712\n\t" \
  "s_nop 0\n\t" \
  "buffer_load_dwordx4 v81, s[36:39], s88 offen  lds\n\t" \
  "v_mfma_f32_16x16x128_f8f6f4 v[172:175], v[112:119], a[216:223], v[172:175]\n\t" \
  "s_waitcnt lgkmcnt(4)\n\t" \
  "v_mfma_f32_16x16x128_f8f6f4 v[176:179], v[120:127], a[192:199], v[176:179]\n\t" \
  "ds_read_b64_tr_b8 v[104:105], v69 offset:32768\n\t" \
  "ds_read_b64_tr_b8 v[106:107], v69 offset:33792\n\t" \
  "v_mfma_f32_16x16x128_f8f6f4 v[180:183], v[120:127], a[200:207], v[180:183]\n\t" \
  "ds_read_b64_tr_b8 v[108:109], v69 offset:40960\n\t" \
  "ds_read_b64_tr_b8 v[110:111], v69 offset:41984\n\t" \
  "s_add_u32 m0, s60, 116736\n\t" \
  "s_nop 0\n\t" \
  "buffer_load_dwordx4 v82, s[36:39], s88 offen  lds\n\t" \
  "v_mfma_f32_16x16x128_f8f6f4 v[184:187], v[120:127], a[208:215], v[184:187]\n\t" \
  "s_add_u32 m0, s60, 117760\n\t" \
  "s_nop 0\n\t" \
  "buffer_load_dwordx4 v83, s[36:39], s88 offen  lds\n\t" \
  "v_mfma_f32_16x16x128_f8f6f4 v[188:191], v[120:127], a[216:223], v[188:191]\n\t" \
  "s_waitcnt lgkmcnt(4)\n\t" \
  "v_mfma_f32_16x16x128_f8f6f4 v[192:195], v[96:103], a[192:199], v[192:195]\n\t" \
  "ds_read_b64_tr_b8 v[112:113], v70 offset:32768\n\t" \
  "ds_read_b64_tr_b8 v[114:115], v70 offset:33792\n\t" \
  "v_mfma_f32_16x16x128_f8f6f4 v[196:199], v[96:103], a[200:207], v[196:199]\n\t" \
  "ds_read_b64_tr_b8 v[116:117], v70 offset:40960\n\t" \
  "ds_read_b64_tr_b8 v[118:119], v70 offset:41984\n\t" \
  "s_add_u32 s89, s62, 655360\n\t" \
  "s_add_u32 s90, s62, 659456\n\t" \
  "buffer_load_dwordx4 a[64:67], v84, s[44:47], s89 offen offset:0 sc0 sc1\n\t" \
  "v_mfma_f32_16x16x128_f8f6f4 v[200:203], v[96:103], a[208:215], v[200:203]\n\t" \
  "buffer_load_dwordx4 a[68:71], v84, s[44:47], s89 offen offset:1024 sc0 sc1\n\t" \
  "v_mfma_f32_16x16x128_f8f6f4 v[204:207], v[96:103], a[216:223], v[204:207]\n\t" \
  "s_waitcnt lgkmcnt(4)\n\t" \
  "v_mfma_f32_16x16x128_f8f6f4 v[208:211], v[104:111], a[192:199], v[208:211]\n\t" \
  "ds_read_b64_tr_b8 v[120:121], v71 offset:32768\n\t" \
  "ds_read_b64_tr_b8 v[122:123], v71 offset:33792\n\t" \
  "v_mfma_f32_16x16x128_f8f6f4 v[212:215], v[104:111], a[200:207], v[212:215]\n\t" \
  "ds_read_b64_tr_b8 v[124:125], v71 offset:40960\n\t" \
  "ds_read_b64_tr_b8 v[126:127], v71 offset:41984\n\t" \
  "buffer_load_dwordx4 a[72:75], v84, s[44:47], s89 offen offset:2048 sc0 sc1\n\t" \
  "v_mfma_f32_16x16x128_f8f6f4 v[216:219], v[104:111], a[208:215], v[216:219]\n\t" \
  "buffer_load_dwordx4 a[76:79], v84, s[44:47], s89 offen offset:3072 sc0 sc1\n\t" \
  "v_mfma_f32_16x16x128_f8f6f4 v[220:223], v[104:111], a[216:223], v[220:223]\n\t" \
  "s_waitcnt vmcnt(17)\n\t" \
  "s_barrier\n\t" \
  "s_waitcnt lgkmcnt(4)\n\t" \
  "v_mfma_f32_16x16x128_f8f6f4 v[224:227], v[112:119], a[192:199], v[224:227]\n\t" \
  "ds_read_b64_tr_b8 v[96:97], v72 offset:0\n\t" \
  "ds_read_b64_tr_b8 v[98:99], v72 offset:1024\n\t" \
  "v_mfma_f32_16x16x128_f8f6f4 v[228:231], v[112:119], a[200:207], v[228:231]\n\t" \
  "ds_read_b64_tr_b8 v[100:101], v72 offset:8192\n\t" \
  "ds_read_b64_tr_b8 v[102:103], v72 offset:9216\n\t" \
  "buffer_load_dwordx4 a[80:83], v84, s[44:47], s90 offen offset:0 sc0 sc1\n\t" \
  "v_mfma_f32_16x16x128_f8f6f4 v[232:235], v[112:119], a[208:215], v[232:235]\n\t" \
  "buffer_load_dwordx4 a[84:87], v84, s[44:47], s90 offen offset:1024 sc0 sc1\n\t" \
  "v_mfma_f32_16x16x128_f8f6f4 v[236:239], v[112:119], a[216:223], v[236:239]\n\t" \
  "s_cmp_lg_u32 s63, 2\n\t" \
  "s_cbranch_scc1 PS64_%=\n\t" \
  "global_store_dwordx4 v84, v[32:35], s[72:73] offset:0 sc0 sc1\n\t" \
  "global_store_dwordx4 v84, v[36:39], s[72:73] offset:1024 sc0 sc1\n\t" \
  "global_store_dwordx4 v84, v[40:43], s[72:73] offset:2048 sc0 sc1\n\t" \
  "global_store_dwordx4 v84, v[44:47], s[72:73] offset:3072 sc0 sc1\n\t" \
  "global_store_dwordx4 v84, v[48:51], s[74:75] offset:0 sc0 sc1\n\t" \
  "global_store_dwordx4 v84, v[52:55], s[74:75] offset:1024 sc0 sc1\n\t" \
  "global_store_dwordx4 v84, v[56:59], s[74:75] offset:2048 sc0 sc1\n\t" \
  "global_store_dwordx4 v84, v[60:63], s[74:75] offset:3072 sc0 sc1\n\t" \
  "s_nop 1\n\t" \
  "PS64_%=:\n\t" \
  "s_waitcnt lgkmcnt(4)\n\t" \
  "v_mfma_f32_16x16x128_f8f6f4 v[240:243], v[120:127], a[192:199], v[240:243]\n\t" \
  "ds_read_b64_tr_b8 v[104:105], v73 offset:0\n\t" \
  "ds_read_b64_tr_b8 v[106:107], v73 offset:1024\n\t" \
  "v_mfma_f32_16x16x128_f8f6f4 v[244:247], v[120:127], a[200:207], v[244:247]\n\t" \
  "ds_read_b64_tr_b8 v[108:109], v73 offset:8192\n\t" \
  "ds_read_b64_tr_b8 v[110:111], v73 offset:9216\n\t" \
  "buffer_load_dwordx4 a[88:91], v84, s[44:47], s90 offen offset:2048 sc0 sc1\n\t" \
  "v_mfma_f32_16x16x128_f8f6f4 v[248:251], v[120:127], a[208:215], v[248:251]\n\t" \
  "buffer_load_dwordx4 a[92:95], v84, s[44:47], s90 offen offset:3072 sc0 sc1\n\t" \
  "v_mfma_f32_16x16x128_f8f6f4 v[252:255], v[120:127], a[216:223], v[252:255]\n\t" \
  "s_waitcnt vmcnt(16)\n\t" \
  "v_cmp_le_u32 vcc, s64, v92\n\t" \
  "s_nop 1\n\t" \
  "s_or_b32 s65, s65, vcc_lo\n\t" \
  "s_bitcmp1_b32 s65, 11\n\t" \
  "s_cbranch_scc1 OK65_%=\n\t" \
  "s_mov_b32 s67, 11\n\t" \
  "s_call_b64 s[68:69], POLL_%=\n\t" \
  "OK65_%=:\n\t" \
  "s_waitcnt lgkmcnt(4)\n\t" \
  "v_mfma_f32_16x16x128_f8f6f4 v[128:131], v[96:103], a[224:231], v[128:131]\n\t" \
  "ds_read_b64_tr_b8 v[112:113], v74 offset:0\n\t" \
  "ds_read_b64_tr_b8 v[114:115], v74 offset:1024\n\t" \
  "v_mfma_f32_16x16x128_f8f6f4 v[132:135], v[96:103], a[232:239], v[132:135]\n\t" \
  "ds_read_b64_tr_b8 v[116:117], v74 offset:8192\n\t" \
  "ds_read_b64_tr_b8 v[118:119], v74 offset:9216\n\t" \
  "v_mfma_f32_16x16x128_f8f6f4 v[136:139], v[96:103], a[240:247], v[136:139]\n\t" \
  "v_mfma_f32_16x16x128_f8f6f4 v[140:143], v[96:103], a[248:255], v[140:143]\n\t" \
  "s_waitcnt lgkmcnt(4)\n\t" \
  "v_mfma_f32_16x16x128_f8f6f4 v[144:147], v[104:111], a[224:231], v[144:147]\n\t" \
  "ds_read_b64_tr_b8 v[120:121], v75 offset:0\n\t" \
  "ds_read_b64_tr_b8 v[122:123], v75 offset:1024\n\t" \
  "v_mfma_f32_16x16x128_f8f6f4 v[148:151], v[104:111], a[232:239], v[148:151]\n\t" \
  "ds_read_b64_tr_b8 v[124:125], v75 offset:8192\n\t" \
  "ds_read_b64_tr_b8 v[126:127], v75 offset:9216\n\t" \
  "v_mfma_f32_16x16x128_f8f6f4 v[152:155], v[104:111], a[240:247], v[152:155]\n\t" \
  "v_mfma_f32_16x16x128_f8f6f4 v[156:159], v[104:111], a[248:255], v[156:159]\n\t" \
  "s_waitcnt lgkmcnt(4)\n\t" \
  "v_mfma_f32_16x16x128_f8f6f4 v[160:163], v[112:119], a[224:231], v[160:163]\n\t" \
  "ds_read_b64_tr_b8 v[96:97], v76 offset:0\n\t" \
  "ds_read_b64_tr_b8 v[98:99], v76 offset:1024\n\t" \
  "v_mfma_f32_16x16x128_f8f6f4 v[164:167], v[112:119], a[232:239], v[164:167]\n\t" \
  "ds_read_b64_tr_b8 v[100:101], v76 offset:8192\n\t" \
  "ds_read_b64_tr_b8 v[102:103], v76 offset:9216\n\t" \
  "v_mfma_f32_16x16x128_f8f6f4 v[168:171], v[112:119], a[240:247], v[168:171]\n\t" \
  "v_mfma_f32_16x16x128_f8f6f4 v[172:175], v[112:119], a[248:255], v[172:175]\n\t" \
  "s_waitcnt lgkmcnt(4)\n\t" \
  "v_mfma_f32_16x16x128_f8f6f4 v[176:179], v[120:127], a[224:231], v[176:179]\n\t" \
  "ds_read_b64_tr_b8 v[104:105], v77 offset:0\n\t" \
  "ds_read_b64_tr_b8 v[106:107], v77 offset:1024\n\t" \
  "v_mfma_f32_16x16x128_f8f6f4 v[180:183], v[120:127], a[232:239], v[180:183]\n\t" \
  "ds_read_b64_tr_b8 v[108:109], v77 offset:8192\n\t" \
  "ds_read_b64_tr_b8 v[110:111], v77 offset:9216\n\t" \
  "v_mfma_f32_16x16x128_f8f6f4 v[184:187], v[120:127], a[240:247], v[184:187]\n\t" \
  "v_mfma_f32_16x16x128_f8f6f4 v[188:191], v[120:127], a[248:255], v[188:191]\n\t" \
  "s_waitcnt lgkmcnt(4)\n\t" \
  "v_mfma_f32_16x16x128_f8f6f4 v[192:195], v[96:103], a[224:231], v[192:195]\n\t" \
  "ds_read_b64_tr_b8 v[112:113], v78 offset:0\n\t" \
  "ds_read_b64_tr_b8 v[114:115], v78 offset:1024\n\t" \
  "v_mfma_f32_16x16x128_f8f6f4 v[196:199], v[96:103], a[232:239], v[196:199]\n\t" \
  "ds_read_b64_tr_b8 v[116:117], v78 offset:8192\n\t" \
  "ds_read_b64_tr_b8 v[118:119], v78 offset:9216\n\t" \
  "v_mfma_f32_16x16x128_f8f6f4 v[200:203], v[96:103], a[240:247], v[200:203]\n\t" \
  "v_mfma_f32_16x16x128_f8f6f4 v[204:207], v[96:103], a[248:255], v[204:207]\n\t" \
  "s_waitcnt lgkmcnt(4)\n\t" \
  "v_mfma_f32_16x16x128_f8f6f4 v[208:211], v[104:111], a[224:231], v[208:211]\n\t" \
  "ds_read_b64_tr_b8 v[120:121], v79 offset:0\n\t" \
  "ds_read_b64_tr_b8 v[122:123], v79 offset:1024\n\t" \
  "v_mfma_f32_16x16x128_f8f6f4 v[212:215], v[104:111], a[232:239], v[212:215]\n\t" \
  "ds_read_b64_tr_b8 v[124:125], v79 offset:8192\n\t" \
  "ds_read_b64_tr_b8 v[126:127], v79 offset:9216\n\t" \
  "v_mfma_f32_16x16x128_f8f6f4 v[216:219], v[104:111], a[240:247], v[216:219]\n\t" \
  "v_mfma_f32_16x16x128_f8f6f4 v[220:223], v[104:111], a[248:255], v[220:223]\n\t" \
  "s_waitcnt vmcnt(17)\n\t" \
  "s_barrier\n\t" \
  "s_waitcnt lgkmcnt(4)\n\t" \
  "v_mfma_f32_16x16x128_f8f6f4 v[224:227], v[112:119], a[224:231], v[224:227]\n\t" \
  "ds_read_b64_tr_b8 v[96:97], v72 offset:16384\n\t" \
  "ds_read_b64_tr_b8 v[98:99], v72 offset:17408\n\t" \
  "v_mfma_f32_16x16x128_f8f6f4 v[228:231], v[112:119], a[232:239], v[228:231]\n\t" \
  "ds_read_b64_tr_b8 v[100:101], v72 offset:24576\n\t" \
  "ds_read_b64_tr_b8 v[102:103], v72 offset:25600\n\t" \
  "s_add_u32 s89, s62, 720896\n\t" \
  "s_add_u32 s90, s62, 724992\n\t" \
  "buffer_load_dwordx4 a[96:99], v84, s[44:47], s89 offen offset:0 sc0 sc1\n\t" \
  "v_mfma_f32_16x16x128_f8f6f4 v[232:235], v[112:119], a[240:247], v[232:235]\n\t" \
  "buffer_load_dwordx4 a[100:103], v84, s[44:47], s89 offen offset:1024 sc0 sc1\n\t" \
  "v_mfma_f32_16x16x128_f8f6f4 v[236:239], v[112:119], a[248:255], v[236:239]\n\t" \
  "s_waitcnt lgkmcnt(4)\n\t" \
  "v_mfma_f32_16x16x128_f8f6f4 v[240:243], v[120:127], a[224:231], v[240:243]\n\t" \
  "ds_read_b64_tr_b8 v[104:105], v73 offset:16384\n\t" \
  "ds_read_b64_tr_b8 v[106:107], v73 offset:17408\n\t" \
  "v_mfma_f32_16x16x128_f8f6f4 v[244:247], v[120:127], a[232:239], v[244:247]\n\t" \
  "ds_read_b64_tr_b8 v[108:109], v73 offset:24576\n\t" \
  "ds_read_b64_tr_b8 v[110:111], v73 offset:25600\n\t" \
  "buffer_load_dwordx4 a[104:107], v84, s[44:47], s89 offen offset:2048 sc0 sc1\n\t" \
  "v_mfma_f32_16x16x128_f8f6f4 v[248:251], v[120:127], a[240:247], v[248:251]\n\t" \
  "buffer_load_dwordx4 a[108:111], v84, s[44:47], s89 offen offset:3072 sc0 sc1\n\t" \
  "v_mfma_f32_16x16x128_f8f6f4 v[252:255], v[120:127], a[248:255], v[252:255]\n\t" \
  "s_waitcnt vmcnt(37)\n\t" \
  "s_bitcmp1_b32 s65, 12\n\t" \
  "s_cbranch_scc1 OK66_%=\n\t" \
  "s_mov_b32 s67, 12\n\t" \
  "s_call_b64 s[68:69], POLL_%=\n\t" \
  "OK66_%=:\n\t" \
  "s_waitcnt lgkmcnt(4)\n\t" \
  "v_mfma_f32_16x16x128_f8f6f4 v[128:131], v[96:103], a[0:7], v[128:131]\n\t" \
  "ds_read_b64_tr_b8 v[112:113], v74 offset:16384\n\t" \
  "ds_read_b64_tr_b8 v[114:115], v74 offset:17408\n\t" \
  "v_mfma_f32_16x16x128_f8f6f4 v[132:135], v[96:103], a[8:15], v[132:135]\n\t" \
  "ds_read_b64_tr_b8 v[116:117], v74 offset:24576\n\t" \
  "ds_read_b64_tr_b8 v[118:119], v74 offset:25600\n\t" \
  "buffer_load_dwordx4 a[112:115], v84, s[44:47], s90 offen offset:0 sc0 sc1\n\t" \
  "v_mfma_f32_16x16x128_f8f6f4 v[136:139], v[96:103], a[16:23], v[136:139]\n\t" \
  "buffer_load_dwordx4 a[116:119], v84, s[44:47], s90 offen offset:1024 sc0 sc1\n\t" \
  "v_mfma_f32_16x16x128_f8f6f4 v[140:143], v[96:103], a[24:31], v[140:143]\n\t" \
  "s_waitcnt lgkmcnt(4)\n\t" \
  "v_mfma_f32_16x16x128_f8f6f4 v[144:147], v[104:111], a[0:7], v[144:147]\n\t" \
  "ds_read_b64_tr_b8 v[120:121], v75 offset:16384\n\t" \
  "ds_read_b64_tr_b8 v[122:123], v75 offset:17408\n\t" \
  "v_mfma_f32_16x16x128_f8f6f4 v[148:151], v[104:111], a[8:15], v[148:151]\n\t" \
  "ds_read_b64_tr_b8 v[124:125], v75 offset:24576\n\t" \
  "ds_read_b64_tr_b8 v[126:127], v75 offset:25600\n\t" \
  "buffer_load_dwordx4 a[120:123], v84, s[44:47], s90 offen offset:2048 sc0 sc1\n\t" \
  "v_mfma_f32_16x16x128_f8f6f4 v[152:155], v[104:111], a[16:23], v[152:155]\n\t" \
  "buffer_load_dwordx4 a[124:127], v84, s[44:47], s90 offen offset:3072 sc0 sc1\n\t" \
  "v_mfma_f32_16x16x128_f8f6f4 v[156:159], v[104:111], a[24:31], v[156:159]\n\t" \
  "s_waitcnt lgkmcnt(4)\n\t" \
  "v_mfma_f32_16x16x128_f8f6f4 v[160:163], v[112:119], a[0:7], v[160:163]\n\t" \
  "ds_read_b64_tr_b8 v[96:97], v76 offset:16384\n\t" \
  "ds_read_b64_tr_b8 v[98:99], v76 offset:17408\n\t" \
  "v_mfma_f32_16x16x128_f8f6f4 v[164:167], v[112:119], a[8:15], v[164:167]\n\t" \
  "ds_read_b64_tr_b8 v[100:101], v76 offset:24576\n\t" \
  "ds_read_b64_tr_b8 v[102:103], v76 offset:25600\n\t" \
  "s_add_u32 s89, s62, 786432\n\t" \
  "s_add_u32 s90, s62, 790528\n\t" \
  "buffer_load_dwordx4 a[128:131], v84, s[44:47], s89 offen offset:0 sc0 sc1\n\t" \
  "v_mfma_f32_16x16x128_f8f6f4 v[168:171], v[112:119], a[16:23], v[168:171]\n\t" \
  "buffer_load_dwordx4 a[132:135], v84, s[44:47], s89 offen offset:1024 sc0 sc1\n\t" \
  "v_mfma_f32_16x16x128_f8f6f4 v[172:175], v[112:119], a[24:31], v[172:175]\n\t" \
  "s_waitcnt lgkmcnt(4)\n\t" \
  "v_mfma_f32_16x16x128_f8f6f4 v[176:179], v[120:127], a[0:7], v[176:179]\n\t" \
  "ds_read_b64_tr_b8 v[104:105], v77 offset:16384\n\t" \
  "ds_read_b64_tr_b8 v[106:107], v77 offset:17408\n\t" \
  "v_mfma_f32_16x16x128_f8f6f4 v[180:183], v[120:127], a[8:15], v[180:183]\n\t" \
  "ds_read_b64_tr_b8 v[108:109], v77 offset:24576\n\t" \
  "ds_read_b64_tr_b8 v[110:111], v77 offset:25600\n\t" \
  "buffer_load_dwordx4 a[136:139], v84, s[44:47], s89 offen offset:2048 sc0 sc1\n\t" \
  "v_mfma_f32_16x16x128_f8f6f4 v[184:187], v[120:127], a[16:23], v[184:187]\n\t" \
  "buffer_load_dwordx4 a[140:143], v84, s[44:47], s89 offen offset:3072 sc0 sc1\n\t" \
  "v_mfma_f32_16x16x128_f8f6f4 v[188:191], v[120:127], a[24:31], v[188:191]\n\t" \
  "s_waitcnt lgkmcnt(4)\n\t" \
  "v_mfma_f32_16x16x128_f8f6f4 v[192:195], v[96:103], a[0:7], v[192:195]\n\t" \
  "ds_read_b64_tr_b8 v[112:113], v78 offset:16384\n\t" \
  "ds_read_b64_tr_b8 v[114:115], v78 offset:17408\n\t" \
  "v_mfma_f32_16x16x128_f8f6f4 v[196:199], v[96:103], a[8:15], v[196:199]\n\t" \
  "ds_read_b64_tr_b8 v[116:117], v78 offset:24576\n\t" \
  "ds_read_b64_tr_b8 v[118:119], v78 offset:25600\n\t" \
  "buffer_load_dwordx4 a[144:147], v84, s[44:47], s90 offen offset:0 sc0 sc1\n\t" \
  "v_mfma_f32_16x16x128_f8f6f4 v[200:203], v[96:103], a[16:23], v[200:203]\n\t" \
  "buffer_load_dwordx4 a[148:151], v84, s[44:47], s90 offen offset:1024 sc0 sc1\n\t" \
  "v_mfma_f32_16x16x128_f8f6f4 v[204:207], v[96:103], a[24:31], v[204:207]\n\t" \
  "s_waitcnt lgkmcnt(4)\n\t" \
  "v_mfma_f32_16x16x128_f8f6f4 v[208:211], v[104:111], a[0:7], v[208:211]\n\t" \
  "ds_read_b64_tr_b8 v[120:121], v79 offset:16384\n\t" \
  "ds_read_b64_tr_b8 v[122:123], v79 offset:17408\n\t" \
  "v_mfma_f32_16x16x128_f8f6f4 v[212:215], v[104:111], a[8:15], v[212:215]\n\t" \
  "ds_read_b64_tr_b8 v[124:125], v79 offset:24576\n\t" \
  "ds_read_b64_tr_b8 v[126:127], v79 offset:25600\n\t" \
  "buffer_load_dwordx4 a[152:155], v84, s[44:47], s90 offen offset:2048 sc0 sc1\n\t" \
  "v_mfma_f32_16x16x128_f8f6f4 v[216:219], v[104:111], a[16:23], v[216:219]\n\t" \
  "buffer_load_dwordx4 a[156:159], v84, s[44:47], s90 offen offset:3072 sc0 sc1\n\t" \
  "v_mfma_f32_16x16x128_f8f6f4 v[220:223], v[104:111], a[24:31], v[220:223]\n\t" \
  "s_waitcnt vmcnt(28)\n\t" \
  "s_barrier\n\t" \
  "s_waitcnt lgkmcnt(4)\n\t" \
  "v_mfma_f32_16x16x128_f8f6f4 v[224:227], v[112:119], a[0:7], v[224:227]\n\t" \
  "ds_read_b64_tr_b8 v[96:97], v72 offset:32768\n\t" \
  "ds_read_b64_tr_b8 v[98:99], v72 offset:33792\n\t" \
  "v_mfma_f32_16x16x128_f8f6f4 v[228:231], v[112:119], a[8:15], v[228:231]\n\t" \
  "ds_read_b64_tr_b8 v[100:101], v72 offset:40960\n\t" \
  "ds_read_b64_tr_b8 v[102:103], v72 offset:41984\n\t" \
  "v_mfma_f32_16x16x128_f8f6f4 v[232:235], v[112:119], a[16:23], v[232:235]\n\t" \
  "v_mfma_f32_16x16x128_f8f6f4 v[236:239], v[112:119], a[24:31], v[236:239]\n\t" \
  "s_waitcnt lgkmcnt(4)\n\t" \
  "v_mfma_f32_16x16x128_f8f6f4 v[240:243], v[120:127], a[0:7], v[240:243]\n\t" \
  "ds_read_b64_tr_b8 v[104:105], v73 offset:32768\n\t" \
  "ds_read_b64_tr_b8 v[106:107], v73 offset:33792\n\t" \
  "v_mfma_f32_16x16x128_f8f6f4 v[244:247], v[120:127], a[8:15], v[244:247]\n\t" \
  "ds_read_b64_tr_b8 v[108:109], v73 offset:40960\n\t" \
  "ds_read_b64_tr_b8 v[110:111], v73 offset:41984\n\t" \
  "v_mfma_f32_16x16x128_f8f6f4 v[248:251], v[120:127], a[16:23], v[248:251]\n\t" \
  "v_mfma_f32_16x16x128_f8f6f4 v[252:255], v[120:127], a[24:31], v[252:255]\n\t" \
  "s_waitcnt vmcnt(41)\n\t" \
  "s_bitcmp1_b32 s65, 13\n\t" \
  "s_cbranch_scc1 OK67_%=\n\t" \
  "s_mov_b32 s67, 13\n\t" \
  "s_call_b64 s[68:69], POLL_%=\n\t" \
  "OK67_%=:\n\t" \
  "s_waitcnt lgkmcnt(4)\n\t" \
  "v_mfma_f32_16x16x128_f8f6f4 v[128:131], v[96:103], a[32:39], v[128:131]\n\t" \
  "ds_read_b64_tr_b8 v[112:113], v74 offset:32768\n\t" \
  "ds_read_b64_tr_b8 v[114:115], v74 offset:33792\n\t" \
  "v_mfma_f32_16x16x128_f8f6f4 v[132:135], v[96:103], a[40:47], v[132:135]\n\t" \
  "ds_read_b64_tr_b8 v[116:117], v74 offset:40960\n\t" \
  "ds_read_b64_tr_b8 v[118:119], v74 offset:41984\n\t" \
  "s_add_u32 s89, s62, 851968\n\t" \
  "s_add_u32 s90, s62, 856064\n\t" \
  "buffer_load_dwordx4 a[160:163], v84, s[44:47], s89 offen offset:0 sc0 sc1\n\t" \
  "v_mfma_f32_16x16x128_f8f6f4 v[136:139], v[96:103], a[48:55], v[136:139]\n\t" \
  "buffer_load_dwordx4 a[164:167], v84, s[44:47], s89 offen offset:1024 sc0 sc1\n\t" \
  "v_mfma_f32_16x16x128_f8f6f4 v[140:143], v[96:103], a[56:63], v[140:143]\n\t" \
  "s_waitcnt lgkmcnt(4)\n\t" \
  "v_mfma_f32_16x16x128_f8f6f4 v[144:147], v[104:111], a[32:39], v[144:147]\n\t" \
  "ds_read_b64_tr_b8 v[120:121], v75 offset:32768\n\t" \
  "ds_read_b64_tr_b8 v[122:123], v75 offset:33792\n\t" \
  "v_mfma_f32_16x16x128_f8f6f4 v[148:151], v[104:111], a[40:47], v[148:151]\n\t" \
  "ds_read_b64_tr_b8 v[124:125], v75 offset:40960\n\t" \
  "ds_read_b64_tr_b8 v[126:127], v75 offset:41984\n\t" \
  "buffer_load_dwordx4 a[168:171], v84, s[44:47], s89 offen offset:2048 sc0 sc1\n\t" \
  "v_mfma_f32_16x16x128_f8f6f4 v[152:155], v[104:111], a[48:55], v[152:155]\n\t" \
  "buffer_load_dwordx4 a[172:175], v84, s[44:47], s89 offen offset:3072 sc0 sc1\n\t" \
  "v_mfma_f32_16x16x128_f8f6f4 v[156:159], v[104:111], a[56:63], v[156:159]\n\t" \
  "s_waitcnt lgkmcnt(4)\n\t" \
  "v_mfma_f32_16x16x128_f8f6f4 v[160:163], v[112:119], a[32:39], v[160:163]\n\t" \
  "ds_read_b64_tr_b8 v[96:97], v76 offset:32768\n\t" \
  "ds_read_b64_tr_b8 v[98:99], v76 offset:33792\n\t" \
  "v_mfma_f32_16x16x128_f8f6f4 v[164:167], v[112:119], a[40:47], v[164:167]\n\t" \
  "ds_read_b64_tr_b8 v[100:101], v76 offset:40960\n\t" \
  "ds_read_b64_tr_b8 v[102:103], v76 offset:41984\n\t" \
  "buffer_load_dwordx4 a[176:179], v84, s[44:47], s90 offen offset:0 sc0 sc1\n\t" \
  "v_mfma_f32_16x16x128_f8f6f4 v[168:171], v[112:119], a[48:55], v[168:171]\n\t" \
  "buffer_load_dwordx4 a[180:183], v84, s[44:47], s90 offen offset:1024 sc0 sc1\n\t" \
  "v_mfma_f32_16x16x128_f8f6f4 v[172:175], v[112:119], a[56:63], v[172:175]\n\t" \
  "s_waitcnt lgkmcnt(4)\n\t" \
  "v_mfma_f32_16x16x128_f8f6f4 v[176:179], v[120:127], a[32:39], v[176:179]\n\t" \
  "ds_read_b64_tr_b8 v[104:105], v77 offset:32768\n\t" \
  "ds_read_b64_tr_b8 v[106:107], v77 offset:33792\n\t" \
  "v_mfma_f32_16x16x128_f8f6f4 v[180:183], v[120:127], a[40:47], v[180:183]\n\t" \
  "ds_read_b64_tr_b8 v[108:109], v77 offset:40960\n\t" \
  "ds_read_b64_tr_b8 v[110:111], v77 offset:41984\n\t" \
  "buffer_load_dwordx4 a[184:187], v84, s[44:47], s90 offen offset:2048 sc0 sc1\n\t" \
  "v_mfma_f32_16x16x128_f8f6f4 v[184:187], v[120:127], a[48:55], v[184:187]\n\t" \
  "buffer_load_dwordx4 a[188:191], v84, s[44:47], s90 offen offset:3072 sc0 sc1\n\t" \
  "v_mfma_f32_16x16x128_f8f6f4 v[188:191], v[120:127], a[56:63], v[188:191]\n\t" \
  "s_waitcnt lgkmcnt(4)\n\t" \
  "v_mfma_f32_16x16x128_f8f6f4 v[192:195], v[96:103], a[32:39], v[192:195]\n\t" \
  "ds_read_b64_tr_b8 v[112:113], v78 offset:32768\n\t" \
  "ds_read_b64_tr_b8 v[114:115], v78 offset:33792\n\t" \
  "v_mfma_f32_16x16x128_f8f6f4 v[196:199], v[96:103], a[40:47], v[196:199]\n\t" \
  "ds_read_b64_tr_b8 v[116:117], v78 offset:40960\n\t" \
  "ds_read_b64_tr_b8 v[118:119], v78 offset:41984\n\t" \
  "s_add_u32 s88, s52, 2097152\n\t" \
  "s_add_u32 m0, s60, 131072\n\t" \
  "s_nop 0\n\t" \
  "buffer_load_dwordx4 v80, s[36:39], s88 offen  lds\n\t" \
  "v_mfma_f32_16x16x128_f8f6f4 v[200:203], v[96:103], a[48:55], v[200:203]\n\t" \
  "s_add_u32 m0, s60, 132096\n\t" \
  "s_nop 0\n\t" \
  "buffer_load_dwordx4 v81, s[36:39], s88 offen  lds\n\t" \
  "v_mfma_f32_16x16x128_f8f6f4 v[204:207], v[96:103], a[56:63], v[204:207]\n\t" \
  "s_waitcnt lgkmcnt(4)\n\t" \
  "v_mfma_f32_16x16x128_f8f6f4 v[208:211], v[104:111], a[32:39], v[208:211]\n\t" \
  "ds_read_b64_tr_b8 v[120:121], v79 offset:32768\n\t" \
  "ds_read_b64_tr_b8 v[122:123], v79 offset:33792\n\t" \
  "v_mfma_f32_16x16x128_f8f6f4 v[212:215], v[104:111], a[40:47], v[212:215]\n\t" \
  "ds_read_b64_tr_b8 v[124:125], v79 offset:40960\n\t" \
  "ds_read_b64_tr_b8 v[126:127], v79 offset:41984\n\t" \
  "s_add_u32 m0, s60, 133120\n\t" \
  "s_nop 0\n\t" \
  "buffer_load_dwordx4 v82, s[36:39], s88 offen  lds\n\t" \
  "v_mfma_f32_16x16x128_f8f6f4 v[216:219], v[104:111], a[48:55], v[216:219]\n\t" \
  "s_add_u32 m0, s60, 134144\n\t" \
  "s_nop 0\n\t" \
  "buffer_load_dwordx4 v83, s[36:39], s88 offen  lds\n\t" \
  "v_mfma_f32_16x16x128_f8f6f4 v[220:223], v[104:111], a[56:63], v[220:223]\n\t" \
  "s_waitcnt vmcnt(36)\n\t" \
  "s_barrier\n\t" \
  "s_waitcnt lgkmcnt(4)\n\t" \
  "v_mfma_f32_16x16x128_f8f6f4 v[224:227], v[112:119], a[32:39], v[224:227]\n\t" \
  "ds_read_b64_tr_b8 v[96:97], v72 offset:49152\n\t" \
  "ds_read_b64_tr_b8 v[98:99], v72 offset:50176\n\t" \
  "v_mfma_f32_16x16x128_f8f6f4 v[228:231], v[112:119], a[40:47], v[228:231]\n\t" \
  "ds_read_b64_tr_b8 v[100:101], v72 offset:57344\n\t" \
  "ds_read_b64_tr_b8 v[102:103], v72 offset:58368\n\t" \
  "s_add_u32 s88, s53, 2097152\n\t" \
  "s_add_u32 m0, s60, 49152\n\t" \
  "s_nop 0\n\t" \
  "buffer_load_dwordx4 v80, s[36:39], s88 offen  lds\n\t" \
  "v_mfma_f32_16x16x128_f8f6f4 v[232:235], v[112:119], a[48:55], v[232:235]\n\t" \
  "s_add_u32 m0, s60, 50176\n\t" \
  "s_nop 0\n\t" \
  "buffer_load_dwordx4 v81, s[36:39], s88 offen  lds\n\t" \
  "v_mfma_f32_16x16x128_f8f6f4 v[236:239], v[112:119], a[56:63], v[236:239]\n\t" \
  "s_waitcnt lgkmcnt(4)\n\t" \
  "v_mfma_f32_16x16x128_f8f6f4 v[240:243], v[120:127], a[32:39], v[240:243]\n\t" \
  "ds_read_b64_tr_b8 v[104:105], v73 offset:49152\n\t" \
  "ds_read_b64_tr_b8 v[106:107], v73 offset:50176\n\t" \
  "v_mfma_f32_16x16x128_f8f6f4 v[244:247], v[120:127], a[40:47], v[244:247]\n\t" \
  "ds_read_b64_tr_b8 v[108:109], v73 offset:57344\n\t" \
  "ds_read_b64_tr_b8 v[110:111], v73 offset:58368\n\t" \
  "s_add_u32 m0, s60, 51200\n\t" \
  "s_nop 0\n\t" \
  "buffer_load_dwordx4 v82, s[36:39], s88 offen  lds\n\t" \
  "v_mfma_f32_16x16x128_f8f6f4 v[248:251], v[120:127], a[48:55], v[248:251]\n\t" \
  "s_add_u32 m0, s60, 52224\n\t" \
  "s_nop 0\n\t" \
  "buffer_load_dwordx4 v83, s[36:39], s88 offen  lds\n\t" \
  "v_mfma_f32_16x16x128_f8f6f4 v[252:255], v[120:127], a[56:63], v[252:255]\n\t" \
  "s_waitcnt vmcnt(32)\n\t" \
  "s_bitcmp1_b32 s65, 14\n\t" \
  "s_cbranch_scc1 OK68_%=\n\t" \
  "s_mov_b32 s67, 14\n\t" \
  "s_call_b64 s[68:69], POLL_%=\n\t" \
  "OK68_%=:\n\t" \
  "s_waitcnt lgkmcnt(4)\n\t" \
  "v_mfma_f32_16x16x128_f8f6f4 v[128:131], v[96:103], a[64:71], v[128:131]\n\t" \
  "ds_read_b64_tr_b8 v[112:113], v74 offset:49152\n\t" \
  "ds_read_b64_tr_b8 v[114:115], v74 offset:50176\n\t" \
  "s_bfe_u32 s96, s65, 262159\n\t" \
  "s_cmp_lg_u32 s96, 15\n\t" \
  "s_cbranch_scc1 XWS69_%=\n\t" \
  "s_add_u32 s96, s62, 983040\n\t" \
  "buffer_load_dword v8, v13, s[44:47], s96 offen sc0 sc1\n\t" \
  "XWS69_%=:\n\t" \
  "v_mfma_f32_16x16x128_f8f6f4 v[132:135], v[96:103], a[72:79], v[132:135]\n\t" \
  "ds_read_b64_tr_b8 v[116:117], v74 offset:57344\n\t" \
  "ds_read_b64_tr_b8 v[118:119], v74 offset:58368\n\t" \
  "s_add_u32 s88, s54, 2097152\n\t" \
  "s_add_u32 m0, s60, 0\n\t" \
  "s_nop 0\n\t" \
  "buffer_load_dwordx4 v80, s[36:39], s88 offen  lds\n\t" \
  "v_mfma_f32_16x16x128_f8f6f4 v[136:139], v[96:103], a[80:87], v[136:139]\n\t" \
  "s_add_u32 m0, s60, 1024\n\t" \
  "s_nop 0\n\t" \
  "buffer_load_dwordx4 v81, s[36:39], s88 offen  lds\n\t" \
  "v_mfma_f32_16x16x128_f8f6f4 v[140:143], v[96:103], a[88:95], v[140:143]\n\t" \
  "s_waitcnt lgkmcnt(4)\n\t" \
  "v_mfma_f32_16x16x128_f8f6f4 v[144:147], v[104:111], a[64:71], v[144:147]\n\t" \
  "ds_read_b64_tr_b8 v[120:121], v75 offset:49152\n\t" \
  "ds_read_b64_tr_b8 v[122:123], v75 offset:50176\n\t" \
  "v_mfma_f32_16x16x128_f8f6f4 v[148:151], v[104:111], a[72:79], v[148:151]\n\t" \
  "ds_read_b64_tr_b8 v[124:125], v75 offset:57344\n\t" \
  "ds_read_b64_tr_b8 v[126:127], v75 offset:58368\n\t" \
  "s_add_u32 m0, s60, 2048\n\t" \
  "s_nop 0\n\t" \
  "buffer_load_dwordx4 v82, s[36:39], s88 offen  lds\n\t" \
  "v_mfma_f32_16x16x128_f8f6f4 v[152:155], v[104:111], a[80:87], v[152:155]\n\t" \
  "s_add_u32 m0, s60, 3072\n\t" \
  "s_nop 0\n\t" \
  "buffer_load_dwordx4 v83, s[36:39], s88 offen  lds\n\t" \
  "v_mfma_f32_16x16x128_f8f6f4 v[156:159], v[104:111], a[88:95], v[156:159]\n\t" \
  "s_waitcnt lgkmcnt(4)\n\t" \
  "v_mfma_f32_16x16x128_f8f6f4 v[160:163], v[112:119], a[64:71], v[160:163]\n\t" \
  "ds_read_b64_tr_b8 v[96:97], v76 offset:49152\n\t" \
  "ds_read_b64_tr_b8 v[98:99], v76 offset:50176\n\t" \
  "v_mfma_f32_16x16x128_f8f6f4 v[164:167], v[112:119], a[72:79], v[164:167]\n\t" \
  "ds_read_b64_tr_b8 v[100:101], v76 offset:57344\n\t" \
  "ds_read_b64_tr_b8 v[102:103], v76 offset:58368\n\t" \
  "s_add_u32 s88, s55, 2097152\n\t" \
  "s_add_u32 m0, s60, 16384\n\t" \
  "s_nop 0\n\t" \
  "buffer_load_dwordx4 v80, s[36:39], s88 offen  lds\n\t" \
  "v_mfma_f32_16x16x128_f8f6f4 v[168:171], v[112:119], a[80:87], v[168:171]\n\t" \
  "s_add_u32 m0, s60, 17408\n\t" \
  "s_nop 0\n\t" \
  "buffer_load_dwordx4 v81, s[36:39], s88 offen  lds\n\t" \
  "v_mfma_f32_16x16x128_f8f6f4 v[172:175], v[112:119], a[88:95], v[172:175]\n\t" \
  "s_waitcnt lgkmcnt(4)\n\t" \
  "v_mfma_f32_16x16x128_f8f6f4 v[176:179], v[120:127], a[64:71], v[176:179]\n\t" \
  "ds_read_b64_tr_b8 v[104:105], v77 offset:49152\n\t" \
  "ds_read_b64_tr_b8 v[106:107], v77 offset:50176\n\t" \
  "v_mfma_f32_16x16x128_f8f6f4 v[180:183], v[120:127], a[72:79], v[180:183]\n\t" \
  "ds_read_b64_tr_b8 v[108:109], v77 offset:57344\n\t" \
  "ds_read_b64_tr_b8 v[110:111], v77 offset:58368\n\t" \
  "s_add_u32 m0, s60, 18432\n\t" \
  "s_nop 0\n\t" \
  "buffer_load_dwordx4 v82, s[36:39], s88 offen  lds\n\t" \
  "v_mfma_f32_16x16x128_f8f6f4 v[184:187], v[120:127], a[80:87], v[184:187]\n\t" \
  "s_add_u32 m0, s60, 19456\n\t" \
  "s_nop 0\n\t" \
  "buffer_load_dwordx4 v83, s[36:39], s88 offen  lds\n\t" \
  "v_mfma_f32_16x16x128_f8f6f4 v[188:191], v[120:127], a[88:95], v[188:191]\n\t" \
  "s_waitcnt lgkmcnt(4)\n\t" \
  "v_mfma_f32_16x16x128_f8f6f4 v[192:195], v[96:103], a[64:71], v[192:195]\n\t" \
  "ds_read_b64_tr_b8 v[112:113], v78 offset:49152\n\t" \
  "ds_read_b64_tr_b8 v[114:115], v78 offset:50176\n\t" \
  "v_mfma_f32_16x16x128_f8f6f4 v[196:199], v[96:103], a[72:79], v[196:199]\n\t" \
  "ds_read_b64_tr_b8 v[116:117], v78 offset:57344\n\t" \
  "ds_read_b64_tr_b8 v[118:119], v78 offset:58368\n\t" \
  "s_add_u32 s89, s62, 917504\n\t" \
  "s_add_u32 s90, s62, 921600\n\t" \
  "buffer_load_dwordx4 a[192:195], v84, s[44:47], s89 offen offset:0 sc0 sc1\n\t" \
  "v_mfma_f32_16x16x128_f8f6f4 v[200:203], v[96:103], a[80:87], v[200:203]\n\t" \
  "buffer_load_dwordx4 a[196:199], v84, s[44:47], s89 offen offset:1024 sc0 sc1\n\t" \
  "v_mfma_f32_16x16x128_f8f6f4 v[204:207], v[96:103], a[88:95], v[204:207]\n\t" \
  "s_waitcnt lgkmcnt(4)\n\t" \
  "v_mfma_f32_16x16x128_f8f6f4 v[208:211], v[104:111], a[64:71], v[208:211]\n\t" \
  "ds_read_b64_tr_b8 v[120:121], v79 offset:49152\n\t" \
  "ds_read_b64_tr_b8 v[122:123], v79 offset:50176\n\t" \
  "v_mfma_f32_16x16x128_f8f6f4 v[212:215], v[104:111], a[72:79], v[212:215]\n\t" \
  "ds_read_b64_tr_b8 v[124:125], v79 offset:57344\n\t" \
  "ds_read_b64_tr_b8 v[126:127], v79 offset:58368\n\t" \
  "buffer_load_dwordx4 a[200:203], v84, s[44:47], s89 offen offset:2048 sc0 sc1\n\t" \
  "v_mfma_f32_16x16x128_f8f6f4 v[216:219], v[104:111], a[80:87], v[216:219]\n\t" \
  "v_mfma_f32_16x16x128_f8f6f4 v[220:223], v[104:111], a[88:95], v[220:223]\n\t" \
  "s_waitcnt vmcnt(15)\n\t" \
  "s_barrier\n\t" \
  "s_waitcnt lgkmcnt(4)\n\t" \
  "v_mfma_f32_16x16x128_f8f6f4 v[224:227], v[112:119], a[64:71], v[224:227]\n\t" \
  "ds_read_b64_tr_b8 v[96:97], v16 offset:0\n\t" \
  "ds_read_b64_tr_b8 v[98:99], v16 offset:1024\n\t" \
  "v_mfma_f32_16x16x128_f8f6f4 v[228:231], v[112:119], a[72:79], v[228:231]\n\t" \
  "ds_read_b64_tr_b8 v[100:101], v16 offset:8192\n\t" \
  "ds_read_b64_tr_b8 v[102:103], v16 offset:9216\n\t" \
  "v_mfma_f32_16x16x128_f8f6f4 v[232:235], v[112:119], a[80:87], v[232:235]\n\t" \
  "v_mfma_f32_16x16x128_f8f6f4 v[236:239], v[112:119], a[88:95], v[236:239]\n\t" \
  "s_cmp_lg_u32 s63, 2\n\t" \
  "s_cbranch_scc1 PS70_%=\n\t" \
  "s_waitcnt vmcnt(45)\n\t" \
  "v_mov_b32 v89, s64\n\t" \
  "s_mov_b64 exec, 1\n\t" \
  "global_store_dword v88, v89, s[76:77] sc0 sc1\n\t" \
  "s_mov_b64 exec, -1\n\t" \
  "PS70_%=:\n\t" \
  "s_cmp_lg_u32 s63, 3\n\t" \
  "s_cbranch_scc1 PS71_%=\n\t" \
  "global_store_dwordx4 v84, v[32:35], s[72:73] offset:0 sc0 sc1\n\t" \
  "global_store_dwordx4 v84, v[36:39], s[72:73] offset:1024 sc0 sc1\n\t" \
  "global_store_dwordx4 v84, v[40:43], s[72:73] offset:2048 sc0 sc1\n\t" \
  "global_store_dwordx4 v84, v[44:47], s[72:73] offset:3072 sc0 sc1\n\t" \
  "global_store_dwordx4 v84, v[48:51], s[74:75] offset:0 sc0 sc1\n\t" \
  "global_store_dwordx4 v84, v[52:55], s[74:75] offset:1024 sc0 sc1\n\t" \
  "global_store_dwordx4 v84, v[56:59], s[74:75] offset:2048 sc0 sc1\n\t" \
  "global_store_dwordx4 v84, v[60:63], s[74:75] offset:3072 sc0 sc1\n\t" \
  "s_nop 1\n\t" \
  "PS71_%=:\n\t" \
  "s_waitcnt lgkmcnt(4)\n\t" \
  "v_mfma_f32_16x16x128_f8f6f4 v[240:243], v[120:127], a[64:71], v[240:243]\n\t" \
  "ds_read_b64_tr_b8 v[104:105], v17 offset:0\n\t" \
  "ds_read_b64_tr_b8 v[106:107], v17 offset:1024\n\t" \
  "v_mfma_f32_16x16x128_f8f6f4 v[244:247], v[120:127], a[72:79], v[244:247]\n\t" \
  "ds_read_b64_tr_b8 v[108:109], v17 offset:8192\n\t" \
  "ds_read_b64_tr_b8 v[110:111], v17 offset:9216\n\t" \
  "v_mfma_f32_16x16x128_f8f6f4 v[248:251], v[120:127], a[80:87], v[248:251]\n\t" \
  "v_mfma_f32_16x16x128_f8f6f4 v[252:255], v[120:127], a[88:95], v[252:255]\n\t" \
  "s_waitcnt vmcnt(35)\n\t" \
  "s_bitcmp1_b32 s65, 15\n\t" \
  "s_cbranch_scc1 OK72_%=\n\t" \
  "s_mov_b32 s67, 15\n\t" \
  "s_call_b64 s[68:69], POLL_%=\n\t" \
  "OK72_%=:\n\t" \
  "s_waitcnt lgkmcnt(4)\n\t" \
  "v_mfma_f32_16x16x128_f8f6f4 v[128:131], v[96:103], a[96:103], v[128:131]\n\t" \
  "ds_read_b64_tr_b8 v[112:113], v18 offset:0\n\t" \
  "ds_read_b64_tr_b8 v[114:115], v18 offset:1024\n\t" \
  "v_mfma_f32_16x16x128_f8f6f4 v[132:135], v[96:103], a[104:111], v[132:135]\n\t" \
  "ds_read_b64_tr_b8 v[116:117], v18 offset:8192\n\t" \
  "ds_read_b64_tr_b8 v[118:119], v18 offset:9216\n\t" \
  "v_mfma_f32_16x16x128_f8f6f4 v[136:139], v[96:103], a[112:119], v[136:139]\n\t" \
  "v_mfma_f32_16x16x128_f8f6f4 v[140:143], v[96:103], a[120:127], v[140:143]\n\t" \
  "s_waitcnt lgkmcnt(4)\n\t" \
  "v_mfma_f32_16x16x128_f8f6f4 v[144:147], v[104:111], a[96:103], v[144:147]\n\t" \
  "ds_read_b64_tr_b8 v[120:121], v19 offset:0\n\t" \
  "ds_read_b64_tr_b8 v[122:123], v19 offset:1024\n\t" \
  "v_mfma_f32_16x16x128_f8f6f4 v[148:151], v[104:111], a[104:111], v[148:151]\n\t" \
  "ds_read_b64_tr_b8 v[124:125], v19 offset:8192\n\t" \
  "ds_read_b64_tr_b8 v[126:127], v19 offset:9216\n\t" \
  "v_mfma_f32_16x16x128_f8f6f4 v[152:155], v[104:111], a[112:119], v[152:155]\n\t" \
  "v_mfma_f32_16x16x128_f8f6f4 v[156:159], v[104:111], a[120:127], v[156:159]\n\t" \
  "s_waitcnt lgkmcnt(4)\n\t" \
  "v_mfma_f32_16x16x128_f8f6f4 v[160:163], v[112:119], a[96:103], v[160:163]\n\t" \
  "ds_read_b64_tr_b8 v[96:97], v20 offset:0\n\t" \
  "ds_read_b64_tr_b8 v[98:99], v20 offset:1024\n\t" \
  "v_mfma_f32_16x16x128_f8f6f4 v[164:167], v[112:119], a[104:111], v[164:167]\n\t" \
  "ds_read_b64_tr_b8 v[100:101], v20 offset:8192\n\t" \
  "ds_read_b64_tr_b8 v[102:103], v20 offset:9216\n\t" \
  "v_mfma_f32_16x16x128_f8f6f4 v[168:171], v[112:119], a[112:119], v[168:171]\n\t" \
  "v_mfma_f32_16x16x128_f8f6f4 v[172:175], v[112:119], a[120:127], v[172:175]\n\t" \
  "s_waitcnt lgkmcnt(4)\n\t" \
  "v_mfma_f32_16x16x128_f8f6f4 v[176:179], v[120:127], a[96:103], v[176:179]\n\t" \
  "ds_read_b64_tr_b8 v[104:105], v21 offset:0\n\t" \
  "ds_read_b64_tr_b8 v[106:107], v21 offset:1024\n\t" \
  "v_mfma_f32_16x16x128_f8f6f4 v[180:183], v[120:127], a[104:111], v[180:183]\n\t" \
  "ds_read_b64_tr_b8 v[108:109], v21 offset:8192\n\t" \
  "ds_read_b64_tr_b8 v[110:111], v21 offset:9216\n\t" \
  "v_mfma_f32_16x16x128_f8f6f4 v[184:187], v[120:127], a[112:119], v[184:187]\n\t" \
  "v_mfma_f32_16x16x128_f8f6f4 v[188:191], v[120:127], a[120:127], v[188:191]\n\t" \
  "s_waitcnt lgkmcnt(4)\n\t" \
  "v_mfma_f32_16x16x128_f8f6f4 v[192:195], v[96:103], a[96:103], v[192:195]\n\t" \
  "ds_read_b64_tr_b8 v[112:113], v22 offset:0\n\t" \
  "ds_read_b64_tr_b8 v[114:115], v22 offset:1024\n\t" \
  "v_mfma_f32_16x16x128_f8f6f4 v[196:199], v[96:103], a[104:111], v[196:199]\n\t" \
  "ds_read_b64_tr_b8 v[116:117], v22 offset:8192\n\t" \
  "ds_read_b64_tr_b8 v[118:119], v22 offset:9216\n\t" \
  "v_mfma_f32_16x16x128_f8f6f4 v[200:203], v[96:103], a[112:119], v[200:203]\n\t" \
  "v_mfma_f32_16x16x128_f8f6f4 v[204:207], v[96:103], a[120:127], v[204:207]\n\t" \
  "s_waitcnt lgkmcnt(4)\n\t" \
  "v_mfma_f32_16x16x128_f8f6f4 v[208:211], v[104:111], a[96:103], v[208:211]\n\t" \
  "ds_read_b64_tr_b8 v[120:121], v23 offset:0\n\t" \
  "ds_read_b64_tr_b8 v[122:123], v23 offset:1024\n\t" \
  "v_mfma_f32_16x16x128_f8f6f4 v[212:215], v[104:111], a[104:111], v[212:215]\n\t" \
  "ds_read_b64_tr_b8 v[124:125], v23 offset:8192\n\t" \
  "ds_read_b64_tr_b8 v[126:127], v23 offset:9216\n\t" \
  "v_mfma_f32_16x16x128_f8f6f4 v[216:219], v[104:111], a[112:119], v[216:219]\n\t" \
  "v_mfma_f32_16x16x128_f8f6f4 v[220:223], v[104:111], a[120:127], v[220:223]\n\t" \
  "s_waitcnt vmcnt(11)\n\t" \
  "s_barrier\n\t" \
  "s_waitcnt lgkmcnt(4)\n\t" \
  "v_mfma_f32_16x16x128_f8f6f4 v[224:227], v[112:119], a[96:103], v[224:227]\n\t" \
  "ds_read_b64_tr_b8 v[96:97], v64 offset:49152\n\t" \
  "ds_read_b64_tr_b8 v[98:99], v64 offset:50176\n\t" \
  "v_mfma_f32_16x16x128_f8f6f4 v[228:231], v[112:119], a[104:111], v[228:231]\n\t" \
  "ds_read_b64_tr_b8 v[100:101], v64 offset:57344\n\t" \
  "ds_read_b64_tr_b8 v[102:103], v64 offset:58368\n\t" \
  "buffer_load_dwordx4 a[204:207], v84, s[44:47], s89 offen offset:3072 sc0 sc1\n\t" \
  "v_mfma_f32_16x16x128_f8f6f4 v[232:235], v[112:119], a[112:119], v[232:235]\n\t" \
  "buffer_load_dwordx4 a[208:211], v84, s[44:47], s90 offen offset:0 sc0 sc1\n\t" \
  "v_mfma_f32_16x16x128_f8f6f4 v[236:239], v[112:119], a[120:127], v[236:239]\n\t" \
  "s_waitcnt lgkmcnt(4)\n\t" \
  "v_mfma_f32_16x16x128_f8f6f4 v[240:243], v[120:127], a[96:103], v[240:243]\n\t" \
  "ds_read_b64_tr_b8 v[104:105], v65 offset:49152\n\t" \
  "ds_read_b64_tr_b8 v[106:107], v65 offset:50176\n\t" \
  "v_mfma_f32_16x16x128_f8f6f4 v[244:247], v[120:127], a[104:111], v[244:247]\n\t" \
  "ds_read_b64_tr_b8 v[108:109], v65 offset:57344\n\t" \
  "ds_read_b64_tr_b8 v[110:111], v65 offset:58368\n\t" \
  "buffer_load_dwordx4 a[212:215], v84, s[44:47], s90 offen offset:1024 sc0 sc1\n\t" \
  "v_mfma_f32_16x16x128_f8f6f4 v[248:251], v[120:127], a[112:119], v[248:251]\n\t" \
  "buffer_load_dwordx4 a[216:219], v84, s[44:47], s90 offen offset:2048 sc0 sc1\n\t" \
  "v_mfma_f32_16x16x128_f8f6f4 v[252:255], v[120:127], a[120:127], v[252:255]\n\t" \
  "s_waitcnt vmcnt(31)\n\t" \
  "s_bitcmp1_b32 s65, 16\n\t" \
  "s_cbranch_scc1 OK73_%=\n\t" \
  "s_mov_b32 s67, 16\n\t" \
  "s_call_b64 s[68:69], POLL_%=\n\t" \
  "OK73_%=:\n\t" \
  "s_waitcnt lgkmcnt(4)\n\t" \
  "v_mfma_f32_16x16x128_f8f6f4 v[128:131], v[96:103], a[128:135], v[128:131]\n\t" \
  "ds_read_b64_tr_b8 v[112:113], v66 offset:49152\n\t" \
  "ds_read_b64_tr_b8 v[114:115], v66 offset:50176\n\t" \
  "v_mfma_f32_16x16x128_f8f6f4 v[132:135], v[96:103], a[136:143], v[132:135]\n\t" \
  "ds_read_b64_tr_b8 v[116:117], v66 offset:57344\n\t" \
  "ds_read_b64_tr_b8 v[118:119], v66 offset:58368\n\t" \
  "buffer_load_dwordx4 a[220:223], v84, s[44:47], s90 offen offset:3072 sc0 sc1\n\t" \
  "v_mfma_f32_16x16x128_f8f6f4 v[136:139], v[96:103], a[144:151], v[136:139]\n\t" \
  "s_add_u32 s89, s62, 983040\n\t" \
  "s_add_u32 s90, s62, 987136\n\t" \
  "buffer_load_dwordx4 a[224:227], v84, s[44:47], s89 offen offset:0 sc0 sc1\n\t" \
  "v_mfma_f32_16x16x128_f8f6f4 v[140:143], v[96:103], a[152:159], v[140:143]\n\t" \
  "s_waitcnt lgkmcnt(4)\n\t" \
  "v_mfma_f32_16x16x128_f8f6f4 v[144:147], v[104:111], a[128:135], v[144:147]\n\t" \
  "ds_read_b64_tr_b8 v[120:121], v67 offset:49152\n\t" \
  "ds_read_b64_tr_b8 v[122:123], v67 offset:50176\n\t" \
  "v_mfma_f32_16x16x128_f8f6f4 v[148:151], v[104:111], a[136:143], v[148:151]\n\t" \
  "ds_read_b64_tr_b8 v[124:125], v67 offset:57344\n\t" \
  "ds_read_b64_tr_b8 v[126:127], v67 offset:58368\n\t" \
  "buffer_load_dwordx4 a[228:231], v84, s[44:47], s89 offen offset:1024 sc0 sc1\n\t" \
  "v_mfma_f32_16x16x128_f8f6f4 v[152:155], v[104:111], a[144:151], v[152:155]\n\t" \
  "buffer_load_dwordx4 a[232:235], v84, s[44:47], s89 offen offset:2048 sc0 sc1\n\t" \
  "v_mfma_f32_16x16x128_f8f6f4 v[156:159], v[104:111], a[152:159], v[156:159]\n\t" \
  "s_waitcnt lgkmcnt(4)\n\t" \
  "v_mfma_f32_16x16x128_f8f6f4 v[160:163], v[112:119], a[128:135], v[160:163]\n\t" \
  "ds_read_b64_tr_b8 v[96:97], v68 offset:49152\n\t" \
  "ds_read_b64_tr_b8 v[98:99], v68 offset:50176\n\t" \
  "v_mfma_f32_16x16x128_f8f6f4 v[164:167], v[112:119], a[136:143], v[164:167]\n\t" \
  "ds_read_b64_tr_b8 v[100:101], v68 offset:57344\n\t" \
  "ds_read_b64_tr_b8 v[102:103], v68 offset:58368\n\t" \
  "buffer_load_dwordx4 a[236:239], v84, s[44:47], s89 offen offset:3072 sc0 sc1\n\t" \
  "v_mfma_f32_16x16x128_f8f6f4 v[168:171], v[112:119], a[144:151], v[168:171]\n\t" \
  "buffer_load_dwordx4 a[240:243], v84, s[44:47], s90 offen offset:0 sc0 sc1\n\t" \
  "v_mfma_f32_16x16x128_f8f6f4 v[172:175], v[112:119], a[152:159], v[172:175]\n\t" \
  "s_waitcnt lgkmcnt(4)\n\t" \
  "v_mfma_f32_16x16x128_f8f6f4 v[176:179], v[120:127], a[128:135], v[176:179]\n\t" \
  "ds_read_b64_tr_b8 v[104:105], v69 offset:49152\n\t" \
  "ds_read_b64_tr_b8 v[106:107], v69 offset:50176\n\t" \
  "v_mfma_f32_16x16x128_f8f6f4 v[180:183], v[120:127], a[136:143], v[180:183]\n\t" \
  "ds_read_b64_tr_b8 v[108:109], v69 offset:57344\n\t" \
  "ds_read_b64_tr_b8 v[110:111], v69 offset:58368\n\t" \
  "buffer_load_dwordx4 a[244:247], v84, s[44:47], s90 offen offset:1024 sc0 sc1\n\t" \
  "v_mfma_f32_16x16x128_f8f6f4 v[184:187], v[120:127], a[144:151], v[184:187]\n\t" \
  "buffer_load_dwordx4 a[248:251], v84, s[44:47], s90 offen offset:2048 sc0 sc1\n\t" \
  "v_mfma_f32_16x16x128_f8f6f4 v[188:191], v[120:127], a[152:159], v[188:191]\n\t" \
  "s_waitcnt lgkmcnt(4)\n\t" \
  "v_mfma_f32_16x16x128_f8f6f4 v[192:195], v[96:103], a[128:135], v[192:195]\n\t" \
  "ds_read_b64_tr_b8 v[112:113], v70 offset:49152\n\t" \
  "ds_read_b64_tr_b8 v[114:115], v70 offset:50176\n\t" \
  "v_mfma_f32_16x16x128_f8f6f4 v[196:199], v[96:103], a[136:143], v[196:199]\n\t" \
  "ds_read_b64_tr_b8 v[116:117], v70 offset:57344\n\t" \
  "ds_read_b64_tr_b8 v[118:119], v70 offset:58368\n\t" \
  "buffer_load_dwordx4 a[252:255], v84, s[44:47], s90 offen offset:3072 sc0 sc1\n\t" \
  "v_mfma_f32_16x16x128_f8f6f4 v[200:203], v[96:103], a[144:151], v[200:203]\n\t" \
  "s_add_u32 s89, s62, 1048576\n\t" \
  "s_add_u32 s90, s62, 1052672\n\t" \
  "buffer_load_dwordx4 a[0:3], v84, s[44:47], s89 offen offset:0 sc0 sc1\n\t" \
  "v_mfma_f32_16x16x128_f8f6f4 v[204:207], v[96:103], a[152:159], v[204:207]\n\t" \
  "s_waitcnt lgkmcnt(4)\n\t" \
  "v_mfma_f32_16x16x128_f8f6f4 v[208:211], v[104:111], a[128:135], v[208:211]\n\t" \
  "ds_read_b64_tr_b8 v[120:121], v71 offset:49152\n\t" \
  "ds_read_b64_tr_b8 v[122:123], v71 offset:50176\n\t" \
  "v_mfma_f32_16x16x128_f8f6f4 v[212:215], v[104:111], a[136:143], v[212:215]\n\t" \
  "ds_read_b64_tr_b8 v[124:125], v71 offset:57344\n\t" \
  "ds_read_b64_tr_b8 v[126:127], v71 offset:58368\n\t" \
  "buffer_load_dwordx4 a[4:7], v84, s[44:47], s89 offen offset:1024 sc0 sc1\n\t" \
  "v_mfma_f32_16x16x128_f8f6f4 v[216:219], v[104:111], a[144:151], v[216:219]\n\t" \
  "buffer_load_dwordx4 a[8:11], v84, s[44:47], s89 offen offset:2048 sc0 sc1\n\t" \
  "v_mfma_f32_16x16x128_f8f6f4 v[220:223], v[104:111], a[152:159], v[220:223]\n\t" \
  "s_waitcnt vmcnt(23)\n\t" \
  "s_barrier\n\t" \
  "s_waitcnt lgkmcnt(4)\n\t" \
  "v_mfma_f32_16x16x128_f8f6f4 v[224:227], v[112:119], a[128:135], v[224:227]\n\t" \
  "ds_read_b64_tr_b8 v[96:97], v64 offset:0\n\t" \
  "ds_read_b64_tr_b8 v[98:99], v64 offset:1024\n\t" \
  "v_mfma_f32_16x16x128_f8f6f4 v[228:231], v[112:119], a[136:143], v[228:231]\n\t" \
  "ds_read_b64_tr_b8 v[100:101], v64 offset:8192\n\t" \
  "ds_read_b64_tr_b8 v[102:103], v64 offset:9216\n\t" \
  "buffer_load_dwordx4 a[12:15], v84, s[44:47], s89 offen offset:3072 sc0 sc1\n\t" \
  "v_mfma_f32_16x16x128_f8f6f4 v[232:235], v[112:119], a[144:151], v[232:235]\n\t" \
  "buffer_load_dwordx4 a[16:19], v84, s[44:47], s90 offen offset:0 sc0 sc1\n\t" \
  "v_mfma_f32_16x16x128_f8f6f4 v[236:239], v[112:119], a[152:159], v[236:239]\n\t" \
  "s_waitcnt lgkmcnt(4)\n\t" \
  "v_mfma_f32_16x16x128_f8f6f4 v[240:243], v[120:127], a[128:135], v[240:243]\n\t" \
  "ds_read_b64_tr_b8 v[104:105], v65 offset:0\n\t" \
  "ds_read_b64_tr_b8 v[106:107], v65 offset:1024\n\t" \
  "v_mfma_f32_16x16x128_f8f6f4 v[244:247], v[120:127], a[136:143], v[244:247]\n\t" \
  "ds_read_b64_tr_b8 v[108:109], v65 offset:8192\n\t" \
  "ds_read_b64_tr_b8 v[110:111], v65 offset:9216\n\t" \
  "buffer_load_dwordx4 a[20:23], v84, s[44:47], s90 offen offset:1024 sc0 sc1\n\t" \
  "v_mfma_f32_16x16x128_f8f6f4 v[248:251], v[120:127], a[144:151], v[248:251]\n\t" \
  "buffer_load_dwordx4 a[24:27], v84, s[44:47], s90 offen offset:2048 sc0 sc1\n\t" \
  "v_mfma_f32_16x16x128_f8f6f4 v[252:255], v[120:127], a[152:159], v[252:255]\n\t" \
  "s_waitcnt vmcnt(39)\n\t" \
  "global_load_dword v91, v87, s[70:71] sc0 sc1\n\t" \
  "s_bitcmp1_b32 s65, 17\n\t" \
  "s_cbranch_scc1 OK74_%=\n\t" \
  "s_mov_b32 s67, 17\n\t" \
  "s_call_b64 s[68:69], POLL_%=\n\t" \
  "OK74_%=:\n\t" \
  "s_waitcnt lgkmcnt(4)\n\t" \
  "v_mfma_f32_16x16x128_f8f6f4 v[128:131], v[96:103], a[160:167], v[128:131]\n\t" \
  "ds_read_b64_tr_b8 v[112:113], v66 offset:0\n\t" \
  "ds_read_b64_tr_b8 v[114:115], v66 offset:1024\n\t" \
  "v_mfma_f32_16x16x128_f8f6f4 v[132:135], v[96:103], a[168:175], v[132:135]\n\t" \
  "ds_read_b64_tr_b8 v[116:117], v66 offset:8192\n\t" \
  "ds_read_b64_tr_b8 v[118:119], v66 offset:9216\n\t" \
  "buffer_load_dwordx4 a[28:31], v84, s[44:47], s90 offen offset:3072 sc0 sc1\n\t" \
  "v_mfma_f32_16x16x128_f8f6f4 v[136:139], v[96:103], a[176:183], v[136:139]\n\t" \
  "s_add_u32 s89, s62, 1114112\n\t" \
  "s_add_u32 s90, s62, 1118208\n\t" \
  "buffer_load_dwordx4 a[32:35], v84, s[44:47], s89 offen offset:0 sc0 sc1\n\t" \
  "v_mfma_f32_16x16x128_f8f6f4 v[140:143], v[96:103], a[184:191], v[140:143]\n\t" \
  "s_waitcnt lgkmcnt(4)\n\t" \
  "v_mfma_f32_16x16x128_f8f6f4 v[144:147], v[104:111], a[160:167], v[144:147]\n\t" \
  "ds_read_b64_tr_b8 v[120:121], v67 offset:0\n\t" \
  "ds_read_b64_tr_b8 v[122:123], v67 offset:1024\n\t" \
  "v_mfma_f32_16x16x128_f8f6f4 v[148:151], v[104:111], a[168:175], v[148:151]\n\t" \
  "ds_read_b64_tr_b8 v[124:125], v67 offset:8192\n\t" \
  "ds_read_b64_tr_b8 v[126:127], v67 offset:9216\n\t" \
  "buffer_load_dwordx4 a[36:39], v84, s[44:47], s89 offen offset:1024 sc0 sc1\n\t" \
  "v_mfma_f32_16x16x128_f8f6f4 v[152:155], v[104:111], a[176:183], v[152:155]\n\t" \
  "buffer_load_dwordx4 a[40:43], v84, s[44:47], s89 offen offset:2048 sc0 sc1\n\t" \
  "v_mfma_f32_16x16x128_f8f6f4 v[156:159], v[104:111], a[184:191], v[156:159]\n\t" \
  "s_waitcnt lgkmcnt(4)\n\t" \
  "v_mfma_f32_16x16x128_f8f6f4 v[160:163], v[112:119], a[160:167], v[160:163]\n\t" \
  "ds_read_b64_tr_b8 v[96:97], v68 offset:0\n\t" \
  "ds_read_b64_tr_b8 v[98:99], v68 offset:1024\n\t" \
  "v_mfma_f32_16x16x128_f8f6f4 v[164:167], v[112:119], a[168:175], v[164:167]\n\t" \
  "ds_read_b64_tr_b8 v[100:101], v68 offset:8192\n\t" \
  "ds_read_b64_tr_b8 v[102:103], v68 offset:9216\n\t" \
  "buffer_load_dwordx4 a[44:47], v84, s[44:47], s89 offen offset:3072 sc0 sc1\n\t" \
  "v_mfma_f32_16x16x128_f8f6f4 v[168:171], v[112:119], a[176:183], v[168:171]\n\t" \
  "buffer_load_dwordx4 a[48:51], v84, s[44:47], s90 offen offset:0 sc0 sc1\n\t" \
  "v_mfma_f32_16x16x128_f8f6f4 v[172:175], v[112:119], a[184:191], v[172:175]\n\t" \
  "s_waitcnt lgkmcnt(4)\n\t" \
  "v_mfma_f32_16x16x128_f8f6f4 v[176:179], v[120:127], a[160:167], v[176:179]\n\t" \
  "ds_read_b64_tr_b8 v[104:105], v69 offset:0\n\t" \
  "ds_read_b64_tr_b8 v[106:107], v69 offset:1024\n\t" \
  "v_mfma_f32_16x16x128_f8f6f4 v[180:183], v[120:127], a[168:175], v[180:183]\n\t" \
  "ds_read_b64_tr_b8 v[108:109], v69 offset:8192\n\t" \
  "ds_read_b64_tr_b8 v[110:111], v69 offset:9216\n\t" \
  "buffer_load_dwordx4 a[52:55], v84, s[44:47], s90 offen offset:1024 sc0 sc1\n\t" \
  "v_mfma_f32_16x16x128_f8f6f4 v[184:187], v[120:127], a[176:183], v[184:187]\n\t" \
  "buffer_load_dwordx4 a[56:59], v84, s[44:47], s90 offen offset:2048 sc0 sc1\n\t" \
  "v_mfma_f32_16x16x128_f8f6f4 v[188:191], v[120:127], a[184:191], v[188:191]\n\t" \
  "s_waitcnt lgkmcnt(4)\n\t" \
  "v_mfma_f32_16x16x128_f8f6f4 v[192:195], v[96:103], a[160:167], v[192:195]\n\t" \
  "ds_read_b64_tr_b8 v[112:113], v70 offset:0\n\t" \
  "ds_read_b64_tr_b8 v[114:115], v70 offset:1024\n\t" \
  "v_mfma_f32_16x16x128_f8f6f4 v[196:199], v[96:103], a[168:175], v[196:199]\n\t" \
  "ds_read_b64_tr_b8 v[116:117], v70 offset:8192\n\t" \
  "ds_read_b64_tr_b8 v[118:119], v70 offset:9216\n\t" \
  "s_add_u32 s88, s56, 2097152\n\t" \
  "s_add_u32 m0, s60, 32768\n\t" \
  "s_nop 0\n\t" \
  "buffer_load_dwordx4 v80, s[36:39], s88 offen  lds\n\t" \
  "v_mfma_f32_16x16x128_f8f6f4 v[200:203], v[96:103], a[176:183], v[200:203]\n\t" \
  "s_add_u32 m0, s60, 33792\n\t" \
  "s_nop 0\n\t" \
  "buffer_load_dwordx4 v81, s[36:39], s88 offen  lds\n\t" \
  "v_mfma_f32_16x16x128_f8f6f4 v[204:207], v[96:103], a[184:191], v[204:207]\n\t" \
  "s_waitcnt lgkmcnt(4)\n\t" \
  "v_mfma_f32_16x16x128_f8f6f4 v[208:211], v[104:111], a[160:167], v[208:211]\n\t" \
  "ds_read_b64_tr_b8 v[120:121], v71 offset:0\n\t" \
  "ds_read_b64_tr_b8 v[122:123], v71 offset:1024\n\t" \
  "v_mfma_f32_16x16x128_f8f6f4 v[212:215], v[104:111], a[168:175], v[212:215]\n\t" \
  "ds_read_b64_tr_b8 v[124:125], v71 offset:8192\n\t" \
  "ds_read_b64_tr_b8 v[126:127], v71 offset:9216\n\t" \
  "s_add_u32 m0, s60, 34816\n\t" \
  "s_nop 0\n\t" \
  "buffer_load_dwordx4 v82, s[36:39], s88 offen  lds\n\t" \
  "v_mfma_f32_16x16x128_f8f6f4 v[216:219], v[104:111], a[176:183], v[216:219]\n\t" \
  "s_add_u32 m0, s60, 35840\n\t" \
  "s_nop 0\n\t" \
  "buffer_load_dwordx4 v83, s[36:39], s88 offen  lds\n\t" \
  "v_mfma_f32_16x16x128_f8f6f4 v[220:223], v[104:111], a[184:191], v[220:223]\n\t" \
  "s_waitcnt vmcnt(36)\n\t" \
  "s_barrier\n\t" \
  "s_waitcnt lgkmcnt(4)\n\t" \
  "v_mfma_f32_16x16x128_f8f6f4 v[224:227], v[112:119], a[160:167], v[224:227]\n\t" \
  "ds_read_b64_tr_b8 v[96:97], v64 offset:16384\n\t" \
  "ds_read_b64_tr_b8 v[98:99], v64 offset:17408\n\t" \
  "v_mfma_f32_16x16x128_f8f6f4 v[228:231], v[112:119], a[168:175], v[228:231]\n\t" \
  "ds_read_b64_tr_b8 v[100:101], v64 offset:24576\n\t" \
  "ds_read_b64_tr_b8 v[102:103], v64 offset:25600\n\t" \
  "s_add_u32 s88, s57, 2097152\n\t" \
  "s_add_u32 m0, s60, 65536\n\t" \
  "s_nop 0\n\t" \
  "buffer_load_dwordx4 v80, s[36:39], s88 offen  lds\n\t" \
  "v_mfma_f32_16x16x128_f8f6f4 v[232:235], v[112:119], a[176:183], v[232:235]\n\t" \
  "s_add_u32 m0, s60, 66560\n\t" \
  "s_nop 0\n\t" \
  "buffer_load_dwordx4 v81, s[36:39], s88 offen  lds\n\t" \
  "v_mfma_f32_16x16x128_f8f6f4 v[236:239], v[112:119], a[184:191], v[236:239]\n\t" \
  "s_waitcnt lgkmcnt(4)\n\t" \
  "v_mfma_f32_16x16x128_f8f6f4 v[240:243], v[120:127], a[160:167], v[240:243]\n\t" \
  "ds_read_b64_tr_b8 v[104:105], v65 offset:16384\n\t" \
  "ds_read_b64_tr_b8 v[106:107], v65 offset:17408\n\t" \
  "v_mfma_f32_16x16x128_f8f6f4 v[244:247], v[120:127], a[168:175], v[244:247]\n\t" \
  "ds_read_b64_tr_b8 v[108:109], v65 offset:24576\n\t" \
  "ds_read_b64_tr_b8 v[110:111], v65 offset:25600\n\t" \
  "s_add_u32 m0, s60, 67584\n\t" \
  "s_nop 0\n\t" \
  "buffer_load_dwordx4 v82, s[36:39], s88 offen  lds\n\t" \
  "v_mfma_f32_16x16x128_f8f6f4 v[248:251], v[120:127], a[176:183], v[248:251]\n\t" \
  "s_add_u32 m0, s60, 68608\n\t" \
  "s_nop 0\n\t" \
  "buffer_load_dwordx4 v83, s[36:39], s88 offen  lds\n\t" \
  "v_mfma_f32_16x16x128_f8f6f4 v[252:255], v[120:127], a[184:191], v[252:255]\n\t" \
  "s_waitcnt vmcnt(16)\n\t" \
  "v_cmp_le_u32 vcc, s64, v91\n\t" \
  "s_nop 1\n\t" \
  "s_or_b32 s65, s65, vcc_lo\n\t" \
  "s_bitcmp1_b32 s65, 18\n\t" \
  "s_cbranch_scc1 OK75_%=\n\t" \
  "s_mov_b32 s67, 18\n\t" \
  "s_call_b64 s[68:69], POLL_%=\n\t" \
  "OK75_%=:\n\t" \
  "s_waitcnt lgkmcnt(4)\n\t" \
  "v_mfma_f32_16x16x128_f8f6f4 v[128:131], v[96:103], a[192:199], v[128:131]\n\t" \
  "ds_read_b64_tr_b8 v[112:113], v66 offset:16384\n\t" \
  "ds_read_b64_tr_b8 v[114:115], v66 offset:17408\n\t" \
  "s_bfe_u32 s96, s65, 262163\n\t" \
  "s_cmp_lg_u32 s96, 15\n\t" \
  "s_cbranch_scc1 XWS76_%=\n\t" \
  "s_add_u32 s96, s62, 1245184\n\t" \
  "buffer_load_dword v8, v13, s[44:47], s96 offen sc0 sc1\n\t" \
  "XWS76_%=:\n\t" \
  "v_mfma_f32_16x16x128_f8f6f4 v[132:135], v[96:103], a[200:207], v[132:135]\n\t" \
  "ds_read_b64_tr_b8 v[116:117], v66 offset:24576\n\t" \
  "ds_read_b64_tr_b8 v[118:119], v66 offset:25600\n\t" \
  "s_add_u32 s88, s58, 2097152\n\t" \
  "s_add_u32 m0, s60, 81920\n\t" \
  "s_nop 0\n\t" \
  "buffer_load_dwordx4 v80, s[36:39], s88 offen  lds\n\t" \
  "v_mfma_f32_16x16x128_f8f6f4 v[136:139], v[96:103], a[208:215], v[136:139]\n\t" \
  "s_add_u32 m0, s60, 82944\n\t" \
  "s_nop 0\n\t" \
  "buffer_load_dwordx4 v81, s[36:39], s88 offen  lds\n\t" \
  "v_mfma_f32_16x16x128_f8f6f4 v[140:143], v[96:103], a[216:223], v[140:143]\n\t" \
  "s_waitcnt lgkmcnt(4)\n\t" \
  "v_mfma_f32_16x16x128_f8f6f4 v[144:147], v[104:111], a[192:199], v[144:147]\n\t" \
  "ds_read_b64_tr_b8 v[120:121], v67 offset:16384\n\t" \
  "ds_read_b64_tr_b8 v[122:123], v67 offset:17408\n\t" \
  "v_mfma_f32_16x16x128_f8f6f4 v[148:151], v[104:111], a[200:207], v[148:151]\n\t" \
  "ds_read_b64_tr_b8 v[124:125], v67 offset:24576\n\t" \
  "ds_read_b64_tr_b8 v[126:127], v67 offset:25600\n\t" \
  "s_add_u32 m0, s60, 83968\n\t" \
  "s_nop 0\n\t" \
  "buffer_load_dwordx4 v82, s[36:39], s88 offen  lds\n\t" \
  "v_mfma_f32_16x16x128_f8f6f4 v[152:155], v[104:111], a[208:215], v[152:155]\n\t" \
  "s_add_u32 m0, s60, 84992\n\t" \
  "s_nop 0\n\t" \
  "buffer_load_dwordx4 v83, s[36:39], s88 offen  lds\n\t" \
  "v_mfma_f32_16x16x128_f8f6f4 v[156:159], v[104:111], a[216:223], v[156:159]\n\t" \
  "s_waitcnt lgkmcnt(4)\n\t" \
  "v_mfma_f32_16x16x128_f8f6f4 v[160:163], v[112:119], a[192:199], v[160:163]\n\t" \
  "ds_read_b64_tr_b8 v[96:97], v68 offset:16384\n\t" \
  "ds_read_b64_tr_b8 v[98:99], v68 offset:17408\n\t" \
  "v_mfma_f32_16x16x128_f8f6f4 v[164:167], v[112:119], a[200:207], v[164:167]\n\t" \
  "ds_read_b64_tr_b8 v[100:101], v68 offset:24576\n\t" \
  "ds_read_b64_tr_b8 v[102:103], v68 offset:25600\n\t" \
  "s_add_u32 s88, s52, 4194304\n\t" \
  "s_add_u32 m0, s60, 98304\n\t" \
  "s_nop 0\n\t" \
  "buffer_load_dwordx4 v80, s[36:39], s88 offen  lds\n\t" \
  "v_mfma_f32_16x16x128_f8f6f4 v[168:171], v[112:119], a[208:215], v[168:171]\n\t" \
  "s_add_u32 m0, s60, 99328\n\t" \
  "s_nop 0\n\t" \
  "buffer_load_dwordx4 v81, s[36:39], s88 offen  lds\n\t" \
  "v_mfma_f32_16x16x128_f8f6f4 v[172:175], v[112:119], a[216:223], v[172:175]\n\t" \
  "s_waitcnt lgkmcnt(4)\n\t" \
  "v_mfma_f32_16x16x128_f8f6f4 v[176:179], v[120:127], a[192:199], v[176:179]\n\t" \
  "ds_read_b64_tr_b8 v[104:105], v69 offset:16384\n\t" \
  "ds_read_b64_tr_b8 v[106:107], v69 offset:17408\n\t" \
  "v_mfma_f32_16x16x128_f8f6f4 v[180:183], v[120:127], a[200:207], v[180:183]\n\t" \
  "ds_read_b64_tr_b8 v[108:109], v69 offset:24576\n\t" \
  "ds_read_b64_tr_b8 v[110:111], v69 offset:25600\n\t" \
  "s_add_u32 m0, s60, 100352\n\t" \
  "s_nop 0\n\t" \
  "buffer_load_dwordx4 v82, s[36:39], s88 offen  lds\n\t" \
  "v_mfma_f32_16x16x128_f8f6f4 v[184:187], v[120:127], a[208:215], v[184:187]\n\t" \
  "s_add_u32 m0, s60, 101376\n\t" \
  "s_nop 0\n\t" \
  "buffer_load_dwordx4 v83, s[36:39], s88 offen  lds\n\t" \
  "v_mfma_f32_16x16x128_f8f6f4 v[188:191], v[120:127], a[216:223], v[188:191]\n\t" \
  "s_waitcnt lgkmcnt(4)\n\t" \
  "v_mfma_f32_16x16x128_f8f6f4 v[192:195], v[96:103], a[192:199], v[192:195]\n\t" \
  "ds_read_b64_tr_b8 v[112:113], v70 offset:16384\n\t" \
  "ds_read_b64_tr_b8 v[114:115], v70 offset:17408\n\t" \
  "v_mfma_f32_16x16x128_f8f6f4 v[196:199], v[96:103], a[200:207], v[196:199]\n\t" \
  "ds_read_b64_tr_b8 v[116:117], v70 offset:24576\n\t" \
  "ds_read_b64_tr_b8 v[118:119], v70 offset:25600\n\t" \
  "buffer_load_dwordx4 a[60:63], v84, s[44:47], s90 offen offset:3072 sc0 sc1\n\t" \
  "v_mfma_f32_16x16x128_f8f6f4 v[200:203], v[96:103], a[208:215], v[200:203]\n\t" \
  "s_add_u32 s89, s62, 1179648\n\t" \
  "s_add_u32 s90, s62, 1183744\n\t" \
  "buffer_load_dwordx4 a[64:67], v84, s[44:47], s89 offen offset:0 sc0 sc1\n\t" \
  "v_mfma_f32_16x16x128_f8f6f4 v[204:207], v[96:103], a[216:223], v[204:207]\n\t" \
  "s_waitcnt lgkmcnt(4)\n\t" \
  "v_mfma_f32_16x16x128_f8f6f4 v[208:211], v[104:111], a[192:199], v[208:211]\n\t" \
  "ds_read_b64_tr_b8 v[120:121], v71 offset:16384\n\t" \
  "ds_read_b64_tr_b8 v[122:123], v71 offset:17408\n\t" \
  "v_mfma_f32_16x16x128_f8f6f4 v[212:215], v[104:111], a[200:207], v[212:215]\n\t" \
  "ds_read_b64_tr_b8 v[124:125], v71 offset:24576\n\t" \
  "ds_read_b64_tr_b8 v[126:127], v71 offset:25600\n\t" \
  "buffer_load_dwordx4 a[68:71], v84, s[44:47], s89 offen offset:1024 sc0 sc1\n\t" \
  "v_mfma_f32_16x16x128_f8f6f4 v[216:219], v[104:111], a[208:215], v[216:219]\n\t" \
  "v_mfma_f32_16x16x128_f8f6f4 v[220:223], v[104:111], a[216:223], v[220:223]\n\t" \
  "s_waitcnt vmcnt(15)\n\t" \
  "s_barrier\n\t" \
  "s_waitcnt lgkmcnt(4)\n\t" \
  "v_mfma_f32_16x16x128_f8f6f4 v[224:227], v[112:119], a[192:199], v[224:227]\n\t" \
  "ds_read_b64_tr_b8 v[96:97], v64 offset:32768\n\t" \
  "ds_read_b64_tr_b8 v[98:99], v64 offset:33792\n\t" \
  "v_mfma_f32_16x16x128_f8f6f4 v[228:231], v[112:119], a[200:207], v[228:231]\n\t" \
  "ds_read_b64_tr_b8 v[100:101], v64 offset:40960\n\t" \
  "ds_read_b64_tr_b8 v[102:103], v64 offset:41984\n\t" \
  "v_mfma_f32_16x16x128_f8f6f4 v[232:235], v[112:119], a[208:215], v[232:235]\n\t" \
  "v_mfma_f32_16x16x128_f8f6f4 v[236:239], v[112:119], a[216:223], v[236:239]\n\t" \
  "s_cmp_lg_u32 s63, 3\n\t" \
  "s_cbranch_scc1 PS77_%=\n\t" \
  "s_waitcnt vmcnt(48)\n\t" \
  "v_mov_b32 v89, s64\n\t" \
  "s_mov_b64 exec, 1\n\t" \
  "global_store_dword v88, v89, s[76:77] sc0 sc1\n\t" \
  "s_mov_b64 exec, -1\n\t" \
  "PS77_%=:\n\t" \
  "s_waitcnt lgkmcnt(4)\n\t" \
  "v_mfma_f32_16x16x128_f8f6f4 v[240:243], v[120:127], a[192:199], v[240:243]\n\t" \
  "ds_read_b64_tr_b8 v[104:105], v65 offset:32768\n\t" \
  "ds_read_b64_tr_b8 v[106:107], v65 offset:33792\n\t" \
  "v_mfma_f32_16x16x128_f8f6f4 v[244:247], v[120:127], a[200:207], v[244:247]\n\t" \
  "ds_read_b64_tr_b8 v[108:109], v65 offset:40960\n\t" \
  "ds_read_b64_tr_b8 v[110:111], v65 offset:41984\n\t" \
  "v_mfma_f32_16x16x128_f8f6f4 v[248:251], v[120:127], a[208:215], v[248:251]\n\t" \
  "v_mfma_f32_16x16x128_f8f6f4 v[252:255], v[120:127], a[216:223], v[252:255]\n\t" \
  "s_waitcnt vmcnt(35)\n\t" \
  "s_bitcmp1_b32 s65, 19\n\t" \
  "s_cbranch_scc1 OK78_%=\n\t" \
  "s_mov_b32 s67, 19\n\t" \
  "s_call_b64 s[68:69], POLL_%=\n\t" \
  "OK78_%=:\n\t" \
  "s_waitcnt lgkmcnt(4)\n\t" \
  "v_mfma_f32_16x16x128_f8f6f4 v[128:131], v[96:103], a[224:231], v[128:131]\n\t" \
  "ds_read_b64_tr_b8 v[112:113], v66 offset:32768\n\t" \
  "ds_read_b64_tr_b8 v[114:115], v66 offset:33792\n\t" \
  "v_mfma_f32_16x16x128_f8f6f4 v[132:135], v[96:103], a[232:239], v[132:135]\n\t" \
  "ds_read_b64_tr_b8 v[116:117], v66 offset:40960\n\t" \
  "ds_read_b64_tr_b8 v[118:119], v66 offset:41984\n\t" \
  "v_mfma_f32_16x16x128_f8f6f4 v[136:139], v[96:103], a[240:247], v[136:139]\n\t" \
  "v_mfma_f32_16x16x128_f8f6f4 v[140:143], v[96:103], a[248:255], v[140:143]\n\t" \
  "s_waitcnt lgkmcnt(4)\n\t" \
  "v_mfma_f32_16x16x128_f8f6f4 v[144:147], v[104:111], a[224:231], v[144:147]\n\t" \
  "ds_read_b64_tr_b8 v[120:121], v67 offset:32768\n\t" \
  "ds_read_b64_tr_b8 v[122:123], v67 offset:33792\n\t" \
  "v_mfma_f32_16x16x128_f8f6f4 v[148:151], v[104:111], a[232:239], v[148:151]\n\t" \
  "ds_read_b64_tr_b8 v[124:125], v67 offset:40960\n\t" \
  "ds_read_b64_tr_b8 v[126:127], v67 offset:41984\n\t" \
  "v_mfma_f32_16x16x128_f8f6f4 v[152:155], v[104:111], a[240:247], v[152:155]\n\t" \
  "v_mfma_f32_16x16x128_f8f6f4 v[156:159], v[104:111], a[248:255], v[156:159]\n\t" \
  "s_waitcnt lgkmcnt(4)\n\t" \
  "v_mfma_f32_16x16x128_f8f6f4 v[160:163], v[112:119], a[224:231], v[160:163]\n\t" \
  "ds_read_b64_tr_b8 v[96:97], v68 offset:32768\n\t" \
  "ds_read_b64_tr_b8 v[98:99], v68 offset:33792\n\t" \
  "v_mfma_f32_16x16x128_f8f6f4 v[164:167], v[112:119], a[232:239], v[164:167]\n\t" \
  "ds_read_b64_tr_b8 v[100:101], v68 offset:40960\n\t" \
  "ds_read_b64_tr_b8 v[102:103], v68 offset:41984\n\t" \
  "v_mfma_f32_16x16x128_f8f6f4 v[168:171], v[112:119], a[240:247], v[168:171]\n\t" \
  "v_mfma_f32_16x16x128_f8f6f4 v[172:175], v[112:119], a[248:255], v[172:175]\n\t" \
  "s_waitcnt lgkmcnt(4)\n\t" \
  "v_mfma_f32_16x16x128_f8f6f4 v[176:179], v[120:127], a[224:231], v[176:179]\n\t" \
  "ds_read_b64_tr_b8 v[104:105], v69 offset:32768\n\t" \
  "ds_read_b64_tr_b8 v[106:107], v69 offset:33792\n\t" \
  "v_mfma_f32_16x16x128_f8f6f4 v[180:183], v[120:127], a[232:239], v[180:183]\n\t" \
  "ds_read_b64_tr_b8 v[108:109], v69 offset:40960\n\t" \
  "ds_read_b64_tr_b8 v[110:111], v69 offset:41984\n\t" \
  "v_mfma_f32_16x16x128_f8f6f4 v[184:187], v[120:127], a[240:247], v[184:187]\n\t" \
  "v_mfma_f32_16x16x128_f8f6f4 v[188:191], v[120:127], a[248:255], v[188:191]\n\t" \
  "s_waitcnt lgkmcnt(4)\n\t" \
  "v_mfma_f32_16x16x128_f8f6f4 v[192:195], v[96:103], a[224:231], v[192:195]\n\t" \
  "ds_read_b64_tr_b8 v[112:113], v70 offset:32768\n\t" \
  "ds_read_b64_tr_b8 v[114:115], v70 offset:33792\n\t" \
  "v_mfma_f32_16x16x128_f8f6f4 v[196:199], v[96:103], a[232:239], v[196:199]\n\t" \
  "ds_read_b64_tr_b8 v[116:117], v70 offset:40960\n\t" \
  "ds_read_b64_tr_b8 v[118:119], v70 offset:41984\n\t" \
  "v_mfma_f32_16x16x128_f8f6f4 v[200:203], v[96:103], a[240:247], v[200:203]\n\t" \
  "v_mfma_f32_16x16x128_f8f6f4 v[204:207], v[96:103], a[248:255], v[204:207]\n\t" \
  "s_waitcnt lgkmcnt(4)\n\t" \
  "v_mfma_f32_16x16x128_f8f6f4 v[208:211], v[104:111], a[224:231], v[208:211]\n\t" \
  "ds_read_b64_tr_b8 v[120:121], v71 offset:32768\n\t" \
  "ds_read_b64_tr_b8 v[122:123], v71 offset:33792\n\t" \
  "v_mfma_f32_16x16x128_f8f6f4 v[212:215], v[104:111], a[232:239], v[212:215]\n\t" \
  "ds_read_b64_tr_b8 v[124:125], v71 offset:40960\n\t" \
  "ds_read_b64_tr_b8 v[126:127], v71 offset:41984\n\t" \
  "v_mfma_f32_16x16x128_f8f6f4 v[216:219], v[104:111], a[240:247], v[216:219]\n\t" \
  "v_mfma_f32_16x16x128_f8f6f4 v[220:223], v[104:111], a[248:255], v[220:223]\n\t" \
  "s_waitcnt vmcnt(11)\n\t" \
  "s_barrier\n\t" \
  "s_waitcnt lgkmcnt(4)\n\t" \
  "v_mfma_f32_16x16x128_f8f6f4 v[224:227], v[112:119], a[224:231], v[224:227]\n\t" \
  "ds_read_b64_tr_b8 v[96:97], v72 offset:0\n\t" \
  "ds_read_b64_tr_b8 v[98:99], v72 offset:1024\n\t" \
  "v_mfma_f32_16x16x128_f8f6f4 v[228:231], v[112:119], a[232:239], v[228:231]\n\t" \
  "ds_read_b64_tr_b8 v[100:101], v72 offset:8192\n\t" \
  "ds_read_b64_tr_b8 v[102:103], v72 offset:9216\n\t" \
  "buffer_load_dwordx4 a[72:75], v84, s[44:47], s89 offen offset:2048 sc0 sc1\n\t" \
  "v_mfma_f32_16x16x128_f8f6f4 v[232:235], v[112:119], a[240:247], v[232:235]\n\t" \
  "buffer_load_dwordx4 a[76:79], v84, s[44:47], s89 offen offset:3072 sc0 sc1\n\t" \
  "v_mfma_f32_16x16x128_f8f6f4 v[236:239], v[112:119], a[248:255], v[236:239]\n\t" \
  "s_waitcnt lgkmcnt(4)\n\t" \
  "v_mfma_f32_16x16x128_f8f6f4 v[240:243], v[120:127], a[224:231], v[240:243]\n\t" \
  "ds_read_b64_tr_b8 v[104:105], v73 offset:0\n\t" \
  "ds_read_b64_tr_b8 v[106:107], v73 offset:1024\n\t" \
  "v_mfma_f32_16x16x128_f8f6f4 v[244:247], v[120:127], a[232:239], v[244:247]\n\t" \
  "ds_read_b64_tr_b8 v[108:109], v73 offset:8192\n\t" \
  "ds_read_b64_tr_b8 v[110:111], v73 offset:9216\n\t" \
  "buffer_load_dwordx4 a[80:83], v84, s[44:47], s90 offen offset:0 sc0 sc1\n\t" \
  "v_mfma_f32_16x16x128_f8f6f4 v[248:251], v[120:127], a[240:247], v[248:251]\n\t" \
  "buffer_load_dwordx4 a[84:87], v84, s[44:47], s90 offen offset:1024 sc0 sc1\n\t" \
  "v_mfma_f32_16x16x128_f8f6f4 v[252:255], v[120:127], a[248:255], v[252:255]\n\t" \
  "s_waitcnt vmcnt(30)\n\t" \
  "s_bitcmp1_b32 s65, 20\n\t" \
  "s_cbranch_scc1 OK79_%=\n\t" \
  "s_mov_b32 s67, 20\n\t" \
  "s_call_b64 s[68:69], POLL_%=\n\t" \
  "OK79_%=:\n\t" \
  "s_waitcnt lgkmcnt(4)\n\t" \
  "v_mfma_f32_16x16x128_f8f6f4 v[128:131], v[96:103], a[0:7], v[128:131]\n\t" \
  "ds_read_b64_tr_b8 v[112:113], v74 offset:0\n\t" \
  "ds_read_b64_tr_b8 v[114:115], v74 offset:1024\n\t" \
  "v_mfma_f32_16x16x128_f8f6f4 v[132:135], v[96:103], a[8:15], v[132:135]\n\t" \
  "ds_read_b64_tr_b8 v[116:117], v74 offset:8192\n\t" \
  "ds_read_b64_tr_b8 v[118:119], v74 offset:9216\n\t" \
  "buffer_load_dwordx4 a[88:91], v84, s[44:47], s90 offen offset:2048 sc0 sc1\n\t" \
  "v_mfma_f32_16x16x128_f8f6f4 v[136:139], v[96:103], a[16:23], v[136:139]\n\t" \
  "buffer_load_dwordx4 a[92:95], v84, s[44:47], s90 offen offset:3072 sc0 sc1\n\t" \
  "v_mfma_f32_16x16x128_f8f6f4 v[140:143], v[96:103], a[24:31], v[140:143]\n\t" \
  "s_waitcnt lgkmcnt(4)\n\t" \
  "v_mfma_f32_16x16x128_f8f6f4 v[144:147], v[104:111], a[0:7], v[144:147]\n\t" \
  "ds_read_b64_tr_b8 v[120:121], v75 offset:0\n\t" \
  "ds_read_b64_tr_b8 v[122:123], v75 offset:1024\n\t" \
  "v_mfma_f32_16x16x128_f8f6f4 v[148:151], v[104:111], a[8:15], v[148:151]\n\t" \
  "ds_read_b64_tr_b8 v[124:125], v75 offset:8192\n\t" \
  "ds_read_b64_tr_b8 v[126:127], v75 offset:9216\n\t" \
  "s_add_u32 s89, s62, 1245184\n\t" \
  "s_add_u32 s90, s62, 1249280\n\t" \
  "buffer_load_dwordx4 a[96:99], v84, s[44:47], s89 offen offset:0 sc0 sc1\n\t" \
  "v_mfma_f32_16x16x128_f8f6f4 v[152:155], v[104:111], a[16:23], v[152:155]\n\t" \
  "buffer_load_dwordx4 a[100:103], v84, s[44:47], s89 offen offset:1024 sc0 sc1\n\t" \
  "v_mfma_f32_16x16x128_f8f6f4 v[156:159], v[104:111], a[24:31], v[156:159]\n\t" \
  "s_waitcnt lgkmcnt(4)\n\t" \
  "v_mfma_f32_16x16x128_f8f6f4 v[160:163], v[112:119], a[0:7], v[160:163]\n\t" \
  "ds_read_b64_tr_b8 v[96:97], v76 offset:0\n\t" \
  "ds_read_b64_tr_b8 v[98:99], v76 offset:1024\n\t" \
  "v_mfma_f32_16x16x128_f8f6f4 v[164:167], v[112:119], a[8:15], v[164:167]\n\t" \
  "ds_read_b64_tr_b8 v[100:101], v76 offset:8192\n\t" \
  "ds_read_b64_tr_b8 v[102:103], v76 offset:9216\n\t" \
  "buffer_load_dwordx4 a[104:107], v84, s[44:47], s89 offen offset:2048 sc0 sc1\n\t" \
  "v_mfma_f32_16x16x128_f8f6f4 v[168:171], v[112:119], a[16:23], v[168:171]\n\t" \
  "buffer_load_dwordx4 a[108:111], v84, s[44:47], s89 offen offset:3072 sc0 sc1\n\t" \
  "v_mfma_f32_16x16x128_f8f6f4 v[172:175], v[112:119], a[24:31], v[172:175]\n\t" \
  "s_waitcnt lgkmcnt(4)\n\t" \
  "v_mfma_f32_16x16x128_f8f6f4 v[176:179], v[120:127], a[0:7], v[176:179]\n\t" \
  "ds_read_b64_tr_b8 v[104:105], v77 offset:0\n\t" \
  "ds_read_b64_tr_b8 v[106:107], v77 offset:1024\n\t" \
  "v_mfma_f32_16x16x128_f8f6f4 v[180:183], v[120:127], a[8:15], v[180:183]\n\t" \
  "ds_read_b64_tr_b8 v[108:109], v77 offset:8192\n\t" \
  "ds_read_b64_tr_b8 v[110:111], v77 offset:9216\n\t" \
  "buffer_load_dwordx4 a[112:115], v84, s[44:47], s90 offen offset:0 sc0 sc1\n\t" \
  "v_mfma_f32_16x16x128_f8f6f4 v[184:187], v[120:127], a[16:23], v[184:187]\n\t" \
  "buffer_load_dwordx4 a[116:119], v84, s[44:47], s90 offen offset:1024 sc0 sc1\n\t" \
  "v_mfma_f32_16x16x128_f8f6f4 v[188:191], v[120:127], a[24:31], v[188:191]\n\t" \
  "s_waitcnt lgkmcnt(4)\n\t" \
  "v_mfma_f32_16x16x128_f8f6f4 v[192:195], v[96:103], a[0:7], v[192:195]\n\t" \
  "ds_read_b64_tr_b8 v[112:113], v78 offset:0\n\t" \
  "ds_read_b64_tr_b8 v[114:115], v78 offset:1024\n\t" \
  "v_mfma_f32_16x16x128_f8f6f4 v[196:199], v[96:103], a[8:15], v[196:199]\n\t" \
  "ds_read_b64_tr_b8 v[116:117], v78 offset:8192\n\t" \
  "ds_read_b64_tr_b8 v[118:119], v78 offset:9216\n\t" \
  "buffer_load_dwordx4 a[120:123], v84, s[44:47], s90 offen offset:2048 sc0 sc1\n\t" \
  "v_mfma_f32_16x16x128_f8f6f4 v[200:203], v[96:103], a[16:23], v[200:203]\n\t" \
  "buffer_load_dwordx4 a[124:127], v84, s[44:47], s90 offen offset:3072 sc0 sc1\n\t" \
  "v_mfma_f32_16x16x128_f8f6f4 v[204:207], v[96:103], a[24:31], v[204:207]\n\t" \
  "s_waitcnt lgkmcnt(4)\n\t" \
  "v_mfma_f32_16x16x128_f8f6f4 v[208:211], v[104:111], a[0:7], v[208:211]\n\t" \
  "ds_read_b64_tr_b8 v[120:121], v79 offset:0\n\t" \
  "ds_read_b64_tr_b8 v[122:123], v79 offset:1024\n\t" \
  "v_mfma_f32_16x16x128_f8f6f4 v[212:215], v[104:111], a[8:15], v[212:215]\n\t" \
  "ds_read_b64_tr_b8 v[124:125], v79 offset:8192\n\t" \
  "ds_read_b64_tr_b8 v[126:127], v79 offset:9216\n\t" \
  "s_add_u32 s89, s62, 1310720\n\t" \
  "s_add_u32 s90, s62, 1314816\n\t" \
  "buffer_load_dwordx4 a[128:131], v84, s[44:47], s89 offen offset:0 sc0 sc1\n\t" \
  "v_mfma_f32_16x16x128_f8f6f4 v[216:219], v[104:111], a[16:23], v[216:219]\n\t" \
  "buffer_load_dwordx4 a[132:135], v84, s[44:47], s89 offen offset:1024 sc0 sc1\n\t" \
  "v_mfma_f32_16x16x128_f8f6f4 v[220:223], v[104:111], a[24:31], v[220:223]\n\t" \
  "s_waitcnt vmcnt(23)\n\t" \
  "s_barrier\n\t" \
  "s_waitcnt lgkmcnt(4)\n\t" \
  "v_mfma_f32_16x16x128_f8f6f4 v[224:227], v[112:119], a[0:7], v[224:227]\n\t" \
  "ds_read_b64_tr_b8 v[96:97], v72 offset:16384\n\t" \
  "ds_read_b64_tr_b8 v[98:99], v72 offset:17408\n\t" \
  "v_mfma_f32_16x16x128_f8f6f4 v[228:231], v[112:119], a[8:15], v[228:231]\n\t" \
  "ds_read_b64_tr_b8 v[100:101], v72 offset:24576\n\t" \
  "ds_read_b64_tr_b8 v[102:103], v72 offset:25600\n\t" \
  "buffer_load_dwordx4 a[136:139], v84, s[44:47], s89 offen offset:2048 sc0 sc1\n\t" \
  "v_mfma_f32_16x16x128_f8f6f4 v[232:235], v[112:119], a[16:23], v[232:235]\n\t" \
  "buffer_load_dwordx4 a[140:143], v84, s[44:47], s89 offen offset:3072 sc0 sc1\n\t" \
  "v_mfma_f32_16x16x128_f8f6f4 v[236:239], v[112:119], a[24:31], v[236:239]\n\t" \
  "s_waitcnt lgkmcnt(4)\n\t" \
  "v_mfma_f32_16x16x128_f8f6f4 v[240:243], v[120:127], a[0:7], v[240:243]\n\t" \
  "ds_read_b64_tr_b8 v[104:105], v73 offset:16384\n\t" \
  "ds_read_b64_tr_b8 v[106:107], v73 offset:17408\n\t" \
  "v_mfma_f32_16x16x128_f8f6f4 v[244:247], v[120:127], a[8:15], v[244:247]\n\t" \
  "ds_read_b64_tr_b8 v[108:109], v73 offset:24576\n\t" \
  "ds_read_b64_tr_b8 v[110:111], v73 offset:25600\n\t" \
  "buffer_load_dwordx4 a[144:147], v84, s[44:47], s90 offen offset:0 sc0 sc1\n\t" \
  "v_mfma_f32_16x16x128_f8f6f4 v[248:251], v[120:127], a[16:23], v[248:251]\n\t" \
  "buffer_load_dwordx4 a[148:151], v84, s[44:47], s90 offen offset:1024 sc0 sc1\n\t" \
  "v_mfma_f32_16x16x128_f8f6f4 v[252:255], v[120:127], a[24:31], v[252:255]\n\t" \
  "s_waitcnt vmcnt(22)\n\t" \
  "s_bitcmp1_b32 s65, 21\n\t" \
  "s_cbranch_scc1 OK80_%=\n\t" \
  "s_mov_b32 s67, 21\n\t" \
  "s_call_b64 s[68:69], POLL_%=\n\t" \
  "OK80_%=:\n\t" \
  "s_waitcnt lgkmcnt(4)\n\t" \
  "v_mfma_f32_16x16x128_f8f6f4 v[128:131], v[96:103], a[32:39], v[128:131]\n\t" \
  "ds_read_b64_tr_b8 v[112:113], v74 offset:16384\n\t" \
  "ds_read_b64_tr_b8 v[114:115], v74 offset:17408\n\t" \
  "v_mfma_f32_16x16x128_f8f6f4 v[132:135], v[96:103], a[40:47], v[132:135]\n\t" \
  "ds_read_b64_tr_b8 v[116:117], v74 offset:24576\n\t" \
  "ds_read_b64_tr_b8 v[118:119], v74 offset:25600\n\t" \
  "buffer_load_dwordx4 a[152:155], v84, s[44:47], s90 offen offset:2048 sc0 sc1\n\t" \
  "v_mfma_f32_16x16x128_f8f6f4 v[136:139], v[96:103], a[48:55], v[136:139]\n\t" \
  "buffer_load_dwordx4 a[156:159], v84, s[44:47], s90 offen offset:3072 sc0 sc1\n\t" \
  "v_mfma_f32_16x16x128_f8f6f4 v[140:143], v[96:103], a[56:63], v[140:143]\n\t" \
  "s_waitcnt lgkmcnt(4)\n\t" \
  "v_mfma_f32_16x16x128_f8f6f4 v[144:147], v[104:111], a[32:39], v[144:147]\n\t" \
  "ds_read_b64_tr_b8 v[120:121], v75 offset:16384\n\t" \
  "ds_read_b64_tr_b8 v[122:123], v75 offset:17408\n\t" \
  "v_mfma_f32_16x16x128_f8f6f4 v[148:151], v[104:111], a[40:47], v[148:151]\n\t" \
  "ds_read_b64_tr_b8 v[124:125], v75 offset:24576\n\t" \
  "ds_read_b64_tr_b8 v[126:127], v75 offset:25600\n\t" \
  "s_add_u32 s89, s62, 1376256\n\t" \
  "s_add_u32 s90, s62, 1380352\n\t" \
  "buffer_load_dwordx4 a[160:163], v84, s[44:47], s89 offen offset:0 sc0 sc1\n\t" \
  "v_mfma_f32_16x16x128_f8f6f4 v[152:155], v[104:111], a[48:55], v[152:155]\n\t" \
  "buffer_load_dwordx4 a[164:167], v84, s[44:47], s89 offen offset:1024 sc0 sc1\n\t" \
  "v_mfma_f32_16x16x128_f8f6f4 v[156:159], v[104:111], a[56:63], v[156:159]\n\t" \
  "s_waitcnt lgkmcnt(4)\n\t" \
  "v_mfma_f32_16x16x128_f8f6f4 v[160:163], v[112:119], a[32:39], v[160:163]\n\t" \
  "ds_read_b64_tr_b8 v[96:97], v76 offset:16384\n\t" \
  "ds_read_b64_tr_b8 v[98:99], v76 offset:17408\n\t" \
  "v_mfma_f32_16x16x128_f8f6f4 v[164:167], v[112:119], a[40:47], v[164:167]\n\t" \
  "ds_read_b64_tr_b8 v[100:101], v76 offset:24576\n\t" \
  "ds_read_b64_tr_b8 v[102:103], v76 offset:25600\n\t" \
  "buffer_load_dwordx4 a[168:171], v84, s[44:47], s89 offen offset:2048 sc0 sc1\n\t" \
  "v_mfma_f32_16x16x128_f8f6f4 v[168:171], v[112:119], a[48:55], v[168:171]\n\t" \
  "buffer_load_dwordx4 a[172:175], v84, s[44:47], s89 offen offset:3072 sc0 sc1\n\t" \
  "v_mfma_f32_16x16x128_f8f6f4 v[172:175], v[112:119], a[56:63], v[172:175]\n\t" \
  "s_waitcnt lgkmcnt(4)\n\t" \
  "v_mfma_f32_16x16x128_f8f6f4 v[176:179], v[120:127], a[32:39], v[176:179]\n\t" \
  "ds_read_b64_tr_b8 v[104:105], v77 offset:16384\n\t" \
  "ds_read_b64_tr_b8 v[106:107], v77 offset:17408\n\t" \
  "v_mfma_f32_16x16x128_f8f6f4 v[180:183], v[120:127], a[40:47], v[180:183]\n\t" \
  "ds_read_b64_tr_b8 v[108:109], v77 offset:24576\n\t" \
  "ds_read_b64_tr_b8 v[110:111], v77 offset:25600\n\t" \
  "buffer_load_dwordx4 a[176:179], v84, s[44:47], s90 offen offset:0 sc0 sc1\n\t" \
  "v_mfma_f32_16x16x128_f8f6f4 v[184:187], v[120:127], a[48:55], v[184:187]\n\t" \
  "buffer_load_dwordx4 a[180:183], v84, s[44:47], s90 offen offset:1024 sc0 sc1\n\t" \
  "v_mfma_f32_16x16x128_f8f6f4 v[188:191], v[120:127], a[56:63], v[188:191]\n\t" \
  "s_waitcnt lgkmcnt(4)\n\t" \
  "v_mfma_f32_16x16x128_f8f6f4 v[192:195], v[96:103], a[32:39], v[192:195]\n\t" \
  "ds_read_b64_tr_b8 v[112:113], v78 offset:16384\n\t" \
  "ds_read_b64_tr_b8 v[114:115], v78 offset:17408\n\t" \
  "v_mfma_f32_16x16x128_f8f6f4 v[196:199], v[96:103], a[40:47], v[196:199]\n\t" \
  "ds_read_b64_tr_b8 v[116:117], v78 offset:24576\n\t" \
  "ds_read_b64_tr_b8 v[118:119], v78 offset:25600\n\t" \
  "s_add_u32 s88, s53, 4194304\n\t" \
  "s_add_u32 m0, s60, 114688\n\t" \
  "s_nop 0\n\t" \
  "buffer_load_dwordx4 v80, s[36:39], s88 offen  lds\n\t" \
  "v_mfma_f32_16x16x128_f8f6f4 v[200:203], v[96:103], a[48:55], v[200:203]\n\t" \
  "s_add_u32 m0, s60, 115712\n\t" \
  "s_nop 0\n\t" \
  "buffer_load_dwordx4 v81, s[36:39], s88 offen  lds\n\t" \
  "v_mfma_f32_16x16x128_f8f6f4 v[204:207], v[96:103], a[56:63], v[204:207]\n\t" \
  "s_add_u32 m0, s60, 116736\n\t" \
  "s_nop 0\n\t" \
  "buffer_load_dwordx4 v82, s[36:39], s88 offen  lds\n\t" \
  "s_waitcnt lgkmcnt(4)\n\t" \
  "v_mfma_f32_16x16x128_f8f6f4 v[208:211], v[104:111], a[32:39], v[208:211]\n\t" \
  "ds_read_b64_tr_b8 v[120:121], v79 offset:16384\n\t" \
  "ds_read_b64_tr_b8 v[122:123], v79 offset:17408\n\t" \
  "v_mfma_f32_16x16x128_f8f6f4 v[212:215], v[104:111], a[40:47], v[212:215]\n\t" \
  "ds_read_b64_tr_b8 v[124:125], v79 offset:24576\n\t" \
  "ds_read_b64_tr_b8 v[126:127], v79 offset:25600\n\t" \
  "s_add_u32 m0, s60, 117760\n\t" \
  "s_nop 0\n\t" \
  "buffer_load_dwordx4 v83, s[36:39], s88 offen  lds\n\t" \
  "v_mfma_f32_16x16x128_f8f6f4 v[216:219], v[104:111], a[48:55], v[216:219]\n\t" \
  "s_add_u32 s88, s54, 4194304\n\t" \
  "s_add_u32 m0, s60, 131072\n\t" \
  "s_nop 0\n\t" \
  "buffer_load_dwordx4 v80, s[36:39], s88 offen  lds\n\t" \
  "v_mfma_f32_16x16x128_f8f6f4 v[220:223], v[104:111], a[56:63], v[220:223]\n\t" \
  "s_add_u32 m0, s60, 132096\n\t" \
  "s_nop 0\n\t" \
  "buffer_load_dwordx4 v81, s[36:39], s88 offen  lds\n\t" \
  "s_waitcnt vmcnt(37)\n\t" \
  "s_barrier\n\t" \
  "s_waitcnt lgkmcnt(4)\n\t" \
  "v_mfma_f32_16x16x128_f8f6f4 v[224:227], v[112:119], a[32:39], v[224:227]\n\t" \
  "ds_read_b64_tr_b8 v[96:97], v72 offset:32768\n\t" \
  "ds_read_b64_tr_b8 v[98:99], v72 offset:33792\n\t" \
  "v_mfma_f32_16x16x128_f8f6f4 v[228:231], v[112:119], a[40:47], v[228:231]\n\t" \
  "ds_read_b64_tr_b8 v[100:101], v72 offset:40960\n\t" \
  "ds_read_b64_tr_b8 v[102:103], v72 offset:41984\n\t" \
  "s_add_u32 m0, s60, 133120\n\t" \
  "s_nop 0\n\t" \
  "buffer_load_dwordx4 v82, s[36:39], s88 offen  lds\n\t" \
  "v_mfma_f32_16x16x128_f8f6f4 v[232:235], v[112:119], a[48:55], v[232:235]\n\t" \
  "s_add_u32 m0, s60, 134144\n\t" \
  "s_nop 0\n\t" \
  "buffer_load_dwordx4 v83, s[36:39], s88 offen  lds\n\t" \
  "v_mfma_f32_16x16x128_f8f6f4 v[236:239], v[112:119], a[56:63], v[236:239]\n\t" \
  "s_add_u32 s88, s55, 4194304\n\t" \
  "s_add_u32 m0, s60, 49152\n\t" \
  "s_nop 0\n\t" \
  "buffer_load_dwordx4 v80, s[36:39], s88 offen  lds\n\t" \
  "s_waitcnt lgkmcnt(4)\n\t" \
  "v_mfma_f32_16x16x128_f8f6f4 v[240:243], v[120:127], a[32:39], v[240:243]\n\t" \
  "ds_read_b64_tr_b8 v[104:105], v73 offset:32768\n\t" \
  "ds_read_b64_tr_b8 v[106:107], v73 offset:33792\n\t" \
  "v_mfma_f32_16x16x128_f8f6f4 v[244:247], v[120:127], a[40:47], v[244:247]\n\t" \
  "ds_read_b64_tr_b8 v[108:109], v73 offset:40960\n\t" \
  "ds_read_b64_tr_b8 v[110:111], v73 offset:41984\n\t" \
  "s_add_u32 m0, s60, 50176\n\t" \
  "s_nop 0\n\t" \
  "buffer_load_dwordx4 v81, s[36:39], s88 offen  lds\n\t" \
  "v_mfma_f32_16x16x128_f8f6f4 v[248:251], v[120:127], a[48:55], v[248:251]\n\t" \
  "s_add_u32 m0, s60, 51200\n\t" \
  "s_nop 0\n\t" \
  "buffer_load_dwordx4 v82, s[36:39], s88 offen  lds\n\t" \
  "v_mfma_f32_16x16x128_f8f6f4 v[252:255], v[120:127], a[56:63], v[252:255]\n\t" \
  "s_add_u32 m0, s60, 52224\n\t" \
  "s_nop 0\n\t" \
  "buffer_load_dwordx4 v83, s[36:39], s88 offen  lds\n\t" \
  "s_waitcnt vmcnt(34)\n\t" \
  "global_load_dword v92, v87, s[70:71] sc0 sc1\n\t" \
  "s_bitcmp1_b32 s65, 22\n\t" \
  "s_cbranch_scc1 OK81_%=\n\t" \
  "s_mov_b32 s67, 22\n\t" \
  "s_call_b64 s[68:69], POLL_%=\n\t" \
  "OK81_%=:\n\t" \
  "s_waitcnt lgkmcnt(4)\n\t" \
  "v_mfma_f32_16x16x128_f8f6f4 v[128:131], v[96:103], a[64:71], v[128:131]\n\t" \
  "ds_read_b64_tr_b8 v[112:113], v74 offset:32768\n\t" \
  "ds_read_b64_tr_b8 v[114:115], v74 offset:33792\n\t" \
  "s_bfe_u32 s96, s65, 262167\n\t" \
  "s_cmp_lg_u32 s96, 15\n\t" \
  "s_cbranch_scc1 XWS82_%=\n\t" \
  "s_add_u32 s96, s62, 1507328\n\t" \
  "buffer_load_dword v8, v13, s[44:47], s96 offen sc0 sc1\n\t" \
  "XWS82_%=:\n\t" \
  "v_mfma_f32_16x16x128_f8f6f4 v[132:135], v[96:103], a[72:79], v[132:135]\n\t" \
  "ds_read_b64_tr_b8 v[116:117], v74 offset:40960\n\t" \
  "ds_read_b64_tr_b8 v[118:119], v74 offset:41984\n\t" \
  "s_add_u32 s88, s56, 4194304\n\t" \
  "s_add_u32 m0, s60, 0\n\t" \
  "s_nop 0\n\t" \
  "buffer_load_dwordx4 v80, s[36:39], s88 offen  lds\n\t" \
  "v_mfma_f32_16x16x128_f8f6f4 v[136:139], v[96:103], a[80:87], v[136:139]\n\t" \
  "s_add_u32 m0, s60, 1024\n\t" \
  "s_nop 0\n\t" \
  "buffer_load_dwordx4 v81, s[36:39], s88 offen  lds\n\t" \
  "v_mfma_f32_16x16x128_f8f6f4 v[140:143], v[96:103], a[88:95], v[140:143]\n\t" \
  "s_waitcnt lgkmcnt(4)\n\t" \
  "v_mfma_f32_16x16x128_f8f6f4 v[144:147], v[104:111], a[64:71], v[144:147]\n\t" \
  "ds_read_b64_tr_b8 v[120:121], v75 offset:32768\n\t" \
  "ds_read_b64_tr_b8 v[122:123], v75 offset:33792\n\t" \
  "v_mfma_f32_16x16x128_f8f6f4 v[148:151], v[104:111], a[72:79], v[148:151]\n\t" \
  "ds_read_b64_tr_b8 v[124:125], v75 offset:40960\n\t" \
  "ds_read_b64_tr_b8 v[126:127], v75 offset:41984\n\t" \
  "s_add_u32 m0, s60, 2048\n\t" \
  "s_nop 0\n\t" \
  "buffer_load_dwordx4 v82, s[36:39], s88 offen  lds\n\t" \
  "v_mfma_f32_16x16x128_f8f6f4 v[152:155], v[104:111], a[80:87], v[152:155]\n\t" \
  "s_add_u32 m0, s60, 3072\n\t" \
  "s_nop 0\n\t" \
  "buffer_load_dwordx4 v83, s[36:39], s88 offen  lds\n\t" \
  "v_mfma_f32_16x16x128_f8f6f4 v[156:159], v[104:111], a[88:95], v[156:159]\n\t" \
  "s_waitcnt lgkmcnt(4)\n\t" \
  "v_mfma_f32_16x16x128_f8f6f4 v[160:163], v[112:119], a[64:71], v[160:163]\n\t" \
  "ds_read_b64_tr_b8 v[96:97], v76 offset:32768\n\t" \
  "ds_read_b64_tr_b8 v[98:99], v76 offset:33792\n\t" \
  "v_mfma_f32_16x16x128_f8f6f4 v[164:167], v[112:119], a[72:79], v[164:167]\n\t" \
  "ds_read_b64_tr_b8 v[100:101], v76 offset:40960\n\t" \
  "ds_read_b64_tr_b8 v[102:103], v76 offset:41984\n\t" \
  "s_add_u32 s88, s57, 4194304\n\t" \
  "s_add_u32 m0, s60, 16384\n\t" \
  "s_nop 0\n\t" \
  "buffer_load_dwordx4 v80, s[36:39], s88 offen  lds\n\t" \
  "v_mfma_f32_16x16x128_f8f6f4 v[168:171], v[112:119], a[80:87], v[168:171]\n\t" \
  "s_add_u32 m0, s60, 17408\n\t" \
  "s_nop 0\n\t" \
  "buffer_load_dwordx4 v81, s[36:39], s88 offen  lds\n\t" \
  "v_mfma_f32_16x16x128_f8f6f4 v[172:175], v[112:119], a[88:95], v[172:175]\n\t" \
  "s_waitcnt lgkmcnt(4)\n\t" \
  "v_mfma_f32_16x16x128_f8f6f4 v[176:179], v[120:127], a[64:71], v[176:179]\n\t" \
  "ds_read_b64_tr_b8 v[104:105], v77 offset:32768\n\t" \
  "ds_read_b64_tr_b8 v[106:107], v77 offset:33792\n\t" \
  "v_mfma_f32_16x16x128_f8f6f4 v[180:183], v[120:127], a[72:79], v[180:183]\n\t" \
  "ds_read_b64_tr_b8 v[108:109], v77 offset:40960\n\t" \
  "ds_read_b64_tr_b8 v[110:111], v77 offset:41984\n\t" \
  "s_add_u32 m0, s60, 18432\n\t" \
  "s_nop 0\n\t" \
  "buffer_load_dwordx4 v82, s[36:39], s88 offen  lds\n\t" \
  "v_mfma_f32_16x16x128_f8f6f4 v[184:187], v[120:127], a[80:87], v[184:187]\n\t" \
  "s_add_u32 m0, s60, 19456\n\t" \
  "s_nop 0\n\t" \
  "buffer_load_dwordx4 v83, s[36:39], s88 offen  lds\n\t" \
  "v_mfma_f32_16x16x128_f8f6f4 v[188:191], v[120:127], a[88:95], v[188:191]\n\t" \
  "s_waitcnt lgkmcnt(4)\n\t" \
  "v_mfma_f32_16x16x128_f8f6f4 v[192:195], v[96:103], a[64:71], v[192:195]\n\t" \
  "ds_read_b64_tr_b8 v[112:113], v78 offset:32768\n\t" \
  "ds_read_b64_tr_b8 v[114:115], v78 offset:33792\n\t" \
  "v_mfma_f32_16x16x128_f8f6f4 v[196:199], v[96:103], a[72:79], v[196:199]\n\t" \
  "ds_read_b64_tr_b8 v[116:117], v78 offset:40960\n\t" \
  "ds_read_b64_tr_b8 v[118:119], v78 offset:41984\n\t" \
  "buffer_load_dwordx4 a[184:187], v84, s[44:47], s90 offen offset:2048 sc0 sc1\n\t" \
  "v_mfma_f32_16x16x128_f8f6f4 v[200:203], v[96:103], a[80:87], v[200:203]\n\t" \
  "buffer_load_dwordx4 a[188:191], v84, s[44:47], s90 offen offset:3072 sc0 sc1\n\t" \
  "v_mfma_f32_16x16x128_f8f6f4 v[204:207], v[96:103], a[88:95], v[204:207]\n\t" \
  "s_waitcnt lgkmcnt(4)\n\t" \
  "v_mfma_f32_16x16x128_f8f6f4 v[208:211], v[104:111], a[64:71], v[208:211]\n\t" \
  "ds_read_b64_tr_b8 v[120:121], v79 offset:32768\n\t" \
  "ds_read_b64_tr_b8 v[122:123], v79 offset:33792\n\t" \
  "v_mfma_f32_16x16x128_f8f6f4 v[212:215], v[104:111], a[72:79], v[212:215]\n\t" \
  "ds_read_b64_tr_b8 v[124:125], v79 offset:40960\n\t" \
  "ds_read_b64_tr_b8 v[126:127], v79 offset:41984\n\t" \
  "s_add_u32 s89, s62, 1441792\n\t" \
  "s_add_u32 s90, s62, 1445888\n\t" \
  "buffer_load_dwordx4 a[192:195], v84, s[44:47], s89 offen offset:0 sc0 sc1\n\t" \
  "v_mfma_f32_16x16x128_f8f6f4 v[216:219], v[104:111], a[80:87], v[216:219]\n\t" \
  "v_mfma_f32_16x16x128_f8f6f4 v[220:223], v[104:111], a[88:95], v[220:223]\n\t" \
  "s_waitcnt vmcnt(20)\n\t" \
  "s_barrier\n\t" \
  "s_waitcnt lgkmcnt(4)\n\t" \
  "v_mfma_f32_16x16x128_f8f6f4 v[224:227], v[112:119], a[64:71], v[224:227]\n\t" \
  "ds_read_b64_tr_b8 v[96:97], v72 offset:49152\n\t" \
  "ds_read_b64_tr_b8 v[98:99], v72 offset:50176\n\t" \
  "v_mfma_f32_16x16x128_f8f6f4 v[228:231], v[112:119], a[72:79], v[228:231]\n\t" \
  "ds_read_b64_tr_b8 v[100:101], v72 offset:57344\n\t" \
  "ds_read_b64_tr_b8 v[102:103], v72 offset:58368\n\t" \
  "v_mfma_f32_16x16x128_f8f6f4 v[232:235], v[112:119], a[80:87], v[232:235]\n\t" \
  "v_mfma_f32_16x16x128_f8f6f4 v[236:239], v[112:119], a[88:95], v[236:239]\n\t" \
  "s_waitcnt lgkmcnt(4)\n\t" \
  "v_mfma_f32_16x16x128_f8f6f4 v[240:243], v[120:127], a[64:71], v[240:243]\n\t" \
  "ds_read_b64_tr_b8 v[104:105], v73 offset:49152\n\t" \
  "ds_read_b64_tr_b8 v[106:107], v73 offset:50176\n\t" \
  "v_mfma_f32_16x16x128_f8f6f4 v[244:247], v[120:127], a[72:79], v[244:247]\n\t" \
  "ds_read_b64_tr_b8 v[108:109], v73 offset:57344\n\t" \
  "ds_read_b64_tr_b8 v[110:111], v73 offset:58368\n\t" \
  "v_mfma_f32_16x16x128_f8f6f4 v[248:251], v[120:127], a[80:87], v[248:251]\n\t" \
  "v_mfma_f32_16x16x128_f8f6f4 v[252:255], v[120:127], a[88:95], v[252:255]\n\t" \
  "s_waitcnt vmcnt(11)\n\t" \
  "v_cmp_le_u32 vcc, s64, v92\n\t" \
  "s_nop 1\n\t" \
  "s_or_b32 s65, s65, vcc_lo\n\t" \
  "s_bitcmp1_b32 s65, 23\n\t" \
  "s_cbranch_scc1 OK83_%=\n\t" \
  "s_mov_b32 s67, 23\n\t" \
  "s_call_b64 s[68:69], POLL_%=\n\t" \
  "OK83_%=:\n\t" \
  "s_waitcnt lgkmcnt(4)\n\t" \
  "v_mfma_f32_16x16x128_f8f6f4 v[128:131], v[96:103], a[96:103], v[128:131]\n\t" \
  "ds_read_b64_tr_b8 v[112:113], v74 offset:49152\n\t" \
  "ds_read_b64_tr_b8 v[114:115], v74 offset:50176\n\t" \
  "v_mfma_f32_16x16x128_f8f6f4 v[132:135], v[96:103], a[104:111], v[132:135]\n\t" \
  "ds_read_b64_tr_b8 v[116:117], v74 offset:57344\n\t" \
  "ds_read_b64_tr_b8 v[118:119], v74 offset:58368\n\t" \
  "v_mfma_f32_16x16x128_f8f6f4 v[136:139], v[96:103], a[112:119], v[136:139]\n\t" \
  "v_mfma_f32_16x16x128_f8f6f4 v[140:143], v[96:103], a[120:127], v[140:143]\n\t" \
  "s_waitcnt lgkmcnt(4)\n\t" \
  "v_mfma_f32_16x16x128_f8f6f4 v[144:147], v[104:111], a[96:103], v[144:147]\n\t" \
  "ds_read_b64_tr_b8 v[120:121], v75 offset:49152\n\t" \
  "ds_read_b64_tr_b8 v[122:123], v75 offset:50176\n\t" \
  "v_mfma_f32_16x16x128_f8f6f4 v[148:151], v[104:111], a[104:111], v[148:151]\n\t" \
  "ds_read_b64_tr_b8 v[124:125], v75 offset:57344\n\t" \
  "ds_read_b64_tr_b8 v[126:127], v75 offset:58368\n\t" \
  "v_mfma_f32_16x16x128_f8f6f4 v[152:155], v[104:111], a[112:119], v[152:155]\n\t" \
  "v_mfma_f32_16x16x128_f8f6f4 v[156:159], v[104:111], a[120:127], v[156:159]\n\t" \
  "s_waitcnt lgkmcnt(4)\n\t" \
  "v_mfma_f32_16x16x128_f8f6f4 v[160:163], v[112:119], a[96:103], v[160:163]\n\t" \
  "ds_read_b64_tr_b8 v[96:97], v76 offset:49152\n\t" \
  "ds_read_b64_tr_b8 v[98:99], v76 offset:50176\n\t" \
  "v_mfma_f32_16x16x128_f8f6f4 v[164:167], v[112:119], a[104:111], v[164:167]\n\t" \
  "ds_read_b64_tr_b8 v[100:101], v76 offset:57344\n\t" \
  "ds_read_b64_tr_b8 v[102:103], v76 offset:58368\n\t" \
  "v_mfma_f32_16x16x128_f8f6f4 v[168:171], v[112:119], a[112:119], v[168:171]\n\t" \
  "v_mfma_f32_16x16x128_f8f6f4 v[172:175], v[112:119], a[120:127], v[172:175]\n\t" \
  "s_waitcnt lgkmcnt(4)\n\t" \
  "v_mfma_f32_16x16x128_f8f6f4 v[176:179], v[120:127], a[96:103], v[176:179]\n\t" \
  "ds_read_b64_tr_b8 v[104:105], v77 offset:49152\n\t" \
  "ds_read_b64_tr_b8 v[106:107], v77 offset:50176\n\t" \
  "v_mfma_f32_16x16x128_f8f6f4 v[180:183], v[120:127], a[104:111], v[180:183]\n\t" \
  "ds_read_b64_tr_b8 v[108:109], v77 offset:57344\n\t" \
  "ds_read_b64_tr_b8 v[110:111], v77 offset:58368\n\t" \
  "v_mfma_f32_16x16x128_f8f6f4 v[184:187], v[120:127], a[112:119], v[184:187]\n\t" \
  "v_mfma_f32_16x16x128_f8f6f4 v[188:191], v[120:127], a[120:127], v[188:191]\n\t" \
  "s_waitcnt lgkmcnt(4)\n\t" \
  "v_mfma_f32_16x16x128_f8f6f4 v[192:195], v[96:103], a[96:103], v[192:195]\n\t" \
  "ds_read_b64_tr_b8 v[112:113], v78 offset:49152\n\t" \
  "ds_read_b64_tr_b8 v[114:115], v78 offset:50176\n\t" \
  "v_mfma_f32_16x16x128_f8f6f4 v[196:199], v[96:103], a[104:111], v[196:199]\n\t" \
  "ds_read_b64_tr_b8 v[116:117], v78 offset:57344\n\t" \
  "ds_read_b64_tr_b8 v[118:119], v78 offset:58368\n\t" \
  "v_mfma_f32_16x16x128_f8f6f4 v[200:203], v[96:103], a[112:119], v[200:203]\n\t" \
  "v_mfma_f32_16x16x128_f8f6f4 v[204:207], v[96:103], a[120:127], v[204:207]\n\t" \
  "s_waitcnt lgkmcnt(4)\n\t" \
  "v_mfma_f32_16x16x128_f8f6f4 v[208:211], v[104:111], a[96:103], v[208:211]\n\t" \
  "ds_read_b64_tr_b8 v[120:121], v79 offset:49152\n\t" \
  "ds_read_b64_tr_b8 v[122:123], v79 offset:50176\n\t" \
  "v_mfma_f32_16x16x128_f8f6f4 v[212:215], v[104:111], a[104:111], v[212:215]\n\t" \
  "ds_read_b64_tr_b8 v[124:125], v79 offset:57344\n\t" \
  "ds_read_b64_tr_b8 v[126:127], v79 offset:58368\n\t" \
  "v_mfma_f32_16x16x128_f8f6f4 v[216:219], v[104:111], a[112:119], v[216:219]\n\t" \
  "v_mfma_f32_16x16x128_f8f6f4 v[220:223], v[104:111], a[120:127], v[220:223]\n\t" \
  "s_waitcnt vmcnt(16)\n\t" \
  "s_barrier\n\t" \
  "s_waitcnt lgkmcnt(4)\n\t" \
  "v_mfma_f32_16x16x128_f8f6f4 v[224:227], v[112:119], a[96:103], v[224:227]\n\t" \
  "ds_read_b64_tr_b8 v[96:97], v16 offset:0\n\t" \
  "ds_read_b64_tr_b8 v[98:99], v16 offset:1024\n\t" \
  "v_mfma_f32_16x16x128_f8f6f4 v[228:231], v[112:119], a[104:111], v[228:231]\n\t" \
  "ds_read_b64_tr_b8 v[100:101], v16 offset:8192\n\t" \
  "ds_read_b64_tr_b8 v[102:103], v16 offset:9216\n\t" \
  "buffer_load_dwordx4 a[196:199], v84, s[44:47], s89 offen offset:1024 sc0 sc1\n\t" \
  "v_mfma_f32_16x16x128_f8f6f4 v[232:235], v[112:119], a[112:119], v[232:235]\n\t" \
  "buffer_load_dwordx4 a[200:203], v84, s[44:47], s89 offen offset:2048 sc0 sc1\n\t" \
  "v_mfma_f32_16x16x128_f8f6f4 v[236:239], v[112:119], a[120:127], v[236:239]\n\t" \
  "s_waitcnt lgkmcnt(4)\n\t" \
  "v_mfma_f32_16x16x128_f8f6f4 v[240:243], v[120:127], a[96:103], v[240:243]\n\t" \
  "ds_read_b64_tr_b8 v[104:105], v17 offset:0\n\t" \
  "ds_read_b64_tr_b8 v[106:107], v17 offset:1024\n\t" \
  "v_mfma_f32_16x16x128_f8f6f4 v[244:247], v[120:127], a[104:111], v[244:247]\n\t" \
  "ds_read_b64_tr_b8 v[108:109], v17 offset:8192\n\t" \
  "ds_read_b64_tr_b8 v[110:111], v17 offset:9216\n\t" \
  "buffer_load_dwordx4 a[204:207], v84, s[44:47], s89 offen offset:3072 sc0 sc1\n\t" \
  "v_mfma_f32_16x16x128_f8f6f4 v[248:251], v[120:127], a[112:119], v[248:251]\n\t" \
  "buffer_load_dwordx4 a[208:211], v84, s[44:47], s90 offen offset:0 sc0 sc1\n\t" \
  "v_mfma_f32_16x16x128_f8f6f4 v[252:255], v[120:127], a[120:127], v[252:255]\n\t" \
  "s_waitcnt vmcnt(34)\n\t" \
  "s_bitcmp1_b32 s65, 24\n\t" \
  "s_cbranch_scc1 OK84_%=\n\t" \
  "s_mov_b32 s67, 24\n\t" \
  "s_call_b64 s[68:69], POLL_%=\n\t" \
  "OK84_%=:\n\t" \
  "s_waitcnt lgkmcnt(4)\n\t" \
  "v_mfma_f32_16x16x128_f8f6f4 v[128:131], v[96:103], a[128:135], v[128:131]\n\t" \
  "ds_read_b64_tr_b8 v[112:113], v18 offset:0\n\t" \
  "ds_read_b64_tr_b8 v[114:115], v18 offset:1024\n\t" \
  "v_mfma_f32_16x16x128_f8f6f4 v[132:135], v[96:103], a[136:143], v[132:135]\n\t" \
  "ds_read_b64_tr_b8 v[116:117], v18 offset:8192\n\t" \
  "ds_read_b64_tr_b8 v[118:119], v18 offset:9216\n\t" \
  "buffer_load_dwordx4 a[212:215], v84, s[44:47], s90 offen offset:1024 sc0 sc1\n\t" \
  "v_mfma_f32_16x16x128_f8f6f4 v[136:139], v[96:103], a[144:151], v[136:139]\n\t" \
  "buffer_load_dwordx4 a[216:219], v84, s[44:47], s90 offen offset:2048 sc0 sc1\n\t" \
  "v_mfma_f32_16x16x128_f8f6f4 v[140:143], v[96:103], a[152:159], v[140:143]\n\t" \
  "s_waitcnt lgkmcnt(4)\n\t" \
  "v_mfma_f32_16x16x128_f8f6f4 v[144:147], v[104:111], a[128:135], v[144:147]\n\t" \
  "ds_read_b64_tr_b8 v[120:121], v19 offset:0\n\t" \
  "ds_read_b64_tr_b8 v[122:123], v19 offset:1024\n\t" \
  "v_mfma_f32_16x16x128_f8f6f4 v[148:151], v[104:111], a[136:143], v[148:151]\n\t" \
  "ds_read_b64_tr_b8 v[124:125], v19 offset:8192\n\t" \
  "ds_read_b64_tr_b8 v[126:127], v19 offset:9216\n\t" \
  "buffer_load_dwordx4 a[220:223], v84, s[44:47], s90 offen offset:3072 sc0 sc1\n\t" \
  "v_mfma_f32_16x16x128_f8f6f4 v[152:155], v[104:111], a[144:151], v[152:155]\n\t" \
  "s_add_u32 s89, s62, 1507328\n\t" \
  "s_add_u32 s90, s62, 1511424\n\t" \
  "buffer_load_dwordx4 a[224:227], v84, s[44:47], s89 offen offset:0 sc0 sc1\n\t" \
  "v_mfma_f32_16x16x128_f8f6f4 v[156:159], v[104:111], a[152:159], v[156:159]\n\t" \
  "s_waitcnt lgkmcnt(4)\n\t" \
  "v_mfma_f32_16x16x128_f8f6f4 v[160:163], v[112:119], a[128:135], v[160:163]\n\t" \
  "ds_read_b64_tr_b8 v[96:97], v20 offset:0\n\t" \
  "ds_read_b64_tr_b8 v[98:99], v20 offset:1024\n\t" \
  "v_mfma_f32_16x16x128_f8f6f4 v[164:167], v[112:119], a[136:143], v[164:167]\n\t" \
  "ds_read_b64_tr_b8 v[100:101], v20 offset:8192\n\t" \
  "ds_read_b64_tr_b8 v[102:103], v20 offset:9216\n\t" \
  "buffer_load_dwordx4 a[228:231], v84, s[44:47], s89 offen offset:1024 sc0 sc1\n\t" \
  "v_mfma_f32_16x16x128_f8f6f4 v[168:171], v[112:119], a[144:151], v[168:171]\n\t" \
  "buffer_load_dwordx4 a[232:235], v84, s[44:47], s89 offen offset:2048 sc0 sc1\n\t" \
  "v_mfma_f32_16x16x128_f8f6f4 v[172:175], v[112:119], a[152:159], v[172:175]\n\t" \
  "s_waitcnt lgkmcnt(4)\n\t" \
  "v_mfma_f32_16x16x128_f8f6f4 v[176:179], v[120:127], a[128:135], v[176:179]\n\t" \
  "ds_read_b64_tr_b8 v[104:105], v21 offset:0\n\t" \
  "ds_read_b64_tr_b8 v[106:107], v21 offset:1024\n\t" \
  "v_mfma_f32_16x16x128_f8f6f4 v[180:183], v[120:127], a[136:143], v[180:183]\n\t" \
  "ds_read_b64_tr_b8 v[108:109], v21 offset:8192\n\t" \
  "ds_read_b64_tr_b8 v[110:111], v21 offset:9216\n\t" \
  "buffer_load_dwordx4 a[236:239], v84, s[44:47], s89 offen offset:3072 sc0 sc1\n\t" \
  "v_mfma_f32_16x16x128_f8f6f4 v[184:187], v[120:127], a[144:151], v[184:187]\n\t" \
  "buffer_load_dwordx4 a[240:243], v84, s[44:47], s90 offen offset:0 sc0 sc1\n\t" \
  "v_mfma_f32_16x16x128_f8f6f4 v[188:191], v[120:127], a[152:159], v[188:191]\n\t" \
  "s_waitcnt lgkmcnt(4)\n\t" \
  "v_mfma_f32_16x16x128_f8f6f4 v[192:195], v[96:103], a[128:135], v[192:195]\n\t" \
  "ds_read_b64_tr_b8 v[112:113], v22 offset:0\n\t" \
  "ds_read_b64_tr_b8 v[114:115], v22 offset:1024\n\t" \
  "v_mfma_f32_16x16x128_f8f6f4 v[196:199], v[96:103], a[136:143], v[196:199]\n\t" \
  "ds_read_b64_tr_b8 v[116:117], v22 offset:8192\n\t" \
  "ds_read_b64_tr_b8 v[118:119], v22 offset:9216\n\t" \
  "buffer_load_dwordx4 a[244:247], v84, s[44:47], s90 offen offset:1024 sc0 sc1\n\t" \
  "v_mfma_f32_16x16x128_f8f6f4 v[200:203], v[96:103], a[144:151], v[200:203]\n\t" \
  "buffer_load_dwordx4 a[248:251], v84, s[44:47], s90 offen offset:2048 sc0 sc1\n\t" \
  "v_mfma_f32_16x16x128_f8f6f4 v[204:207], v[96:103], a[152:159], v[204:207]\n\t" \
  "s_waitcnt lgkmcnt(4)\n\t" \
  "v_mfma_f32_16x16x128_f8f6f4 v[208:211], v[104:111], a[128:135], v[208:211]\n\t" \
  "ds_read_b64_tr_b8 v[120:121], v23 offset:0\n\t" \
  "ds_read_b64_tr_b8 v[122:123], v23 offset:1024\n\t" \
  "v_mfma_f32_16x16x128_f8f6f4 v[212:215], v[104:111], a[136:143], v[212:215]\n\t" \
  "ds_read_b64_tr_b8 v[124:125], v23 offset:8192\n\t" \
  "ds_read_b64_tr_b8 v[126:127], v23 offset:9216\n\t" \
  "buffer_load_dwordx4 a[252:255], v84, s[44:47], s90 offen offset:3072 sc0 sc1\n\t" \
  "v_mfma_f32_16x16x128_f8f6f4 v[216:219], v[104:111], a[144:151], v[216:219]\n\t" \
  "s_add_u32 s89, s62, 1572864\n\t" \
  "s_add_u32 s90, s62, 1576960\n\t" \
  "buffer_load_dwordx4 a[0:3], v84, s[44:47], s89 offen offset:0 sc0 sc1\n\t" \
  "v_mfma_f32_16x16x128_f8f6f4 v[220:223], v[104:111], a[152:159], v[220:223]\n\t" \
  "s_waitcnt vmcnt(28)\n\t" \
  "s_barrier\n\t" \
  "s_waitcnt lgkmcnt(4)\n\t" \
  "v_mfma_f32_16x16x128_f8f6f4 v[224:227], v[112:119], a[128:135], v[224:227]\n\t" \
  "ds_read_b64_tr_b8 v[96:97], v64 offset:49152\n\t" \
  "ds_read_b64_tr_b8 v[98:99], v64 offset:50176\n\t" \
  "v_mfma_f32_16x16x128_f8f6f4 v[228:231], v[112:119], a[136:143], v[228:231]\n\t" \
  "ds_read_b64_tr_b8 v[100:101], v64 offset:57344\n\t" \
  "ds_read_b64_tr_b8 v[102:103], v64 offset:58368\n\t" \
  "buffer_load_dwordx4 a[4:7], v84, s[44:47], s89 offen offset:1024 sc0 sc1\n\t" \
  "v_mfma_f32_16x16x128_f8f6f4 v[232:235], v[112:119], a[144:151], v[232:235]\n\t" \
  "buffer_load_dwordx4 a[8:11], v84, s[44:47], s89 offen offset:2048 sc0 sc1\n\t" \
  "v_mfma_f32_16x16x128_f8f6f4 v[236:239], v[112:119], a[152:159], v[236:239]\n\t" \
  "s_waitcnt lgkmcnt(4)\n\t" \
  "v_mfma_f32_16x16x128_f8f6f4 v[240:243], v[120:127], a[128:135], v[240:243]\n\t" \
  "ds_read_b64_tr_b8 v[104:105], v65 offset:49152\n\t" \
  "ds_read_b64_tr_b8 v[106:107], v65 offset:50176\n\t" \
  "v_mfma_f32_16x16x128_f8f6f4 v[244:247], v[120:127], a[136:143], v[244:247]\n\t" \
  "ds_read_b64_tr_b8 v[108:109], v65 offset:57344\n\t" \
  "ds_read_b64_tr_b8 v[110:111], v65 offset:58368\n\t" \
  "buffer_load_dwordx4 a[12:15], v84, s[44:47], s89 offen offset:3072 sc0 sc1\n\t" \
  "v_mfma_f32_16x16x128_f8f6f4 v[248:251], v[120:127], a[144:151], v[248:251]\n\t" \
  "buffer_load_dwordx4 a[16:19], v84, s[44:47], s90 offen offset:0 sc0 sc1\n\t" \
  "v_mfma_f32_16x16x128_f8f6f4 v[252:255], v[120:127], a[152:159], v[252:255]\n\t" \
  "s_waitcnt vmcnt(21)\n\t" \
  "s_bitcmp1_b32 s65, 25\n\t" \
  "s_cbranch_scc1 OK85_%=\n\t" \
  "s_mov_b32 s67, 25\n\t" \
  "s_call_b64 s[68:69], POLL_%=\n\t" \
  "OK85_%=:\n\t" \
  "s_waitcnt lgkmcnt(4)\n\t" \
  "v_mfma_f32_16x16x128_f8f6f4 v[128:131], v[96:103], a[160:167], v[128:131]\n\t" \
  "ds_read_b64_tr_b8 v[112:113], v66 offset:49152\n\t" \
  "ds_read_b64_tr_b8 v[114:115], v66 offset:50176\n\t" \
  "v_mfma_f32_16x16x128_f8f6f4 v[132:135], v[96:103], a[168:175], v[132:135]\n\t" \
  "ds_read_b64_tr_b8 v[116:117], v66 offset:57344\n\t" \
  "ds_read_b64_tr_b8 v[118:119], v66 offset:58368\n\t" \
  "buffer_load_dwordx4 a[20:23], v84, s[44:47], s90 offen offset:1024 sc0 sc1\n\t" \
  "v_mfma_f32_16x16x128_f8f6f4 v[136:139], v[96:103], a[176:183], v[136:139]\n\t" \
  "buffer_load_dwordx4 a[24:27], v84, s[44:47], s90 offen offset:2048 sc0 sc1\n\t" \
  "v_mfma_f32_16x16x128_f8f6f4 v[140:143], v[96:103], a[184:191], v[140:143]\n\t" \
  "s_waitcnt lgkmcnt(4)\n\t" \
  "v_mfma_f32_16x16x128_f8f6f4 v[144:147], v[104:111], a[160:167], v[144:147]\n\t" \
  "ds_read_b64_tr_b8 v[120:121], v67 offset:49152\n\t" \
  "ds_read_b64_tr_b8 v[122:123], v67 offset:50176\n\t" \
  "v_mfma_f32_16x16x128_f8f6f4 v[148:151], v[104:111], a[168:175], v[148:151]\n\t" \
  "ds_read_b64_tr_b8 v[124:125], v67 offset:57344\n\t" \
  "ds_read_b64_tr_b8 v[126:127], v67 offset:58368\n\t" \
  "buffer_load_dwordx4 a[28:31], v84, s[44:47], s90 offen offset:3072 sc0 sc1\n\t" \
  "v_mfma_f32_16x16x128_f8f6f4 v[152:155], v[104:111], a[176:183], v[152:155]\n\t" \
  "s_add_u32 s89, s62, 1638400\n\t" \
  "s_add_u32 s90, s62, 1642496\n\t" \
  "buffer_load_dwordx4 a[32:35], v84, s[44:47], s89 offen offset:0 sc0 sc1\n\t" \
  "v_mfma_f32_16x16x128_f8f6f4 v[156:159], v[104:111], a[184:191], v[156:159]\n\t" \
  "s_waitcnt lgkmcnt(4)\n\t" \
  "v_mfma_f32_16x16x128_f8f6f4 v[160:163], v[112:119], a[160:167], v[160:163]\n\t" \
  "ds_read_b64_tr_b8 v[96:97], v68 offset:49152\n\t" \
  "ds_read_b64_tr_b8 v[98:99], v68 offset:50176\n\t" \
  "v_mfma_f32_16x16x128_f8f6f4 v[164:167], v[112:119], a[168:175], v[164:167]\n\t" \
  "ds_read_b64_tr_b8 v[100:101], v68 offset:57344\n\t" \
  "ds_read_b64_tr_b8 v[102:103], v68 offset:58368\n\t" \
  "buffer_load_dwordx4 a[36:39], v84, s[44:47], s89 offen offset:1024 sc0 sc1\n\t" \
  "v_mfma_f32_16x16x128_f8f6f4 v[168:171], v[112:119], a[176:183], v[168:171]\n\t" \
  "buffer_load_dwordx4 a[40:43], v84, s[44:47], s89 offen offset:2048 sc0 sc1\n\t" \
  "v_mfma_f32_16x16x128_f8f6f4 v[172:175], v[112:119], a[184:191], v[172:175]\n\t" \
  "s_waitcnt lgkmcnt(4)\n\t" \
  "v_mfma_f32_16x16x128_f8f6f4 v[176:179], v[120:127], a[160:167], v[176:179]\n\t" \
  "ds_read_b64_tr_b8 v[104:105], v69 offset:49152\n\t" \
  "ds_read_b64_tr_b8 v[106:107], v69 offset:50176\n\t" \
  "v_mfma_f32_16x16x128_f8f6f4 v[180:183], v[120:127], a[168:175], v[180:183]\n\t" \
  "ds_read_b64_tr_b8 v[108:109], v69 offset:57344\n\t" \
  "ds_read_b64_tr_b8 v[110:111], v69 offset:58368\n\t" \
  "buffer_load_dwordx4 a[44:47], v84, s[44:47], s89 offen offset:3072 sc0 sc1\n\t" \
  "v_mfma_f32_16x16x128_f8f6f4 v[184:187], v[120:127], a[176:183], v[184:187]\n\t" \
  "buffer_load_dwordx4 a[48:51], v84, s[44:47], s90 offen offset:0 sc0 sc1\n\t" \
  "v_mfma_f32_16x16x128_f8f6f4 v[188:191], v[120:127], a[184:191], v[188:191]\n\t" \
  "s_waitcnt lgkmcnt(4)\n\t" \
  "v_mfma_f32_16x16x128_f8f6f4 v[192:195], v[96:103], a[160:167], v[192:195]\n\t" \
  "ds_read_b64_tr_b8 v[112:113], v70 offset:49152\n\t" \
  "ds_read_b64_tr_b8 v[114:115], v70 offset:50176\n\t" \
  "v_mfma_f32_16x16x128_f8f6f4 v[196:199], v[96:103], a[168:175], v[196:199]\n\t" \
  "ds_read_b64_tr_b8 v[116:117], v70 offset:57344\n\t" \
  "ds_read_b64_tr_b8 v[118:119], v70 offset:58368\n\t" \
  "s_add_u32 s88, s58, 4194304\n\t" \
  "s_add_u32 m0, s60, 32768\n\t" \
  "s_nop 0\n\t" \
  "buffer_load_dwordx4 v80, s[36:39], s88 offen  lds\n\t" \
  "v_mfma_f32_16x16x128_f8f6f4 v[200:203], v[96:103], a[176:183], v[200:203]\n\t" \
  "s_add_u32 m0, s60, 33792\n\t" \
  "s_nop 0\n\t" \
  "buffer_load_dwordx4 v81, s[36:39], s88 offen  lds\n\t" \
  "v_mfma_f32_16x16x128_f8f6f4 v[204:207], v[96:103], a[184:191], v[204:207]\n\t" \
  "s_add_u32 m0, s60, 34816\n\t" \
  "s_nop 0\n\t" \
  "buffer_load_dwordx4 v82, s[36:39], s88 offen  lds\n\t" \
  "s_waitcnt lgkmcnt(4)\n\t" \
  "v_mfma_f32_16x16x128_f8f6f4 v[208:211], v[104:111], a[160:167], v[208:211]\n\t" \
  "ds_read_b64_tr_b8 v[120:121], v71 offset:49152\n\t" \
  "ds_read_b64_tr_b8 v[122:123], v71 offset:50176\n\t" \
  "v_mfma_f32_16x16x128_f8f6f4 v[212:215], v[104:111], a[168:175], v[212:215]\n\t" \
  "ds_read_b64_tr_b8 v[124:125], v71 offset:57344\n\t" \
  "ds_read_b64_tr_b8 v[126:127], v71 offset:58368\n\t" \
  "s_add_u32 m0, s60, 35840\n\t" \
  "s_nop 0\n\t" \
  "buffer_load_dwordx4 v83, s[36:39], s88 offen  lds\n\t" \
  "v_mfma_f32_16x16x128_f8f6f4 v[216:219], v[104:111], a[176:183], v[216:219]\n\t" \
  "s_add_u32 s88, s52, 6291456\n\t" \
  "s_add_u32 m0, s60, 65536\n\t" \
  "s_nop 0\n\t" \
  "buffer_load_dwordx4 v80, s[36:39], s88 offen  lds\n\t" \
  "v_mfma_f32_16x16x128_f8f6f4 v[220:223], v[104:111], a[184:191], v[220:223]\n\t" \
  "s_add_u32 m0, s60, 66560\n\t" \
  "s_nop 0\n\t" \
  "buffer_load_dwordx4 v81, s[36:39], s88 offen  lds\n\t" \
  "s_waitcnt vmcnt(41)\n\t" \
  "s_barrier\n\t" \
  "s_waitcnt lgkmcnt(4)\n\t" \
  "v_mfma_f32_16x16x128_f8f6f4 v[224:227], v[112:119], a[160:167], v[224:227]\n\t" \
  "ds_read_b64_tr_b8 v[96:97], v64 offset:0\n\t" \
  "ds_read_b64_tr_b8 v[98:99], v64 offset:1024\n\t" \
  "v_mfma_f32_16x16x128_f8f6f4 v[228:231], v[112:119], a[168:175], v[228:231]\n\t" \
  "ds_read_b64_tr_b8 v[100:101], v64 offset:8192\n\t" \
  "ds_read_b64_tr_b8 v[102:103], v64 offset:9216\n\t" \
  "s_add_u32 m0, s60, 67584\n\t" \
  "s_nop 0\n\t" \
  "buffer_load_dwordx4 v82, s[36:39], s88 offen  lds\n\t" \
  "v_mfma_f32_16x16x128_f8f6f4 v[232:235], v[112:119], a[176:183], v[232:235]\n\t" \
  "s_add_u32 m0, s60, 68608\n\t" \
  "s_nop 0\n\t" \
  "buffer_load_dwordx4 v83, s[36:39], s88 offen  lds\n\t" \
  "v_mfma_f32_16x16x128_f8f6f4 v[236:239], v[112:119], a[184:191], v[236:239]\n\t" \
  "s_add_u32 s88, s53, 6291456\n\t" \
  "s_add_u32 m0, s60, 81920\n\t" \
  "s_nop 0\n\t" \
  "buffer_load_dwordx4 v80, s[36:39], s88 offen  lds\n\t" \
  "s_waitcnt lgkmcnt(4)\n\t" \
  "v_mfma_f32_16x16x128_f8f6f4 v[240:243], v[120:127], a[160:167], v[240:243]\n\t" \
  "ds_read_b64_tr_b8 v[104:105], v65 offset:0\n\t" \
  "ds_read_b64_tr_b8 v[106:107], v65 offset:1024\n\t" \
  "v_mfma_f32_16x16x128_f8f6f4 v[244:247], v[120:127], a[168:175], v[244:247]\n\t" \
  "ds_read_b64_tr_b8 v[108:109], v65 offset:8192\n\t" \
  "ds_read_b64_tr_b8 v[110:111], v65 offset:9216\n\t" \
  "s_add_u32 m0, s60, 82944\n\t" \
  "s_nop 0\n\t" \
  "buffer_load_dwordx4 v81, s[36:39], s88 offen  lds\n\t" \
  "v_mfma_f32_16x16x128_f8f6f4 v[248:251], v[120:127], a[176:183], v[248:251]\n\t" \
  "s_add_u32 m0, s60, 83968\n\t" \
  "s_nop 0\n\t" \
  "buffer_load_dwordx4 v82, s[36:39], s88 offen  lds\n\t" \
  "v_mfma_f32_16x16x128_f8f6f4 v[252:255], v[120:127], a[184:191], v[252:255]\n\t" \
  "s_add_u32 m0, s60, 84992\n\t" \
  "s_nop 0\n\t" \
  "buffer_load_dwordx4 v83, s[36:39], s88 offen  lds\n\t" \
  "s_waitcnt vmcnt(33)\n\t" \
  "s_bitcmp1_b32 s65, 26\n\t" \
  "s_cbranch_scc1 OK86_%=\n\t" \
  "s_mov_b32 s67, 26\n\t" \
  "s_call_b64 s[68:69], POLL_%=\n\t" \
  "OK86_%=:\n\t" \
  "s_waitcnt lgkmcnt(4)\n\t" \
  "v_mfma_f32_16x16x128_f8f6f4 v[128:131], v[96:103], a[192:199], v[128:131]\n\t" \
  "ds_read_b64_tr_b8 v[112:113], v66 offset:0\n\t" \
  "ds_read_b64_tr_b8 v[114:115], v66 offset:1024\n\t" \
  "s_bfe_u32 s96, s65, 262171\n\t" \
  "s_cmp_lg_u32 s96, 15\n\t" \
  "s_cbranch_scc1 XWS87_%=\n\t" \
  "s_add_u32 s96, s62, 1769472\n\t" \
  "buffer_load_dword v8, v13, s[44:47], s96 offen sc0 sc1\n\t" \
  "XWS87_%=:\n\t" \
  "v_mfma_f32_16x16x128_f8f6f4 v[132:135], v[96:103], a[200:207], v[132:135]\n\t" \
  "ds_read_b64_tr_b8 v[116:117], v66 offset:8192\n\t" \
  "ds_read_b64_tr_b8 v[118:119], v66 offset:9216\n\t" \
  "s_add_u32 s88, s55, 6291456\n\t" \
  "s_add_u32 m0, s60, 98304\n\t" \
  "s_nop 0\n\t" \
  "buffer_load_dwordx4 v80, s[36:39], s88 offen  lds\n\t" \
  "v_mfma_f32_16x16x128_f8f6f4 v[136:139], v[96:103], a[208:215], v[136:139]\n\t" \
  "s_add_u32 m0, s60, 99328\n\t" \
  "s_nop 0\n\t" \
  "buffer_load_dwordx4 v81, s[36:39], s88 offen  lds\n\t" \
  "v_mfma_f32_16x16x128_f8f6f4 v[140:143], v[96:103], a[216:223], v[140:143]\n\t" \
  "s_waitcnt lgkmcnt(4)\n\t" \
  "v_mfma_f32_16x16x128_f8f6f4 v[144:147], v[104:111], a[192:199], v[144:147]\n\t" \
  "ds_read_b64_tr_b8 v[120:121], v67 offset:0\n\t" \
  "ds_read_b64_tr_b8 v[122:123], v67 offset:1024\n\t" \
  "v_mfma_f32_16x16x128_f8f6f4 v[148:151], v[104:111], a[200:207], v[148:151]\n\t" \
  "ds_read_b64_tr_b8 v[124:125], v67 offset:8192\n\t" \
  "ds_read_b64_tr_b8 v[126:127], v67 offset:9216\n\t" \
  "s_add_u32 m0, s60, 100352\n\t" \
  "s_nop 0\n\t" \
  "buffer_load_dwordx4 v82, s[36:39], s88 offen  lds\n\t" \
  "v_mfma_f32_16x16x128_f8f6f4 v[152:155], v[104:111], a[208:215], v[152:155]\n\t" \
  "s_add_u32 m0, s60, 101376\n\t" \
  "s_nop 0\n\t" \
  "buffer_load_dwordx4 v83, s[36:39], s88 offen  lds\n\t" \
  "v_mfma_f32_16x16x128_f8f6f4 v[156:159], v[104:111], a[216:223], v[156:159]\n\t" \
  "s_waitcnt lgkmcnt(4)\n\t" \
  "v_mfma_f32_16x16x128_f8f6f4 v[160:163], v[112:119], a[192:199], v[160:163]\n\t" \
  "ds_read_b64_tr_b8 v[96:97], v68 offset:0\n\t" \
  "ds_read_b64_tr_b8 v[98:99], v68 offset:1024\n\t" \
  "v_mfma_f32_16x16x128_f8f6f4 v[164:167], v[112:119], a[200:207], v[164:167]\n\t" \
  "ds_read_b64_tr_b8 v[100:101], v68 offset:8192\n\t" \
  "ds_read_b64_tr_b8 v[102:103], v68 offset:9216\n\t" \
  "s_add_u32 s88, s56, 6291456\n\t" \
  "s_add_u32 m0, s60, 114688\n\t" \
  "s_nop 0\n\t" \
  "buffer_load_dwordx4 v80, s[36:39], s88 offen  lds\n\t" \
  "v_mfma_f32_16x16x128_f8f6f4 v[168:171], v[112:119], a[208:215], v[168:171]\n\t" \
  "s_add_u32 m0, s60, 115712\n\t" \
  "s_nop 0\n\t" \
  "buffer_load_dwordx4 v81, s[36:39], s88 offen  lds\n\t" \
  "v_mfma_f32_16x16x128_f8f6f4 v[172:175], v[112:119], a[216:223], v[172:175]\n\t" \
  "s_waitcnt lgkmcnt(4)\n\t" \
  "v_mfma_f32_16x16x128_f8f6f4 v[176:179], v[120:127], a[192:199], v[176:179]\n\t" \
  "ds_read_b64_tr_b8 v[104:105], v69 offset:0\n\t" \
  "ds_read_b64_tr_b8 v[106:107], v69 offset:1024\n\t" \
  "v_mfma_f32_16x16x128_f8f6f4 v[180:183], v[120:127], a[200:207], v[180:183]\n\t" \
  "ds_read_b64_tr_b8 v[108:109], v69 offset:8192\n\t" \
  "ds_read_b64_tr_b8 v[110:111], v69 offset:9216\n\t" \
  "s_add_u32 m0, s60, 116736\n\t" \
  "s_nop 0\n\t" \
  "buffer_load_dwordx4 v82, s[36:39], s88 offen  lds\n\t" \
  "v_mfma_f32_16x16x128_f8f6f4 v[184:187], v[120:127], a[208:215], v[184:187]\n\t" \
  "s_add_u32 m0, s60, 117760\n\t" \
  "s_nop 0\n\t" \
  "buffer_load_dwordx4 v83, s[36:39], s88 offen  lds\n\t" \
  "v_mfma_f32_16x16x128_f8f6f4 v[188:191], v[120:127], a[216:223], v[188:191]\n\t" \
  "s_waitcnt lgkmcnt(4)\n\t" \
  "v_mfma_f32_16x16x128_f8f6f4 v[192:195], v[96:103], a[192:199], v[192:195]\n\t" \
  "ds_read_b64_tr_b8 v[112:113], v70 offset:0\n\t" \
  "ds_read_b64_tr_b8 v[114:115], v70 offset:1024\n\t" \
  "v_mfma_f32_16x16x128_f8f6f4 v[196:199], v[96:103], a[200:207], v[196:199]\n\t" \
  "ds_read_b64_tr_b8 v[116:117], v70 offset:8192\n\t" \
  "ds_read_b64_tr_b8 v[118:119], v70 offset:9216\n\t" \
  "buffer_load_dwordx4 a[52:55], v84, s[44:47], s90 offen offset:1024 sc0 sc1\n\t" \
  "v_mfma_f32_16x16x128_f8f6f4 v[200:203], v[96:103], a[208:215], v[200:203]\n\t" \
  "buffer_load_dwordx4 a[56:59], v84, s[44:47], s90 offen offset:2048 sc0 sc1\n\t" \
  "v_mfma_f32_16x16x128_f8f6f4 v[204:207], v[96:103], a[216:223], v[204:207]\n\t" \
  "s_waitcnt lgkmcnt(4)\n\t" \
  "v_mfma_f32_16x16x128_f8f6f4 v[208:211], v[104:111], a[192:199], v[208:211]\n\t" \
  "ds_read_b64_tr_b8 v[120:121], v71 offset:0\n\t" \
  "ds_read_b64_tr_b8 v[122:123], v71 offset:1024\n\t" \
  "v_mfma_f32_16x16x128_f8f6f4 v[212:215], v[104:111], a[200:207], v[212:215]\n\t" \
  "ds_read_b64_tr_b8 v[124:125], v71 offset:8192\n\t" \
  "ds_read_b64_tr_b8 v[126:127], v71 offset:9216\n\t" \
  "buffer_load_dwordx4 a[60:63], v84, s[44:47], s90 offen offset:3072 sc0 sc1\n\t" \
  "v_mfma_f32_16x16x128_f8f6f4 v[216:219], v[104:111], a[208:215], v[216:219]\n\t" \
  "v_mfma_f32_16x16x128_f8f6f4 v[220:223], v[104:111], a[216:223], v[220:223]\n\t" \
  "s_waitcnt vmcnt(54)\n\t" \
  "s_barrier\n\t" \
  "s_waitcnt lgkmcnt(4)\n\t" \
  "v_mfma_f32_16x16x128_f8f6f4 v[224:227], v[112:119], a[192:199], v[224:227]\n\t" \
  "ds_read_b64_tr_b8 v[96:97], v64 offset:16384\n\t" \
  "ds_read_b64_tr_b8 v[98:99], v64 offset:17408\n\t" \
  "v_mfma_f32_16x16x128_f8f6f4 v[228:231], v[112:119], a[200:207], v[228:231]\n\t" \
  "ds_read_b64_tr_b8 v[100:101], v64 offset:24576\n\t" \
  "ds_read_b64_tr_b8 v[102:103], v64 offset:25600\n\t" \
  "v_mfma_f32_16x16x128_f8f6f4 v[232:235], v[112:119], a[208:215], v[232:235]\n\t" \
  "v_mfma_f32_16x16x128_f8f6f4 v[236:239], v[112:119], a[216:223], v[236:239]\n\t" \
  "s_waitcnt lgkmcnt(4)\n\t" \
  "v_mfma_f32_16x16x128_f8f6f4 v[240:243], v[120:127], a[192:199], v[240:243]\n\t" \
  "ds_read_b64_tr_b8 v[104:105], v65 offset:16384\n\t" \
  "ds_read_b64_tr_b8 v[106:107], v65 offset:17408\n\t" \
  "v_mfma_f32_16x16x128_f8f6f4 v[244:247], v[120:127], a[200:207], v[244:247]\n\t" \
  "ds_read_b64_tr_b8 v[108:109], v65 offset:24576\n\t" \
  "ds_read_b64_tr_b8 v[110:111], v65 offset:25600\n\t" \
  "v_mfma_f32_16x16x128_f8f6f4 v[248:251], v[120:127], a[208:215], v[248:251]\n\t" \
  "v_mfma_f32_16x16x128_f8f6f4 v[252:255], v[120:127], a[216:223], v[252:255]\n\t" \
  "s_waitcnt vmcnt(36)\n\t" \
  "s_bitcmp1_b32 s65, 27\n\t" \
  "s_cbranch_scc1 OK88_%=\n\t" \
  "s_mov_b32 s67, 27\n\t" \
  "s_call_b64 s[68:69], POLL_%=\n\t" \
  "OK88_%=:\n\t" \
  "s_waitcnt lgkmcnt(4)\n\t" \
  "v_mfma_f32_16x16x128_f8f6f4 v[128:131], v[96:103], a[224:231], v[128:131]\n\t" \
  "ds_read_b64_tr_b8 v[112:113], v66 offset:16384\n\t" \
  "ds_read_b64_tr_b8 v[114:115], v66 offset:17408\n\t" \
  "v_mfma_f32_16x16x128_f8f6f4 v[132:135], v[96:103], a[232:239], v[132:135]\n\t" \
  "ds_read_b64_tr_b8 v[116:117], v66 offset:24576\n\t" \
  "ds_read_b64_tr_b8 v[118:119], v66 offset:25600\n\t" \
  "v_mfma_f32_16x16x128_f8f6f4 v[136:139], v[96:103], a[240:247], v[136:139]\n\t" \
  "v_mfma_f32_16x16x128_f8f6f4 v[140:143], v[96:103], a[248:255], v[140:143]\n\t" \
  "s_waitcnt lgkmcnt(4)\n\t" \
  "v_mfma_f32_16x16x128_f8f6f4 v[144:147], v[104:111], a[224:231], v[144:147]\n\t" \
  "ds_read_b64_tr_b8 v[120:121], v67 offset:16384\n\t" \
  "ds_read_b64_tr_b8 v[122:123], v67 offset:17408\n\t" \
  "v_mfma_f32_16x16x128_f8f6f4 v[148:151], v[104:111], a[232:239], v[148:151]\n\t" \
  "ds_read_b64_tr_b8 v[124:125], v67 offset:24576\n\t" \
  "ds_read_b64_tr_b8 v[126:127], v67 offset:25600\n\t" \
  "v_mfma_f32_16x16x128_f8f6f4 v[152:155], v[104:111], a[240:247], v[152:155]\n\t" \
  "v_mfma_f32_16x16x128_f8f6f4 v[156:159], v[104:111], a[248:255], v[156:159]\n\t" \
  "s_waitcnt lgkmcnt(4)\n\t" \
  "v_mfma_f32_16x16x128_f8f6f4 v[160:163], v[112:119], a[224:231], v[160:163]\n\t" \
  "ds_read_b64_tr_b8 v[96:97], v68 offset:16384\n\t" \
  "ds_read_b64_tr_b8 v[98:99], v68 offset:17408\n\t" \
  "v_mfma_f32_16x16x128_f8f6f4 v[164:167], v[112:119], a[232:239], v[164:167]\n\t" \
  "ds_read_b64_tr_b8 v[100:101], v68 offset:24576\n\t" \
  "ds_read_b64_tr_b8 v[102:103], v68 offset:25600\n\t" \
  "v_mfma_f32_16x16x128_f8f6f4 v[168:171], v[112:119], a[240:247], v[168:171]\n\t" \
  "v_mfma_f32_16x16x128_f8f6f4 v[172:175], v[112:119], a[248:255], v[172:175]\n\t" \
  "s_waitcnt lgkmcnt(4)\n\t" \
  "v_mfma_f32_16x16x128_f8f6f4 v[176:179], v[120:127], a[224:231], v[176:179]\n\t" \
  "ds_read_b64_tr_b8 v[104:105], v69 offset:16384\n\t" \
  "ds_read_b64_tr_b8 v[106:107], v69 offset:17408\n\t" \
  "v_mfma_f32_16x16x128_f8f6f4 v[180:183], v[120:127], a[232:239], v[180:183]\n\t" \
  "ds_read_b64_tr_b8 v[108:109], v69 offset:24576\n\t" \
  "ds_read_b64_tr_b8 v[110:111], v69 offset:25600\n\t" \
  "v_mfma_f32_16x16x128_f8f6f4 v[184:187], v[120:127], a[240:247], v[184:187]\n\t" \
  "v_mfma_f32_16x16x128_f8f6f4 v[188:191], v[120:127], a[248:255], v[188:191]\n\t" \
  "s_waitcnt lgkmcnt(4)\n\t" \
  "v_mfma_f32_16x16x128_f8f6f4 v[192:195], v[96:103], a[224:231], v[192:195]\n\t" \
  "ds_read_b64_tr_b8 v[112:113], v70 offset:16384\n\t" \
  "ds_read_b64_tr_b8 v[114:115], v70 offset:17408\n\t" \
  "v_mfma_f32_16x16x128_f8f6f4 v[196:199], v[96:103], a[232:239], v[196:199]\n\t" \
  "ds_read_b64_tr_b8 v[116:117], v70 offset:24576\n\t" \
  "ds_read_b64_tr_b8 v[118:119], v70 offset:25600\n\t" \
  "v_mfma_f32_16x16x128_f8f6f4 v[200:203], v[96:103], a[240:247], v[200:203]\n\t" \
  "v_mfma_f32_16x16x128_f8f6f4 v[204:207], v[96:103], a[248:255], v[204:207]\n\t" \
  "s_waitcnt lgkmcnt(4)\n\t" \
  "v_mfma_f32_16x16x128_f8f6f4 v[208:211], v[104:111], a[224:231], v[208:211]\n\t" \
  "ds_read_b64_tr_b8 v[120:121], v71 offset:16384\n\t" \
  "ds_read_b64_tr_b8 v[122:123], v71 offset:17408\n\t" \
  "v_mfma_f32_16x16x128_f8f6f4 v[212:215], v[104:111], a[232:239], v[212:215]\n\t" \
  "ds_read_b64_tr_b8 v[124:125], v71 offset:24576\n\t" \
  "ds_read_b64_tr_b8 v[126:127], v71 offset:25600\n\t" \
  "v_mfma_f32_16x16x128_f8f6f4 v[216:219], v[104:111], a[240:247], v[216:219]\n\t" \
  "v_mfma_f32_16x16x128_f8f6f4 v[220:223], v[104:111], a[248:255], v[220:223]\n\t" \
  "s_waitcnt vmcnt(19)\n\t" \
  "s_barrier\n\t" \
  "s_waitcnt lgkmcnt(4)\n\t" \
  "v_mfma_f32_16x16x128_f8f6f4 v[224:227], v[112:119], a[224:231], v[224:227]\n\t" \
  "ds_read_b64_tr_b8 v[96:97], v64 offset:32768\n\t" \
  "ds_read_b64_tr_b8 v[98:99], v64 offset:33792\n\t" \
  "v_mfma_f32_16x16x128_f8f6f4 v[228:231], v[112:119], a[232:239], v[228:231]\n\t" \
  "ds_read_b64_tr_b8 v[100:101], v64 offset:40960\n\t" \
  "ds_read_b64_tr_b8 v[102:103], v64 offset:41984\n\t" \
  "s_add_u32 s89, s62, 1703936\n\t" \
  "s_add_u32 s90, s62, 1708032\n\t" \
  "buffer_load_dwordx4 a[64:67], v84, s[44:47], s89 offen offset:0 sc0 sc1\n\t" \
  "v_mfma_f32_16x16x128_f8f6f4 v[232:235], v[112:119], a[240:247], v[232:235]\n\t" \
  "buffer_load_dwordx4 a[68:71], v84, s[44:47], s89 offen offset:1024 sc0 sc1\n\t" \
  "v_mfma_f32_16x16x128_f8f6f4 v[236:239], v[112:119], a[248:255], v[236:239]\n\t" \
  "s_waitcnt lgkmcnt(4)\n\t" \
  "v_mfma_f32_16x16x128_f8f6f4 v[240:243], v[120:127], a[224:231], v[240:243]\n\t" \
  "ds_read_b64_tr_b8 v[104:105], v65 offset:32768\n\t" \
  "ds_read_b64_tr_b8 v[106:107], v65 offset:33792\n\t" \
  "v_mfma_f32_16x16x128_f8f6f4 v[244:247], v[120:127], a[232:239], v[244:247]\n\t" \
  "ds_read_b64_tr_b8 v[108:109], v65 offset:40960\n\t" \
  "ds_read_b64_tr_b8 v[110:111], v65 offset:41984\n\t" \
  "buffer_load_dwordx4 a[72:75], v84, s[44:47], s89 offen offset:2048 sc0 sc1\n\t" \
  "v_mfma_f32_16x16x128_f8f6f4 v[248:251], v[120:127], a[240:247], v[248:251]\n\t" \
  "buffer_load_dwordx4 a[76:79], v84, s[44:47], s89 offen offset:3072 sc0 sc1\n\t" \
  "v_mfma_f32_16x16x128_f8f6f4 v[252:255], v[120:127], a[248:255], v[252:255]\n\t" \
  "s_waitcnt vmcnt(32)\n\t" \
  "s_bitcmp1_b32 s65, 28\n\t" \
  "s_cbranch_scc1 OK89_%=\n\t" \
  "s_mov_b32 s67, 28\n\t" \
  "s_call_b64 s[68:69], POLL_%=\n\t" \
  "OK89_%=:\n\t" \
  "s_bitcmp1_b32 s65, 29\n\t" \
  "s_cbranch_scc1 OK90_%=\n\t" \
  "s_mov_b32 s67, 29\n\t" \
  "s_call_b64 s[68:69], POLL_%=\n\t" \
  "OK90_%=:\n\t" \
  "s_waitcnt lgkmcnt(4)\n\t" \
  "v_mfma_f32_16x16x128_f8f6f4 v[128:131], v[96:103], a[0:7], v[128:131]\n\t" \
  "ds_read_b64_tr_b8 v[112:113], v66 offset:32768\n\t" \
  "ds_read_b64_tr_b8 v[114:115], v66 offset:33792\n\t" \
  "v_mfma_f32_16x16x128_f8f6f4 v[132:135], v[96:103], a[8:15], v[132:135]\n\t" \
  "ds_read_b64_tr_b8 v[116:117], v66 offset:40960\n\t" \
  "ds_read_b64_tr_b8 v[118:119], v66 offset:41984\n\t" \
  "buffer_load_dwordx4 a[80:83], v84, s[44:47], s90 offen offset:0 sc0 sc1\n\t" \
  "v_mfma_f32_16x16x128_f8f6f4 v[136:139], v[96:103], a[16:23], v[136:139]\n\t" \
  "buffer_load_dwordx4 a[84:87], v84, s[44:47], s90 offen offset:1024 sc0 sc1\n\t" \
  "v_mfma_f32_16x16x128_f8f6f4 v[140:143], v[96:103], a[24:31], v[140:143]\n\t" \
  "buffer_load_dwordx4 a[88:91], v84, s[44:47], s90 offen offset:2048 sc0 sc1\n\t" \
  "s_waitcnt lgkmcnt(4)\n\t" \
  "v_mfma_f32_16x16x128_f8f6f4 v[144:147], v[104:111], a[0:7], v[144:147]\n\t" \
  "ds_read_b64_tr_b8 v[120:121], v67 offset:32768\n\t" \
  "ds_read_b64_tr_b8 v[122:123], v67 offset:33792\n\t" \
  "v_mfma_f32_16x16x128_f8f6f4 v[148:151], v[104:111], a[8:15], v[148:151]\n\t" \
  "ds_read_b64_tr_b8 v[124:125], v67 offset:40960\n\t" \
  "ds_read_b64_tr_b8 v[126:127], v67 offset:41984\n\t" \
  "buffer_load_dwordx4 a[92:95], v84, s[44:47], s90 offen offset:3072 sc0 sc1\n\t" \
  "v_mfma_f32_16x16x128_f8f6f4 v[152:155], v[104:111], a[16:23], v[152:155]\n\t" \
  "s_add_u32 s89, s62, 1769472\n\t" \
  "s_add_u32 s90, s62, 1773568\n\t" \
  "buffer_load_dwordx4 a[96:99], v84, s[44:47], s89 offen offset:0 sc0 sc1\n\t" \
  "v_mfma_f32_16x16x128_f8f6f4 v[156:159], v[104:111], a[24:31], v[156:159]\n\t" \
  "buffer_load_dwordx4 a[100:103], v84, s[44:47], s89 offen offset:1024 sc0 sc1\n\t" \
  "s_waitcnt lgkmcnt(4)\n\t" \
  "v_mfma_f32_16x16x128_f8f6f4 v[160:163], v[112:119], a[0:7], v[160:163]\n\t" \
  "ds_read_b64_tr_b8 v[96:97], v68 offset:32768\n\t" \
  "ds_read_b64_tr_b8 v[98:99], v68 offset:33792\n\t" \
  "v_mfma_f32_16x16x128_f8f6f4 v[164:167], v[112:119], a[8:15], v[164:167]\n\t" \
  "ds_read_b64_tr_b8 v[100:101], v68 offset:40960\n\t" \
  "ds_read_b64_tr_b8 v[102:103], v68 offset:41984\n\t" \
  "buffer_load_dwordx4 a[104:107], v84, s[44:47], s89 offen offset:2048 sc0 sc1\n\t" \
  "v_mfma_f32_16x16x128_f8f6f4 v[168:171], v[112:119], a[16:23], v[168:171]\n\t" \
  "buffer_load_dwordx4 a[108:111], v84, s[44:47], s89 offen offset:3072 sc0 sc1\n\t" \
  "v_mfma_f32_16x16x128_f8f6f4 v[172:175], v[112:119], a[24:31], v[172:175]\n\t" \
  "buffer_load_dwordx4 a[112:115], v84, s[44:47], s90 offen offset:0 sc0 sc1\n\t" \
  "s_waitcnt lgkmcnt(4)\n\t" \
  "v_mfma_f32_16x16x128_f8f6f4 v[176:179], v[120:127], a[0:7], v[176:179]\n\t" \
  "ds_read_b64_tr_b8 v[104:105], v69 offset:32768\n\t" \
  "ds_read_b64_tr_b8 v[106:107], v69 offset:33792\n\t" \
  "v_mfma_f32_16x16x128_f8f6f4 v[180:183], v[120:127], a[8:15], v[180:183]\n\t" \
  "ds_read_b64_tr_b8 v[108:109], v69 offset:40960\n\t" \
  "ds_read_b64_tr_b8 v[110:111], v69 offset:41984\n\t" \
  "buffer_load_dwordx4 a[116:119], v84, s[44:47], s90 offen offset:1024 sc0 sc1\n\t" \
  "v_mfma_f32_16x16x128_f8f6f4 v[184:187], v[120:127], a[16:23], v[184:187]\n\t" \
  "buffer_load_dwordx4 a[120:123], v84, s[44:47], s90 offen offset:2048 sc0 sc1\n\t" \
  "v_mfma_f32_16x16x128_f8f6f4 v[188:191], v[120:127], a[24:31], v[188:191]\n\t" \
  "buffer_load_dwordx4 a[124:127], v84, s[44:47], s90 offen offset:3072 sc0 sc1\n\t" \
  "s_waitcnt lgkmcnt(4)\n\t" \
  "v_mfma_f32_16x16x128_f8f6f4 v[192:195], v[96:103], a[0:7], v[192:195]\n\t" \
  "ds_read_b64_tr_b8 v[112:113], v70 offset:32768\n\t" \
  "ds_read_b64_tr_b8 v[114:115], v70 offset:33792\n\t" \
  "v_mfma_f32_16x16x128_f8f6f4 v[196:199], v[96:103], a[8:15], v[196:199]\n\t" \
  "ds_read_b64_tr_b8 v[116:117], v70 offset:40960\n\t" \
  "ds_read_b64_tr_b8 v[118:119], v70 offset:41984\n\t" \
  "s_add_u32 s89, s62, 1835008\n\t" \
  "s_add_u32 s90, s62, 1839104\n\t" \
  "buffer_load_dwordx4 a[128:131], v84, s[44:47], s89 offen offset:0 sc0 sc1\n\t" \
  "v_mfma_f32_16x16x128_f8f6f4 v[200:203], v[96:103], a[16:23], v[200:203]\n\t" \
  "buffer_load_dwordx4 a[132:135], v84, s[44:47], s89 offen offset:1024 sc0 sc1\n\t" \
  "v_mfma_f32_16x16x128_f8f6f4 v[204:207], v[96:103], a[24:31], v[204:207]\n\t" \
  "buffer_load_dwordx4 a[136:139], v84, s[44:47], s89 offen offset:2048 sc0 sc1\n\t" \
  "s_waitcnt lgkmcnt(4)\n\t" \
  "v_mfma_f32_16x16x128_f8f6f4 v[208:211], v[104:111], a[0:7], v[208:211]\n\t" \
  "ds_read_b64_tr_b8 v[120:121], v71 offset:32768\n\t" \
  "ds_read_b64_tr_b8 v[122:123], v71 offset:33792\n\t" \
  "v_mfma_f32_16x16x128_f8f6f4 v[212:215], v[104:111], a[8:15], v[212:215]\n\t" \
  "ds_read_b64_tr_b8 v[124:125], v71 offset:40960\n\t" \
  "ds_read_b64_tr_b8 v[126:127], v71 offset:41984\n\t" \
  "buffer_load_dwordx4 a[140:143], v84, s[44:47], s89 offen offset:3072 sc0 sc1\n\t" \
  "v_mfma_f32_16x16x128_f8f6f4 v[216:219], v[104:111], a[16:23], v[216:219]\n\t" \
  "buffer_load_dwordx4 a[144:147], v84, s[44:47], s90 offen offset:0 sc0 sc1\n\t" \
  "v_mfma_f32_16x16x128_f8f6f4 v[220:223], v[104:111], a[24:31], v[220:223]\n\t" \
  "buffer_load_dwordx4 a[148:151], v84, s[44:47], s90 offen offset:1024 sc0 sc1\n\t" \
  "s_waitcnt vmcnt(37)\n\t" \
  "s_barrier\n\t" \
  "s_waitcnt lgkmcnt(4)\n\t" \
  "v_mfma_f32_16x16x128_f8f6f4 v[224:227], v[112:119], a[0:7], v[224:227]\n\t" \
  "ds_read_b64_tr_b8 v[96:97], v72 offset:0\n\t" \
  "ds_read_b64_tr_b8 v[98:99], v72 offset:1024\n\t" \
  "v_mfma_f32_16x16x128_f8f6f4 v[228:231], v[112:119], a[8:15], v[228:231]\n\t" \
  "ds_read_b64_tr_b8 v[100:101], v72 offset:8192\n\t" \
  "ds_read_b64_tr_b8 v[102:103], v72 offset:9216\n\t" \
  "buffer_load_dwordx4 a[152:155], v84, s[44:47], s90 offen offset:2048 sc0 sc1\n\t" \
  "v_mfma_f32_16x16x128_f8f6f4 v[232:235], v[112:119], a[16:23], v[232:235]\n\t" \
  "buffer_load_dwordx4 a[156:159], v84, s[44:47], s90 offen offset:3072 sc0 sc1\n\t" \
  "v_mfma_f32_16x16x128_f8f6f4 v[236:239], v[112:119], a[24:31], v[236:239]\n\t" \
  "s_waitcnt lgkmcnt(4)\n\t" \
  "v_mfma_f32_16x16x128_f8f6f4 v[240:243], v[120:127], a[0:7], v[240:243]\n\t" \
  "ds_read_b64_tr_b8 v[104:105], v73 offset:0\n\t" \
  "ds_read_b64_tr_b8 v[106:107], v73 offset:1024\n\t" \
  "v_mfma_f32_16x16x128_f8f6f4 v[244:247], v[120:127], a[8:15], v[244:247]\n\t" \
  "ds_read_b64_tr_b8 v[108:109], v73 offset:8192\n\t" \
  "ds_read_b64_tr_b8 v[110:111], v73 offset:9216\n\t" \
  "s_add_u32 s89, s62, 1900544\n\t" \
  "s_add_u32 s90, s62, 1904640\n\t" \
  "buffer_load_dwordx4 a[160:163], v84, s[44:47], s89 offen offset:0 sc0 sc1\n\t" \
  "v_mfma_f32_16x16x128_f8f6f4 v[248:251], v[120:127], a[16:23], v[248:251]\n\t" \
  "buffer_load_dwordx4 a[164:167], v84, s[44:47], s89 offen offset:1024 sc0 sc1\n\t" \
  "v_mfma_f32_16x16x128_f8f6f4 v[252:255], v[120:127], a[24:31], v[252:255]\n\t" \
  "s_waitcnt vmcnt(26)\n\t" \
  "s_bitcmp1_b32 s65, 30\n\t" \
  "s_cbranch_scc1 OK91_%=\n\t" \
  "s_mov_b32 s67, 30\n\t" \
  "s_call_b64 s[68:69], POLL_%=\n\t" \
  "OK91_%=:\n\t" \
  "s_waitcnt lgkmcnt(4)\n\t" \
  "v_mfma_f32_16x16x128_f8f6f4 v[128:131], v[96:103], a[32:39], v[128:131]\n\t" \
  "ds_read_b64_tr_b8 v[112:113], v74 offset:0\n\t" \
  "ds_read_b64_tr_b8 v[114:115], v74 offset:1024\n\t" \
  "v_mfma_f32_16x16x128_f8f6f4 v[132:135], v[96:103], a[40:47], v[132:135]\n\t" \
  "ds_read_b64_tr_b8 v[116:117], v74 offset:8192\n\t" \
  "ds_read_b64_tr_b8 v[118:119], v74 offset:9216\n\t" \
  "buffer_load_dwordx4 a[168:171], v84, s[44:47], s89 offen offset:2048 sc0 sc1\n\t" \
  "v_mfma_f32_16x16x128_f8f6f4 v[136:139], v[96:103], a[48:55], v[136:139]\n\t" \
  "buffer_load_dwordx4 a[172:175], v84, s[44:47], s89 offen offset:3072 sc0 sc1\n\t" \
  "v_mfma_f32_16x16x128_f8f6f4 v[140:143], v[96:103], a[56:63], v[140:143]\n\t" \
  "buffer_load_dwordx4 a[176:179], v84, s[44:47], s90 offen offset:0 sc0 sc1\n\t" \
  "s_waitcnt lgkmcnt(4)\n\t" \
  "v_mfma_f32_16x16x128_f8f6f4 v[144:147], v[104:111], a[32:39], v[144:147]\n\t" \
  "ds_read_b64_tr_b8 v[120:121], v75 offset:0\n\t" \
  "ds_read_b64_tr_b8 v[122:123], v75 offset:1024\n\t" \
  "v_mfma_f32_16x16x128_f8f6f4 v[148:151], v[104:111], a[40:47], v[148:151]\n\t" \
  "ds_read_b64_tr_b8 v[124:125], v75 offset:8192\n\t" \
  "ds_read_b64_tr_b8 v[126:127], v75 offset:9216\n\t" \
  "buffer_load_dwordx4 a[180:183], v84, s[44:47], s90 offen offset:1024 sc0 sc1\n\t" \
  "v_mfma_f32_16x16x128_f8f6f4 v[152:155], v[104:111], a[48:55], v[152:155]\n\t" \
  "buffer_load_dwordx4 a[184:187], v84, s[44:47], s90 offen offset:2048 sc0 sc1\n\t" \
  "v_mfma_f32_16x16x128_f8f6f4 v[156:159], v[104:111], a[56:63], v[156:159]\n\t" \
  "buffer_load_dwordx4 a[188:191], v84, s[44:47], s90 offen offset:3072 sc0 sc1\n\t" \
  "s_waitcnt lgkmcnt(4)\n\t" \
  "v_mfma_f32_16x16x128_f8f6f4 v[160:163], v[112:119], a[32:39], v[160:163]\n\t" \
  "ds_read_b64_tr_b8 v[96:97], v76 offset:0\n\t" \
  "ds_read_b64_tr_b8 v[98:99], v76 offset:1024\n\t" \
  "v_mfma_f32_16x16x128_f8f6f4 v[164:167], v[112:119], a[40:47], v[164:167]\n\t" \
  "ds_read_b64_tr_b8 v[100:101], v76 offset:8192\n\t" \
  "ds_read_b64_tr_b8 v[102:103], v76 offset:9216\n\t" \
  "s_add_u32 s89, s62, 1966080\n\t" \
  "s_add_u32 s90, s62, 1970176\n\t" \
  "buffer_load_dwordx4 a[192:195], v84, s[44:47], s89 offen offset:0 sc0 sc1\n\t" \
  "v_mfma_f32_16x16x128_f8f6f4 v[168:171], v[112:119], a[48:55], v[168:171]\n\t" \
  "buffer_load_dwordx4 a[196:199], v84, s[44:47], s89 offen offset:1024 sc0 sc1\n\t" \
  "v_mfma_f32_16x16x128_f8f6f4 v[172:175], v[112:119], a[56:63], v[172:175]\n\t" \
  "s_waitcnt lgkmcnt(4)\n\t" \
  "v_mfma_f32_16x16x128_f8f6f4 v[176:179], v[120:127], a[32:39], v[176:179]\n\t" \
  "ds_read_b64_tr_b8 v[104:105], v77 offset:0\n\t" \
  "ds_read_b64_tr_b8 v[106:107], v77 offset:1024\n\t" \
  "v_mfma_f32_16x16x128_f8f6f4 v[180:183], v[120:127], a[40:47], v[180:183]\n\t" \
  "ds_read_b64_tr_b8 v[108:109], v77 offset:8192\n\t" \
  "ds_read_b64_tr_b8 v[110:111], v77 offset:9216\n\t" \
  "buffer_load_dwordx4 a[200:203], v84, s[44:47], s89 offen offset:2048 sc0 sc1\n\t" \
  "v_mfma_f32_16x16x128_f8f6f4 v[184:187], v[120:127], a[48:55], v[184:187]\n\t" \
  "buffer_load_dwordx4 a[204:207], v84, s[44:47], s89 offen offset:3072 sc0 sc1\n\t" \
  "v_mfma_f32_16x16x128_f8f6f4 v[188:191], v[120:127], a[56:63], v[188:191]\n\t" \
  "s_waitcnt lgkmcnt(4)\n\t" \
  "v_mfma_f32_16x16x128_f8f6f4 v[192:195], v[96:103], a[32:39], v[192:195]\n\t" \
  "ds_read_b64_tr_b8 v[112:113], v78 offset:0\n\t" \
  "ds_read_b64_tr_b8 v[114:115], v78 offset:1024\n\t" \
  "v_mfma_f32_16x16x128_f8f6f4 v[196:199], v[96:103], a[40:47], v[196:199]\n\t" \
  "ds_read_b64_tr_b8 v[116:117], v78 offset:8192\n\t" \
  "ds_read_b64_tr_b8 v[118:119], v78 offset:9216\n\t" \
  "s_add_u32 s88, s54, 6291456\n\t" \
  "s_add_u32 m0, s60, 131072\n\t" \
  "s_nop 0\n\t" \
  "buffer_load_dwordx4 v80, s[36:39], s88 offen  lds\n\t" \
  "v_mfma_f32_16x16x128_f8f6f4 v[200:203], v[96:103], a[48:55], v[200:203]\n\t" \
  "s_add_u32 m0, s60, 132096\n\t" \
  "s_nop 0\n\t" \
  "buffer_load_dwordx4 v81, s[36:39], s88 offen  lds\n\t" \
  "v_mfma_f32_16x16x128_f8f6f4 v[204:207], v[96:103], a[56:63], v[204:207]\n\t" \
  "s_add_u32 m0, s60, 133120\n\t" \
  "s_nop 0\n\t" \
  "buffer_load_dwordx4 v82, s[36:39], s88 offen  lds\n\t" \
  "s_waitcnt lgkmcnt(4)\n\t" \
  "v_mfma_f32_16x16x128_f8f6f4 v[208:211], v[104:111], a[32:39], v[208:211]\n\t" \
  "ds_read_b64_tr_b8 v[120:121], v79 offset:0\n\t" \
  "ds_read_b64_tr_b8 v[122:123], v79 offset:1024\n\t" \
  "v_mfma_f32_16x16x128_f8f6f4 v[212:215], v[104:111], a[40:47], v[212:215]\n\t" \
  "ds_read_b64_tr_b8 v[124:125], v79 offset:8192\n\t" \
  "ds_read_b64_tr_b8 v[126:127], v79 offset:9216\n\t" \
  "s_add_u32 m0, s60, 134144\n\t" \
  "s_nop 0\n\t" \
  "buffer_load_dwordx4 v83, s[36:39], s88 offen  lds\n\t" \
  "v_mfma_f32_16x16x128_f8f6f4 v[216:219], v[104:111], a[48:55], v[216:219]\n\t" \
  "s_add_u32 s88, s57, 6291456\n\t" \
  "s_add_u32 m0, s60, 49152\n\t" \
  "s_nop 0\n\t" \
  "buffer_load_dwordx4 v80, s[36:39], s88 offen  lds\n\t" \
  "v_mfma_f32_16x16x128_f8f6f4 v[220:223], v[104:111], a[56:63], v[220:223]\n\t" \
  "s_add_u32 m0, s60, 50176\n\t" \
  "s_nop 0\n\t" \
  "buffer_load_dwordx4 v81, s[36:39], s88 offen  lds\n\t" \
  "s_waitcnt vmcnt(53)\n\t" \
  "s_barrier\n\t" \
  "s_waitcnt lgkmcnt(4)\n\t" \
  "v_mfma_f32_16x16x128_f8f6f4 v[224:227], v[112:119], a[32:39], v[224:227]\n\t" \
  "ds_read_b64_tr_b8 v[96:97], v72 offset:16384\n\t" \
  "ds_read_b64_tr_b8 v[98:99], v72 offset:17408\n\t" \
  "v_mfma_f32_16x16x128_f8f6f4 v[228:231], v[112:119], a[40:47], v[228:231]\n\t" \
  "ds_read_b64_tr_b8 v[100:101], v72 offset:24576\n\t" \
  "ds_read_b64_tr_b8 v[102:103], v72 offset:25600\n\t" \
  "s_add_u32 m0, s60, 51200\n\t" \
  "s_nop 0\n\t" \
  "buffer_load_dwordx4 v82, s[36:39], s88 offen  lds\n\t" \
  "v_mfma_f32_16x16x128_f8f6f4 v[232:235], v[112:119], a[48:55], v[232:235]\n\t" \
  "s_add_u32 m0, s60, 52224\n\t" \
  "s_nop 0\n\t" \
  "buffer_load_dwordx4 v83, s[36:39], s88 offen  lds\n\t" \
  "v_mfma_f32_16x16x128_f8f6f4 v[236:239], v[112:119], a[56:63], v[236:239]\n\t" \
  "s_add_u32 s88, s58, 6291456\n\t" \
  "s_add_u32 m0, s60, 0\n\t" \
  "s_nop 0\n\t" \
  "buffer_load_dwordx4 v80, s[36:39], s88 offen  lds\n\t" \
  "s_waitcnt lgkmcnt(4)\n\t" \
  "v_mfma_f32_16x16x128_f8f6f4 v[240:243], v[120:127], a[32:39], v[240:243]\n\t" \
  "ds_read_b64_tr_b8 v[104:105], v73 offset:16384\n\t" \
  "ds_read_b64_tr_b8 v[106:107], v73 offset:17408\n\t" \
  "v_mfma_f32_16x16x128_f8f6f4 v[244:247], v[120:127], a[40:47], v[244:247]\n\t" \
  "ds_read_b64_tr_b8 v[108:109], v73 offset:24576\n\t" \
  "ds_read_b64_tr_b8 v[110:111], v73 offset:25600\n\t" \
  "s_add_u32 m0, s60, 1024\n\t" \
  "s_nop 0\n\t" \
  "buffer_load_dwordx4 v81, s[36:39], s88 offen  lds\n\t" \
  "v_mfma_f32_16x16x128_f8f6f4 v[248:251], v[120:127], a[48:55], v[248:251]\n\t" \
  "s_add_u32 m0, s60, 2048\n\t" \
  "s_nop 0\n\t" \
  "buffer_load_dwordx4 v82, s[36:39], s88 offen  lds\n\t" \
  "v_mfma_f32_16x16x128_f8f6f4 v[252:255], v[120:127], a[56:63], v[252:255]\n\t" \
  "s_add_u32 m0, s60, 3072\n\t" \
  "s_nop 0\n\t" \
  "buffer_load_dwordx4 v83, s[36:39], s88 offen  lds\n\t" \
  "s_waitcnt vmcnt(40)\n\t" \
  "s_bitcmp1_b32 s65, 31\n\t" \
  "s_cbranch_scc1 OK92_%=\n\t" \
  "s_mov_b32 s67, 31\n\t" \
  "s_call_b64 s[68:69], POLL_%=\n\t" \
  "OK92_%=:\n\t" \
  "global_load_dword v94, v88, s[98:99] sc1\n\t" \
  "s_waitcnt lgkmcnt(4)\n\t" \
  "v_mfma_f32_16x16x128_f8f6f4 v[128:131], v[96:103], a[64:71], v[128:131]\n\t" \
  "ds_read_b64_tr_b8 v[112:113], v74 offset:16384\n\t" \
  "ds_read_b64_tr_b8 v[114:115], v74 offset:17408\n\t" \
  "s_bfe_u32 s96, s65, 65567\n\t" \
  "s_cmp_lg_u32 s96, 1\n\t" \
  "s_cbranch_scc1 XWS93_%=\n\t" \
  "s_add_u32 s96, s62, 2031616\n\t" \
  "buffer_load_dword v8, v9, s[44:47], s96 offen sc0 sc1\n\t" \
  "XWS93_%=:\n\t" \
  "v_mfma_f32_16x16x128_f8f6f4 v[132:135], v[96:103], a[72:79], v[132:135]\n\t" \
  "ds_read_b64_tr_b8 v[116:117], v74 offset:24576\n\t" \
  "ds_read_b64_tr_b8 v[118:119], v74 offset:25600\n\t" \
  "v_mfma_f32_16x16x128_f8f6f4 v[136:139], v[96:103], a[80:87], v[136:139]\n\t" \
  "v_mfma_f32_16x16x128_f8f6f4 v[140:143], v[96:103], a[88:95], v[140:143]\n\t" \
  "s_waitcnt lgkmcnt(4)\n\t" \
  "v_mfma_f32_16x16x128_f8f6f4 v[144:147], v[104:111], a[64:71], v[144:147]\n\t" \
  "ds_read_b64_tr_b8 v[120:121], v75 offset:16384\n\t" \
  "ds_read_b64_tr_b8 v[122:123], v75 offset:17408\n\t" \
  "v_mfma_f32_16x16x128_f8f6f4 v[148:151], v[104:111], a[72:79], v[148:151]\n\t" \
  "ds_read_b64_tr_b8 v[124:125], v75 offset:24576\n\t" \
  "ds_read_b64_tr_b8 v[126:127], v75 offset:25600\n\t" \
  "v_mfma_f32_16x16x128_f8f6f4 v[152:155], v[104:111], a[80:87], v[152:155]\n\t" \
  "v_mfma_f32_16x16x128_f8f6f4 v[156:159], v[104:111], a[88:95], v[156:159]\n\t" \
  "s_waitcnt lgkmcnt(4)\n\t" \
  "v_mfma_f32_16x16x128_f8f6f4 v[160:163], v[112:119], a[64:71], v[160:163]\n\t" \
  "ds_read_b64_tr_b8 v[96:97], v76 offset:16384\n\t" \
  "ds_read_b64_tr_b8 v[98:99], v76 offset:17408\n\t" \
  "v_mfma_f32_16x16x128_f8f6f4 v[164:167], v[112:119], a[72:79], v[164:167]\n\t" \
  "ds_read_b64_tr_b8 v[100:101], v76 offset:24576\n\t" \
  "ds_read_b64_tr_b8 v[102:103], v76 offset:25600\n\t" \
  "v_mfma_f32_16x16x128_f8f6f4 v[168:171], v[112:119], a[80:87], v[168:171]\n\t" \
  "v_mfma_f32_16x16x128_f8f6f4 v[172:175], v[112:119], a[88:95], v[172:175]\n\t" \
  "s_waitcnt lgkmcnt(4)\n\t" \
  "v_mfma_f32_16x16x128_f8f6f4 v[176:179], v[120:127], a[64:71], v[176:179]\n\t" \
  "ds_read_b64_tr_b8 v[104:105], v77 offset:16384\n\t" \
  "ds_read_b64_tr_b8 v[106:107], v77 offset:17408\n\t" \
  "v_mfma_f32_16x16x128_f8f6f4 v[180:183], v[120:127], a[72:79], v[180:183]\n\t" \
  "ds_read_b64_tr_b8 v[108:109], v77 offset:24576\n\t" \
  "ds_read_b64_tr_b8 v[110:111], v77 offset:25600\n\t" \
  "v_mfma_f32_16x16x128_f8f6f4 v[184:187], v[120:127], a[80:87], v[184:187]\n\t" \
  "v_mfma_f32_16x16x128_f8f6f4 v[188:191], v[120:127], a[88:95], v[188:191]\n\t" \
  "s_waitcnt lgkmcnt(4)\n\t" \
  "v_mfma_f32_16x16x128_f8f6f4 v[192:195], v[96:103], a[64:71], v[192:195]\n\t" \
  "ds_read_b64_tr_b8 v[112:113], v78 offset:16384\n\t" \
  "ds_read_b64_tr_b8 v[114:115], v78 offset:17408\n\t" \
  "v_mfma_f32_16x16x128_f8f6f4 v[196:199], v[96:103], a[72:79], v[196:199]\n\t" \
  "ds_read_b64_tr_b8 v[116:117], v78 offset:24576\n\t" \
  "ds_read_b64_tr_b8 v[118:119], v78 offset:25600\n\t" \
  "buffer_load_dwordx4 a[208:211], v84, s[44:47], s90 offen offset:0 sc0 sc1\n\t" \
  "v_mfma_f32_16x16x128_f8f6f4 v[200:203], v[96:103], a[80:87], v[200:203]\n\t" \
  "buffer_load_dwordx4 a[212:215], v84, s[44:47], s90 offen offset:1024 sc0 sc1\n\t" \
  "v_mfma_f32_16x16x128_f8f6f4 v[204:207], v[96:103], a[88:95], v[204:207]\n\t" \
  "buffer_load_dwordx4 a[216:219], v84, s[44:47], s90 offen offset:2048 sc0 sc1\n\t" \
  "s_waitcnt lgkmcnt(4)\n\t" \
  "v_mfma_f32_16x16x128_f8f6f4 v[208:211], v[104:111], a[64:71], v[208:211]\n\t" \
  "ds_read_b64_tr_b8 v[120:121], v79 offset:16384\n\t" \
  "ds_read_b64_tr_b8 v[122:123], v79 offset:17408\n\t" \
  "v_mfma_f32_16x16x128_f8f6f4 v[212:215], v[104:111], a[72:79], v[212:215]\n\t" \
  "ds_read_b64_tr_b8 v[124:125], v79 offset:24576\n\t" \
  "ds_read_b64_tr_b8 v[126:127], v79 offset:25600\n\t" \
  "buffer_load_dwordx4 a[220:223], v84, s[44:47], s90 offen offset:3072 sc0 sc1\n\t" \
  "v_mfma_f32_16x16x128_f8f6f4 v[216:219], v[104:111], a[80:87], v[216:219]\n\t" \
  "v_mfma_f32_16x16x128_f8f6f4 v[220:223], v[104:111], a[88:95], v[220:223]\n\t" \
  "s_waitcnt vmcnt(13)\n\t" \
  "s_barrier\n\t" \
  "s_waitcnt lgkmcnt(4)\n\t" \
  "v_mfma_f32_16x16x128_f8f6f4 v[224:227], v[112:119], a[64:71], v[224:227]\n\t" \
  "ds_read_b64_tr_b8 v[96:97], v16 offset:0\n\t" \
  "ds_read_b64_tr_b8 v[98:99], v16 offset:1024\n\t" \
  "v_mfma_f32_16x16x128_f8f6f4 v[228:231], v[112:119], a[72:79], v[228:231]\n\t" \
  "ds_read_b64_tr_b8 v[100:101], v16 offset:8192\n\t" \
  "ds_read_b64_tr_b8 v[102:103], v16 offset:9216\n\t" \
  "v_mfma_f32_16x16x128_f8f6f4 v[232:235], v[112:119], a[80:87], v[232:235]\n\t" \
  "v_mfma_f32_16x16x128_f8f6f4 v[236:239], v[112:119], a[88:95], v[236:239]\n\t" \
  "s_waitcnt lgkmcnt(4)\n\t" \
  "v_mfma_f32_16x16x128_f8f6f4 v[240:243], v[120:127], a[64:71], v[240:243]\n\t" \
  "ds_read_b64_tr_b8 v[104:105], v17 offset:0\n\t" \
  "ds_read_b64_tr_b8 v[106:107], v17 offset:1024\n\t" \
  "v_mfma_f32_16x16x128_f8f6f4 v[244:247], v[120:127], a[72:79], v[244:247]\n\t" \
  "ds_read_b64_tr_b8 v[108:109], v17 offset:8192\n\t" \
  "ds_read_b64_tr_b8 v[110:111], v17 offset:9216\n\t" \
  "v_mfma_f32_16x16x128_f8f6f4 v[248:251], v[120:127], a[80:87], v[248:251]\n\t" \
  "v_mfma_f32_16x16x128_f8f6f4 v[252:255], v[120:127], a[88:95], v[252:255]\n\t" \
  "s_waitcnt vmcnt(37)\n\t" \
  "s_waitcnt lgkmcnt(4)\n\t" \
  "v_mfma_f32_16x16x128_f8f6f4 v[128:131], v[96:103], a[96:103], v[128:131]\n\t" \
  "ds_read_b64_tr_b8 v[112:113], v18 offset:0\n\t" \
  "ds_read_b64_tr_b8 v[114:115], v18 offset:1024\n\t" \
  "v_mfma_f32_16x16x128_f8f6f4 v[132:135], v[96:103], a[104:111], v[132:135]\n\t" \
  "ds_read_b64_tr_b8 v[116:117], v18 offset:8192\n\t" \
  "ds_read_b64_tr_b8 v[118:119], v18 offset:9216\n\t" \
  "v_mfma_f32_16x16x128_f8f6f4 v[136:139], v[96:103], a[112:119], v[136:139]\n\t" \
  "v_mfma_f32_16x16x128_f8f6f4 v[140:143], v[96:103], a[120:127], v[140:143]\n\t" \
  "s_waitcnt lgkmcnt(4)\n\t" \
  "v_mfma_f32_16x16x128_f8f6f4 v[144:147], v[104:111], a[96:103], v[144:147]\n\t" \
  "ds_read_b64_tr_b8 v[120:121], v19 offset:0\n\t" \
  "ds_read_b64_tr_b8 v[122:123], v19 offset:1024\n\t" \
  "v_mfma_f32_16x16x128_f8f6f4 v[148:151], v[104:111], a[104:111], v[148:151]\n\t" \
  "ds_read_b64_tr_b8 v[124:125], v19 offset:8192\n\t" \
  "ds_read_b64_tr_b8 v[126:127], v19 offset:9216\n\t" \
  "v_mfma_f32_16x16x128_f8f6f4 v[152:155], v[104:111], a[112:119], v[152:155]\n\t" \
  "v_mfma_f32_16x16x128_f8f6f4 v[156:159], v[104:111], a[120:127], v[156:159]\n\t" \
  "s_waitcnt lgkmcnt(4)\n\t" \
  "v_mfma_f32_16x16x128_f8f6f4 v[160:163], v[112:119], a[96:103], v[160:163]\n\t" \
  "ds_read_b64_tr_b8 v[96:97], v20 offset:0\n\t" \
  "ds_read_b64_tr_b8 v[98:99], v20 offset:1024\n\t" \
  "v_mfma_f32_16x16x128_f8f6f4 v[164:167], v[112:119], a[104:111], v[164:167]\n\t" \
  "ds_read_b64_tr_b8 v[100:101], v20 offset:8192\n\t" \
  "ds_read_b64_tr_b8 v[102:103], v20 offset:9216\n\t" \
  "s_add_u32 s89, s62, 2031616\n\t" \
  "s_add_u32 s90, s62, 2035712\n\t" \
  "buffer_load_dwordx4 a[224:227], v84, s[44:47], s89 offen offset:0 sc0 sc1\n\t" \
  "v_mfma_f32_16x16x128_f8f6f4 v[168:171], v[112:119], a[112:119], v[168:171]\n\t" \
  "buffer_load_dwordx4 a[228:231], v84, s[44:47], s89 offen offset:1024 sc0 sc1\n\t" \
  "v_mfma_f32_16x16x128_f8f6f4 v[172:175], v[112:119], a[120:127], v[172:175]\n\t" \
  "buffer_load_dwordx4 a[232:235], v84, s[44:47], s89 offen offset:2048 sc0 sc1\n\t" \
  "s_waitcnt lgkmcnt(4)\n\t" \
  "v_mfma_f32_16x16x128_f8f6f4 v[176:179], v[120:127], a[96:103], v[176:179]\n\t" \
  "ds_read_b64_tr_b8 v[104:105], v21 offset:0\n\t" \
  "ds_read_b64_tr_b8 v[106:107], v21 offset:1024\n\t" \
  "v_mfma_f32_16x16x128_f8f6f4 v[180:183], v[120:127], a[104:111], v[180:183]\n\t" \
  "ds_read_b64_tr_b8 v[108:109], v21 offset:8192\n\t" \
  "ds_read_b64_tr_b8 v[110:111], v21 offset:9216\n\t" \
  "buffer_load_dwordx4 a[236:239], v84, s[44:47], s89 offen offset:3072 sc0 sc1\n\t" \
  "v_mfma_f32_16x16x128_f8f6f4 v[184:187], v[120:127], a[112:119], v[184:187]\n\t" \
  "buffer_load_dwordx4 a[240:243], v84, s[44:47], s90 offen offset:0 sc0 sc1\n\t" \
  "v_mfma_f32_16x16x128_f8f6f4 v[188:191], v[120:127], a[120:127], v[188:191]\n\t" \
  "buffer_load_dwordx4 a[244:247], v84, s[44:47], s90 offen offset:1024 sc0 sc1\n\t" \
  "s_waitcnt lgkmcnt(4)\n\t" \
  "v_mfma_f32_16x16x128_f8f6f4 v[192:195], v[96:103], a[96:103], v[192:195]\n\t" \
  "ds_read_b64_tr_b8 v[112:113], v22 offset:0\n\t" \
  "ds_read_b64_tr_b8 v[114:115], v22 offset:1024\n\t" \
  "v_mfma_f32_16x16x128_f8f6f4 v[196:199], v[96:103], a[104:111], v[196:199]\n\t" \
  "ds_read_b64_tr_b8 v[116:117], v22 offset:8192\n\t" \
  "ds_read_b64_tr_b8 v[118:119], v22 offset:9216\n\t" \
  "buffer_load_dwordx4 a[248:251], v84, s[44:47], s90 offen offset:2048 sc0 sc1\n\t" \
  "v_mfma_f32_16x16x128_f8f6f4 v[200:203], v[96:103], a[112:119], v[200:203]\n\t" \
  "buffer_load_dwordx4 a[252:255], v84, s[44:47], s90 offen offset:3072 sc0 sc1\n\t" \
  "v_mfma_f32_16x16x128_f8f6f4 v[204:207], v[96:103], a[120:127], v[204:207]\n\t" \
  "s_waitcnt lgkmcnt(4)\n\t" \
  "v_mfma_f32_16x16x128_f8f6f4 v[208:211], v[104:111], a[96:103], v[208:211]\n\t" \
  "ds_read_b64_tr_b8 v[120:121], v23 offset:0\n\t" \
  "ds_read_b64_tr_b8 v[122:123], v23 offset:1024\n\t" \
  "v_mfma_f32_16x16x128_f8f6f4 v[212:215], v[104:111], a[104:111], v[212:215]\n\t" \
  "ds_read_b64_tr_b8 v[124:125], v23 offset:8192\n\t" \
  "ds_read_b64_tr_b8 v[126:127], v23 offset:9216\n\t" \
  "v_mfma_f32_16x16x128_f8f6f4 v[216:219], v[104:111], a[112:119], v[216:219]\n\t" \
  "v_mfma_f32_16x16x128_f8f6f4 v[220:223], v[104:111], a[120:127], v[220:223]\n\t" \
  "s_waitcnt vmcnt(13)\n\t" \
  "s_barrier\n\t" \
  "s_waitcnt lgkmcnt(4)\n\t" \
  "v_mfma_f32_16x16x128_f8f6f4 v[224:227], v[112:119], a[96:103], v[224:227]\n\t" \
  "ds_read_b64_tr_b8 v[96:97], v72 offset:32768\n\t" \
  "ds_read_b64_tr_b8 v[98:99], v72 offset:33792\n\t" \
  "v_mfma_f32_16x16x128_f8f6f4 v[228:231], v[112:119], a[104:111], v[228:231]\n\t" \
  "ds_read_b64_tr_b8 v[100:101], v72 offset:40960\n\t" \
  "ds_read_b64_tr_b8 v[102:103], v72 offset:41984\n\t" \
  "v_mfma_f32_16x16x128_f8f6f4 v[232:235], v[112:119], a[112:119], v[232:235]\n\t" \
  "v_mfma_f32_16x16x128_f8f6f4 v[236:239], v[112:119], a[120:127], v[236:239]\n\t" \
  "s_waitcnt lgkmcnt(4)\n\t" \
  "v_mfma_f32_16x16x128_f8f6f4 v[240:243], v[120:127], a[96:103], v[240:243]\n\t" \
  "ds_read_b64_tr_b8 v[104:105], v72 offset:49152\n\t" \
  "ds_read_b64_tr_b8 v[106:107], v72 offset:50176\n\t" \
  "v_mfma_f32_16x16x128_f8f6f4 v[244:247], v[120:127], a[104:111], v[244:247]\n\t" \
  "ds_read_b64_tr_b8 v[108:109], v72 offset:57344\n\t" \
  "ds_read_b64_tr_b8 v[110:111], v72 offset:58368\n\t" \
  "v_mfma_f32_16x16x128_f8f6f4 v[248:251], v[120:127], a[112:119], v[248:251]\n\t" \
  "v_mfma_f32_16x16x128_f8f6f4 v[252:255], v[120:127], a[120:127], v[252:255]\n\t" \
  "s_waitcnt vmcnt(12)\n\t" \
  "v_readfirstlane_b32 s80, v94\n\t" \
  "s_nop 3\n\t" \
  "s_cmp_eq_u32 s80, s64\n\t" \
  "s_cselect_b32 s80, 1, 0\n\t" \
  "s_or_b32 s66, s66, s80\n\t" \
  "s_cmp_lg_u32 s66, 0\n\t" \
  "s_cbranch_scc0 GOOD94_%=\n\t" \
  "s_branch BADOUT_%=\n\t" \
  "GOOD94_%=:\n\t" \
  "s_mov_b32 s84, 0\n\t" \
  "s_mov_b32 s85, 1048576\n\t" \
  "s_mov_b32 s86, 2097152\n\t" \
  "s_mov_b32 s87, 3145728\n\t" \
  "s_waitcnt vmcnt(37)\n\t" \
  "s_waitcnt lgkmcnt(4)\n\t" \
  "v_mfma_f32_16x16x128_f8f6f4 v[128:131], v[96:103], a[128:135], v[128:131]\n\t" \
  "ds_read_b64_tr_b8 v[112:113], v64 offset:49152\n\t" \
  "ds_read_b64_tr_b8 v[114:115], v64 offset:50176\n\t" \
  "v_mfma_f32_16x16x128_f8f6f4 v[132:135], v[96:103], a[136:143], v[132:135]\n\t" \
  "ds_read_b64_tr_b8 v[116:117], v64 offset:57344\n\t" \
  "ds_read_b64_tr_b8 v[118:119], v64 offset:58368\n\t" \
  "v_mfma_f32_16x16x128_f8f6f4 v[136:139], v[96:103], a[144:151], v[136:139]\n\t" \
  "v_mfma_f32_16x16x128_f8f6f4 v[140:143], v[96:103], a[152:159], v[140:143]\n\t" \
  "s_waitcnt vmcnt(29)\n\t" \
  "s_waitcnt lgkmcnt(4)\n\t" \
  "v_mfma_f32_16x16x128_f8f6f4 v[128:131], v[104:111], a[160:167], v[128:131]\n\t" \
  "ds_read_b64_tr_b8 v[120:121], v64 offset:0\n\t" \
  "ds_read_b64_tr_b8 v[122:123], v64 offset:1024\n\t" \
  "v_mfma_f32_16x16x128_f8f6f4 v[132:135], v[104:111], a[168:175], v[132:135]\n\t" \
  "ds_read_b64_tr_b8 v[124:125], v64 offset:8192\n\t" \
  "ds_read_b64_tr_b8 v[126:127], v64 offset:9216\n\t" \
  "v_mfma_f32_16x16x128_f8f6f4 v[136:139], v[104:111], a[176:183], v[136:139]\n\t" \
  "v_mfma_f32_16x16x128_f8f6f4 v[140:143], v[104:111], a[184:191], v[140:143]\n\t" \
  "s_waitcnt vmcnt(8)\n\t" \
  "s_waitcnt lgkmcnt(4)\n\t" \
  "v_mfma_f32_16x16x128_f8f6f4 v[128:131], v[112:119], a[192:199], v[128:131]\n\t" \
  "ds_read_b64_tr_b8 v[96:97], v73 offset:32768\n\t" \
  "ds_read_b64_tr_b8 v[98:99], v73 offset:33792\n\t" \
  "v_mfma_f32_16x16x128_f8f6f4 v[132:135], v[112:119], a[200:207], v[132:135]\n\t" \
  "ds_read_b64_tr_b8 v[100:101], v73 offset:40960\n\t" \
  "ds_read_b64_tr_b8 v[102:103], v73 offset:41984\n\t" \
  "v_mfma_f32_16x16x128_f8f6f4 v[136:139], v[112:119], a[208:215], v[136:139]\n\t" \
  "v_mfma_f32_16x16x128_f8f6f4 v[140:143], v[112:119], a[216:223], v[140:143]\n\t" \
  "s_waitcnt vmcnt(0)\n\t" \
  "s_waitcnt lgkmcnt(4)\n\t" \
  "v_mfma_f32_16x16x128_f8f6f4 v[128:131], v[120:127], a[224:231], v[128:131]\n\t" \
  "ds_read_b64_tr_b8 v[104:105], v73 offset:49152\n\t" \
  "ds_read_b64_tr_b8 v[106:107], v73 offset:50176\n\t" \
  "v_mfma_f32_16x16x128_f8f6f4 v[132:135], v[120:127], a[232:239], v[132:135]\n\t" \
  "ds_read_b64_tr_b8 v[108:109], v73 offset:57344\n\t" \
  "ds_read_b64_tr_b8 v[110:111], v73 offset:58368\n\t" \
  "v_mfma_f32_16x16x128_f8f6f4 v[136:139], v[120:127], a[240:247], v[136:139]\n\t" \
  "v_mfma_f32_16x16x128_f8f6f4 v[140:143], v[120:127], a[248:255], v[140:143]\n\t" \
  "s_waitcnt lgkmcnt(4)\n\t" \
  "v_mfma_f32_16x16x128_f8f6f4 v[144:147], v[96:103], a[128:135], v[144:147]\n\t" \
  "ds_read_b64_tr_b8 v[112:113], v65 offset:49152\n\t" \
  "ds_read_b64_tr_b8 v[114:115], v65 offset:50176\n\t" \
  "v_mfma_f32_16x16x128_f8f6f4 v[148:151], v[96:103], a[136:143], v[148:151]\n\t" \
  "ds_read_b64_tr_b8 v[116:117], v65 offset:57344\n\t" \
  "ds_read_b64_tr_b8 v[118:119], v65 offset:58368\n\t" \
  "v_mfma_f32_16x16x128_f8f6f4 v[152:155], v[96:103], a[144:151], v[152:155]\n\t" \
  "v_mfma_f32_16x16x128_f8f6f4 v[156:159], v[96:103], a[152:159], v[156:159]\n\t" \
  "v_pk_mul_f32 v[128:129], v[128:129], s[78:79]\n\t" \
  "v_pk_mul_f32 v[130:131], v[130:131], s[78:79]\n\t" \
  "v_max_f32 v128, 0, v128\n\t" \
  "s_waitcnt lgkmcnt(4)\n\t" \
  "v_mfma_f32_16x16x128_f8f6f4 v[144:147], v[104:111], a[160:167], v[144:147]\n\t" \
  "ds_read_b64_tr_b8 v[120:121], v65 offset:0\n\t" \
  "ds_read_b64_tr_b8 v[122:123], v65 offset:1024\n\t" \
  "v_max_f32 v129, 0, v129\n\t" \
  "v_max_f32 v130, 0, v130\n\t" \
  "v_max_f32 v131, 0, v131\n\t" \
  "v_mfma_f32_16x16x128_f8f6f4 v[148:151], v[104:111], a[168:175], v[148:151]\n\t" \
  "ds_read_b64_tr_b8 v[124:125], v65 offset:8192\n\t" \
  "ds_read_b64_tr_b8 v[126:127], v65 offset:9216\n\t" \
  "buffer_store_dwordx4 v[128:131], v85, s[48:51], s84 offen offset:0 sc1\n\t" \
  "v_pk_mul_f32 v[132:133], v[132:133], s[78:79]\n\t" \
  "v_mfma_f32_16x16x128_f8f6f4 v[152:155], v[104:111], a[176:183], v[152:155]\n\t" \
  "v_pk_mul_f32 v[134:135], v[134:135], s[78:79]\n\t" \
  "v_max_f32 v132, 0, v132\n\t" \
  "v_mfma_f32_16x16x128_f8f6f4 v[156:159], v[104:111], a[184:191], v[156:159]\n\t" \
  "v_max_f32 v133, 0, v133\n\t" \
  "v_max_f32 v134, 0, v134\n\t" \
  "s_waitcnt lgkmcnt(4)\n\t" \
  "v_mfma_f32_16x16x128_f8f6f4 v[144:147], v[112:119], a[192:199], v[144:147]\n\t" \
  "ds_read_b64_tr_b8 v[96:97], v74 offset:32768\n\t" \
  "ds_read_b64_tr_b8 v[98:99], v74 offset:33792\n\t" \
  "v_max_f32 v135, 0, v135\n\t" \
  "buffer_store_dwordx4 v[132:135], v85, s[48:51], s85 offen offset:0 sc1\n\t" \
  "v_mfma_f32_16x16x128_f8f6f4 v[148:151], v[112:119], a[200:207], v[148:151]\n\t" \
  "ds_read_b64_tr_b8 v[100:101], v74 offset:40960\n\t" \
  "ds_read_b64_tr_b8 v[102:103], v74 offset:41984\n\t" \
  "v_pk_mul_f32 v[136:137], v[136:137], s[78:79]\n\t" \
  "v_pk_mul_f32 v[138:139], v[138:139], s[78:79]\n\t" \
  "v_mfma_f32_16x16x128_f8f6f4 v[152:155], v[112:119], a[208:215], v[152:155]\n\t" \
  "v_max_f32 v136, 0, v136\n\t" \
  "v_max_f32 v137, 0, v137\n\t" \
  "v_mfma_f32_16x16x128_f8f6f4 v[156:159], v[112:119], a[216:223], v[156:159]\n\t" \
  "v_max_f32 v138, 0, v138\n\t" \
  "v_max_f32 v139, 0, v139\n\t" \
  "s_waitcnt lgkmcnt(4)\n\t" \
  "v_mfma_f32_16x16x128_f8f6f4 v[144:147], v[120:127], a[224:231], v[144:147]\n\t" \
  "ds_read_b64_tr_b8 v[104:105], v74 offset:49152\n\t" \
  "ds_read_b64_tr_b8 v[106:107], v74 offset:50176\n\t" \
  "buffer_store_dwordx4 v[136:139], v85, s[48:51], s86 offen offset:0 sc1\n\t" \
  "v_pk_mul_f32 v[140:141], v[140:141], s[78:79]\n\t" \
  "v_mfma_f32_16x16x128_f8f6f4 v[148:151], v[120:127], a[232:239], v[148:151]\n\t" \
  "ds_read_b64_tr_b8 v[108:109], v74 offset:57344\n\t" \
  "ds_read_b64_tr_b8 v[110:111], v74 offset:58368\n\t" \
  "v_pk_mul_f32 v[142:143], v[142:143], s[78:79]\n\t" \
  "v_max_f32 v140, 0, v140\n\t" \
  "v_mfma_f32_16x16x128_f8f6f4 v[152:155], v[120:127], a[240:247], v[152:155]\n\t" \
  "v_max_f32 v141, 0, v141\n\t" \
  "v_max_f32 v142, 0, v142\n\t" \
  "v_mfma_f32_16x16x128_f8f6f4 v[156:159], v[120:127], a[248:255], v[156:159]\n\t" \
  "v_max_f32 v143, 0, v143\n\t" \
  "buffer_store_dwordx4 v[140:143], v85, s[48:51], s87 offen offset:0 sc1\n\t" \
  "s_waitcnt lgkmcnt(4)\n\t" \
  "v_mfma_f32_16x16x128_f8f6f4 v[160:163], v[96:103], a[128:135], v[160:163]\n\t" \
  "ds_read_b64_tr_b8 v[112:113], v66 offset:49152\n\t" \
  "ds_read_b64_tr_b8 v[114:115], v66 offset:50176\n\t" \
  "v_mfma_f32_16x16x128_f8f6f4 v[164:167], v[96:103], a[136:143], v[164:167]\n\t" \
  "ds_read_b64_tr_b8 v[116:117], v66 offset:57344\n\t" \
  "ds_read_b64_tr_b8 v[118:119], v66 offset:58368\n\t" \
  "v_mfma_f32_16x16x128_f8f6f4 v[168:171], v[96:103], a[144:151], v[168:171]\n\t" \
  "v_mfma_f32_16x16x128_f8f6f4 v[172:175], v[96:103], a[152:159], v[172:175]\n\t" \
  "v_pk_mul_f32 v[144:145], v[144:145], s[78:79]\n\t" \
  "v_pk_mul_f32 v[146:147], v[146:147], s[78:79]\n\t" \
  "v_max_f32 v144, 0, v144\n\t" \
  "s_waitcnt lgkmcnt(4)\n\t" \
  "v_mfma_f32_16x16x128_f8f6f4 v[160:163], v[104:111], a[160:167], v[160:163]\n\t" \
  "ds_read_b64_tr_b8 v[120:121], v66 offset:0\n\t" \
  "ds_read_b64_tr_b8 v[122:123], v66 offset:1024\n\t" \
  "v_max_f32 v145, 0, v145\n\t" \
  "v_max_f32 v146, 0, v146\n\t" \
  "v_max_f32 v147, 0, v147\n\t" \
  "v_mfma_f32_16x16x128_f8f6f4 v[164:167], v[104:111], a[168:175], v[164:167]\n\t" \
  "ds_read_b64_tr_b8 v[124:125], v66 offset:8192\n\t" \
  "ds_read_b64_tr_b8 v[126:127], v66 offset:9216\n\t" \
  "buffer_store_dwordx4 v[144:147], v85, s[48:51], s84 offen offset:64 sc1\n\t" \
  "v_pk_mul_f32 v[148:149], v[148:149], s[78:79]\n\t" \
  "v_mfma_f32_16x16x128_f8f6f4 v[168:171], v[104:111], a[176:183], v[168:171]\n\t" \
  "v_pk_mul_f32 v[150:151], v[150:151], s[78:79]\n\t" \
  "v_max_f32 v148, 0, v148\n\t" \
  "v_mfma_f32_16x16x128_f8f6f4 v[172:175], v[104:111], a[184:191], v[172:175]\n\t" \
  "v_max_f32 v149, 0, v149\n\t" \
  "v_max_f32 v150, 0, v150\n\t" \
  "s_waitcnt lgkmcnt(4)\n\t" \
  "v_mfma_f32_16x16x128_f8f6f4 v[160:163], v[112:119], a[192:199], v[160:163]\n\t" \
  "ds_read_b64_tr_b8 v[96:97], v75 offset:32768\n\t" \
  "ds_read_b64_tr_b8 v[98:99], v75 offset:33792\n\t" \
  "v_max_f32 v151, 0, v151\n\t" \
  "buffer_store_dwordx4 v[148:151], v85, s[48:51], s85 offen offset:64 sc1\n\t" \
  "v_mfma_f32_16x16x128_f8f6f4 v[164:167], v[112:119], a[200:207], v[164:167]\n\t" \
  "ds_read_b64_tr_b8 v[100:101], v75 offset:40960\n\t" \
  "ds_read_b64_tr_b8 v[102:103], v75 offset:41984\n\t" \
  "v_pk_mul_f32 v[152:153], v[152:153], s[78:79]\n\t" \
  "v_pk_mul_f32 v[154:155], v[154:155], s[78:79]\n\t" \
  "v_mfma_f32_16x16x128_f8f6f4 v[168:171], v[112:119], a[208:215], v[168:171]\n\t" \
  "v_max_f32 v152, 0, v152\n\t" \
  "v_max_f32 v153, 0, v153\n\t" \
  "v_mfma_f32_16x16x128_f8f6f4 v[172:175], v[112:119], a[216:223], v[172:175]\n\t" \
  "v_max_f32 v154, 0, v154\n\t" \
  "v_max_f32 v155, 0, v155\n\t" \
  "s_waitcnt lgkmcnt(4)\n\t" \
  "v_mfma_f32_16x16x128_f8f6f4 v[160:163], v[120:127], a[224:231], v[160:163]\n\t" \
  "ds_read_b64_tr_b8 v[104:105], v75 offset:49152\n\t" \
  "ds_read_b64_tr_b8 v[106:107], v75 offset:50176\n\t" \
  "buffer_store_dwordx4 v[152:155], v85, s[48:51], s86 offen offset:64 sc1\n\t" \
  "v_pk_mul_f32 v[156:157], v[156:157], s[78:79]\n\t" \
  "v_mfma_f32_16x16x128_f8f6f4 v[164:167], v[120:127], a[232:239], v[164:167]\n\t" \
  "ds_read_b64_tr_b8 v[108:109], v75 offset:57344\n\t" \
  "ds_read_b64_tr_b8 v[110:111], v75 offset:58368\n\t" \
  "v_pk_mul_f32 v[158:159], v[158:159], s[78:79]\n\t" \
  "v_max_f32 v156, 0, v156\n\t" \
  "v_mfma_f32_16x16x128_f8f6f4 v[168:171], v[120:127], a[240:247], v[168:171]\n\t" \
  "v_max_f32 v157, 0, v157\n\t" \
  "v_max_f32 v158, 0, v158\n\t" \
  "v_mfma_f32_16x16x128_f8f6f4 v[172:175], v[120:127], a[248:255], v[172:175]\n\t" \
  "v_max_f32 v159, 0, v159\n\t" \
  "buffer_store_dwordx4 v[156:159], v85, s[48:51], s87 offen offset:64 sc1\n\t" \
  "s_waitcnt lgkmcnt(4)\n\t" \
  "v_mfma_f32_16x16x128_f8f6f4 v[176:179], v[96:103], a[128:135], v[176:179]\n\t" \
  "ds_read_b64_tr_b8 v[112:113], v67 offset:49152\n\t" \
  "ds_read_b64_tr_b8 v[114:115], v67 offset:50176\n\t" \
  "v_mfma_f32_16x16x128_f8f6f4 v[180:183], v[96:103], a[136:143], v[180:183]\n\t" \
  "ds_read_b64_tr_b8 v[116:117], v67 offset:57344\n\t" \
  "ds_read_b64_tr_b8 v[118:119], v67 offset:58368\n\t" \
  "v_mfma_f32_16x16x128_f8f6f4 v[184:187], v[96:103], a[144:151], v[184:187]\n\t" \
  "v_mfma_f32_16x16x128_f8f6f4 v[188:191], v[96:103], a[152:159], v[188:191]\n\t" \
  "v_pk_mul_f32 v[160:161], v[160:161], s[78:79]\n\t" \
  "v_pk_mul_f32 v[162:163], v[162:163], s[78:79]\n\t" \
  "v_max_f32 v160, 0, v160\n\t" \
  "s_waitcnt lgkmcnt(4)\n\t" \
  "v_mfma_f32_16x16x128_f8f6f4 v[176:179], v[104:111], a[160:167], v[176:179]\n\t" \
  "ds_read_b64_tr_b8 v[120:121], v67 offset:0\n\t" \
  "ds_read_b64_tr_b8 v[122:123], v67 offset:1024\n\t" \
  "v_max_f32 v161, 0, v161\n\t" \
  "v_max_f32 v162, 0, v162\n\t" \
  "v_max_f32 v163, 0, v163\n\t" \
  "v_mfma_f32_16x16x128_f8f6f4 v[180:183], v[104:111], a[168:175], v[180:183]\n\t" \
  "ds_read_b64_tr_b8 v[124:125], v67 offset:8192\n\t" \
  "ds_read_b64_tr_b8 v[126:127], v67 offset:9216\n\t" \
  "buffer_store_dwordx4 v[160:163], v85, s[48:51], s84 offen offset:128 sc1\n\t" \
  "v_pk_mul_f32 v[164:165], v[164:165], s[78:79]\n\t" \
  "v_mfma_f32_16x16x128_f8f6f4 v[184:187], v[104:111], a[176:183], v[184:187]\n\t" \
  "v_pk_mul_f32 v[166:167], v[166:167], s[78:79]\n\t" \
  "v_max_f32 v164, 0, v164\n\t" \
  "v_mfma_f32_16x16x128_f8f6f4 v[188:191], v[104:111], a[184:191], v[188:191]\n\t" \
  "v_max_f32 v165, 0, v165\n\t" \
  "v_max_f32 v166, 0, v166\n\t" \
  "s_waitcnt lgkmcnt(4)\n\t" \
  "v_mfma_f32_16x16x128_f8f6f4 v[176:179], v[112:119], a[192:199], v[176:179]\n\t" \
  "ds_read_b64_tr_b8 v[96:97], v76 offset:32768\n\t" \
  "ds_read_b64_tr_b8 v[98:99], v76 offset:33792\n\t" \
  "v_max_f32 v167, 0, v167\n\t" \
  "buffer_store_dwordx4 v[164:167], v85, s[48:51], s85 offen offset:128 sc1\n\t" \
  "v_mfma_f32_16x16x128_f8f6f4 v[180:183], v[112:119], a[200:207], v[180:183]\n\t" \
  "ds_read_b64_tr_b8 v[100:101], v76 offset:40960\n\t" \
  "ds_read_b64_tr_b8 v[102:103], v76 offset:41984\n\t" \
  "v_pk_mul_f32 v[168:169], v[168:169], s[78:79]\n\t" \
  "v_pk_mul_f32 v[170:171], v[170:171], s[78:79]\n\t" \
  "v_mfma_f32_16x16x128_f8f6f4 v[184:187], v[112:119], a[208:215], v[184:187]\n\t" \
  "v_max_f32 v168, 0, v168\n\t" \
  "v_max_f32 v169, 0, v169\n\t" \
  "v_mfma_f32_16x16x128_f8f6f4 v[188:191], v[112:119], a[216:223], v[188:191]\n\t" \
  "v_max_f32 v170, 0, v170\n\t" \
  "v_max_f32 v171, 0, v171\n\t" \
  "s_waitcnt lgkmcnt(4)\n\t" \
  "v_mfma_f32_16x16x128_f8f6f4 v[176:179], v[120:127], a[224:231], v[176:179]\n\t" \
  "ds_read_b64_tr_b8 v[104:105], v76 offset:49152\n\t" \
  "ds_read_b64_tr_b8 v[106:107], v76 offset:50176\n\t" \
  "buffer_store_dwordx4 v[168:171], v85, s[48:51], s86 offen offset:128 sc1\n\t" \
  "v_pk_mul_f32 v[172:173], v[172:173], s[78:79]\n\t" \
  "v_mfma_f32_16x16x128_f8f6f4 v[180:183], v[120:127], a[232:239], v[180:183]\n\t" \
  "ds_read_b64_tr_b8 v[108:109], v76 offset:57344\n\t" \
  "ds_read_b64_tr_b8 v[110:111], v76 offset:58368\n\t" \
  "v_pk_mul_f32 v[174:175], v[174:175], s[78:79]\n\t" \
  "v_max_f32 v172, 0, v172\n\t" \
  "v_mfma_f32_16x16x128_f8f6f4 v[184:187], v[120:127], a[240:247], v[184:187]\n\t" \
  "v_max_f32 v173, 0, v173\n\t" \
  "v_max_f32 v174, 0, v174\n\t" \
  "v_mfma_f32_16x16x128_f8f6f4 v[188:191], v[120:127], a[248:255], v[188:191]\n\t" \
  "v_max_f32 v175, 0, v175\n\t" \
  "buffer_store_dwordx4 v[172:175], v85, s[48:51], s87 offen offset:128 sc1\n\t" \
  "s_waitcnt lgkmcnt(4)\n\t" \
  "v_mfma_f32_16x16x128_f8f6f4 v[192:195], v[96:103], a[128:135], v[192:195]\n\t" \
  "ds_read_b64_tr_b8 v[112:113], v68 offset:49152\n\t" \
  "ds_read_b64_tr_b8 v[114:115], v68 offset:50176\n\t" \
  "v_mfma_f32_16x16x128_f8f6f4 v[196:199], v[96:103], a[136:143], v[196:199]\n\t" \
  "ds_read_b64_tr_b8 v[116:117], v68 offset:57344\n\t" \
  "ds_read_b64_tr_b8 v[118:119], v68 offset:58368\n\t" \
  "v_mfma_f32_16x16x128_f8f6f4 v[200:203], v[96:103], a[144:151], v[200:203]\n\t" \
  "v_mfma_f32_16x16x128_f8f6f4 v[204:207], v[96:103], a[152:159], v[204:207]\n\t" \
  "v_pk_mul_f32 v[176:177], v[176:177], s[78:79]\n\t" \
  "v_pk_mul_f32 v[178:179], v[178:179], s[78:79]\n\t" \
  "v_max_f32 v176, 0, v176\n\t" \
  "s_waitcnt lgkmcnt(4)\n\t" \
  "v_mfma_f32_16x16x128_f8f6f4 v[192:195], v[104:111], a[160:167], v[192:195]\n\t" \
  "ds_read_b64_tr_b8 v[120:121], v68 offset:0\n\t" \
  "ds_read_b64_tr_b8 v[122:123], v68 offset:1024\n\t" \
  "v_max_f32 v177, 0, v177\n\t" \
  "v_max_f32 v178, 0, v178\n\t" \
  "v_max_f32 v179, 0, v179\n\t" \
  "v_mfma_f32_16x16x128_f8f6f4 v[196:199], v[104:111], a[168:175], v[196:199]\n\t" \
  "ds_read_b64_tr_b8 v[124:125], v68 offset:8192\n\t" \
  "ds_read_b64_tr_b8 v[126:127], v68 offset:9216\n\t" \
  "buffer_store_dwordx4 v[176:179], v85, s[48:51], s84 offen offset:192 sc1\n\t" \
  "v_pk_mul_f32 v[180:181], v[180:181], s[78:79]\n\t" \
  "v_mfma_f32_16x16x128_f8f6f4 v[200:203], v[104:111], a[176:183], v[200:203]\n\t" \
  "v_pk_mul_f32 v[182:183], v[182:183], s[78:79]\n\t" \
  "v_max_f32 v180, 0, v180\n\t" \
  "v_mfma_f32_16x16x128_f8f6f4 v[204:207], v[104:111], a[184:191], v[204:207]\n\t" \
  "v_max_f32 v181, 0, v181\n\t" \
  "v_max_f32 v182, 0, v182\n\t" \
  "s_waitcnt lgkmcnt(4)\n\t" \
  "v_mfma_f32_16x16x128_f8f6f4 v[192:195], v[112:119], a[192:199], v[192:195]\n\t" \
  "ds_read_b64_tr_b8 v[96:97], v77 offset:32768\n\t" \
  "ds_read_b64_tr_b8 v[98:99], v77 offset:33792\n\t" \
  "v_max_f32 v183, 0, v183\n\t" \
  "buffer_store_dwordx4 v[180:183], v85, s[48:51], s85 offen offset:192 sc1\n\t" \
  "v_mfma_f32_16x16x128_f8f6f4 v[196:199], v[112:119], a[200:207], v[196:199]\n\t" \
  "ds_read_b64_tr_b8 v[100:101], v77 offset:40960\n\t" \
  "ds_read_b64_tr_b8 v[102:103], v77 offset:41984\n\t" \
  "v_pk_mul_f32 v[184:185], v[184:185], s[78:79]\n\t" \
  "v_pk_mul_f32 v[186:187], v[186:187], s[78:79]\n\t" \
  "v_mfma_f32_16x16x128_f8f6f4 v[200:203], v[112:119], a[208:215], v[200:203]\n\t" \
  "v_max_f32 v184, 0, v184\n\t" \
  "v_max_f32 v185, 0, v185\n\t" \
  "v_mfma_f32_16x16x128_f8f6f4 v[204:207], v[112:119], a[216:223], v[204:207]\n\t" \
  "v_max_f32 v186, 0, v186\n\t" \
  "v_max_f32 v187, 0, v187\n\t" \
  "s_waitcnt lgkmcnt(4)\n\t" \
  "v_mfma_f32_16x16x128_f8f6f4 v[192:195], v[120:127], a[224:231], v[192:195]\n\t" \
  "ds_read_b64_tr_b8 v[104:105], v77 offset:49152\n\t" \
  "ds_read_b64_tr_b8 v[106:107], v77 offset:50176\n\t" \
  "buffer_store_dwordx4 v[184:187], v85, s[48:51], s86 offen offset:192 sc1\n\t" \
  "v_pk_mul_f32 v[188:189], v[188:189], s[78:79]\n\t" \
  "v_mfma_f32_16x16x128_f8f6f4 v[196:199], v[120:127], a[232:239], v[196:199]\n\t" \
  "ds_read_b64_tr_b8 v[108:109], v77 offset:57344\n\t" \
  "ds_read_b64_tr_b8 v[110:111], v77 offset:58368\n\t" \
  "v_pk_mul_f32 v[190:191], v[190:191], s[78:79]\n\t" \
  "v_max_f32 v188, 0, v188\n\t" \
  "v_mfma_f32_16x16x128_f8f6f4 v[200:203], v[120:127], a[240:247], v[200:203]\n\t" \
  "v_max_f32 v189, 0, v189\n\t" \
  "v_max_f32 v190, 0, v190\n\t" \
  "v_mfma_f32_16x16x128_f8f6f4 v[204:207], v[120:127], a[248:255], v[204:207]\n\t" \
  "v_max_f32 v191, 0, v191\n\t" \
  "buffer_store_dwordx4 v[188:191], v85, s[48:51], s87 offen offset:192 sc1\n\t" \
  "s_waitcnt lgkmcnt(4)\n\t" \
  "v_mfma_f32_16x16x128_f8f6f4 v[208:211], v[96:103], a[128:135], v[208:211]\n\t" \
  "ds_read_b64_tr_b8 v[112:113], v69 offset:49152\n\t" \
  "ds_read_b64_tr_b8 v[114:115], v69 offset:50176\n\t" \
  "v_mfma_f32_16x16x128_f8f6f4 v[212:215], v[96:103], a[136:143], v[212:215]\n\t" \
  "ds_read_b64_tr_b8 v[116:117], v69 offset:57344\n\t" \
  "ds_read_b64_tr_b8 v[118:119], v69 offset:58368\n\t" \
  "v_mfma_f32_16x16x128_f8f6f4 v[216:219], v[96:103], a[144:151], v[216:219]\n\t" \
  "v_mfma_f32_16x16x128_f8f6f4 v[220:223], v[96:103], a[152:159], v[220:223]\n\t" \
  "v_pk_mul_f32 v[192:193], v[192:193], s[78:79]\n\t" \
  "v_pk_mul_f32 v[194:195], v[194:195], s[78:79]\n\t" \
  "v_max_f32 v192, 0, v192\n\t" \
  "s_waitcnt lgkmcnt(4)\n\t" \
  "v_mfma_f32_16x16x128_f8f6f4 v[208:211], v[104:111], a[160:167], v[208:211]\n\t" \
  "ds_read_b64_tr_b8 v[120:121], v69 offset:0\n\t" \
  "ds_read_b64_tr_b8 v[122:123], v69 offset:1024\n\t" \
  "v_max_f32 v193, 0, v193\n\t" \
  "v_max_f32 v194, 0, v194\n\t" \
  "v_max_f32 v195, 0, v195\n\t" \
  "v_mfma_f32_16x16x128_f8f6f4 v[212:215], v[104:111], a[168:175], v[212:215]\n\t" \
  "ds_read_b64_tr_b8 v[124:125], v69 offset:8192\n\t" \
  "ds_read_b64_tr_b8 v[126:127], v69 offset:9216\n\t" \
  "buffer_store_dwordx4 v[192:195], v85, s[48:51], s84 offen offset:256 sc1\n\t" \
  "v_pk_mul_f32 v[196:197], v[196:197], s[78:79]\n\t" \
  "v_mfma_f32_16x16x128_f8f6f4 v[216:219], v[104:111], a[176:183], v[216:219]\n\t" \
  "v_pk_mul_f32 v[198:199], v[198:199], s[78:79]\n\t" \
  "v_max_f32 v196, 0, v196\n\t" \
  "v_mfma_f32_16x16x128_f8f6f4 v[220:223], v[104:111], a[184:191], v[220:223]\n\t" \
  "v_max_f32 v197, 0, v197\n\t" \
  "v_max_f32 v198, 0, v198\n\t" \
  "s_waitcnt lgkmcnt(4)\n\t" \
  "v_mfma_f32_16x16x128_f8f6f4 v[208:211], v[112:119], a[192:199], v[208:211]\n\t" \
  "ds_read_b64_tr_b8 v[96:97], v78 offset:32768\n\t" \
  "ds_read_b64_tr_b8 v[98:99], v78 offset:33792\n\t" \
  "v_max_f32 v199, 0, v199\n\t" \
  "buffer_store_dwordx4 v[196:199], v85, s[48:51], s85 offen offset:256 sc1\n\t" \
  "v_mfma_f32_16x16x128_f8f6f4 v[212:215], v[112:119], a[200:207], v[212:215]\n\t" \
  "ds_read_b64_tr_b8 v[100:101], v78 offset:40960\n\t" \
  "ds_read_b64_tr_b8 v[102:103], v78 offset:41984\n\t" \
  "v_pk_mul_f32 v[200:201], v[200:201], s[78:79]\n\t" \
  "v_pk_mul_f32 v[202:203], v[202:203], s[78:79]\n\t" \
  "v_mfma_f32_16x16x128_f8f6f4 v[216:219], v[112:119], a[208:215], v[216:219]\n\t" \
  "v_max_f32 v200, 0, v200\n\t" \
  "v_max_f32 v201, 0, v201\n\t" \
  "v_mfma_f32_16x16x128_f8f6f4 v[220:223], v[112:119], a[216:223], v[220:223]\n\t" \
  "v_max_f32 v202, 0, v202\n\t" \
  "v_max_f32 v203, 0, v203\n\t" \
  "s_waitcnt lgkmcnt(4)\n\t" \
  "v_mfma_f32_16x16x128_f8f6f4 v[208:211], v[120:127], a[224:231], v[208:211]\n\t" \
  "ds_read_b64_tr_b8 v[104:105], v78 offset:49152\n\t" \
  "ds_read_b64_tr_b8 v[106:107], v78 offset:50176\n\t" \
  "buffer_store_dwordx4 v[200:203], v85, s[48:51], s86 offen offset:256 sc1\n\t" \
  "v_pk_mul_f32 v[204:205], v[204:205], s[78:79]\n\t" \
  "v_mfma_f32_16x16x128_f8f6f4 v[212:215], v[120:127], a[232:239], v[212:215]\n\t" \
  "ds_read_b64_tr_b8 v[108:109], v78 offset:57344\n\t" \
  "ds_read_b64_tr_b8 v[110:111], v78 offset:58368\n\t" \
  "v_pk_mul_f32 v[206:207], v[206:207], s[78:79]\n\t" \
  "v_max_f32 v204, 0, v204\n\t" \
  "v_mfma_f32_16x16x128_f8f6f4 v[216:219], v[120:127], a[240:247], v[216:219]\n\t" \
  "v_max_f32 v205, 0, v205\n\t" \
  "v_max_f32 v206, 0, v206\n\t" \
  "v_mfma_f32_16x16x128_f8f6f4 v[220:223], v[120:127], a[248:255], v[220:223]\n\t" \
  "v_max_f32 v207, 0, v207\n\t" \
  "buffer_store_dwordx4 v[204:207], v85, s[48:51], s87 offen offset:256 sc1\n\t" \
  "s_waitcnt lgkmcnt(4)\n\t" \
  "v_mfma_f32_16x16x128_f8f6f4 v[224:227], v[96:103], a[128:135], v[224:227]\n\t" \
  "ds_read_b64_tr_b8 v[112:113], v70 offset:49152\n\t" \
  "ds_read_b64_tr_b8 v[114:115], v70 offset:50176\n\t" \
  "v_mfma_f32_16x16x128_f8f6f4 v[228:231], v[96:103], a[136:143], v[228:231]\n\t" \
  "ds_read_b64_tr_b8 v[116:117], v70 offset:57344\n\t" \
  "ds_read_b64_tr_b8 v[118:119], v70 offset:58368\n\t" \
  "v_mfma_f32_16x16x128_f8f6f4 v[232:235], v[96:103], a[144:151], v[232:235]\n\t" \
  "v_mfma_f32_16x16x128_f8f6f4 v[236:239], v[96:103], a[152:159], v[236:239]\n\t" \
  "v_pk_mul_f32 v[208:209], v[208:209], s[78:79]\n\t" \
  "v_pk_mul_f32 v[210:211], v[210:211], s[78:79]\n\t" \
  "v_max_f32 v208, 0, v208\n\t" \
  "s_waitcnt lgkmcnt(4)\n\t" \
  "v_mfma_f32_16x16x128_f8f6f4 v[224:227], v[104:111], a[160:167], v[224:227]\n\t" \
  "ds_read_b64_tr_b8 v[120:121], v70 offset:0\n\t" \
  "ds_read_b64_tr_b8 v[122:123], v70 offset:1024\n\t" \
  "v_max_f32 v209, 0, v209\n\t" \
  "v_max_f32 v210, 0, v210\n\t" \
  "v_max_f32 v211, 0, v211\n\t" \
  "v_mfma_f32_16x16x128_f8f6f4 v[228:231], v[104:111], a[168:175], v[228:231]\n\t" \
  "ds_read_b64_tr_b8 v[124:125], v70 offset:8192\n\t" \
  "ds_read_b64_tr_b8 v[126:127], v70 offset:9216\n\t" \
  "buffer_store_dwordx4 v[208:211], v85, s[48:51], s84 offen offset:320 sc1\n\t" \
  "v_pk_mul_f32 v[212:213], v[212:213], s[78:79]\n\t" \
  "v_mfma_f32_16x16x128_f8f6f4 v[232:235], v[104:111], a[176:183], v[232:235]\n\t" \
  "v_pk_mul_f32 v[214:215], v[214:215], s[78:79]\n\t" \
  "v_max_f32 v212, 0, v212\n\t" \
  "v_mfma_f32_16x16x128_f8f6f4 v[236:239], v[104:111], a[184:191], v[236:239]\n\t" \
  "v_max_f32 v213, 0, v213\n\t" \
  "v_max_f32 v214, 0, v214\n\t" \
  "s_waitcnt lgkmcnt(4)\n\t" \
  "v_mfma_f32_16x16x128_f8f6f4 v[224:227], v[112:119], a[192:199], v[224:227]\n\t" \
  "ds_read_b64_tr_b8 v[96:97], v79 offset:32768\n\t" \
  "ds_read_b64_tr_b8 v[98:99], v79 offset:33792\n\t" \
  "v_max_f32 v215, 0, v215\n\t" \
  "buffer_store_dwordx4 v[212:215], v85, s[48:51], s85 offen offset:320 sc1\n\t" \
  "v_mfma_f32_16x16x128_f8f6f4 v[228:231], v[112:119], a[200:207], v[228:231]\n\t" \
  "ds_read_b64_tr_b8 v[100:101], v79 offset:40960\n\t" \
  "ds_read_b64_tr_b8 v[102:103], v79 offset:41984\n\t" \
  "v_pk_mul_f32 v[216:217], v[216:217], s[78:79]\n\t" \
  "v_pk_mul_f32 v[218:219], v[218:219], s[78:79]\n\t" \
  "v_mfma_f32_16x16x128_f8f6f4 v[232:235], v[112:119], a[208:215], v[232:235]\n\t" \
  "v_max_f32 v216, 0, v216\n\t" \
  "v_max_f32 v217, 0, v217\n\t" \
  "v_mfma_f32_16x16x128_f8f6f4 v[236:239], v[112:119], a[216:223], v[236:239]\n\t" \
  "v_max_f32 v218, 0, v218\n\t" \
  "v_max_f32 v219, 0, v219\n\t" \
  "s_waitcnt lgkmcnt(4)\n\t" \
  "v_mfma_f32_16x16x128_f8f6f4 v[224:227], v[120:127], a[224:231], v[224:227]\n\t" \
  "ds_read_b64_tr_b8 v[104:105], v79 offset:49152\n\t" \
  "ds_read_b64_tr_b8 v[106:107], v79 offset:50176\n\t" \
  "buffer_store_dwordx4 v[216:219], v85, s[48:51], s86 offen offset:320 sc1\n\t" \
  "v_pk_mul_f32 v[220:221], v[220:221], s[78:79]\n\t" \
  "v_mfma_f32_16x16x128_f8f6f4 v[228:231], v[120:127], a[232:239], v[228:231]\n\t" \
  "ds_read_b64_tr_b8 v[108:109], v79 offset:57344\n\t" \
  "ds_read_b64_tr_b8 v[110:111], v79 offset:58368\n\t" \
  "v_pk_mul_f32 v[222:223], v[222:223], s[78:79]\n\t" \
  "v_max_f32 v220, 0, v220\n\t" \
  "v_mfma_f32_16x16x128_f8f6f4 v[232:235], v[120:127], a[240:247], v[232:235]\n\t" \
  "v_max_f32 v221, 0, v221\n\t" \
  "v_max_f32 v222, 0, v222\n\t" \
  "v_mfma_f32_16x16x128_f8f6f4 v[236:239], v[120:127], a[248:255], v[236:239]\n\t" \
  "v_max_f32 v223, 0, v223\n\t" \
  "buffer_store_dwordx4 v[220:223], v85, s[48:51], s87 offen offset:320 sc1\n\t" \
  "s_waitcnt lgkmcnt(4)\n\t" \
  "v_mfma_f32_16x16x128_f8f6f4 v[240:243], v[96:103], a[128:135], v[240:243]\n\t" \
  "ds_read_b64_tr_b8 v[112:113], v71 offset:49152\n\t" \
  "ds_read_b64_tr_b8 v[114:115], v71 offset:50176\n\t" \
  "v_mfma_f32_16x16x128_f8f6f4 v[244:247], v[96:103], a[136:143], v[244:247]\n\t" \
  "ds_read_b64_tr_b8 v[116:117], v71 offset:57344\n\t" \
  "ds_read_b64_tr_b8 v[118:119], v71 offset:58368\n\t" \
  "v_mfma_f32_16x16x128_f8f6f4 v[248:251], v[96:103], a[144:151], v[248:251]\n\t" \
  "v_mfma_f32_16x16x128_f8f6f4 v[252:255], v[96:103], a[152:159], v[252:255]\n\t" \
  "v_pk_mul_f32 v[224:225], v[224:225], s[78:79]\n\t" \
  "v_pk_mul_f32 v[226:227], v[226:227], s[78:79]\n\t" \
  "v_max_f32 v224, 0, v224\n\t" \
  "s_waitcnt lgkmcnt(4)\n\t" \
  "v_mfma_f32_16x16x128_f8f6f4 v[240:243], v[104:111], a[160:167], v[240:243]\n\t" \
  "ds_read_b64_tr_b8 v[120:121], v71 offset:0\n\t" \
  "ds_read_b64_tr_b8 v[122:123], v71 offset:1024\n\t" \
  "v_max_f32 v225, 0, v225\n\t" \
  "v_max_f32 v226, 0, v226\n\t" \
  "v_max_f32 v227, 0, v227\n\t" \
  "v_mfma_f32_16x16x128_f8f6f4 v[244:247], v[104:111], a[168:175], v[244:247]\n\t" \
  "ds_read_b64_tr_b8 v[124:125], v71 offset:8192\n\t" \
  "ds_read_b64_tr_b8 v[126:127], v71 offset:9216\n\t" \
  "buffer_store_dwordx4 v[224:227], v85, s[48:51], s84 offen offset:384 sc1\n\t" \
  "v_pk_mul_f32 v[228:229], v[228:229], s[78:79]\n\t" \
  "v_mfma_f32_16x16x128_f8f6f4 v[248:251], v[104:111], a[176:183], v[248:251]\n\t" \
  "v_pk_mul_f32 v[230:231], v[230:231], s[78:79]\n\t" \
  "v_max_f32 v228, 0, v228\n\t" \
  "v_mfma_f32_16x16x128_f8f6f4 v[252:255], v[104:111], a[184:191], v[252:255]\n\t" \
  "v_max_f32 v229, 0, v229\n\t" \
  "v_max_f32 v230, 0, v230\n\t" \
  "s_waitcnt lgkmcnt(4)\n\t" \
  "v_mfma_f32_16x16x128_f8f6f4 v[240:243], v[112:119], a[192:199], v[240:243]\n\t" \
  "v_max_f32 v231, 0, v231\n\t" \
  "buffer_store_dwordx4 v[228:231], v85, s[48:51], s85 offen offset:384 sc1\n\t" \
  "v_mfma_f32_16x16x128_f8f6f4 v[244:247], v[112:119], a[200:207], v[244:247]\n\t" \
  "v_pk_mul_f32 v[232:233], v[232:233], s[78:79]\n\t" \
  "v_pk_mul_f32 v[234:235], v[234:235], s[78:79]\n\t" \
  "v_mfma_f32_16x16x128_f8f6f4 v[248:251], v[112:119], a[208:215], v[248:251]\n\t" \
  "v_max_f32 v232, 0, v232\n\t" \
  "v_max_f32 v233, 0, v233\n\t" \
  "v_mfma_f32_16x16x128_f8f6f4 v[252:255], v[112:119], a[216:223], v[252:255]\n\t" \
  "v_max_f32 v234, 0, v234\n\t" \
  "v_max_f32 v235, 0, v235\n\t" \
  "s_waitcnt lgkmcnt(0)\n\t" \
  "v_mfma_f32_16x16x128_f8f6f4 v[240:243], v[120:127], a[224:231], v[240:243]\n\t" \
  "buffer_store_dwordx4 v[232:235], v85, s[48:51], s86 offen offset:384 sc1\n\t" \
  "v_pk_mul_f32 v[236:237], v[236:237], s[78:79]\n\t" \
  "v_mfma_f32_16x16x128_f8f6f4 v[244:247], v[120:127], a[232:239], v[244:247]\n\t" \
  "v_pk_mul_f32 v[238:239], v[238:239], s[78:79]\n\t" \
  "v_max_f32 v236, 0, v236\n\t" \
  "v_mfma_f32_16x16x128_f8f6f4 v[248:251], v[120:127], a[240:247], v[248:251]\n\t" \
  "v_max_f32 v237, 0, v237\n\t" \
  "v_max_f32 v238, 0, v238\n\t" \
  "v_mfma_f32_16x16x128_f8f6f4 v[252:255], v[120:127], a[248:255], v[252:255]\n\t" \
  "v_max_f32 v239, 0, v239\n\t" \
  "buffer_store_dwordx4 v[236:239], v85, s[48:51], s87 offen offset:384 sc1\n\t" \
  "s_nop 15\n\t" \
  "v_pk_mul_f32 v[240:241], v[240:241], s[78:79]\n\t" \
  "v_pk_mul_f32 v[242:243], v[242:243], s[78:79]\n\t" \
  "v_max_f32 v240, 0, v240\n\t" \
  "v_max_f32 v241, 0, v241\n\t" \
  "v_max_f32 v242, 0, v242\n\t" \
  "v_max_f32 v243, 0, v243\n\t" \
  "buffer_store_dwordx4 v[240:243], v85, s[48:51], s84 offen offset:448 sc1\n\t" \
  "v_pk_mul_f32 v[244:245], v[244:245], s[78:79]\n\t" \
  "v_pk_mul_f32 v[246:247], v[246:247], s[78:79]\n\t" \
  "v_max_f32 v244, 0, v244\n\t" \
  "v_max_f32 v245, 0, v245\n\t" \
  "v_max_f32 v246, 0, v246\n\t" \
  "v_max_f32 v247, 0, v247\n\t" \
  "buffer_store_dwordx4 v[244:247], v85, s[48:51], s85 offen offset:448 sc1\n\t" \
  "v_pk_mul_f32 v[248:249], v[248:249], s[78:79]\n\t" \
  "v_pk_mul_f32 v[250:251], v[250:251], s[78:79]\n\t" \
  "v_max_f32 v248, 0, v248\n\t" \
  "v_max_f32 v249, 0, v249\n\t" \
  "v_max_f32 v250, 0, v250\n\t" \
  "v_max_f32 v251, 0, v251\n\t" \
  "buffer_store_dwordx4 v[248:251], v85, s[48:51], s86 offen offset:448 sc1\n\t" \
  "v_pk_mul_f32 v[252:253], v[252:253], s[78:79]\n\t" \
  "v_pk_mul_f32 v[254:255], v[254:255], s[78:79]\n\t" \
  "v_max_f32 v252, 0, v252\n\t" \
  "v_max_f32 v253, 0, v253\n\t" \
  "v_max_f32 v254, 0, v254\n\t" \
  "v_max_f32 v255, 0, v255\n\t" \
  "buffer_store_dwordx4 v[252:255], v85, s[48:51], s87 offen offset:448 sc1\n\t" \
  "s_branch FIN_%=\n\t" \
  "BADOUT_%=:\n\t" \
  "s_waitcnt vmcnt(0) lgkmcnt(0)\n\t" \
  "s_nop 15\n\t" \
  "v_mov_b32 v128, 0x7fc00000\n\t" \
  "v_mov_b32 v129, 0x7fc00000\n\t" \
  "v_mov_b32 v130, 0x7fc00000\n\t" \
  "v_mov_b32 v131, 0x7fc00000\n\t" \
  "s_mov_b32 s84, 0\n\t" \
  "buffer_store_dwordx4 v[128:131], v85, s[48:51], s84 offen offset:0 sc1\n\t" \
  "buffer_store_dwordx4 v[128:131], v85, s[48:51], s84 offen offset:64 sc1\n\t" \
  "buffer_store_dwordx4 v[128:131], v85, s[48:51], s84 offen offset:128 sc1\n\t" \
  "buffer_store_dwordx4 v[128:131], v85, s[48:51], s84 offen offset:192 sc1\n\t" \
  "buffer_store_dwordx4 v[128:131], v85, s[48:51], s84 offen offset:256 sc1\n\t" \
  "buffer_store_dwordx4 v[128:131], v85, s[48:51], s84 offen offset:320 sc1\n\t" \
  "buffer_store_dwordx4 v[128:131], v85, s[48:51], s84 offen offset:384 sc1\n\t" \
  "buffer_store_dwordx4 v[128:131], v85, s[48:51], s84 offen offset:448 sc1\n\t" \
  "s_mov_b32 s84, 1048576\n\t" \
  "buffer_store_dwordx4 v[128:131], v85, s[48:51], s84 offen offset:0 sc1\n\t" \
  "buffer_store_dwordx4 v[128:131], v85, s[48:51], s84 offen offset:64 sc1\n\t" \
  "buffer_store_dwordx4 v[128:131], v85, s[48:51], s84 offen offset:128 sc1\n\t" \
  "buffer_store_dwordx4 v[128:131], v85, s[48:51], s84 offen offset:192 sc1\n\t" \
  "buffer_store_dwordx4 v[128:131], v85, s[48:51], s84 offen offset:256 sc1\n\t" \
  "buffer_store_dwordx4 v[128:131], v85, s[48:51], s84 offen offset:320 sc1\n\t" \
  "buffer_store_dwordx4 v[128:131], v85, s[48:51], s84 offen offset:384 sc1\n\t" \
  "buffer_store_dwordx4 v[128:131], v85, s[48:51], s84 offen offset:448 sc1\n\t" \
  "s_mov_b32 s84, 2097152\n\t" \
  "buffer_store_dwordx4 v[128:131], v85, s[48:51], s84 offen offset:0 sc1\n\t" \
  "buffer_store_dwordx4 v[128:131], v85, s[48:51], s84 offen offset:64 sc1\n\t" \
  "buffer_store_dwordx4 v[128:131], v85, s[48:51], s84 offen offset:128 sc1\n\t" \
  "buffer_store_dwordx4 v[128:131], v85, s[48:51], s84 offen offset:192 sc1\n\t" \
  "buffer_store_dwordx4 v[128:131], v85, s[48:51], s84 offen offset:256 sc1\n\t" \
  "buffer_store_dwordx4 v[128:131], v85, s[48:51], s84 offen offset:320 sc1\n\t" \
  "buffer_store_dwordx4 v[128:131], v85, s[48:51], s84 offen offset:384 sc1\n\t" \
  "buffer_store_dwordx4 v[128:131], v85, s[48:51], s84 offen offset:448 sc1\n\t" \
  "s_mov_b32 s84, 3145728\n\t" \
  "buffer_store_dwordx4 v[128:131], v85, s[48:51], s84 offen offset:0 sc1\n\t" \
  "buffer_store_dwordx4 v[128:131], v85, s[48:51], s84 offen offset:64 sc1\n\t" \
  "buffer_store_dwordx4 v[128:131], v85, s[48:51], s84 offen offset:128 sc1\n\t" \
  "buffer_store_dwordx4 v[128:131], v85, s[48:51], s84 offen offset:192 sc1\n\t" \
  "buffer_store_dwordx4 v[128:131], v85, s[48:51], s84 offen offset:256 sc1\n\t" \
  "buffer_store_dwordx4 v[128:131], v85, s[48:51], s84 offen offset:320 sc1\n\t" \
  "buffer_store_dwordx4 v[128:131], v85, s[48:51], s84 offen offset:384 sc1\n\t" \
  "buffer_store_dwordx4 v[128:131], v85, s[48:51], s84 offen offset:448 sc1\n\t" \
  "s_branch FIN_%=\n\t" \
  "POLL_%=:\n\t" \
  "s_memrealtime s[80:81]\n\t" \
  "PLOOP_%=:\n\t" \
  "global_load_dword v89, v87, s[70:71] sc0 sc1\n\t" \
  "global_load_dword v94, v88, s[98:99] sc1\n\t" \
  "s_waitcnt vmcnt(0) lgkmcnt(0)\n\t" \
  "v_cmp_le_u32 vcc, s64, v89\n\t" \
  "v_readfirstlane_b32 s96, v94\n\t" \
  "s_nop 3\n\t" \
  "s_or_b32 s65, s65, vcc_lo\n\t" \
  "s_bitcmp1_b32 s65, s67\n\t" \
  "s_cbranch_scc1 PDONE_%=\n\t" \
  "s_cmp_eq_u32 s96, s64\n\t" \
  "s_cbranch_scc1 PTMO_%=\n\t" \
  "s_memrealtime s[82:83]\n\t" \
  "s_waitcnt lgkmcnt(0)\n\t" \
  "s_sub_u32 s96, s82, s80\n\t" \
  "s_subb_u32 s97, s83, s81\n\t" \
  "s_cmp_lg_u32 s97, 0\n\t" \
  "s_cbranch_scc1 PTMO_%=\n\t" \
  "s_cmp_gt_u32 s96, 30000000\n\t" \
  "s_cbranch_scc1 PTMO_%=\n\t" \
  "s_sleep 2\n\t" \
  "s_branch PLOOP_%=\n\t" \
  "PTMO_%=:\n\t" \
  "v_mov_b32 v89, s64\n\t" \
  "s_mov_b64 exec, 1\n\t" \
  "global_store_dword v88, v89, s[98:99] sc1\n\t" \
  "s_mov_b64 exec, -1\n\t" \
  "s_mov_b32 s66, 1\n\t" \
  "s_mov_b32 s65, -1\n\t" \
  "PDONE_%=:\n\t" \
  "s_setpc_b64 s[68:69]\n\t" \
  "FIN_%=:\n\t" \
  "s_cmp_lg_u32 s4, 0\n\t" \
  "s_cbranch_scc1 DONE_%=\n\t" \
  "v_mov_b32 v89, s64\n\t" \
  "s_mov_b64 exec, 1\n\t" \
  "global_store_dword v88, v89, s[92:93] sc1\n\t" \
  "s_mov_b64 exec, -1\n\t" \
  "DONE_%=:\n\t"
#define GEMM_ASM_CLOB "v2", "v3", "v4", "v5", "v6", "v7", "v8", "v9", "v10", "v11", "v12", "v13", "v14", "v15", "v16", "v17", "v18", "v19", "v20", "v21", "v22", "v23", "v24", "v25", "v26", "v27", "v28", "v29", "v30", "v31", "v32", "v33", "v34", "v35", "v36", "v37", "v38", "v39", "v40", "v41", "v42", "v43", "v44", "v45", "v46", "v47", "v48", "v49", "v50", "v51", "v52", "v53", "v54", "v55", "v56", "v57", "v58", "v59", "v60", "v61", "v62", "v63", "v64", "v65", "v66", "v67", "v68", "v69", "v70", "v71", "v72", "v73", "v74", "v75", "v76", "v77", "v78", "v79", "v80", "v81", "v82", "v83", "v84", "v85", "v86", "v87", "v88", "v89", "v90", "v91", "v92", "v93", "v94", "v95", "v96", "v97", "v98", "v99", "v100", "v101", "v102", "v103", "v104", "v105", "v106", "v107", "v108", "v109", "v110", "v111", "v112", "v113", "v114", "v115", "v116", "v117", "v118", "v119", "v120", "v121", "v122", "v123", "v124", "v125", "v126", "v127", "v128", "v129", "v130", "v131", "v132", "v133", "v134", "v135", "v136", "v137", "v138", "v139", "v140", "v141", "v142", "v143", "v144", "v145", "v146", "v147", "v148", "v149", "v150", "v151", "v152", "v153", "v154", "v155", "v156", "v157", "v158", "v159", "v160", "v161", "v162", "v163", "v164", "v165", "v166", "v167", "v168", "v169", "v170", "v171", "v172", "v173", "v174", "v175", "v176", "v177", "v178", "v179", "v180", "v181", "v182", "v183", "v184", "v185", "v186", "v187", "v188", "v189", "v190", "v191", "v192", "v193", "v194", "v195", "v196", "v197", "v198", "v199", "v200", "v201", "v202", "v203", "v204", "v205", "v206", "v207", "v208", "v209", "v210", "v211", "v212", "v213", "v214", "v215", "v216", "v217", "v218", "v219", "v220", "v221", "v222", "v223", "v224", "v225", "v226", "v227", "v228", "v229", "v230", "v231", "v232", "v233", "v234", "v235", "v236", "v237", "v238", "v239", "v240", "v241", "v242", "v243", "v244", "v245", "v246", "v247", "v248", "v249", "v250", "v251", "v252", "v253", "v254", "v255", "a0", "a1", "a2", "a3", "a4", "a5", "a6", "a7", "a8", "a9", "a10", "a11", "a12", "a13", "a14", "a15", "a16", "a17", "a18", "a19", "a20", "a21", "a22", "a23", "a24", "a25", "a26", "a27", "a28", "a29", "a30", "a31", "a32", "a33", "a34", "a35", "a36", "a37", "a38", "a39", "a40", "a41", "a42", "a43", "a44", "a45", "a46", "a47", "a48", "a49", "a50", "a51", "a52", "a53", "a54", "a55", "a56", "a57", "a58", "a59", "a60", "a61", "a62", "a63", "a64", "a65", "a66", "a67", "a68", "a69", "a70", "a71", "a72", "a73", "a74", "a75", "a76", "a77", "a78", "a79", "a80", "a81", "a82", "a83", "a84", "a85", "a86", "a87", "a88", "a89", "a90", "a91", "a92", "a93", "a94", "a95", "a96", "a97", "a98", "a99", "a100", "a101", "a102", "a103", "a104", "a105", "a106", "a107", "a108", "a109", "a110", "a111", "a112", "a113", "a114", "a115", "a116", "a117", "a118", "a119", "a120", "a121", "a122", "a123", "a124", "a125", "a126", "a127", "a128", "a129", "a130", "a131", "a132", "a133", "a134", "a135", "a136", "a137", "a138", "a139", "a140", "a141", "a142", "a143", "a144", "a145", "a146", "a147", "a148", "a149", "a150", "a151", "a152", "a153", "a154", "a155", "a156", "a157", "a158", "a159", "a160", "a161", "a162", "a163", "a164", "a165", "a166", "a167", "a168", "a169", "a170", "a171", "a172", "a173", "a174", "a175", "a176", "a177", "a178", "a179", "a180", "a181", "a182", "a183", "a184", "a185", "a186", "a187", "a188", "a189", "a190", "a191", "a192", "a193", "a194", "a195", "a196", "a197", "a198", "a199", "a200", "a201", "a202", "a203", "a204", "a205", "a206", "a207", "a208", "a209", "a210", "a211", "a212", "a213", "a214", "a215", "a216", "a217", "a218", "a219", "a220", "a221", "a222", "a223", "a224", "a225", "a226", "a227", "a228", "a229", "a230", "a231", "a232", "a233", "a234", "a235", "a236", "a237", "a238", "a239", "a240", "a241", "a242", "a243", "a244", "a245", "a246", "a247", "a248", "a249", "a250", "a251", "a252", "a253", "a254", "a255", "s3", "s4", "s5", "s6", "s7", "s8", "s9", "s10", "s11", "s12", "s13", "s14", "s15", "s16", "s17", "s18", "s19", "s20", "s21", "s22", "s23", "s24", "s25", "s26", "s27", "s28", "s29", "s30", "s31", "s34", "s35", "s36", "s37", "s38", "s39", "s40", "s41", "s42", "s43", "s44", "s45", "s46", "s47", "s48", "s49", "s50", "s51", "s52", "s53", "s54", "s55", "s56", "s57", "s58", "s59", "s60", "s61", "s62", "s63", "s64", "s65", "s66", "s67", "s68", "s69", "s70", "s71", "s72", "s73", "s74", "s75", "s76", "s77", "s78", "s79", "s80", "s81", "s82", "s83", "s84", "s85", "s86", "s87", "s88", "s89", "s90", "s91", "s92", "s93", "s94", "s95", "s96", "s97", "s98", "s99", "memory", "scc", "vcc"

__global__ __launch_bounds__(256, 1) void a2a_gemm_kernel(Args a) {
  extern __shared__ __attribute__((aligned(16))) unsigned char smem[];
  if ((unsigned)(size_t)(LAS unsigned char*)smem != 0u) return;
  const unsigned long long karg = (unsigned long long)__builtin_amdgcn_kernarg_segment_ptr();
  const unsigned bid = blockIdx.x, tid = threadIdx.x;
  asm volatile(GEMM_ASM_TEXT
               :
               : [karg] "s"(karg), [bid] "s"(bid), [tid] "v"(tid)
               : GEMM_ASM_CLOB);
}

extern "C" void kernel_launch(int rank, int nranks, ncclComm_t comm, hipStream_t stream,
                              void* const* d_in, const int* in_sizes, int n_in,
                              void* d_out, int out_size,
                              const int* in_dtypes, int out_dtype,
                              void* const* d_out_all,
                              void* const* d_ws_all, size_t ws_bytes) {
  (void)comm; (void)in_dtypes; (void)out_dtype; (void)d_out_all;
  if (nranks != NRANKS || n_in != 4 || rank < 0 || rank >= NRANKS || ws_bytes < WS_NEEDED || out_size != MLOC * NN ||
      in_sizes[0] != KK * MLOC || in_sizes[1] != KK * NN) {
    fprintf(stderr, "kernel_launch: unexpected problem shape\n");
    return;
  }
  static bool attr_done = false;
  if (!attr_done) {
    hipError_t e = hipFuncSetAttribute((const void*)a2a_gemm_kernel, hipFuncAttributeMaxDynamicSharedMemorySize, LDS_BYTES);
    if (e != hipSuccess) { fprintf(stderr, "hipFuncSetAttribute: %s\n", hipGetErrorString(e)); return; }
    attr_done = true;
  }
  Args a;
  a.x = (const unsigned char*)d_in[0]; a.w = (const unsigned char*)d_in[1];
  a.sx = (const float*)d_in[2]; a.sw = (const float*)d_in[3]; a.out = (float*)d_out;
  for (int p = 0; p < NRANKS; ++p) a.ws[p] = (unsigned char*)d_ws_all[(rank + p) % NRANKS];
  a.pad0 = nullptr; a.rank = rank; a.pad1 = 0;
  hipLaunchKernelGGL(a2a_gemm_kernel, dim3(256), dim3(256), LDS_BYTES, stream, a);
  hipError_t e = hipGetLastError();
  if (e != hipSuccess) fprintf(stderr, "launch: %s\n", hipGetErrorString(e));
}
```
